```python
import numpy as np

kernel_name = "gpu_gemm_ar_m4096_k4096_n32768_f32_silu"


ABSMAX_RTOL = 0.001

COMPUTE_DTYPE = "float32"
EPILOGUE = "silu"
IN_QUANT = None
OUT_QUANT = None
IN_DTYPES = {"x": "float32", "w_mat": "float32"}
WORLD_SIZE = 8

def setup_inputs():

    x = np.random.randn(4096, 4096).astype(np.float32)
    w_mat = (np.random.randn(4096, 32768) / np.sqrt(4096)).astype(np.float32)
    return {"x": x, "w_mat": w_mat}

def _epilogue(y):
    return (y / (1.0 + np.exp(-np.clip(y, -60.0, 60.0)))).astype(np.float32)

def reference(x, w_mat):
    y = (x @ w_mat).astype(np.float32)
    return _epilogue(y)
```

```cpp
#include <hip/hip_runtime.h>
#include <rccl/rccl.h>
#include <cstdio>

#define HIP_CHECK(expr) do { hipError_t _e = (expr); if (_e != hipSuccess) { \
    fprintf(stderr, "HIP error %s at %s:%d\n", hipGetErrorString(_e), __FILE__, __LINE__); \
    return; } } while (0)

#define LAS __attribute__((address_space(3)))
#define GAS __attribute__((address_space(1)))
#define DI __device__ __forceinline__
typedef _Float16 f16x8 __attribute__((ext_vector_type(8)));
typedef float f32x4 __attribute__((ext_vector_type(4)));
typedef unsigned u32x4 __attribute__((ext_vector_type(4)));
typedef GAS unsigned gu32;

#define NG 30
#define DBG 0
namespace {
constexpr int MM = 4096, NN = 32768, KS = 512, NR = 8;
constexpr int NCOMM = 8 * (32 - NG);
constexpr unsigned STEPB = 16384, BLKB = 32768, PANEL = 2u << 20;
constexpr int NPAN = 144;
constexpr unsigned BUFB = NPAN * PANEL;
constexpr int NCB = 16;
constexpr int NBB = 128 - NCB;
constexpr int NSLOT = NBB / 2 + 1;
constexpr int NTILE = 4 * NSLOT;
constexpr size_t OFF_CALL = 0;
constexpr size_t OFF_TMO = 32768;
constexpr size_t OFF_F = 65536;
constexpr size_t OFF_CT = 131072;
constexpr size_t OFF_DONE = 196608;
constexpr size_t OFF_DBG = 262144;
constexpr size_t OFF_DATA = 1u << 20;
constexpr size_t WS_NEED = OFF_DATA + 2 * (size_t)BUFB;
constexpr int LDS_BYTES = 131072 + 4096;
constexpr unsigned long long DEADLINE = 50000000ull;
constexpr int AUX_SYS = 17;

struct Args {
    const float* x; const float* w;
    char* ws[NR]; float* out[NR];
    int rank, pad;
};
typedef const __attribute__((address_space(4))) Args Params;

constexpr int WCNT(int vm, int lgkm) { return (vm & 15) | (7 << 4) | ((lgkm & 15) << 8) | ((vm >> 4) << 14); }
#define SCHED __builtin_amdgcn_sched_barrier(0)
#define WAITVM(n) asm volatile("s_waitcnt vmcnt(" #n ")" ::: "memory")
#define WAITLG0 asm volatile("s_waitcnt lgkmcnt(0)" ::: "memory")
#define BARRIER do { asm volatile("" ::: "memory"); __builtin_amdgcn_s_barrier(); asm volatile("" ::: "memory"); } while (0)

DI unsigned ld_sys(const char* p) { return __hip_atomic_load((gu32*)p, __ATOMIC_RELAXED, __HIP_MEMORY_SCOPE_SYSTEM); }
DI void st_sys(char* p, unsigned v) { __hip_atomic_store((gu32*)p, v, __ATOMIC_RELAXED, __HIP_MEMORY_SCOPE_SYSTEM); }
DI unsigned ld_ag(const char* p) { return __hip_atomic_load((gu32*)p, __ATOMIC_RELAXED, __HIP_MEMORY_SCOPE_AGENT); }
DI void st_ag(char* p, unsigned v) { __hip_atomic_store((gu32*)p, v, __ATOMIC_RELAXED, __HIP_MEMORY_SCOPE_AGENT); }
DI unsigned rfl(unsigned x) { return __builtin_amdgcn_readfirstlane(x); }
DI int lane_now() { int l; asm volatile("v_mbcnt_lo_u32_b32 %0, -1, 0\n\tv_mbcnt_hi_u32_b32 %0, -1, %0" : "=v"(l)); return l; }
DI bool give_up(char* ws, unsigned call, unsigned long long t0) {
    if (rfl(ld_ag(ws + OFF_TMO)) == call || __builtin_amdgcn_s_memrealtime() - t0 > DEADLINE) { st_ag(ws + OFF_TMO, call); return true; }
    return false;
}
#define STAMP(ws, i) do {} while (0)

struct Src { __amdgpu_buffer_rsrc_t rx, rw; unsigned vx, vw; };
DI Src make_src(Params& p, int q, int lane) {
    Src s;
    s.rx = __builtin_amdgcn_make_buffer_rsrc((void*)p.x, 0, MM * KS * 4, 0x00020000);
    s.rw = __builtin_amdgcn_make_buffer_rsrc((void*)p.w, 0, KS * NN * 4, 0x00020000);
    s.vx = ((64 * q + (lane & 15)) * KS + 8 * (lane >> 4)) * 4;
    s.vw = (8 * (lane >> 4) * NN + 64 * q + 4 * (lane & 15)) * 4;
    return s;
}
DI void unit_load(const Src& s, int panel, int st, LAS unsigned char* slot) {
    if (panel < 16) {
        const unsigned s0 = (panel * 256 * KS + 32 * st) * 4;
#pragma unroll
        for (int e = 0; e < 8; ++e) __builtin_amdgcn_raw_ptr_buffer_load_lds(s.rx, (LAS void*)(slot + e * 1024), 16, s.vx, s0 + (e >> 1) * (16 * KS * 4) + (e & 1) * 16, 0, 0);
    } else {
        const unsigned s0 = (32 * st * NN + (panel - 16) * 256) * 4;
#pragma unroll
        for (int e = 0; e < 8; ++e) __builtin_amdgcn_raw_ptr_buffer_load_lds(s.rw, (LAS void*)(slot + e * 1024), 16, s.vw, s0 + e * (NN * 4), 0, 2);
    }
}
DI void unit_cvt(int panel, const LAS unsigned char* slot, int lane, u32x4 (&f)[4]) {
    f32x4 v[8];
#pragma unroll
    for (int e = 0; e < 8; ++e) v[e] = *(const LAS f32x4*)(slot + e * 1024 + lane * 16);
    if (panel < 16) {
#pragma unroll
        for (int j = 0; j < 4; ++j) { f16x8 h;
#pragma unroll
            for (int e = 0; e < 8; ++e) h[e] = (_Float16)v[2 * j + (e >> 2)][e & 3];
            f[j] = __builtin_bit_cast(u32x4, h); }
    } else {
#pragma unroll
        for (int j = 0; j < 4; ++j) { f16x8 h;
#pragma unroll
            for (int e = 0; e < 8; ++e) h[e] = (_Float16)v[e][j];
            f[j] = __builtin_bit_cast(u32x4, h); }
    }
}
DI void store4(const char* base, unsigned voff, const u32x4 (&f)[4]) {
    asm volatile("global_store_dwordx4 %0, %1, %5 sc0 sc1\n\t"
                 "global_store_dwordx4 %0, %2, %5 offset:1024 sc0 sc1\n\t"
                 "global_store_dwordx4 %0, %3, %5 offset:2048 sc0 sc1\n\t"
                 "global_store_dwordx4 %0, %4, %5 offset:3072 sc0 sc1\n\ts_nop 1"
                 :: "v"(voff), "v"(f[0]), "v"(f[1]), "v"(f[2]), "v"(f[3]), "s"(base) : "memory");
}
DI void store1(const char* base, unsigned voff, const u32x4& f) {
    asm volatile("global_store_dwordx4 %0, %1, %2 sc0 sc1\n\ts_nop 1" :: "v"(voff), "v"(f), "s"(base) : "memory");
}
constexpr int G0P = 16 + 14, G0B = G0P * 8, NBLK = (16 + NBB) * 8;
DI void block_of(int b, int& panel, int& kt) {
    if (b < G0B) { kt = b / G0P; panel = b - kt * G0P; return; }
    const int r = b - G0B, q = r >> 7, i = r & 127;
    if (14 + 16 * q + 16 <= NBB) { kt = i >> 4; panel = 16 + 14 + 16 * q + (i & 15); }
    else { constexpr int rem = (NBB - 14) & 15; kt = i / rem; panel = 16 + 14 + 16 * q + (i - kt * rem); }
}

DI void comm_wg(Params& p, LAS unsigned char* lds, unsigned call, int wid, int cw) {
    const int lane = lane_now(), q = wid, rank = p.rank;
    char* const ws = p.ws[rank];
    const Src src = make_src(p, q, lane);
    LAS unsigned char* ring = lds + q * 3 * 8192;
    const unsigned dbase = (unsigned)OFF_DATA + (call & 1) * BUFB + q * 4096 + lane * 16;
    const char* dst[NR];
#pragma unroll
    for (int j = 0; j < NR; ++j) dst[j] = p.ws[(rank + 1 + j + cw + q * 2) & 7];
    char* const myflagdst = p.ws[lane & 7];
    const int nb = (NBLK - cw + NCOMM - 1) / NCOMM, nu = 2 * nb;
    { int pa, kt; block_of(cw, pa, kt); unit_load(src, pa, 2 * kt, ring); unit_load(src, pa, 2 * kt + 1, ring + 8192); }
    WAITVM(8);
    int sl = 0;
    for (int u = 0; u < nu; ++u) {
        int pa, kt, pn, kn;
        block_of(cw + (u >> 1) * NCOMM, pa, kt);
        const int un = u + 2 < nu ? u + 2 : u;
        block_of(cw + (un >> 1) * NCOMM, pn, kn);
        const int sl2 = sl == 0 ? 2 : sl - 1;
        unit_load(src, pn, 2 * kn + (un & 1), ring + sl2 * 8192);
        u32x4 f[4];
        unit_cvt(pa, ring + sl * 8192, lane, f);
        const unsigned voff = dbase + pa * PANEL + (kt * 8 + rank) * BLKB + (u & 1) * STEPB;
#pragma unroll
        for (int j = 0; j < NR; ++j) store4(dst[j], voff, f);
        WAITVM(40);
        BARRIER;
        if (u >= 2 && (u & 1) == 0 && q == 0 && lane < NR) {
            int pp, kp; block_of(cw + ((u >> 1) - 1) * NCOMM, pp, kp);
            st_sys(myflagdst + OFF_F + pp * 256 + (kp * 8 + rank) * 4, call);
        }
        sl = sl == 2 ? 0 : sl + 1;
    }
    WAITVM(0);
    BARRIER;
    if (q == 0 && lane < NR) { int pp, kp; block_of(cw + (nb - 1) * NCOMM, pp, kp); st_sys(myflagdst + OFF_F + pp * 256 + (kp * 8 + rank) * 4, call); }
    STAMP(ws, 1);
    bool bad = false;
    {
        const unsigned long long t0 = __builtin_amdgcn_s_memrealtime();
        while (!__all(ld_ag(ws + OFF_CT + (lane & 31) * 4) >= call)) { if (give_up(ws, call, t0)) { bad = true; break; } __builtin_amdgcn_s_sleep(8); }
        __builtin_amdgcn_fence(__ATOMIC_ACQUIRE, "agent");
        WAITVM(0);
    }
    STAMP(ws, 2);
    {
        const __amdgpu_buffer_rsrc_t ro = __builtin_amdgcn_make_buffer_rsrc((void*)p.out[rank], 0, (unsigned)MM * NN * 4u, 0x00020000);
        const char* od[NR - 1];
#pragma unroll
        for (int j = 0; j < NR - 1; ++j) { int t = j + cw + q * 2; t = t % (NR - 1); od[j] = (const char*)p.out[(rank + 1 + t) & 7]; }
        const int nw = NCOMM * 4, w0 = cw * 4 + q, ni = (MM / 2 - w0 + nw - 1) / nw;
        const unsigned col0 = (NBB + 2 * rank) * 256 * 4;
        auto issue = [&](int i, int s) {
            const unsigned s0 = (unsigned)(w0 + i * nw) * 2u * NN * 4u + col0;
#pragma unroll
            for (int e = 0; e < 4; ++e) __builtin_amdgcn_raw_ptr_buffer_load_lds(ro, (LAS void*)(ring + s * 8192 + e * 1024), 16, lane * 16, s0 + (e >> 1) * (NN * 4) + (e & 1) * 1024, 0, AUX_SYS);
        };
        issue(0, 0); issue(ni > 1 ? 1 : 0, 1);
        WAITVM(4);
        int s = 0;
        for (int i = 0; i < ni; ++i) {
            issue(i + 2 < ni ? i + 2 : i, s == 0 ? 2 : s - 1);
            u32x4 f[4];
#pragma unroll
            for (int e = 0; e < 4; ++e) f[e] = *(const LAS u32x4*)(ring + s * 8192 + e * 1024 + lane * 16);
            if (bad) {
#pragma unroll
                for (int e = 0; e < 4; ++e) f[e] = (u32x4){0x7fc00000u, 0x7fc00000u, 0x7fc00000u, 0x7fc00000u};
            }
            const unsigned v0 = (unsigned)(w0 + i * nw) * 2u * NN * 4u + col0 + lane * 16, v1 = v0 + NN * 4;
#pragma unroll
            for (int j = 0; j < NR - 1; ++j)
                asm volatile("global_store_dwordx4 %0, %2, %6 sc0 sc1\n\t"
                             "global_store_dwordx4 %0, %3, %6 offset:1024 sc0 sc1\n\t"
                             "global_store_dwordx4 %1, %4, %6 sc0 sc1\n\t"
                             "global_store_dwordx4 %1, %5, %6 offset:1024 sc0 sc1\n\ts_nop 1"
                             :: "v"(v0), "v"(v1), "v"(f[0]), "v"(f[1]), "v"(f[2]), "v"(f[3]), "s"(od[j]) : "memory");
            WAITVM(32);
            s = s == 2 ? 0 : s + 1;
        }
    }
    WAITVM(0);
    BARRIER;
    STAMP(ws, 3);
    if (q == 0) {
        if (lane < NR && lane != rank) st_sys(myflagdst + OFF_DONE + (rank * 32 + cw) * 128, call);
        const unsigned long long t0 = __builtin_amdgcn_s_memrealtime();
        const int from = (lane & 7) == rank ? (rank + 1) & 7 : lane & 7;
        while (!__all(ld_sys(ws + OFF_DONE + (from * 32 + cw) * 128) >= call)) {
            if (give_up(ws, call, t0)) { if (lane == 0) p.out[rank][(size_t)(NBB + 2 * rank) * 256] = __builtin_nanf(""); break; }
            __builtin_amdgcn_s_sleep(8);
        }
    }
    STAMP(ws, 4);
}

struct G4 {
    LAS unsigned char* lds;
    __amdgpu_buffer_rsrc_t rD;
    unsigned voff, ldsw;
    unsigned alo, ahi, blo, bhi;
    f32x4 acc[8][8];
    f16x8 A[2][8], B[2][8];
};
template <int ST> DI f16x8 rdA(const G4& g, int i) { return *(const LAS f16x8*)(size_t)((ST < 2 ? g.alo : g.ahi) + (ST & 1) * 32768 + i * 1024); }
template <int ST> DI f16x8 rdB(const G4& g, int i) { return *(const LAS f16x8*)(size_t)((ST < 2 ? g.blo : g.bhi) + (ST & 1) * 32768 + i * 1024); }
template <int ST, int I> DI void dma(const G4& g, unsigned sA, unsigned sB) {
    if constexpr (I < 4) __builtin_amdgcn_raw_ptr_buffer_load_lds(g.rD, (LAS void*)(g.lds + ST * 32768 + g.ldsw), 16, g.voff, sA, I * 1024, AUX_SYS);
    else __builtin_amdgcn_raw_ptr_buffer_load_lds(g.rD, (LAS void*)(g.lds + ST * 32768 + 16384 + g.ldsw), 16, g.voff, sB, (I - 4) * 1024, AUX_SYS);
}
template <int J, bool ZERO, int VM> DI void step(G4& g, unsigned sA, unsigned sB) {
    constexpr int c = J & 1, NX = (J + 1) & 3;
#pragma unroll
    for (int m = 0; m < 8; ++m) {
#pragma unroll
        for (int n = 0; n < 8; ++n) {
            const int q = m * 8 + n;
            if constexpr (ZERO) g.acc[m][n] = __builtin_amdgcn_mfma_f32_16x16x32_f16(g.A[c][m], g.B[c][n], (f32x4){0.f, 0.f, 0.f, 0.f}, 0, 0, 0);
            else g.acc[m][n] = __builtin_amdgcn_mfma_f32_16x16x32_f16(g.A[c][m], g.B[c][n], g.acc[m][n], 0, 0, 0);
            SCHED;
            if (q % 3 == 0 && q / 3 < 16) { const int i = q / 3;
                if (i == 0) g.A[c ^ 1][0] = rdA<NX>(g, 0);
                else if (i <= 8) g.B[c ^ 1][i - 1] = rdB<NX>(g, i - 1);
                else g.A[c ^ 1][i - 8] = rdA<NX>(g, i - 8);
                SCHED; }
            if (q == 1) { dma<J, 4>(g, sA, sB); SCHED; }
            if (q == 8) { dma<J, 5>(g, sA, sB); SCHED; }
            if (q == 16) { dma<J, 6>(g, sA, sB); SCHED; }
            if (q == 23) { dma<J, 7>(g, sA, sB); SCHED; }
            if (q == 31) { dma<J, 0>(g, sA, sB); SCHED; }
            if (q == 38) { dma<J, 1>(g, sA, sB); SCHED; }
            if (q == 46) { dma<J, 2>(g, sA, sB); SCHED; }
            if (q == 53) { dma<J, 3>(g, sA, sB); SCHED; }
        }
    }
    __builtin_amdgcn_s_waitcnt(WCNT(VM, 0));
    __builtin_amdgcn_s_barrier();
    SCHED;
}
DI int lead_ones(bool ok) { const unsigned long long m = ~__ballot(ok); return m ? __builtin_ctzll(m) : 64; }

struct Ready {
    char* ws; unsigned call;
    __amdgpu_buffer_rsrc_t rF; LAS unsigned char* zone;
    int pa, pb, n; bool pending; float one;
    DI void poll(int need) {
        const unsigned long long t0 = __builtin_amdgcn_s_memrealtime();
        const int lane = lane_now();
        for (;;) {
            const unsigned a = ld_sys(ws + OFF_F + pa * 256 + lane * 4), b = ld_sys(ws + OFF_F + pb * 256 + lane * 4);
            n = lead_ones(a >= call && b >= call);
            if (n >= need) break;
            if (give_up(ws, call, t0)) { n = 64; one = __builtin_nanf(""); break; }
            __builtin_amdgcn_s_sleep(16);
        }
    }
    DI void request() {
        const int lane = lane_now();
        __builtin_amdgcn_raw_ptr_buffer_load_lds(rF, (LAS void*)zone, 4, lane * 4, pa * 256, 0, AUX_SYS);
        __builtin_amdgcn_raw_ptr_buffer_load_lds(rF, (LAS void*)(zone + 256), 4, lane * 4, pb * 256, 0, AUX_SYS);
        pending = true;
    }
    DI void check(int need) {
        if (pending) {
            const int lane = lane_now();
            const unsigned a = *(const volatile LAS unsigned*)(zone + lane * 4), b = *(const volatile LAS unsigned*)(zone + 256 + lane * 4);
            n = lead_ones(a >= call && b >= call); pending = false;
        }
        if (n < need) poll(need);
        asm volatile("" ::: "memory");
    }
};

DI float silu1(float y, float one) {
    float t = y * -1.44269504088896f;
    t = __builtin_amdgcn_fmed3f(t, -86.5617024533378f, 86.5617024533378f);
    const float e = __builtin_amdgcn_exp2f(t);
    return y * __builtin_amdgcn_rcpf(one + e);
}

DI void gemm_wg(Params& p, LAS unsigned char* lds, unsigned call, int wid, int grp, int idx) {
    const int lane = lane_now(), wr = wid >> 1, wc = wid & 1, rank = p.rank;
    char* const ws = p.ws[rank];
    {
        const int j = idx * 8 + grp;
        if (j < NCB * 8) {
            const int d = (rank + j) & 7, h = (j >> 3) & 1, kt = j >> 4, pa = 16 + NBB + 2 * d + h;
            const Src src = make_src(p, wid, lane);
            LAS unsigned char* slot = lds + wid * 16384;
            unit_load(src, pa, 2 * kt, slot); unit_load(src, pa, 2 * kt + 1, slot + 8192);
            WAITVM(0);
            const unsigned voff = (unsigned)OFF_DATA + (call & 1) * BUFB + wid * 4096 + lane * 16 + pa * PANEL + (kt * 8 + rank) * BLKB;
            u32x4 f[4];
            unit_cvt(pa, slot, lane, f); store4(p.ws[d], voff, f);
            unit_cvt(pa, slot + 8192, lane, f); store4(p.ws[d], voff + STEPB, f);
            WAITVM(0);
            BARRIER;
            if (wid == 0 && lane == 0) st_sys(p.ws[d] + OFF_F + pa * 256 + (kt * 8 + rank) * 4, call);
        }
    }
    G4 g;
    g.lds = lds;
    g.rD = __builtin_amdgcn_make_buffer_rsrc((void*)(ws + OFF_DATA + (size_t)(call & 1) * BUFB), 0, BUFB, 0x00020000);
    g.voff = wid * 4096 + lane * 16; g.ldsw = wid * 4096;
    const unsigned lbase = (unsigned)(size_t)lds;
    g.alo = lbase + wr * 8192 + lane * 16; g.ahi = g.alo + 65536; g.blo = lbase + 16384 + wc * 8192 + lane * 16; g.bhi = g.blo + 65536;
    asm volatile("" : "+v"(g.alo), "+v"(g.ahi), "+v"(g.blo), "+v"(g.bhi));
    Ready R;
    R.ws = ws; R.call = call; R.rF = __builtin_amdgcn_make_buffer_rsrc((void*)(ws + OFF_F), 0, NPAN * 256, 0x00020000);
    R.zone = lds + 131072 + wid * 512; R.pending = false; R.one = 1.0f;
    const int rg = grp & 3, h = grp >> 2;
    auto tile_of = [&](int j, int& rb, int& cb) { const int s = j >> 2; rb = 4 * rg + (j & 3); cb = s == 0 ? NBB + 2 * rank + h : 2 * (s - 1) + h; };
    int j = idx, pm, pn;
    tile_of(j, pm, pn);
    unsigned cA = pm * PANEL, cB = (16 + pn) * PANEL;
    R.pa = pm; R.pb = 16 + pn;
    R.poll(2);
    asm volatile("" ::: "memory");
    STAMP(ws, 1);
#define DMA8(ST, a, b) dma<ST, 0>(g, a, b); dma<ST, 1>(g, a, b); dma<ST, 2>(g, a, b); dma<ST, 3>(g, a, b); dma<ST, 4>(g, a, b); dma<ST, 5>(g, a, b); dma<ST, 6>(g, a, b); dma<ST, 7>(g, a, b)
    DMA8(0, cA, cB); DMA8(1, cA + STEPB, cB + STEPB); DMA8(2, cA + 2 * STEPB, cB + 2 * STEPB); DMA8(3, cA + 3 * STEPB, cB + 3 * STEPB);
    __builtin_amdgcn_s_waitcnt(WCNT(0, 15)); __builtin_amdgcn_s_barrier(); SCHED;
#pragma unroll
    for (int i = 0; i < 8; ++i) { g.A[0][i] = rdA<0>(g, i); g.B[0][i] = rdB<0>(g, i); }
    __builtin_amdgcn_s_waitcnt(WCNT(63, 0)); __builtin_amdgcn_s_barrier(); SCHED;
    for (;;) {
        const bool has_next = j + NG < NTILE;
        int qm = pm, qn = pn;
        if (has_next) tile_of(j + NG, qm, qn);
        const unsigned nA = qm * PANEL, nB = (16 + qn) * PANEL;
        R.check(4);
        if (R.n < 64) R.request();
        SCHED;
        step<0, true, 60>(g, cA + 4 * STEPB, cB + 4 * STEPB); step<1, false, 60>(g, cA + 5 * STEPB, cB + 5 * STEPB);
        step<2, false, 16>(g, cA + 6 * STEPB, cB + 6 * STEPB); step<3, false, 16>(g, cA + 7 * STEPB, cB + 7 * STEPB);
        for (int it = 1; it < 32; ++it) {
            const bool last = it == 31;
            R.check(last ? 2 : 2 * it + 4);
            if (it == 30) { R.pa = qm; R.pb = 16 + qn; }
            if (R.n < 64 || it == 30) R.request();
            SCHED;
            const unsigned dA = last ? nA : cA + (4 * it + 4) * STEPB, dB = last ? nB : cB + (4 * it + 4) * STEPB;
            step<0, false, 16>(g, dA, dB); step<1, false, 16>(g, dA + STEPB, dB + STEPB);
            step<2, false, 16>(g, dA + 2 * STEPB, dB + 2 * STEPB); step<3, false, 16>(g, dA + 3 * STEPB, dB + 3 * STEPB);
        }
        {
            const int lz = lane_now();
            float* o = p.out[rank] + (size_t)(pm * 256 + wr * 128 + 4 * (lz >> 4)) * NN + pn * 256 + wc * 128 + 4 * (lz & 15);
            const float one = R.one;
#pragma unroll
            for (int m = 0; m < 8; ++m)
#pragma unroll
                for (int r = 0; r < 4; ++r)
#pragma unroll
                    for (int q = 0; q < 2; ++q) {
                        const f32x4 v = {silu1(g.acc[m][4 * q][r], one), silu1(g.acc[m][4 * q + 1][r], one), silu1(g.acc[m][4 * q + 2][r], one), silu1(g.acc[m][4 * q + 3][r], one)};
                        *(GAS f32x4*)(o + (size_t)(m * 16 + r) * NN + q * 64) = v;
                        if (q == 1 && (r & 1)) SCHED;
                    }
        }
        if (pn >= NBB) {
            WAITVM(0);
            BARRIER;
            if (wid == 0 && lane_now() == 0) {
                __builtin_amdgcn_fence(__ATOMIC_RELEASE, "agent");
                WAITVM(0);
                st_ag(ws + OFF_CT + (h * 16 + pm) * 4, call);
            }
        }
        if (!has_next) break;
        pm = qm; pn = qn; cA = nA; cB = nB; j += NG;
    }
    __builtin_amdgcn_s_waitcnt(WCNT(0, 0));
}

__global__ __launch_bounds__(256, 1) void fused(Args) {
    extern __shared__ __attribute__((aligned(16))) unsigned char shm[];
    Params& p = *(Params*)__builtin_amdgcn_kernarg_segment_ptr();
    LAS unsigned char* lds = (LAS unsigned char*)shm;
    char* const ws = p.ws[p.rank];
    const int wid = rfl(threadIdx.x >> 6);
    STAMP(ws, 0);
    const unsigned call = rfl(ld_ag(ws + OFF_CALL + blockIdx.x * 128)) + 1;
    WAITVM(0);
    BARRIER;
    if (threadIdx.x == 0) st_ag(ws + OFF_CALL + blockIdx.x * 128, call);
    const int grp = blockIdx.x & 7, idx = blockIdx.x >> 3;
    if (idx < NG) gemm_wg(p, lds, call, wid, grp, idx);
    else comm_wg(p, lds, call, wid, (idx - NG) * 8 + grp);
}
}

extern "C" void kernel_launch(int rank, int nranks, ncclComm_t comm, hipStream_t stream,
                              void* const* d_in, const int* in_sizes, int n_in,
                              void* d_out, int out_size,
                              const int* in_dtypes, int out_dtype,
                              void* const* d_out_all,
                              void* const* d_ws_all, size_t ws_bytes) {
    if (nranks != NR || (unsigned)rank >= (unsigned)NR || n_in != 2 || ws_bytes < WS_NEED || in_sizes[0] != MM * KS || in_sizes[1] != KS * NN || out_size != MM * NN) {
        fprintf(stderr, "kernel_launch: unexpected problem shape\n");
        return;
    }
    static bool attr_set[NR];
    if (!attr_set[rank]) {
        HIP_CHECK(hipFuncSetAttribute((const void*)fused, hipFuncAttributeMaxDynamicSharedMemorySize, LDS_BYTES));
        attr_set[rank] = true;
    }
    Args p;
    p.x = (const float*)d_in[0]; p.w = (const float*)d_in[1];
    for (int i = 0; i < NR; ++i) { p.ws[i] = (char*)d_ws_all[i]; p.out[i] = (float*)d_out_all[i]; }
    p.rank = rank; p.pad = 0;
    hipLaunchKernelGGL(fused, dim3(256), dim3(256), LDS_BYTES, stream, p);
    HIP_CHECK(hipGetLastError());
}
```

```cpp
#include <hip/hip_runtime.h>
#include <rccl/rccl.h>
#include <cstdio>

#define HIP_CHECK(expr) do { hipError_t _e = (expr); if (_e != hipSuccess) { \
    fprintf(stderr, "HIP error %s at %s:%d\n", hipGetErrorString(_e), __FILE__, __LINE__); \
    return; } } while (0)

#define LAS __attribute__((address_space(3)))
#define GAS __attribute__((address_space(1)))
#define DI __device__ __forceinline__
typedef _Float16 f16x8 __attribute__((ext_vector_type(8)));
typedef float f32x4 __attribute__((ext_vector_type(4)));
typedef unsigned u32x4 __attribute__((ext_vector_type(4)));
typedef GAS unsigned gu32;

#define NG 28
#define RS 7
#define CPR 3
#define DBG 0
namespace {
constexpr int MM = 4096, NN = 32768, KS = 512, NR = 8;
constexpr int NGEMM = 8 * NG, NCOMM = 256 - NGEMM, NCW = 4 * NCOMM;
constexpr int NPW = NCW / 7, NLW = NCW - 7 * NPW;
constexpr unsigned STEPB = 16384, BLKB = 32768, PANEL = 2u << 20;
constexpr unsigned PKB = 29696, PKTAIL = 24576;
constexpr int NPAN = 144;
constexpr unsigned OUTB = (unsigned)MM * NN * 4u;
constexpr int NCB = 8 * CPR, NBB = 128 - NCB;
constexpr int NL = NBB + CPR;
constexpr int GS = 2 * ((NG + 3) / 4);
constexpr int NT0 = 4 * ((NL + 1) / 2), NT1 = 4 * (NL / 2);
static_assert(NT1 >= RS * NG && NT0 <= (RS + 1) * NG && NLW <= NCOMM, "one pool round");
constexpr unsigned OFF_CALL = 0;
constexpr unsigned OFF_TMO = 32768;
constexpr unsigned OFF_POOL = 32768 + 2048;
constexpr unsigned OFF_PROG = 32768 + 128;
constexpr unsigned OFF_TICK = 32768 + 4096;
constexpr unsigned OFF_F = 65536;
constexpr unsigned OFF_AVA = 131072;
constexpr unsigned OFF_AVB = OFF_AVA + 8192;
constexpr unsigned OFF_AVP = OFF_AVB + 8192;
constexpr unsigned OFF_END = OFF_AVP + 2048;
constexpr unsigned OFF_DONE = 196608;
constexpr unsigned OFF_DBG = 262144, OFF_DBG2 = 524288;
#define KR 4
constexpr unsigned OFF_DATA = 1u << 20;
constexpr unsigned WLB = 8 * BLKB, REL_WL = 16 * PANEL;
constexpr unsigned WRB = 56 * BLKB, REL_WR = REL_WL + 128 * WLB;
constexpr unsigned BUFB = REL_WR + KR * GS * WRB;
constexpr unsigned OFF_PK16 = OFF_DATA + BUFB;
constexpr int NCI = CPR * (MM / 32);
constexpr unsigned OFF_CST = OFF_PK16 + NPAN * 8 * PKB, CSTB = NCI * BLKB;
constexpr unsigned OFF_PK32 = OFF_CST + CSTB;
constexpr unsigned WS_NEED = OFF_PK32 + NCI * PKB;
static_assert(NCW * 128 <= 65536, "done flags");
constexpr int LDS_B = 65536, LDS_ZONE = LDS_B + 5 * 16384, LDS_BYTES = LDS_ZONE + 8192;
constexpr unsigned long long DEADLINE = 50000000ull;
constexpr int AUX_SYS = 17;

struct Args {
    const float* x; const float* w;
    char* ws[NR]; float* out[NR];
    int rank, pad;
};
typedef const __attribute__((address_space(4))) Args Params;

constexpr int WCNT(int vm, int lgkm) { return (vm & 15) | (7 << 4) | ((lgkm & 15) << 8) | ((vm >> 4) << 14); }
#define SCHED __builtin_amdgcn_sched_barrier(0)
#define WAITVM(n) asm volatile("s_waitcnt vmcnt(" #n ")" ::: "memory")
#define WAITLG0 asm volatile("s_waitcnt lgkmcnt(0)" ::: "memory")
#define BARRIER do { asm volatile("" ::: "memory"); __builtin_amdgcn_s_barrier(); asm volatile("" ::: "memory"); } while (0)

DI unsigned ld_sys(const char* p) { return __hip_atomic_load((gu32*)p, __ATOMIC_RELAXED, __HIP_MEMORY_SCOPE_SYSTEM); }
DI void st_sys(char* p, unsigned v) { __hip_atomic_store((gu32*)p, v, __ATOMIC_RELAXED, __HIP_MEMORY_SCOPE_SYSTEM); }
DI unsigned ld_ag(const char* p) { return __hip_atomic_load((gu32*)p, __ATOMIC_RELAXED, __HIP_MEMORY_SCOPE_AGENT); }
DI void st_ag(char* p, unsigned v) { __hip_atomic_store((gu32*)p, v, __ATOMIC_RELAXED, __HIP_MEMORY_SCOPE_AGENT); }
DI unsigned rfl(unsigned x) { return __builtin_amdgcn_readfirstlane(x); }
DI int lane_now() { int l; asm volatile("v_mbcnt_lo_u32_b32 %0, -1, 0\n\tv_mbcnt_hi_u32_b32 %0, -1, %0" : "=v"(l)); return l; }
DI bool give_up(char* ws, unsigned call, unsigned long long t0) {
    if (rfl(ld_ag(ws + OFF_TMO)) == call || __builtin_amdgcn_s_memrealtime() - t0 > DEADLINE) { st_ag(ws + OFF_TMO, call); return true; }
    return false;
}
DI bool wait_words(char* ws, const char* f, int n, unsigned call) {
    const int lane = lane_now();
    const unsigned long long t0 = __builtin_amdgcn_s_memrealtime();
    for (;;) {
        bool ok = true;
#pragma unroll
        for (int i = 0; i < 4; ++i) { const int k = lane + 64 * i; if (64 * i < n) ok &= ld_sys(f + (k < n ? k : 0) * 4) >= call; }
        if (__all(ok)) break;
        if (give_up(ws, call, t0)) return false;
        __builtin_amdgcn_s_sleep(4);
    }
    asm volatile("" ::: "memory");
    return true;
}
#define STAMP(ws, i) do {} while (0)
#define STAMPW(ws, i) do {} while (0)
#define STAMPV(ws, i, v) do {} while (0)

#define XAUX 2
struct Src { __amdgpu_buffer_rsrc_t rx, rw; unsigned vx, vw; };
DI Src make_src(Params& p, int q, int lane) {
    Src s;
    s.rx = __builtin_amdgcn_make_buffer_rsrc((void*)p.x, 0, MM * KS * 4, 0x00020000);
    s.rw = __builtin_amdgcn_make_buffer_rsrc((void*)p.w, 0, KS * NN * 4, 0x00020000);
    s.vx = ((64 * q + (lane & 15)) * KS + 8 * (lane >> 4)) * 4;
    s.vw = (8 * (lane >> 4) * NN + 64 * q + 4 * (lane & 15)) * 4;
    return s;
}
DI void unit_load(const Src& s, int panel, int st, LAS unsigned char* slot) {
    if (panel < 16) {
        const unsigned s0 = (panel * 256 * KS + 32 * st) * 4;
#pragma unroll
        for (int e = 0; e < 8; ++e) __builtin_amdgcn_raw_ptr_buffer_load_lds(s.rx, (LAS void*)(slot + e * 1024), 16, s.vx, s0 + (e >> 1) * (16 * KS * 4) + (e & 1) * 16, 0, XAUX);
    } else {
        const unsigned s0 = (32 * st * NN + (panel - 16) * 256) * 4;
#pragma unroll
        for (int e = 0; e < 8; ++e) __builtin_amdgcn_raw_ptr_buffer_load_lds(s.rw, (LAS void*)(slot + e * 1024), 16, s.vw, s0 + e * (NN * 4), 0, 2);
    }
}
DI void unit_cvt(int panel, const LAS unsigned char* slot, int lane, u32x4 (&f)[4]) {
    f32x4 v[8];
#pragma unroll
    for (int e = 0; e < 8; ++e) v[e] = *(const LAS f32x4*)(slot + e * 1024 + lane * 16);
    if (panel < 16) {
#pragma unroll
        for (int j = 0; j < 4; ++j) { f16x8 h;
#pragma unroll
            for (int e = 0; e < 8; ++e) h[e] = (_Float16)v[2 * j + (e >> 2)][e & 3];
            f[j] = __builtin_bit_cast(u32x4, h); }
    } else {
#pragma unroll
        for (int j = 0; j < 4; ++j) { f16x8 h;
#pragma unroll
            for (int e = 0; e < 8; ++e) h[e] = (_Float16)v[e][j];
            f[j] = __builtin_bit_cast(u32x4, h); }
    }
}
DI void store1(const char* base, unsigned v, const u32x4& f) {
    asm volatile("global_store_dwordx4 %0, %1, %2 sc0 sc1\n\ts_nop 1" :: "v"(v), "v"(f), "s"(base) : "memory");
}
DI void store3(const char* base, unsigned v, const u32x4& a, const u32x4& b, const u32x4& c) {
    asm volatile("global_store_dwordx4 %0, %1, %4 sc0 sc1\n\t"
                 "global_store_dwordx4 %0, %2, %4 offset:1024 sc0 sc1\n\t"
                 "global_store_dwordx4 %0, %3, %4 offset:2048 sc0 sc1\n\ts_nop 1"
                 :: "v"(v), "v"(a), "v"(b), "v"(c), "s"(base) : "memory");
}
#define LBITS "sc1"
DI void store4l(const char* base, unsigned v, unsigned stride, const u32x4 (&f)[4]) {
    asm volatile("global_store_dwordx4 %0, %4, %8 " LBITS "\n\t"
                 "global_store_dwordx4 %1, %5, %8 " LBITS "\n\t"
                 "global_store_dwordx4 %2, %6, %8 " LBITS "\n\t"
                 "global_store_dwordx4 %3, %7, %8 " LBITS "\n\ts_nop 1"
                 :: "v"(v), "v"(v + stride), "v"(v + 2 * stride), "v"(v + 3 * stride), "v"(f[0]), "v"(f[1]), "v"(f[2]), "v"(f[3]), "s"(base) : "memory");
}

#define OBITS "sc1 nt"
DI void store4o(const char* base, unsigned v, unsigned stride, const u32x4 (&f)[4]) {
    asm volatile("global_store_dwordx4 %0, %4, %8 " OBITS "\n\t"
                 "global_store_dwordx4 %1, %5, %8 " OBITS "\n\t"
                 "global_store_dwordx4 %2, %6, %8 " OBITS "\n\t"
                 "global_store_dwordx4 %3, %7, %8 " OBITS "\n\ts_nop 1"
                 :: "v"(v), "v"(v + stride), "v"(v + 2 * stride), "v"(v + 3 * stride), "v"(f[0]), "v"(f[1]), "v"(f[2]), "v"(f[3]), "s"(base) : "memory");
}

DI unsigned perm(unsigned hi, unsigned lo, unsigned sel) { return __builtin_amdgcn_perm(hi, lo, sel); }
template <int CTRL, int ROWMASK> DI unsigned dpp0(unsigned x) { return __builtin_amdgcn_update_dpp(0u, x, CTRL, ROWMASK, 0xf, false); }
DI unsigned wave_sum(unsigned x) {
    x += dpp0<0x111, 0xf>(x); x += dpp0<0x112, 0xf>(x); x += dpp0<0x114, 0xf>(x); x += dpp0<0x118, 0xf>(x); x += dpp0<0x142, 0xa>(x); x += dpp0<0x143, 0xc>(x);
    return __builtin_amdgcn_readlane(x, 63);
}
DI unsigned umax(unsigned a, unsigned b) { return a > b ? a : b; }
DI unsigned umin(unsigned a, unsigned b) { return a < b ? a : b; }
DI unsigned wave_max(unsigned x) {
    x = umax(x, dpp0<0x111, 0xf>(x)); x = umax(x, dpp0<0x112, 0xf>(x)); x = umax(x, dpp0<0x114, 0xf>(x)); x = umax(x, dpp0<0x118, 0xf>(x));
    x = umax(x, dpp0<0x142, 0xa>(x)); x = umax(x, dpp0<0x143, 0xc>(x));
    return __builtin_amdgcn_readlane(x, 63);
}
struct ExcList {
    volatile LAS unsigned short* e; unsigned cnt;
    DI void begin(LAS unsigned char* tail, int lane) { e = (volatile LAS unsigned short*)(tail + 512); e[lane] = 0; cnt = 1; }
    DI bool add(bool act, unsigned entry) {
        const unsigned long long bal = __ballot(act);
        const unsigned slot = cnt + __builtin_amdgcn_mbcnt_hi((unsigned)(bal >> 32), __builtin_amdgcn_mbcnt_lo((unsigned)bal, 0u));
        if (act && slot < 64) e[slot] = (unsigned short)entry;
        cnt += __builtin_popcountll(bal);
        return cnt <= 64;
    }
};
DI bool pack_finish(const char* base, unsigned blk, int c, int lane, LAS unsigned char* tail, ExcList& x, unsigned ebase,
                    const u32x4& p0, const u32x4& p1, const u32x4& p2, unsigned t0, unsigned t1) {
    typedef unsigned u32x2 __attribute__((ext_vector_type(2)));
    if (lane == 0) x.e[0] = (unsigned short)(ebase | (x.cnt > 64 ? 0x8000u : 0u));
    *(volatile LAS u32x2*)(tail + lane * 8) = (u32x2){t0, t1};
    store3(base, blk + c * 3072 + lane * 16, p0, p1, p2);
    const u32x4 t = *(volatile LAS u32x4*)(tail + (lane < 40 ? lane : 0) * 16);
    if (lane < 40) store1(base, blk + PKTAIL + c * 640 + lane * 16, t);
    return x.cnt <= 64;
}
DI void pack16(const u32x4 (&f)[4], const char* base, unsigned blk, int c, int lane, LAS unsigned char* tail) {
    unsigned lo[8], h[8], g[8], sum = 0;
#pragma unroll
    for (int i = 0; i < 8; ++i) {
        const unsigned a = f[i >> 1][2 * (i & 1)], b = f[i >> 1][2 * (i & 1) + 1];
        lo[i] = perm(b, a, 0x06040200u); h[i] = perm(b, a, 0x07050301u);
        sum = __builtin_amdgcn_udot4(h[i] & 0x7c7c7c7cu, 0x01010101u, sum, false);
    }
    const int top = (int)((wave_sum(sum) + 15155u + 8191u) >> 13);
    const unsigned eb = top < 7 ? 0u : top > 31 ? 24u : (unsigned)(top - 7), b4 = eb * 0x04040404u;
    ExcList x; x.begin(tail, lane);
    unsigned m1 = 0, m0 = 0;
#pragma unroll
    for (int i = 0; i < 8; ++i) {
        g[i] = (h[i] & 0x83838383u) | (((h[i] | 0x80808080u) - b4) & 0x1c1c1c1cu);
        const unsigned xr = (g[i] + b4) ^ h[i];
        m1 |= ((xr >> 6) & 0x01010101u) << i; m0 |= ((xr >> 5) & 0x01010101u) << i;
    }
    for (unsigned m = m1 | m0; __any(m != 0); m &= m - 1) {
        const int k = m ? __builtin_ctz(m) : 0, i = k & 7;
        if (!x.add(m != 0, ((i >> 2) * 1024 + lane * 16 + (i & 3) * 4 + (k >> 3)) | ((m0 >> k) & 1u) << 11 | ((m1 >> k) & 1u) << 12)) break;
    }
    unsigned d[6];
#pragma unroll
    for (int q = 0; q < 2; ++q) {
        const unsigned s = g[4 * q + 3];
        d[3 * q] = g[4 * q] | (s & 0x03030303u) << 5; d[3 * q + 1] = g[4 * q + 1] | (s & 0x0c0c0c0cu) << 3;
        d[3 * q + 2] = g[4 * q + 2] | (s & 0x10101010u) << 1 | (s & 0x80808080u) >> 1;
    }
    pack_finish(base, blk, c, lane, tail, x, eb, (u32x4){lo[0], lo[1], lo[2], lo[3]}, (u32x4){lo[4], lo[5], lo[6], lo[7]}, (u32x4){d[0], d[1], d[2], d[3]}, d[4], d[5]);
}
DI bool pack32(const u32x4 (&f)[4], const char* base, unsigned blk, int c, int lane, LAS unsigned char* tail) {
    unsigned t[12], h[4], mx = 0;
#pragma unroll
    for (int j = 0; j < 4; ++j) {
        t[3 * j] = perm(f[j][3], f[j][0], 0x04020100u); t[3 * j + 1] = perm(f[j][3], f[j][1], 0x05020100u); t[3 * j + 2] = perm(f[j][3], f[j][2], 0x06020100u);
        h[j] = perm(perm(f[j][3], f[j][2], 0x07030703u), perm(f[j][1], f[j][0], 0x07030703u), 0x05040100u);
#pragma unroll
        for (int d = 0; d < 4; ++d) mx = umax(mx, f[j][d] & 0x7fffffffu);
    }
    const unsigned top = wave_max(mx) >> 24, eb = top < 7 ? 0u : top - 7, b1 = eb * 0x01010101u;
    ExcList x; x.begin(tail, lane);
    unsigned cd[4], xn[2] = {0, 0};
#pragma unroll
    for (int j = 0; j < 4; ++j) {
        cd[j] = ((h[j] | 0x80808080u) - b1) & 0x07070707u;
        const unsigned xr = ((cd[j] + b1) ^ h[j]) & 0x78787878u;
        xn[j >> 1] |= (j & 1) ? xr << 1 : xr >> 3;
    }
    for (unsigned long long m = (unsigned long long)xn[1] << 32 | xn[0]; __any(m != 0);) {
        const int k = m ? __builtin_ctzll(m) >> 2 : 0, j = 2 * (k >> 3) + (k & 1);
        const unsigned v = (unsigned)(m >> (4 * k)) & 15u;
        if (!x.add(m != 0, (lane * 16 + j * 4 + ((k >> 1) & 3)) | v << 11)) break;
        m &= ~(15ull << (4 * k));
    }
    const unsigned n0 = cd[0] | (h[0] & 0x80808080u) >> 4 | cd[1] << 4 | (h[1] & 0x80808080u), n1 = cd[2] | (h[2] & 0x80808080u) >> 4 | cd[3] << 4 | (h[3] & 0x80808080u);
    return pack_finish(base, blk, c, lane, tail, x, eb, (u32x4){t[0], t[1], t[2], t[3]}, (u32x4){t[4], t[5], t[6], t[7]}, (u32x4){t[8], t[9], t[10], t[11]}, n0, n1);
}
DI bool unpack(bool is32, const LAS unsigned char* main3, const LAS unsigned char* tail, LAS unsigned char* patch, int lane, u32x4 (&f)[4]) {
    typedef unsigned u32x2 __attribute__((ext_vector_type(2)));
    unsigned e = *(const volatile LAS unsigned short*)(tail + 512 + lane * 2);
    const u32x2 tt = *(const volatile LAS u32x2*)(tail + lane * 8);
    const u32x4 p0 = *(const volatile LAS u32x4*)(main3 + lane * 16), p1 = *(const volatile LAS u32x4*)(main3 + 1024 + lane * 16), p2 = *(const volatile LAS u32x4*)(main3 + 2048 + lane * 16);
    const unsigned hdr = rfl(e);
    if (lane == 0) e = 0;
    LAS unsigned* const pw = (LAS unsigned*)(patch + (e & 0x7fcu));
    __hip_atomic_fetch_or(pw, (e >> 11) << ((is32 ? 3 : 5) + 8 * (e & 3u)), __ATOMIC_RELAXED, __HIP_MEMORY_SCOPE_WAVEFRONT);
    const u32x4 x0 = *(const volatile LAS u32x4*)(patch + lane * 16), x1 = *(const volatile LAS u32x4*)(patch + 1024 + lane * 16);
    *(volatile LAS unsigned*)pw = 0;
    if (is32) {
        const unsigned b1 = (hdr & 127u) * 0x01010101u, t[12] = {p0[0], p0[1], p0[2], p0[3], p1[0], p1[1], p1[2], p1[3], p2[0], p2[1], p2[2], p2[3]};
#pragma unroll
        for (int j = 0; j < 4; ++j) {
            const unsigned n = tt[j >> 1];
            const unsigned h = ((j & 1) ? ((((n >> 4) & 0x07070707u) + b1) | (n & 0x80808080u)) : (((n & 0x07070707u) + b1) | ((n << 4) & 0x80808080u))) ^ x0[j];
            f[j][0] = perm(h, t[3 * j], 0x04020100u); f[j][1] = perm(h, t[3 * j + 1], 0x05020100u); f[j][2] = perm(h, t[3 * j + 2], 0x06020100u);
            f[j][3] = perm(perm(h, t[3 * j + 2], 0x07030703u), perm(t[3 * j + 1], t[3 * j], 0x07030703u), 0x05040100u);
        }
    } else {
        const unsigned b4 = (hdr & 31u) * 0x04040404u, d[6] = {p2[0], p2[1], p2[2], p2[3], tt[0], tt[1]};
#pragma unroll
        for (int q = 0; q < 2; ++q) {
            const unsigned a = d[3 * q], b = d[3 * q + 1], c = d[3 * q + 2];
            const unsigned g[4] = {a & 0x9f9f9f9fu, b & 0x9f9f9f9fu, c & 0x9f9f9f9fu,
                                   ((a >> 5) & 0x03030303u) | ((b >> 3) & 0x0c0c0c0cu) | ((c >> 1) & 0x10101010u) | ((c << 1) & 0x80808080u)};
#pragma unroll
            for (int i = 0; i < 4; ++i) {
                const unsigned h = (g[i] + b4) ^ (q ? x1[i] : x0[i]), lo = q ? p1[i] : p0[i];
                f[2 * q + (i >> 1)][2 * (i & 1)] = perm(h, lo, 0x05010400u); f[2 * q + (i >> 1)][2 * (i & 1) + 1] = perm(h, lo, 0x07030602u);
            }
        }
    }
    return hdr < 0x8000u;
}

DI unsigned own_off(int panel) { return panel < 16 ? panel * PANEL : REL_WL + (panel - 16) * WLB; }
DI int order_of(int cb, int rank) { return cb >= NBB ? cb - NBB - CPR * rank : cb + CPR; }
DI unsigned ring_off(int cb, int rank) { return REL_WR + order_of(cb, rank) % (KR * GS) * WRB; }
constexpr int B0 = GS - CPR, P0P = 16 + B0 + NCB, P0B = P0P * 8, NBLK = NPAN * 8;
constexpr int KTA = NGEMM / P0P;
DI void conv_block(int b, int& panel, int& kt) {
    if (b < P0B) { kt = b / P0P; const int i = b - kt * P0P; panel = i < 16 + B0 ? i : i - B0 + NBB; }
    else { const int r = b - P0B; panel = 16 + B0 + (r >> 3); kt = r & 7; }
}
DI unsigned draw_call(char* ws, int wid) {
    unsigned c;
    asm volatile("global_load_dword %0, %1, %2 sc1\n\ts_waitcnt vmcnt(0)" : "=&v"(c) : "v"(OFF_CALL + blockIdx.x * 128), "s"(ws) : "memory");
    c = rfl(c) + 1;
    BARRIER;
    if (wid == 0 && lane_now() == 0) st_ag(ws + OFF_CALL + blockIdx.x * 128, c);
    return c;
}
DI unsigned convert_shards(Params& p, LAS unsigned char* lds, int wid, int gi) {
    const int lane = lane_now(), rank = p.rank;
    char* const ws = p.ws[rank];
    const Src src = make_src(p, wid, lane);
    LAS unsigned char* const ring = lds + wid * 32768;
    LAS unsigned char* const tail = lds + 131072 + wid * 1024;
    const int nu = 2 * ((NBLK - gi + NGEMM - 1) / NGEMM);
    auto load = [&](int u) { int pa, kt; const int v = u < nu ? u : (u & 1); conv_block(gi + (v >> 1) * NGEMM, pa, kt); unit_load(src, pa, 2 * kt + (v & 1), ring + (u & 3) * 8192); };
    load(0); load(1);
    char* peer_ws = p.ws[lane & 7];
    const unsigned call = draw_call(ws, wid);
    asm volatile("" : "+v"(peer_ws));
    const unsigned vb = OFF_DATA + wid * 4096 + lane * 16, pb = OFF_PK16;
    auto emit = [&](int u) { int pa, kt; conv_block(gi + (u >> 1) * NGEMM, pa, kt); u32x4 f[4]; unit_cvt(pa, ring + (u & 3) * 8192, lane, f);
                             pack16(f, ws, pb + (pa * 8 + kt) * PKB, 4 * (u & 1) + wid, lane, tail);
                             const unsigned v = vb + own_off(pa) + (2 * kt + (u & 1)) * STEPB; store3(ws, v, f[0], f[1], f[2]); store1(ws, v + 3072, f[3]); };
    for (int u = 0; u < nu; ++u) {
        if (u >= 2) { load(u + 3); WAITVM(24); }
        emit(u);
        if (u == 1) {
            load(2); load(3); load(4);
            WAITVM(24);
            BARRIER;
            if (wid == 0 && lane < NR) st_sys(peer_ws + OFF_AVA + (rank * 256 + gi) * 4, call);
        }
    }
    WAITVM(0);
    BARRIER;
    if (wid == 0) {
        if (lane < NR) st_sys(peer_ws + OFF_AVB + (rank * 256 + gi) * 4, call);
        if (2 * lane < nu) { int pa, kt; conv_block(gi + lane * NGEMM, pa, kt); st_sys(ws + OFF_F + pa * 256 + kt * 4, call); }
    }
    return call;
}

constexpr int G0 = 16 + GS, NBI = (16 + NL) * 8, NITEM = NBI + NCI;
struct Item { const char* dst; unsigned size, pk, raw, dof, stride; int flag; };
constexpr int PD = 3;
constexpr int N01 = 8 * (G0 + GS);
static_assert(KR >= 2, "ring");
DI bool gate_open(unsigned t, unsigned probe, unsigned call) {
    if (t < (unsigned)N01) return true;
    const unsigned q = (t - 8u * G0) / (8u * GS) + 1;
    return q < KR || __builtin_amdgcn_readlane(probe, 48 + q - KR) >= call * (16u * GS);
}
#define XLAG 1
constexpr int xfirst(int s) { return s < 8 || XLAG == 0 ? (s < 8 ? 0 : 4) : (s - 8) / (XLAG ? XLAG : 1) + 1; }
constexpr int xcount(int s) { const int hi = XLAG == 0 ? 4 : (s / (XLAG ? XLAG : 1) + 1 < 4 ? s / (XLAG ? XLAG : 1) + 1 : 4); return hi > xfirst(s) ? hi - xfirst(s) : 0; }
constexpr int step_start(int s) { int n = 0; for (int i = 0; i < s; ++i) n += 4 * xcount(i) + GS; return n; }
static_assert(step_start(16) == N01 && NL >= 2 * GS, "order");
DI int colblock(int i, int rank) { return i < CPR ? NBB + CPR * rank + i : i - CPR; }
DI void block_of(unsigned t, int rank, int& panel, int& kt) {
    if (t < (unsigned)N01) {
        int s = 0, i = t, x0 = xfirst(0), nx = 4 * xcount(0);
#pragma unroll
        for (int k = 1; k < 16; ++k) if (t >= (unsigned)step_start(k)) { s = k; i = t - step_start(k); x0 = xfirst(k); nx = 4 * xcount(k); }
        if (i < nx) { panel = 4 * x0 + i; kt = s - XLAG * (panel >> 2); }
        else { panel = 16 + colblock(GS * (s >> 3) + i - nx, rank); kt = s & 7; }
    } else {
        constexpr int NFULL = (NL - GS) / GS, REM = (NL - GS) % GS ? (NL - GS) % GS : 1;
        const int r = t - 8 * G0, q = r / (8 * GS), i = r - q * (8 * GS);
        if (q < NFULL) { kt = i / GS; panel = 16 + B0 + GS * q + (i - kt * GS); }
        else { kt = i / REM; panel = 16 + B0 + GS * q + (i - kt * REM); }
    }
}
DI unsigned slab_out(unsigned s, int from) { return ((s & 127u) * 32u * NN + (NBB + CPR * from + (s >> 7)) * 256u) * 4u; }
DI Item make_item(Params& p, unsigned t, unsigned call, int peer, int pj) {
    const int rank = p.rank;
    Item it;
    it.size = WS_NEED;
    if (t < (unsigned)NBI) {
        int panel, kt;
        block_of(t, rank, panel, kt);
        it.dst = p.ws[rank]; it.stride = 1024;
        it.pk = OFF_PK16 + (panel * 8 + kt) * PKB; it.raw = OFF_DATA + own_off(panel) + kt * BLKB;
        it.dof = OFF_DATA + (panel < 16 ? panel * PANEL + WLB : ring_off(panel - 16, rank)) + (7 * kt + pj) * BLKB; it.flag = panel * 64 + 8 + 7 * kt + pj;
    } else {
        const unsigned s = t < (unsigned)NITEM ? t - NBI : 0;
        if (t >= (unsigned)NITEM) it.size = 0;
        it.dst = (const char*)p.out[rank]; it.stride = NN * 4;
        it.pk = OFF_PK32 + s * PKB; it.raw = OFF_CST + s * BLKB; it.dof = slab_out(s, peer); it.flag = -1;
    }
    return it;
}
template <int N> DI void lds_load(const char* base, unsigned size, unsigned off, LAS unsigned char* to, int lane) {
    const __amdgpu_buffer_rsrc_t r = __builtin_amdgcn_make_buffer_rsrc((void*)base, 0, size, 0x00020000);
    const unsigned m = (unsigned)(size_t)to, v = lane * 16;
    asm volatile("s_mov_b32 m0, %3\n\ts_nop 4\n\tbuffer_load_dwordx4 %0, %1, %2 offen sc0 sc1 lds\n\tbuffer_load_dwordx4 %0, %1, %2 offen offset:1024 sc0 sc1 lds\n\t"
                 "buffer_load_dwordx4 %0, %1, %2 offen offset:2048 sc0 sc1 lds" :: "v"(v), "s"(r), "s"(off), "s"(m) : "memory");
    if (N == 4) asm volatile("s_mov_b32 m0, %3\n\ts_nop 0\n\tbuffer_load_dwordx4 %0, %1, %2 offen offset:3072 sc0 sc1 lds" :: "v"(v), "s"(r), "s"(off), "s"(m) : "memory");
    if (N == 5) asm volatile("s_mov_b32 m0, %3\n\ts_nop 0\n\tbuffer_load_dwordx4 %0, %1, %2 offen sc0 sc1 lds\n\tbuffer_load_dwordx4 %0, %1, %2 offen offset:1024 sc0 sc1 lds"
                             :: "v"(v), "s"(r), "s"(off + 3072), "s"(m + 3072) : "memory");
}
constexpr unsigned NANS = 0x7fc07fc0u;
DI void fetch_wave(Params& p, LAS unsigned char* lds, unsigned call, int wid, int wv) {
    const int lane = lane_now(), rank = p.rank;
    char* const ws = p.ws[rank];
    const int pj = wv % 7, k0 = wv / 7, peer = (rank + 1 + pj) & 7;
    const char* const src = p.ws[peer];
    LAS unsigned char* const ring = lds;
    LAS unsigned char* const tails = lds + 24576;
    LAS unsigned char* const patch = lds + 24576 + 10240;
    *(LAS u32x4*)(patch + lane * 16) = (u32x4){0, 0, 0, 0}; *(LAS u32x4*)(patch + 1024 + lane * 16) = (u32x4){0, 0, 0, 0};
    const unsigned wbase = (call - 1) * NBI - NPW, cbase = (call - 1) * NCI - NPW;
    static_assert(CPR * 16 <= 48 && NL <= 15 * GS, "probe lanes");
    const unsigned vprobe = lane >= 48 ? OFF_PROG + (lane - 48) * 4 : OFF_AVP + peer * 256 + (lane < CPR * 16 ? lane : 0) * 4;
    static_assert(NPW <= step_start(KTA), "first items");
    int pa0, kt0;
    block_of(k0, rank, pa0, kt0);
    bool ok = wait_words(ws, ws + OFF_AVA + peer * 1024 + (kt0 * P0P + (pa0 < 16 + B0 ? pa0 : pa0 + B0 - NBB)) * 4, 1, call);
    STAMP(ws, 1);
    unsigned tw = k0, tc = k0;
    bool alla = false, allb = false;
    auto choose = [&](unsigned probe) -> unsigned {
        if (tw < (unsigned)NBI && gate_open(tw, probe, call)) {
            if (!alla && tw >= (unsigned)NPW) { ok &= wait_words(ws, ws + OFF_AVA + peer * 1024, KTA * P0P, call); alla = true; }
            if (!allb && tw >= (unsigned)step_start(KTA)) { ok &= wait_words(ws, ws + OFF_AVB + peer * 1024, NGEMM, call); allb = true; }
            return tw;
        }
        if (tc < (unsigned)NCI && (~__ballot(probe >= call) << 16) == 0) return NBI + tc;
        return NITEM;
    };
    while (tw < (unsigned)NBI || tc < (unsigned)NCI) {
        unsigned t;
        {
            const unsigned long long t0 = __builtin_amdgcn_s_memrealtime();
            while ((t = choose(ld_sys(ws + vprobe))) == (unsigned)NITEM) {
                if (give_up(ws, call, t0)) { ok = false; t = tw < (unsigned)NBI ? tw : NBI + tc; break; }
                __builtin_amdgcn_s_sleep(32);
            }
        }
        Item cur = make_item(p, t, call, peer, pj);
        unsigned tb = 0;
        lds_load<5>(src, cur.size, cur.pk + PKTAIL, tails, lane);
#pragma unroll
        for (int c = 0; c < PD; ++c) lds_load<3>(src, cur.size, cur.pk + c * 3072, ring + c * 3072, lane);
        WAITVM(0);
        int pflag = -1;
        for (;;) {
            unsigned tk, probe; Item nxt;
#pragma unroll
            for (int c = 0; c < 8; ++c) {
                if (c != 8 - PD) asm volatile("s_waitcnt vmcnt(%0)" :: "i"(7 * PD - 3) : "memory");
                else {
                    asm volatile("s_waitcnt vmcnt(%2)" : "+v"(tk), "+v"(probe) : "i"(7 * PD - 3) : "memory");
                    if (cur.flag >= 0) tw = rfl(tk) - wbase; else tc = rfl(tk) - cbase;
                    t = choose(probe);
                    nxt = make_item(p, t, call, peer, pj);
                    lds_load<5>(src, nxt.size, nxt.pk + PKTAIL, tails + (5120 - tb), lane);
                }
                if (c == PD && pflag >= 0 && lane == 0) st_sys(ws + OFF_F + pflag * 4, call);
                if (c == 0) {
                    unsigned long long sv;
                    asm volatile("global_load_dword %1, %6, %5 sc0 sc1\n\t"
                                 "s_mov_b64 %2, exec\n\ts_mov_b64 exec, 1\n\tglobal_atomic_add %0, %3, %4, %5 sc0\n\ts_mov_b64 exec, %2"
                                 : "=&v"(tk), "=&v"(probe), "=&s"(sv) : "v"(OFF_TICK + (cur.flag < 0 ? 1024u : 0u) + pj * 128u), "v"(1u), "s"(ws), "v"(vprobe) : "memory");
                }
                if (c < 8 - PD) lds_load<3>(src, cur.size, cur.pk + (c + PD) * 3072, ring + (c + PD) * 3072, lane);
                else lds_load<3>(src, nxt.size, nxt.pk + (c + PD - 8) * 3072, ring + (c + PD - 8) * 3072, lane);
                u32x4 f[4];
                if (!unpack(cur.flag < 0, ring + c * 3072, tails + tb + c * 640, patch, lane, f)) {
                    const char* q = src + (size_t)(cur.raw + c * 4096 + lane * 16);
                    asm volatile("global_load_dwordx4 %0, %4, off sc0 sc1\n\tglobal_load_dwordx4 %1, %4, off offset:1024 sc0 sc1\n\t"
                                 "global_load_dwordx4 %2, %4, off offset:2048 sc0 sc1\n\tglobal_load_dwordx4 %3, %4, off offset:3072 sc0 sc1\n\ts_waitcnt vmcnt(0)"
                                 : "=&v"(f[0]), "=&v"(f[1]), "=&v"(f[2]), "=&v"(f[3]) : "v"(q) : "memory");
                }
                if (!ok) f[0] = f[1] = f[2] = f[3] = (u32x4){NANS, NANS, NANS, NANS};
                if (cur.flag >= 0) store4l(cur.dst, cur.dof + 4 * c * 1024 + lane * 16, 1024, f);
                else store4o(cur.dst, cur.dof + 4 * c * (NN * 4) + lane * 16, NN * 4, f);
            }
            pflag = cur.flag;
            if (t >= (unsigned)NITEM) break;
            cur = nxt; tb = 5120 - tb;
        }
        WAITVM(0);
        if (pflag >= 0 && lane == 0) st_sys(ws + OFF_F + pflag * 4, call);
    }
    STAMPW(ws, 44 + wid);
    if (lane == 0) st_sys(p.ws[peer] + OFF_DONE + wv * 128, call);
    if (!wait_words(ws, ws + OFF_DONE + wv * 128, 1, call) && lane == 0) p.out[rank][(NBB + CPR * peer) * 256] = __builtin_nanf("");
    STAMPW(ws, 48 + wid);
    if (wv == 0) {
        constexpr int NGRP = (NL + GS - 1) / GS;
        const unsigned want = call * 16u * (lane < NGRP - 1 ? GS : NL - (NGRP - 1) * GS);
        const unsigned long long t0 = __builtin_amdgcn_s_memrealtime();
        while (!__all(lane >= NGRP || ld_sys(ws + OFF_PROG + (lane & 15) * 4) >= want) && !give_up(ws, call, t0)) __builtin_amdgcn_s_sleep(8);
        if (lane < NR) st_sys(p.ws[lane] + OFF_END + rank * 4, call);
        wait_words(ws, ws + OFF_END, NR, call);
    }
}
DI void pack_tile(Params& p, LAS unsigned char* tail, int wid, unsigned slab0) {
    const int lane = lane_now(), rank = p.rank;
    char* const ws = p.ws[rank];
    for (int i = 0; i < 16; i += 2) {
        const unsigned s = slab0 + 2 * wid + (i >> 3);
        const char* q = ws + (size_t)(OFF_CST + s * BLKB + (i & 7) * 4096 + lane * 16);
        u32x4 f[4], h[4];
        asm volatile("global_load_dwordx4 %0, %8, off sc1 nt\n\tglobal_load_dwordx4 %1, %8, off offset:1024 sc1 nt\n\t"
                     "global_load_dwordx4 %2, %8, off offset:2048 sc1 nt\n\tglobal_load_dwordx4 %3, %8, off offset:3072 sc1 nt\n\t"
                     "global_load_dwordx4 %4, %9, off sc1 nt\n\tglobal_load_dwordx4 %5, %9, off offset:1024 sc1 nt\n\t"
                     "global_load_dwordx4 %6, %9, off offset:2048 sc1 nt\n\tglobal_load_dwordx4 %7, %9, off offset:3072 sc1 nt\n\ts_waitcnt vmcnt(0)"
                     : "=&v"(f[0]), "=&v"(f[1]), "=&v"(f[2]), "=&v"(f[3]), "=&v"(h[0]), "=&v"(h[1]), "=&v"(h[2]), "=&v"(h[3]) : "v"(q), "v"(q + 4096) : "memory");
        const unsigned o = slab_out(s, rank) + 4 * (i & 7) * (NN * 4) + lane * 16;
        store4o((const char*)p.out[rank], o, NN * 4, f);
        store4o((const char*)p.out[rank], o + 4 * (NN * 4), NN * 4, h);
        const unsigned v = OFF_CST + s * BLKB + (i & 7) * 4096 + lane * 16;
        if (!pack32(f, ws, OFF_PK32 + s * PKB, i & 7, lane, tail)) { store3(ws, v, f[0], f[1], f[2]); store1(ws, v + 3072, f[3]); }
        if (!pack32(h, ws, OFF_PK32 + s * PKB, (i & 7) + 1, lane, tail)) { store3(ws, v + 4096, h[0], h[1], h[2]); store1(ws, v + 7168, h[3]); }
    }
}

struct G4 {
    unsigned lds;
    __amdgpu_buffer_rsrc_t rD;
    unsigned voff;
    unsigned ard, brd;
    unsigned bn;
    unsigned sA, sB;
    unsigned ringB, nextA, nextB;
    unsigned sb, sn;
    f32x4 acc[8][8];
    f16x8 A[2][8], B[2][8];
};
#define OAUX 2
#define NBS 5
#define GBITS "sc0 sc1"
DI f16x8 rdfrag(unsigned base, int off) { return *(const LAS f16x8*)(size_t)(base + off); }
DI void dma_to(unsigned ldsaddr) { asm volatile("s_mov_b32 m0, %0" :: "s"(ldsaddr) : "memory"); }
template <int I> DI void dma(const G4& g, unsigned src) {
    asm volatile("buffer_load_dwordx4 %0, %1, %2 offen offset:%3 " GBITS " lds" :: "v"(g.voff), "s"(g.rD), "s"(src), "i"(I * 1024) : "memory");
}
DI void dma4(const G4& g, unsigned ldsoff, unsigned src) {
    dma_to(g.lds + ldsoff); asm volatile("s_nop 0"); dma<0>(g, src); dma<1>(g, src); dma<2>(g, src); dma<3>(g, src);
}
#define PIN_S(x) asm volatile("" : "+s"(x))
#define RD0 0
#define RDS 3
#define DPOS {2, 8, 16, 23, 32, 38, 46, 53}
#define ABL 16
template <int J, bool ZERO, int VM> DI void step(G4& g, int it) {
    constexpr int c = J & 1, NX = (J + 1) & 3;
#pragma unroll
    for (int m = 0; m < 8; ++m) {
#pragma unroll
        for (int n = 0; n < 8; ++n) {
            const int q = m * 8 + n;
            { const int n2 = (m & 1) ? 7 - n : n;
            if constexpr (ZERO) asm volatile("v_mfma_f32_16x16x32_f16 %0, %1, %2, 0" : "=a"(g.acc[m][n2]) : "v"(g.A[c][m]), "v"(g.B[c][n2]));
            else asm volatile("v_mfma_f32_16x16x32_f16 %0, %1, %2, %0" : "+a"(g.acc[m][n2]) : "v"(g.A[c][m]), "v"(g.B[c][n2])); }
            SCHED;
            if (!(ABL & 4) && q >= RD0 && (q - RD0) % RDS == 0 && (q - RD0) / RDS < 16) { const int i = (q - RD0) / RDS;
                if (i == 0) g.A[c ^ 1][0] = rdfrag(g.ard, NX * 16384);
                else if (i <= 8) g.B[c ^ 1][i - 1] = rdfrag(g.bn, (i - 1) * 1024);
                else g.A[c ^ 1][i - 8] = rdfrag(g.ard, NX * 16384 + (i - 8) * 1024);
                SCHED; }
            constexpr int D[8] = DPOS;
#if !(ABL & 1)
            if (q == D[0] - 1) { dma_to(g.lds + J * 16384); SCHED; }
            if (q == D[0]) { dma<0>(g, g.sA); SCHED; }
            if (q == D[1]) { dma<1>(g, g.sA); SCHED; }
            if (q == D[2]) { dma<2>(g, g.sA); SCHED; }
            if (q == D[3]) { dma<3>(g, g.sA); SCHED; }
            if (q == D[4] - 1) { PIN_S(g.sb); dma_to(g.lds + LDS_B + g.sb); SCHED; }
            if (q == D[4]) { dma<0>(g, g.sB); SCHED; }
            if (q == D[5]) { dma<1>(g, g.sB); SCHED; }
            if (q == D[6]) { dma<2>(g, g.sB); SCHED; }
            if (q == D[7]) { dma<3>(g, g.sB); SCHED; }
#endif
            if (q == 26) { PIN_S(g.sA); g.sA += STEPB; PIN_S(g.sA); SCHED; }
            if (q == 28 && J == 3) { g.sA = it == 30 ? g.nextA : g.sA; PIN_S(g.sA); SCHED; }
            if (q == 55) { PIN_S(g.sB); g.sB += STEPB; PIN_S(g.sB); SCHED; }
            if (q == 56 && J == 2) { g.sB = it == 2 ? g.ringB : g.sB; PIN_S(g.sB); SCHED; }
            if (q == 57 && J == 2) { g.sB = it == 30 ? g.nextB : g.sB; PIN_S(g.sB); SCHED; }
            if (q == 58) { PIN_S(g.sn); g.sb = g.sn; g.sn = g.sn == (NBS - 1) * 16384 ? 0 : g.sn + 16384; PIN_S(g.sn); SCHED; }
            if (q == 60) { g.bn = g.brd + g.sn; asm volatile("" : "+v"(g.bn)); SCHED; }
        }
    }
    __builtin_amdgcn_s_waitcnt(WCNT(VM, 0));
#if !(ABL & 2)
    __builtin_amdgcn_s_barrier();
#endif
    SCHED;
}
constexpr int VMK = NBS == 5 ? 20 : 16;
DI void g4_init(G4& g, LAS unsigned char* lds, const char* data, int wid) {
    const int lane = lane_now(), wr = wid >> 1, wc = wid & 1;
    const unsigned lbase = (unsigned)(size_t)lds;
    g.rD = __builtin_amdgcn_make_buffer_rsrc((void*)data, 0, BUFB, 0x00020000);
    g.lds = lbase + wid * 4096; g.voff = wid * 4096 + lane * 16;
    g.ard = lbase + wr * 8192 + lane * 16; g.brd = lbase + LDS_B + wc * 8192 + lane * 16; g.sb = 0; g.sn = 16384; g.bn = g.brd + 16384;
    asm volatile("" : "+v"(g.ard), "+v"(g.brd));
}
DI void g4_prologue(G4& g, unsigned dA, unsigned dB) {
#pragma unroll
    for (int i = 0; i < NBS; ++i) { if (i < 4) dma4(g, i * 16384, dA + i * STEPB); dma4(g, LDS_B + i * 16384, dB + i * STEPB); }
    g.sA = dA + 4 * STEPB; g.sB = dB + NBS * STEPB;
    WAITVM(0); __builtin_amdgcn_s_barrier(); SCHED;
#pragma unroll
    for (int i = 0; i < 8; ++i) { g.A[0][i] = rdfrag(g.ard, i * 1024); g.B[0][i] = rdfrag(g.brd, i * 1024); }
    __builtin_amdgcn_s_waitcnt(WCNT(63, 0)); __builtin_amdgcn_s_barrier(); SCHED;
}
DI float rd_acc(float a) { float v; asm volatile("v_accvgpr_read_b32 %0, %1" : "=v"(v) : "a"(a)); return v; }
DI f32x4 silu4(f32x4 y, float one) {
    f32x4 t = y * -1.44269504088896f;
#pragma unroll
    for (int i = 0; i < 4; ++i) t[i] = __builtin_amdgcn_exp2f(__builtin_amdgcn_fmed3f(t[i], -86.5617024533378f, 86.5617024533378f));
    t = t + one;
#pragma unroll
    for (int i = 0; i < 4; ++i) t[i] = __builtin_amdgcn_rcpf(t[i]);
    return y * t;
}
DI void g4_epilogue(G4& g, __amdgpu_buffer_rsrc_t rO, unsigned so, unsigned rowb, float one) {
    const int lz = lane_now();
    const unsigned vo = 4 * (lz >> 4) * rowb + 16 * (lz & 15);
#pragma unroll
    for (int m = 0; m < 8; ++m) {
#pragma unroll
        for (int r = 0; r < 4; ++r) {
#pragma unroll
            for (int q = 0; q < 2; ++q) {
                const f32x4 y = {rd_acc(g.acc[m][4 * q][r]), rd_acc(g.acc[m][4 * q + 1][r]), rd_acc(g.acc[m][4 * q + 2][r]), rd_acc(g.acc[m][4 * q + 3][r])};
                const f32x4 v = silu4(y, one);
                __builtin_amdgcn_raw_buffer_store_b128(__builtin_bit_cast(u32x4, v), rO, vo + q * 256, so + r * rowb, OAUX);
            }
            if (r & 1) SCHED;
        }
        so += 16 * rowb;
    }
}
DI int lead_ones(bool ok) { const unsigned long long m = ~__ballot(ok); return m ? __builtin_ctzll(m) : 64; }

struct Ready {
    char* ws; unsigned call;
    __amdgpu_buffer_rsrc_t rF; LAS unsigned char* zone;
    int pa, pb, n; bool pending; float one;
    DI void poll(int need) {
        const unsigned long long t0 = __builtin_amdgcn_s_memrealtime();
        const int lane = lane_now();
        for (;;) {
            const unsigned a = ld_sys(ws + OFF_F + pa * 256 + lane * 4), b = ld_sys(ws + OFF_F + pb * 256 + lane * 4);
            n = lead_ones(a >= call && b >= call);
            if (n >= need) break;
            if (give_up(ws, call, t0)) { n = 64; one = __builtin_nanf(""); break; }
            __builtin_amdgcn_s_sleep(16);
        }
    }
    DI void request() {
        const int lane = lane_now();
        __builtin_amdgcn_raw_ptr_buffer_load_lds(rF, (LAS void*)zone, 4, lane * 4, pa * 256, 0, AUX_SYS);
        __builtin_amdgcn_raw_ptr_buffer_load_lds(rF, (LAS void*)(zone + 256), 4, lane * 4, pb * 256, 0, AUX_SYS);
        pending = true;
    }
    DI int plan(int it, int stop) {
        if (n == 64) return stop;
        const int safe = (n - 5) >> 1;
        if (safe <= it + 1) { request(); return it + 1; }
        return safe - 1 < stop ? safe - 1 : stop;
    }
    DI void check(int need) {
        if (pending) {
            const int lane = lane_now();
            const unsigned a = *(const volatile LAS unsigned*)(zone + lane * 4), b = *(const volatile LAS unsigned*)(zone + 256 + lane * 4);
            n = lead_ones(a >= call && b >= call); pending = false;
        }
        if (n < need) poll(need);
        asm volatile("" ::: "memory");
    }
};

DI void gemm_wg(Params& p, LAS unsigned char* lds, int wid, int grp, int idx) {
    const int wr = wid >> 1, wc = wid & 1, rank = p.rank;
    char* const ws = p.ws[rank];
    const unsigned call = convert_shards(p, lds, wid, idx * 8 + grp);
    STAMP(ws, 2);
    G4 g;
    g4_init(g, lds, ws + OFF_DATA, wid);
    Ready R;
    R.ws = ws; R.call = call; R.rF = __builtin_amdgcn_make_buffer_rsrc((void*)(ws + OFF_F), 0, NPAN * 256, 0x00020000);
    R.zone = lds + LDS_ZONE + wid * 512; R.pending = false; R.one = 1.0f;
    volatile LAS unsigned* const pool_word = (volatile LAS unsigned*)(lds + LDS_ZONE + 2048);
    auto tile_of = [&](int l, int j, int& rb, int& cb) { rb = 4 * (l & 3) + (j & 3); cb = colblock(2 * (j >> 2) + (l >> 2), rank); };
    int pm, pn;
    tile_of(grp, idx, pm, pn);
    if (!wait_words(ws, ws + OFF_AVB + rank * 1024, NGEMM, call)) R.one = __builtin_nanf("");
    R.pa = pm; R.pb = 16 + pn; R.n = 8;
    STAMP(ws, 1);
    g4_prologue(g, pm * PANEL, own_off(16 + pn));
    for (int ti = 0;; ++ti) {
        int qm = pm, qn = pn;
        bool has_next = ti + 1 < RS;
        if (has_next) tile_of(grp, idx + (ti + 1) * NG, qm, qn);
        g.ringB = rfl(ring_off(pn, rank)); g.nextA = rfl(qm * PANEL); g.nextB = rfl(own_off(16 + qn));
        R.check(5);
        int lim = R.plan(0, ti == RS - 1 ? 28 : 32);
        SCHED;
        step<0, true, 60>(g, 0); step<1, false, 60>(g, 0); step<2, false, 20>(g, 0); step<3, false, 20>(g, 0);
        for (int it = 1; it < 32;) {
            if (it >= lim) {
                R.check(it < 30 ? 2 * it + 5 : 64);
                int stop = 32;
                if (ti == RS - 1) {
                    stop = it < 28 ? 28 : it == 28 ? 29 : 32;
                    if (it == 28 && wid == 0 && lane_now() == 0) {
                        unsigned got = ~0u;
                        for (int k = 0; k < 8 && got == ~0u; ++k) {
                            const int l = grp ^ k;
                            gu32* const c = (gu32*)(ws + OFF_POOL + l * 128);
                            __hip_atomic_fetch_max(c, call << 10, __ATOMIC_RELAXED, __HIP_MEMORY_SCOPE_AGENT);
                            const unsigned t = __hip_atomic_fetch_add(c, 1u, __ATOMIC_RELAXED, __HIP_MEMORY_SCOPE_AGENT) - (call << 10);
                            if (t < (unsigned)((l < 4 ? NT0 : NT1) - RS * NG)) got = l << 8 | t;
                        }
                        *pool_word = got;
                    }
                    if (it == 29) {
                        const unsigned t = rfl(*pool_word);
                        if (t != ~0u) { has_next = true; tile_of(t >> 8, RS * NG + (t & 255), qm, qn); g.nextA = rfl(qm * PANEL); g.nextB = rfl(own_off(16 + qn)); }
                    }
                }
                lim = R.plan(it, stop);
            }
            do {
                SCHED;
                step<0, false, 20>(g, it); step<1, false, 20>(g, it); step<2, false, 20>(g, it); step<3, false, 20>(g, it);
            } while (++it < lim);
        }
        STAMP(ws, 8 + 2 * ti);
        R.pa = qm; R.pb = 16 + qn;
        R.request();
        {
            const bool ctile = pn >= NBB;
            const unsigned rowb = ctile ? 1024 : NN * 4, colb = ctile ? (pn - NBB - CPR * rank) * (MM * 1024u) : pn * 1024u;
            const __amdgpu_buffer_rsrc_t rO = __builtin_amdgcn_make_buffer_rsrc(ctile ? (void*)(ws + OFF_CST) : (void*)p.out[rank], 0, ctile ? CSTB : OUTB, 0x00020000);
            g4_epilogue(g, rO, (pm * 256 + wr * 128) * rowb + colb + wc * 512, rowb, R.one);
        }
        WAITVM(60);
        if (wid == 0 && lane_now() == 0) __hip_atomic_fetch_add((gu32*)(ws + OFF_PROG + order_of(pn, rank) / GS * 4), 1u, __ATOMIC_RELAXED, __HIP_MEMORY_SCOPE_AGENT);
        if (pn >= NBB) {
            const int ci = pn - NBB - CPR * rank;
            WAITVM(0);
            BARRIER;
            pack_tile(p, lds + LDS_ZONE + 4096 + wid * 1024, wid, ci * 128 + pm * 8);
            WAITVM(0);
            BARRIER;
            if (wid == 0) {
                const int lane = lane_now();
                if (lane < NR) st_sys(p.ws[lane] + OFF_AVP + (rank * 64 + ci * 16 + pm) * 4, call);
            }
        }
        STAMP(ws, 9 + 2 * ti);
        if (!has_next) break;
        pm = qm; pn = qn;
    }
    __builtin_amdgcn_s_waitcnt(WCNT(0, 0));
}

__global__ __launch_bounds__(256, 1) void fused(Args) {
    extern __shared__ __attribute__((aligned(16))) unsigned char shm[];
    Params& p = *(Params*)__builtin_amdgcn_kernarg_segment_ptr();
    LAS unsigned char* lds = (LAS unsigned char*)shm;
    char* const ws = p.ws[p.rank];
    const int wid = rfl(threadIdx.x >> 6);
    STAMP(ws, 0);
    const int grp = blockIdx.x & 7, idx = blockIdx.x >> 3;
    if (idx < NG) gemm_wg(p, lds, wid, grp, idx);
    else {
        const int cw = (idx - NG) * 8 + grp;
        const unsigned call = draw_call(ws, wid);
        if (wid < 3 || cw >= NLW) fetch_wave(p, lds + wid * 37376, call, wid, cw * 4 + wid - (cw < NLW ? cw : NLW));
    }
}
}

extern "C" void kernel_launch(int rank, int nranks, ncclComm_t comm, hipStream_t stream,
                              void* const* d_in, const int* in_sizes, int n_in,
                              void* d_out, int out_size,
                              const int* in_dtypes, int out_dtype,
                              void* const* d_out_all,
                              void* const* d_ws_all, size_t ws_bytes) {
    if (nranks != NR || (unsigned)rank >= (unsigned)NR || n_in != 2 || ws_bytes < WS_NEED || in_sizes[0] != MM * KS || in_sizes[1] != KS * NN || out_size != MM * NN) {
        fprintf(stderr, "kernel_launch: unexpected problem shape\n");
        return;
    }
    static bool attr_set[NR];
    if (!attr_set[rank]) {
        HIP_CHECK(hipFuncSetAttribute((const void*)fused, hipFuncAttributeMaxDynamicSharedMemorySize, LDS_BYTES));
        attr_set[rank] = true;
    }
    Args p;
    p.x = (const float*)d_in[0]; p.w = (const float*)d_in[1];
    for (int i = 0; i < NR; ++i) { p.ws[i] = (char*)d_ws_all[i]; p.out[i] = (float*)d_out_all[i]; }
    p.rank = rank; p.pad = 0;
    hipLaunchKernelGGL(fused, dim3(256), dim3(256), LDS_BYTES, stream, p);
    HIP_CHECK(hipGetLastError());
}
```

```cpp
#include <hip/hip_runtime.h>
#include <rccl/rccl.h>
#include <cstdio>

#define HIP_CHECK(expr) do { hipError_t _e = (expr); if (_e != hipSuccess) { \
    fprintf(stderr, "HIP error %s at %s:%d\n", hipGetErrorString(_e), __FILE__, __LINE__); \
    return; } } while (0)

#define LAS __attribute__((address_space(3)))
#define GAS __attribute__((address_space(1)))
#define DI __device__ __forceinline__
typedef _Float16 f16x8 __attribute__((ext_vector_type(8)));
typedef float f32x4 __attribute__((ext_vector_type(4)));
typedef unsigned u32x4 __attribute__((ext_vector_type(4)));
typedef unsigned u32x2 __attribute__((ext_vector_type(2)));
typedef GAS unsigned gu32;

#define NG 29
#define DBG 0
namespace {
constexpr int MM = 4096, NN = 32768, KS = 512, NR = 8;
constexpr int NGEMM = 8 * NG, NCOMM = 256 - NGEMM, NCW = 4 * NCOMM;
constexpr int NPW = NCW / 7, NIDLE = NCW - 7 * NPW;
constexpr int TW = 384;
constexpr int CPR = 2, NCB = NR * CPR;
constexpr int BCOLS = NN - NCB * TW, NBB = (BCOLS + TW - 1) / TW;
constexpr int NWP = NBB + NCB, NPAN = 16 + NWP;
constexpr int NL = CPR + NBB;
constexpr int NROUND = 5, PSLOT = 4 * NG - CPR;
static_assert(NROUND * 2 * NG == 4 * NL + CPR, "slots");
constexpr unsigned STEPX = 16384, STEPW = 24576, XBLK = 2 * STEPX, WBLK = 2 * STEPW, XPANEL = 64 * XBLK;
constexpr unsigned QB = 14848, QMAIN = 2560;
constexpr unsigned OUTB = (unsigned)MM * NN * 4u, ROWB = NN * 4;
constexpr unsigned OFF_CALL = 0;
constexpr unsigned OFF_TMO = 32768;
constexpr unsigned OFF_FIN = 32768 + 1024;
constexpr unsigned OFF_TICK = 32768 + 4096;
constexpr unsigned OFF_F = 65536;
constexpr unsigned OFF_AV = 131072;
constexpr unsigned OFF_AVP = OFF_AV + 8192;
constexpr unsigned OFF_AVC = OFF_AVP + 2048;
constexpr unsigned OFF_DONE = 196608;
constexpr unsigned OFF_DBG = 262144, OFF_DBG2 = 524288;
constexpr int RING = NL - 32;
constexpr unsigned OFF_DATA = 1u << 20;
constexpr unsigned REL_WL = 16 * XPANEL;
constexpr unsigned REL_WR = REL_WL + NWP * 8 * WBLK;
constexpr unsigned DATAB = REL_WR + RING * 56 * WBLK;
constexpr int NQ16 = 16 * 8 * 2 + NWP * 8 * 3;
constexpr unsigned OFF_PK16 = OFF_DATA + DATAB;
constexpr int NCI = CPR * (MM / 32);
constexpr unsigned OFF_PK32 = OFF_PK16 + NQ16 * QB;
constexpr unsigned OFF_RAW32 = OFF_PK32 + NCI * 3 * QB;
constexpr unsigned WS_NEED = OFF_RAW32 + NCI * WBLK;
static_assert(NCW * 128 <= 65536 && NPAN * 256 <= 65536 && NIDLE <= NCOMM, "flags");
constexpr int LDS_A = 4 * STEPW, LDS_ZONE = LDS_A + 3 * STEPX, LDS_BYTES = LDS_ZONE + 8192;
constexpr unsigned long long DEADLINE = 50000000ull;
constexpr int AUX_SYS = 17, AUX_NT = 2;

struct Args {
    const float* x; const float* w;
    char* ws[NR]; float* out[NR];
    int rank, pad;
};
typedef const __attribute__((address_space(4))) Args Params;

constexpr int WCNT(int vm, int lgkm) { return (vm & 15) | (7 << 4) | ((lgkm & 15) << 8) | ((vm >> 4) << 14); }
#define SCHED __builtin_amdgcn_sched_barrier(0)
#define WAITVM(n) asm volatile("s_waitcnt vmcnt(" #n ")" ::: "memory")
#define WAITLG0 asm volatile("s_waitcnt lgkmcnt(0)" ::: "memory")
#define BARRIER do { asm volatile("" ::: "memory"); __builtin_amdgcn_s_barrier(); asm volatile("" ::: "memory"); } while (0)

DI unsigned ld_sys(const char* p) { return __hip_atomic_load((gu32*)p, __ATOMIC_RELAXED, __HIP_MEMORY_SCOPE_SYSTEM); }
DI void st_sys(char* p, unsigned v) { __hip_atomic_store((gu32*)p, v, __ATOMIC_RELAXED, __HIP_MEMORY_SCOPE_SYSTEM); }
DI unsigned ld_ag(const char* p) { return __hip_atomic_load((gu32*)p, __ATOMIC_RELAXED, __HIP_MEMORY_SCOPE_AGENT); }
DI void st_ag(char* p, unsigned v) { __hip_atomic_store((gu32*)p, v, __ATOMIC_RELAXED, __HIP_MEMORY_SCOPE_AGENT); }
DI unsigned rfl(unsigned x) { return __builtin_amdgcn_readfirstlane(x); }
DI int lane_now() { int l; asm volatile("v_mbcnt_lo_u32_b32 %0, -1, 0\n\tv_mbcnt_hi_u32_b32 %0, -1, %0" : "=v"(l)); return l; }
DI bool give_up(char* ws, unsigned call, unsigned long long t0) {
    if (rfl(ld_ag(ws + OFF_TMO)) == call || __builtin_amdgcn_s_memrealtime() - t0 > DEADLINE) { st_ag(ws + OFF_TMO, call); return true; }
    return false;
}
DI bool wait_words(char* ws, const char* f, int n, int step, unsigned want, unsigned call) {
    const int lane = lane_now();
    const unsigned long long t0 = __builtin_amdgcn_s_memrealtime();
    for (;;) {
        bool ok = true;
#pragma unroll
        for (int i = 0; i < 4; ++i) { const int k = lane + 64 * i; if (64 * i < n) ok &= ld_sys(f + (k < n ? k : 0) * step) >= want; }
        if (__all(ok)) break;
        if (give_up(ws, call, t0)) return false;
        __builtin_amdgcn_s_sleep(4);
    }
    asm volatile("" ::: "memory");
    return true;
}
#define STAMP(ws, i) do {} while (0)
#define STAMPW(ws, i) do {} while (0)
#define STAMPV(ws, i, v) do {} while (0)

DI void store1(const char* base, unsigned v, const u32x4& f) {
    asm volatile("s_nop 4\n\tglobal_store_dwordx4 %0, %1, %2 sc0 sc1\n\ts_nop 1" :: "v"(v), "v"(f), "s"(base) : "memory");
}
DI void store3(const char* base, unsigned v, const u32x4& a, const u32x4& b, const u32x4& c) {
    asm volatile("s_nop 4\n\tglobal_store_dwordx4 %0, %1, %4 sc0 sc1\n\t"
                 "global_store_dwordx4 %0, %2, %4 offset:1024 sc0 sc1\n\t"
                 "global_store_dwordx4 %0, %3, %4 offset:2048 sc0 sc1\n\ts_nop 1"
                 :: "v"(v), "v"(a), "v"(b), "v"(c), "s"(base) : "memory");
}
DI void store4(const char* base, unsigned v, const u32x4 (&f)[4]) {
    asm volatile("s_nop 4\n\tglobal_store_dwordx4 %0, %1, %5 sc0 sc1\n\t"
                 "global_store_dwordx4 %0, %2, %5 offset:1024 sc0 sc1\n\t"
                 "global_store_dwordx4 %0, %3, %5 offset:2048 sc0 sc1\n\t"
                 "global_store_dwordx4 %0, %4, %5 offset:3072 sc0 sc1\n\ts_nop 1"
                 :: "v"(v), "v"(f[0]), "v"(f[1]), "v"(f[2]), "v"(f[3]), "s"(base) : "memory");
}
DI void store4o(const char* base, unsigned v, const u32x4 (&f)[4]) {
    asm volatile("s_nop 4\n\tglobal_store_dwordx4 %0, %4, %8 sc1 nt\n\t"
                 "global_store_dwordx4 %1, %5, %8 sc1 nt\n\t"
                 "global_store_dwordx4 %2, %6, %8 sc1 nt\n\t"
                 "global_store_dwordx4 %3, %7, %8 sc1 nt\n\ts_nop 1"
                 :: "v"(v), "v"(v + 2 * ROWB), "v"(v + 4 * ROWB), "v"(v + 6 * ROWB), "v"(f[0]), "v"(f[1]), "v"(f[2]), "v"(f[3]), "s"(base) : "memory");
}

DI unsigned perm(unsigned hi, unsigned lo, unsigned sel) { return __builtin_amdgcn_perm(hi, lo, sel); }
template <int CTRL, int ROWMASK> DI unsigned dpp0(unsigned x) { return __builtin_amdgcn_update_dpp(0u, x, CTRL, ROWMASK, 0xf, false); }
DI unsigned wave_sum(unsigned x) {
    x += dpp0<0x111, 0xf>(x); x += dpp0<0x112, 0xf>(x); x += dpp0<0x114, 0xf>(x); x += dpp0<0x118, 0xf>(x); x += dpp0<0x142, 0xa>(x); x += dpp0<0x143, 0xc>(x);
    return __builtin_amdgcn_readlane(x, 63);
}
DI unsigned umax(unsigned a, unsigned b) { return a > b ? a : b; }
DI unsigned umin(unsigned a, unsigned b) { return a < b ? a : b; }
DI unsigned wave_max(unsigned x) {
    x = umax(x, dpp0<0x111, 0xf>(x)); x = umax(x, dpp0<0x112, 0xf>(x)); x = umax(x, dpp0<0x114, 0xf>(x)); x = umax(x, dpp0<0x118, 0xf>(x));
    x = umax(x, dpp0<0x142, 0xa>(x)); x = umax(x, dpp0<0x143, 0xc>(x));
    return __builtin_amdgcn_readlane(x, 63);
}
struct ExcList {
    volatile LAS unsigned short* e; unsigned cnt;
    DI void begin(LAS unsigned char* tail, int lane) { e = (volatile LAS unsigned short*)(tail + 512); e[lane] = 0; cnt = 1; }
    DI bool add(bool act, unsigned entry) {
        const unsigned long long bal = __ballot(act);
        const unsigned slot = cnt + __builtin_amdgcn_mbcnt_hi((unsigned)(bal >> 32), __builtin_amdgcn_mbcnt_lo((unsigned)bal, 0u));
        if (act && slot < 64) e[slot] = (unsigned short)entry;
        cnt += __builtin_popcountll(bal);
        return cnt <= 64;
    }
};
DI bool pack_finish(const char* base, unsigned blk, int c, int lane, LAS unsigned char* tail, ExcList& x, unsigned ebase,
                    const u32x4& p0, const u32x4& p1, const u32x4& p2, unsigned t0, unsigned t1) {
    if (lane == 0) x.e[0] = (unsigned short)(ebase | (x.cnt > 64 ? 0x8000u : 0u));
    *(volatile LAS u32x2*)(tail + lane * 8) = (u32x2){t0, t1};
    store3(base, blk + QMAIN + c * 3072 + lane * 16, p0, p1, p2);
    const u32x4 t = *(volatile LAS u32x4*)(tail + (lane < 40 ? lane : 0) * 16);
    if (lane < 40) store1(base, blk + c * 640 + lane * 16, t);
    return x.cnt <= 64;
}
DI void pack16(const u32x4 (&f)[4], const char* base, unsigned blk, int c, int lane, LAS unsigned char* tail) {
    unsigned lo[8], h[8], g[8], sum = 0;
#pragma unroll
    for (int i = 0; i < 8; ++i) {
        const unsigned a = f[i >> 1][2 * (i & 1)], b = f[i >> 1][2 * (i & 1) + 1];
        lo[i] = perm(b, a, 0x06040200u); h[i] = perm(b, a, 0x07050301u);
        sum = __builtin_amdgcn_udot4(h[i] & 0x7c7c7c7cu, 0x01010101u, sum, false);
    }
    const int top = (int)((wave_sum(sum) + 15155u + 8191u) >> 13);
    const unsigned eb = top < 7 ? 0u : top > 31 ? 24u : (unsigned)(top - 7), b4 = eb * 0x04040404u;
    ExcList x; x.begin(tail, lane);
    unsigned m1 = 0, m0 = 0;
#pragma unroll
    for (int i = 0; i < 8; ++i) {
        g[i] = (h[i] & 0x83838383u) | (((h[i] | 0x80808080u) - b4) & 0x1c1c1c1cu);
        const unsigned xr = (g[i] + b4) ^ h[i];
        m1 |= ((xr >> 6) & 0x01010101u) << i; m0 |= ((xr >> 5) & 0x01010101u) << i;
    }
    for (unsigned m = m1 | m0; __any(m != 0); m &= m - 1) {
        const int k = m ? __builtin_ctz(m) : 0, i = k & 7;
        if (!x.add(m != 0, ((i >> 2) * 1024 + lane * 16 + (i & 3) * 4 + (k >> 3)) | ((m0 >> k) & 1u) << 11 | ((m1 >> k) & 1u) << 12)) break;
    }
    unsigned d[6];
#pragma unroll
    for (int q = 0; q < 2; ++q) {
        const unsigned s = g[4 * q + 3];
        d[3 * q] = g[4 * q] | (s & 0x03030303u) << 5; d[3 * q + 1] = g[4 * q + 1] | (s & 0x0c0c0c0cu) << 3;
        d[3 * q + 2] = g[4 * q + 2] | (s & 0x10101010u) << 1 | (s & 0x80808080u) >> 1;
    }
    pack_finish(base, blk, c, lane, tail, x, eb, (u32x4){lo[0], lo[1], lo[2], lo[3]}, (u32x4){lo[4], lo[5], lo[6], lo[7]}, (u32x4){d[0], d[1], d[2], d[3]}, d[4], d[5]);
}
DI bool pack32(const u32x4 (&f)[4], const char* base, unsigned blk, int c, int lane, LAS unsigned char* tail) {
    unsigned t[12], h[4], mx = 0;
#pragma unroll
    for (int j = 0; j < 4; ++j) {
        t[3 * j] = perm(f[j][3], f[j][0], 0x04020100u); t[3 * j + 1] = perm(f[j][3], f[j][1], 0x05020100u); t[3 * j + 2] = perm(f[j][3], f[j][2], 0x06020100u);
        h[j] = perm(perm(f[j][3], f[j][2], 0x07030703u), perm(f[j][1], f[j][0], 0x07030703u), 0x05040100u);
#pragma unroll
        for (int d = 0; d < 4; ++d) mx = umax(mx, f[j][d] & 0x7fffffffu);
    }
    const unsigned top = wave_max(mx) >> 24, eb = top < 7 ? 0u : top - 7, b1 = eb * 0x01010101u;
    ExcList x; x.begin(tail, lane);
    unsigned cd[4], xn[2] = {0, 0};
#pragma unroll
    for (int j = 0; j < 4; ++j) {
        cd[j] = ((h[j] | 0x80808080u) - b1) & 0x07070707u;
        const unsigned xr = ((cd[j] + b1) ^ h[j]) & 0x78787878u;
        xn[j >> 1] |= (j & 1) ? xr << 1 : xr >> 3;
    }
    for (unsigned long long m = (unsigned long long)xn[1] << 32 | xn[0]; __any(m != 0);) {
        const int k = m ? __builtin_ctzll(m) >> 2 : 0, j = 2 * (k >> 3) + (k & 1);
        const unsigned v = (unsigned)(m >> (4 * k)) & 15u;
        if (!x.add(m != 0, (lane * 16 + j * 4 + ((k >> 1) & 3)) | v << 11)) break;
        m &= ~(15ull << (4 * k));
    }
    const unsigned n0 = cd[0] | (h[0] & 0x80808080u) >> 4 | cd[1] << 4 | (h[1] & 0x80808080u), n1 = cd[2] | (h[2] & 0x80808080u) >> 4 | cd[3] << 4 | (h[3] & 0x80808080u);
    return pack_finish(base, blk, c, lane, tail, x, eb, (u32x4){t[0], t[1], t[2], t[3]}, (u32x4){t[4], t[5], t[6], t[7]}, (u32x4){t[8], t[9], t[10], t[11]}, n0, n1);
}
DI bool unpack(bool is32, const LAS unsigned char* main3, const LAS unsigned char* tail, LAS unsigned char* patch, int lane, u32x4 (&f)[4]) {
    unsigned e = *(const volatile LAS unsigned short*)(tail + 512 + lane * 2);
    const u32x2 tt = *(const volatile LAS u32x2*)(tail + lane * 8);
    const u32x4 p0 = *(const volatile LAS u32x4*)(main3 + lane * 16), p1 = *(const volatile LAS u32x4*)(main3 + 1024 + lane * 16), p2 = *(const volatile LAS u32x4*)(main3 + 2048 + lane * 16);
    const unsigned hdr = rfl(e);
    if (lane == 0) e = 0;
    LAS unsigned* const pw = (LAS unsigned*)(patch + (e & 0x7fcu));
    __hip_atomic_fetch_or(pw, (e >> 11) << ((is32 ? 3 : 5) + 8 * (e & 3u)), __ATOMIC_RELAXED, __HIP_MEMORY_SCOPE_WAVEFRONT);
    const u32x4 x0 = *(const volatile LAS u32x4*)(patch + lane * 16), x1 = *(const volatile LAS u32x4*)(patch + 1024 + lane * 16);
    *(volatile LAS unsigned*)pw = 0;
    if (is32) {
        const unsigned b1 = (hdr & 127u) * 0x01010101u, t[12] = {p0[0], p0[1], p0[2], p0[3], p1[0], p1[1], p1[2], p1[3], p2[0], p2[1], p2[2], p2[3]};
#pragma unroll
        for (int j = 0; j < 4; ++j) {
            const unsigned n = tt[j >> 1];
            const unsigned h = ((j & 1) ? ((((n >> 4) & 0x07070707u) + b1) | (n & 0x80808080u)) : (((n & 0x07070707u) + b1) | ((n << 4) & 0x80808080u))) ^ x0[j];
            f[j][0] = perm(h, t[3 * j], 0x04020100u); f[j][1] = perm(h, t[3 * j + 1], 0x05020100u); f[j][2] = perm(h, t[3 * j + 2], 0x06020100u);
            f[j][3] = perm(perm(h, t[3 * j + 2], 0x07030703u), perm(t[3 * j + 1], t[3 * j], 0x07030703u), 0x05040100u);
        }
    } else {
        const unsigned b4 = (hdr & 31u) * 0x04040404u, d[6] = {p2[0], p2[1], p2[2], p2[3], tt[0], tt[1]};
#pragma unroll
        for (int q = 0; q < 2; ++q) {
            const unsigned a = d[3 * q], b = d[3 * q + 1], c = d[3 * q + 2];
            const unsigned g[4] = {a & 0x9f9f9f9fu, b & 0x9f9f9f9fu, c & 0x9f9f9f9fu,
                                   ((a >> 5) & 0x03030303u) | ((b >> 3) & 0x0c0c0c0cu) | ((c >> 1) & 0x10101010u) | ((c << 1) & 0x80808080u)};
#pragma unroll
            for (int i = 0; i < 4; ++i) {
                const unsigned h = (g[i] + b4) ^ (q ? x1[i] : x0[i]), lo = q ? p1[i] : p0[i];
                f[2 * q + (i >> 1)][2 * (i & 1)] = perm(h, lo, 0x05010400u); f[2 * q + (i >> 1)][2 * (i & 1) + 1] = perm(h, lo, 0x07030602u);
            }
        }
    }
    return hdr < 0x8000u;
}

DI int col0(int cb) { return cb >= NBB ? BCOLS + (cb - NBB) * TW : cb == NBB - 1 ? BCOLS - TW : cb * TW; }
DI int colblock(int li, int rank) { return li < CPR ? NBB + CPR * rank + li : li - CPR; }
DI unsigned own_off(int panel, int kt) { return panel < 16 ? panel * XPANEL + kt * XBLK : REL_WL + ((panel - 16) * 8 + kt) * WBLK; }
DI unsigned ring_off(int li) { return REL_WR + (li >= RING ? li - RING : li) * (56 * WBLK); }
DI unsigned pk16_off(int panel, int kt) { return OFF_PK16 + (panel < 16 ? (panel * 8 + kt) * 2 : 256 + ((panel - 16) * 8 + kt) * 3) * QB; }
static_assert(2 * RING >= NL, "ring");
constexpr int round_end(int r) { return (2 * NG * (r + 1) - (r ? CPR : 0) + 3) / 4; }
constexpr int B0 = round_end(0) - CPR, QK0 = 32 + 3 * (B0 + NCB);
constexpr int ST1 = 3, KT1 = ST1 * NGEMM / QK0;
static_assert(QK0 <= NGEMM && NQ16 > ST1 * NGEMM + NGEMM, "stages");
DI void conv_quad(int b, int& panel, int& kt, int& qd) {
    if (b < 8 * QK0) {
        kt = b / QK0; const int i = b - kt * QK0;
        if (i < 32) { panel = i >> 1; qd = i & 1; }
        else { const int w = (i - 32) / 3; qd = i - 32 - 3 * w; panel = 16 + (w < B0 ? w : w - B0 + NBB); }
    } else { const int r = b - 8 * QK0, k = r / 3; qd = r - 3 * k; panel = 16 + B0 + (k >> 3); kt = k & 7; }
}
DI unsigned draw_call(char* ws, int wid) {
    unsigned c;
    asm volatile("s_nop 4\n\tglobal_load_dword %0, %1, %2 sc1\n\ts_waitcnt vmcnt(0)" : "=&v"(c) : "v"(OFF_CALL + blockIdx.x * 128), "s"(ws) : "memory");
    c = rfl(c) + 1;
    BARRIER;
    if (wid == 0 && lane_now() == 0) st_ag(ws + OFF_CALL + blockIdx.x * 128, c);
    return c;
}
DI unsigned convert_shards(Params& p, LAS unsigned char* lds, int wid, int gi) {
    const int lane = lane_now(), rank = p.rank;
    char* const ws = p.ws[rank];
    const __amdgpu_buffer_rsrc_t rx = __builtin_amdgcn_make_buffer_rsrc((void*)p.x, 0, MM * KS * 4, 0x00020000);
    const __amdgpu_buffer_rsrc_t rw = __builtin_amdgcn_make_buffer_rsrc((void*)p.w, 0, KS * NN * 4, 0x00020000);
    const unsigned vx = ((lane & 15) * KS + 8 * (lane >> 4)) * 4, vw = (8 * (lane >> 4) * NN + 4 * (lane & 15)) * 4;
    LAS unsigned char* const ring = lds + wid * 32768;
    LAS unsigned char* const tail = lds + 131072 + wid * 1024;
    const int nu = (NQ16 - gi + NGEMM - 1) / NGEMM;
    auto load = [&](int u) {
        int pa, kt, qd; conv_quad(gi + (u < nu ? u : 0) * NGEMM, pa, kt, qd);
        LAS unsigned char* const slot = ring + (u & 3) * 8192;
        if (pa < 16) {
            const unsigned s0 = ((pa * 256 + 64 * wid) * KS + 32 * (2 * kt + qd)) * 4;
#pragma unroll
            for (int e = 0; e < 8; ++e) __builtin_amdgcn_raw_ptr_buffer_load_lds(rx, (LAS void*)(slot + e * 1024), 16, vx, s0 + (e >> 1) * (16 * KS * 4) + (e & 1) * 16, 0, AUX_NT);
        } else {
            const int c = 4 * qd + wid, hi = c >= 6;
            const unsigned s0 = (32 * (2 * kt + hi) * NN + col0(pa - 16) + 64 * (c - 6 * hi)) * 4;
#pragma unroll
            for (int e = 0; e < 8; ++e) __builtin_amdgcn_raw_ptr_buffer_load_lds(rw, (LAS void*)(slot + e * 1024), 16, vw, s0 + e * ROWB, 0, AUX_NT);
        }
    };
    load(0);
    char* peer_ws = p.ws[lane & 7];
    const unsigned call = draw_call(ws, wid);
    asm volatile("" : "+v"(peer_ws));
    for (int u = 0; u < nu; ++u) {
        if (u > 0) { load(u + 3); WAITVM(24); }
        if (u == ST1 + 1) {
            BARRIER;
            if (wid == 0 && lane < NR) st_sys(peer_ws + OFF_AV + (rank * 256 + gi) * 4, 4 * call + 2);
        }
        int pa, kt, qd; conv_quad(gi + u * NGEMM, pa, kt, qd);
        f32x4 v[8]; u32x4 f[4];
#pragma unroll
        for (int e = 0; e < 8; ++e) v[e] = *(const LAS f32x4*)(ring + (u & 3) * 8192 + e * 1024 + lane * 16);
#pragma unroll
        for (int j = 0; j < 4; ++j) { f16x8 h;
#pragma unroll
            for (int e = 0; e < 8; ++e) h[e] = (_Float16)(pa < 16 ? v[2 * j + (e >> 2)][e & 3] : v[e][j]);
            f[j] = __builtin_bit_cast(u32x4, h); }
        pack16(f, ws, pk16_off(pa, kt) + qd * QB, wid, lane, tail);
        store4(ws, OFF_DATA + own_off(pa, kt) + (4 * qd + wid) * 4096 + lane * 16, f);
        if (u == 0) {
            load(1); load(2); load(3);
            WAITVM(24);
            BARRIER;
            if (wid == 0 && lane < NR) st_sys(peer_ws + OFF_AV + (rank * 256 + gi) * 4, 4 * call + 1);
        }
    }
    WAITVM(0);
    BARRIER;
    if (wid == 0 && lane < NR) st_sys(peer_ws + OFF_AV + (rank * 256 + gi) * 4, 4 * call + 3);
    return call;
}

constexpr int G0 = 16 + round_end(0), NBI = (16 + NL) * 8, NITEM = NBI + NCI;
struct Item { unsigned size, pk, raw, dof; int flag; bool three; };
#define XLAGS {0, 0, 3, 3}
#define BL 0
constexpr int XL[4] = XLAGS, NSTEP0 = 8 + (XL[3] > BL ? XL[3] : BL);
constexpr int x_lo(int s) { int g = 0; while (g < 4 && s - XL[g] >= 8) ++g; return 4 * g; }
constexpr int x_hi(int s) { int g = 0; while (g < 4 && s >= XL[g]) ++g; return 4 * g; }
constexpr int w_lo(int s) { return s < 8 ? 16 : 16 + CPR; }
constexpr int w_hi(int s) { return s < BL || s >= 8 + BL ? 16 + CPR : G0; }
constexpr int step_start(int s) { int n = 0; for (int k = 0; k < s; ++k) n += x_hi(k) - x_lo(k) + w_hi(k) - w_lo(k); return n; }
static_assert(step_start(NSTEP0) == 8 * G0, "order");
DI void block_of(unsigned t, int& i, int& kt) {
    if (t < 8u * G0) {
        int s = 0, hi = x_hi(0), skip = w_lo(0) - x_hi(0); i = t;
#pragma unroll
        for (int k = 1; k < NSTEP0; ++k) if (t >= (unsigned)step_start(k)) { s = k; i = t - step_start(k) + x_lo(k); hi = x_hi(k); skip = w_lo(k) - x_hi(k); }
        if (i >= hi) i += skip;
        kt = s - (i < 4 ? XL[0] : i < 8 ? XL[1] : i < 12 ? XL[2] : i < 16 ? XL[3] : i < 16 + CPR ? 0 : BL);
    } else {
        unsigned u = 0, n = 1, b = 0;
#pragma unroll
        for (int r = 1; r < NROUND; ++r)
            if (t >= 8u * (16 + round_end(r - 1))) { b = 16 + round_end(r - 1); u = t - 8u * b; n = round_end(r) - round_end(r - 1); }
        kt = u / n; i = b + (u - kt * n);
    }
}
DI unsigned slab_out(unsigned s, int from) { return (s & 127u) * 32u * ROWB + (BCOLS + (CPR * from + (s >> 7)) * TW) * 4u; }
DI Item make_item(unsigned t, int rank, int peer, int pj) {
    Item it;
    it.size = WS_NEED;
    if (t < (unsigned)NBI) {
        int i, kt; block_of(t, i, kt);
        const int panel = i < 16 ? i : 16 + colblock(i - 16, rank);
        it.three = i >= 16;
        it.pk = pk16_off(panel, kt); it.raw = OFF_DATA + own_off(panel, kt);
        it.dof = OFF_DATA + (i < 16 ? i * XPANEL + (8 + 7 * kt + pj) * XBLK : ring_off(i - 16) + (7 * kt + pj) * WBLK);
        it.flag = panel * 64 + 8 + 7 * kt + pj;
    } else {
        const unsigned s = t < (unsigned)NITEM ? t - NBI : 0;
        if (t >= (unsigned)NITEM) it.size = 0;
        it.three = true;
        it.pk = OFF_PK32 + s * (3 * QB); it.raw = OFF_RAW32 + s * WBLK; it.dof = slab_out(s, peer); it.flag = -1;
    }
    return it;
}
DI void load_main(const char* base, unsigned size, unsigned off, LAS unsigned char* to, int lane) {
    const __amdgpu_buffer_rsrc_t r = __builtin_amdgcn_make_buffer_rsrc((void*)base, 0, size, 0x00020000);
    asm volatile("s_mov_b32 m0, %3\n\ts_nop 4\n\tbuffer_load_dwordx4 %0, %1, %2 offen sc0 sc1 lds\n\tbuffer_load_dwordx4 %0, %1, %2 offen offset:1024 sc0 sc1 lds\n\t"
                 "buffer_load_dwordx4 %0, %1, %2 offen offset:2048 sc0 sc1 lds" :: "v"(lane * 16), "s"(r), "s"(off), "s"((unsigned)(size_t)to) : "memory");
}
DI void load_tails(const char* base, unsigned size, unsigned off, LAS unsigned char* to, int lane) {
    const __amdgpu_buffer_rsrc_t r = __builtin_amdgcn_make_buffer_rsrc((void*)base, 0, size, 0x00020000);
    asm volatile("s_mov_b32 m0, %3\n\ts_nop 4\n\tbuffer_load_dwordx4 %0, %1, %2 offen sc0 sc1 lds\n\tbuffer_load_dwordx4 %0, %1, %2 offen offset:1024 sc0 sc1 lds\n\t"
                 "s_mov_b32 exec_hi, 0\n\tbuffer_load_dwordx4 %0, %1, %2 offen offset:2048 sc0 sc1 lds\n\ts_mov_b32 exec_hi, -1"
                 :: "v"(lane * 16), "s"(r), "s"(off), "s"((unsigned)(size_t)to) : "memory");
}
constexpr unsigned NANS = 0x7fc07fc0u;
struct Fetch {
    const char* src; const char* ws; const char* out;
    LAS unsigned char* lds;
    unsigned slot; int lane; bool ok;
    template <int C0, bool WAITED> DI void quad(const Item& cur, unsigned qd, unsigned nsize, unsigned npk) {
        LAS unsigned char* const q = lds + slot; LAS unsigned char* const nq = lds + (QB - slot);
#pragma unroll
        for (int c = 0; c < 4; ++c) {
            if (c > 0 || !WAITED) WAITVM(25);
            if (c == 0) load_tails(src, nsize, npk, nq, lane);
            load_main(src, nsize, npk + QMAIN + c * 3072, nq + QMAIN + c * 3072, lane);
            u32x4 f[4];
            if (!unpack(cur.flag < 0, q + QMAIN + c * 3072, q + c * 640, lds + 2 * QB, lane, f)) {
                const char* a = src + (size_t)(cur.raw + (4 * qd + c) * 4096 + lane * 16);
                asm volatile("global_load_dwordx4 %0, %4, off sc0 sc1\n\tglobal_load_dwordx4 %1, %4, off offset:1024 sc0 sc1\n\t"
                             "global_load_dwordx4 %2, %4, off offset:2048 sc0 sc1\n\tglobal_load_dwordx4 %3, %4, off offset:3072 sc0 sc1\n\ts_waitcnt vmcnt(0)"
                             : "=&v"(f[0]), "=&v"(f[1]), "=&v"(f[2]), "=&v"(f[3]) : "v"(a) : "memory");
            }
            if (!ok) f[0] = f[1] = f[2] = f[3] = (u32x4){NANS, NANS, NANS, NANS};
            if (cur.flag >= 0) store4(ws, cur.dof + (4 * qd + c) * 4096 + lane * 16, f);
            else store4o(out, cur.dof + ((C0 + c) / 3 * 8 + (lane >> 5)) * ROWB + (C0 + c) % 3 * 512 + (lane & 31) * 16, f);
        }
        slot = QB - slot;
    }
};
DI void fetch_wave(Params& p, LAS unsigned char* lds, unsigned call, int wid, int wv) {
    const int lane = lane_now(), rank = p.rank;
    char* const ws = p.ws[rank];
    const int pj = wv % 7, k0 = wv / 7, peer = (rank + 1 + pj) & 7;
    Fetch F;
    F.src = p.ws[peer]; F.ws = ws; F.out = (const char*)p.out[rank]; F.lds = lds; F.lane = lane;
    *(LAS u32x4*)(lds + 2 * QB + lane * 16) = (u32x4){0, 0, 0, 0}; *(LAS u32x4*)(lds + 2 * QB + 1024 + lane * 16) = (u32x4){0, 0, 0, 0};
    const unsigned wbase = (call - 1) * NBI - NPW, cbase = (call - 1) * NCI - NPW;
    static_assert(CPR * 16 == 32 && NL - RING == 32, "probe lanes");
    const unsigned vprobe = lane < 32 ? OFF_AVP + peer * 256 + lane * 4 : OFF_FIN + (lane - 32) * 4;
    static_assert(NPW <= step_start(1), "first items");
    int i0, kt0; block_of(k0, i0, kt0);
    const int q0 = i0 < 16 ? 2 * i0 : 32 + 3 * (i0 < 16 + CPR ? B0 + CPR * rank + i0 - 16 : i0 - 16 - CPR);
    F.ok = wait_words(ws, ws + OFF_AV + q0 * 4, NR, 1024, 4 * call + 1, call) && wait_words(ws, ws + OFF_AV + peer * 1024 + q0 * 4, 3, 4, 4 * call + 1, call);
    STAMP(ws, 1);
    unsigned tw = k0, tc = k0;
    unsigned stage = 0;
    auto choose = [&](unsigned probe) -> unsigned {
        if (tw < (unsigned)NBI) {
            int i, kt; block_of(tw, i, kt);
            if (i < 16 + RING || __builtin_amdgcn_readlane(probe, 32 + i - 16 - RING) >= 16 * call) {
                const unsigned need = tw < (unsigned)NPW ? 0 : tw >= 8u * G0 || kt >= KT1 ? 3 : kt > 0 ? 2 : 1;
                if (stage < need) { F.ok &= wait_words(ws, ws + OFF_AV + peer * 1024, NGEMM, 4, 4 * call + need, call); stage = need; }
                return tw;
            }
        }
        if (tc < (unsigned)NCI && (unsigned)__ballot(probe >= call) == ~0u) return NBI + tc;
        return NITEM;
    };
    while (tw < (unsigned)NBI || tc < (unsigned)NCI) {
        unsigned t;
        {
            const unsigned long long t0 = __builtin_amdgcn_s_memrealtime();
            while ((t = choose(ld_sys(ws + vprobe))) == (unsigned)NITEM) {
                if (give_up(ws, call, t0)) { F.ok = false; t = tw < (unsigned)NBI ? tw : NBI + tc; break; }
                __builtin_amdgcn_s_sleep(32);
            }
        }
        Item cur = make_item(t, rank, peer, pj);
        F.slot = 0;
        load_tails(F.src, cur.size, cur.pk, lds, lane);
#pragma unroll
        for (int c = 0; c < 4; ++c) load_main(F.src, cur.size, cur.pk + QMAIN + c * 3072, lds + QMAIN + c * 3072, lane);
        WAITVM(0);
        int pflag = -1;
        for (;;) {
            unsigned tk, probe; unsigned long long sv;
            asm volatile("s_nop 3\n\ts_waitcnt vmcnt(25)\n\tglobal_load_dword %1, %6, %5 sc0 sc1\n\t"
                         "s_mov_b64 %2, exec\n\ts_mov_b64 exec, 1\n\tglobal_atomic_add %0, %3, %4, %5 sc0\n\ts_mov_b64 exec, %2"
                         : "=&v"(tk), "=&v"(probe), "=&s"(sv) : "v"(OFF_TICK + (cur.flag < 0 ? 1024u : 0u) + pj * 128u), "v"(1u), "s"(ws), "v"(vprobe) : "memory");
            F.quad<0, true>(cur, 0, cur.size, cur.pk + QB);
            asm volatile("s_waitcnt vmcnt(25)" : "+v"(tk), "+v"(probe) :: "memory");
            if (pflag >= 0 && lane == 0) st_sys(ws + OFF_F + pflag * 4, call);
            if (cur.flag >= 0) tw = rfl(tk) - wbase; else tc = rfl(tk) - cbase;
            if (cur.three) F.quad<4, true>(cur, 1, cur.size, cur.pk + 2 * QB);
            t = choose(probe);
            const Item nxt = make_item(t, rank, peer, pj);
            F.quad<8, false>(cur, cur.three ? 2 : 1, nxt.size, nxt.pk);
            pflag = cur.flag;
            if (t >= (unsigned)NITEM) break;
            cur = nxt;
        }
        WAITVM(0);
        if (pflag >= 0 && lane == 0) st_sys(ws + OFF_F + pflag * 4, call);
    }
    STAMPW(ws, 44 + wid);
    if (lane == 0) st_sys(p.ws[peer] + OFF_DONE + wv * 128, call);
    if (!wait_words(ws, ws + OFF_DONE + wv * 128, 1, 4, call, call) && lane == 0) p.out[rank][BCOLS + CPR * peer * TW] = __builtin_nanf("");
    STAMPW(ws, 48 + wid);
}
#define PT_SET(A0, A1, A2, A3, R) \
    asm volatile("s_waitcnt vmcnt(60)\n\t" R \
                 "global_load_dwordx4 a[" A0 "], %16, %20 sc1 nt\n\tglobal_load_dwordx4 a[" A1 "], %17, %20 sc1 nt\n\t" \
                 "global_load_dwordx4 a[" A2 "], %18, %20 sc1 nt\n\tglobal_load_dwordx4 a[" A3 "], %19, %20 sc1 nt" \
                 : "=&v"(r[0]), "=&v"(r[1]), "=&v"(r[2]), "=&v"(r[3]), "=&v"(r[4]), "=&v"(r[5]), "=&v"(r[6]), "=&v"(r[7]), \
                   "=&v"(r[8]), "=&v"(r[9]), "=&v"(r[10]), "=&v"(r[11]), "=&v"(r[12]), "=&v"(r[13]), "=&v"(r[14]), "=&v"(r[15]) \
                 : "v"(v), "v"(v + 2 * ROWB), "v"(v + 4 * ROWB), "v"(v + 6 * ROWB), "s"(out) : "memory", PT_CLOB)
#define PT_CLOB "a0", "a1", "a2", "a3", "a4", "a5", "a6", "a7", "a8", "a9", "a10", "a11", "a12", "a13", "a14", "a15", "a16", "a17", "a18", "a19", "a20", "a21", "a22", "a23", "a24", "a25", "a26", "a27", "a28", "a29", "a30", "a31", "a32", "a33", "a34", "a35", "a36", "a37", "a38", "a39", "a40", "a41", "a42", "a43", "a44", "a45", "a46", "a47", "a48", "a49", "a50", "a51", "a52", "a53", "a54", "a55", "a56", "a57", "a58", "a59", "a60", "a61", "a62", "a63", "a64", "a65", "a66", "a67", "a68", "a69", "a70", "a71", "a72", "a73", "a74", "a75", "a76", "a77", "a78", "a79", "a80", "a81", "a82", "a83", "a84", "a85", "a86", "a87", "a88", "a89", "a90", "a91", "a92", "a93", "a94", "a95", "a96", "a97", "a98", "a99", "a100", "a101", "a102", "a103", "a104", "a105", "a106", "a107", "a108", "a109", "a110", "a111", "a112", "a113", "a114", "a115", "a116", "a117", "a118", "a119", "a120", "a121", "a122", "a123", "a124", "a125", "a126", "a127"
#define PT_R0 "v_accvgpr_read_b32 %0, a0\n\t" "v_accvgpr_read_b32 %1, a1\n\t" "v_accvgpr_read_b32 %2, a2\n\t" "v_accvgpr_read_b32 %3, a3\n\t" "v_accvgpr_read_b32 %4, a4\n\t" "v_accvgpr_read_b32 %5, a5\n\t" "v_accvgpr_read_b32 %6, a6\n\t" "v_accvgpr_read_b32 %7, a7\n\t" "v_accvgpr_read_b32 %8, a8\n\t" "v_accvgpr_read_b32 %9, a9\n\t" "v_accvgpr_read_b32 %10, a10\n\t" "v_accvgpr_read_b32 %11, a11\n\t" "v_accvgpr_read_b32 %12, a12\n\t" "v_accvgpr_read_b32 %13, a13\n\t" "v_accvgpr_read_b32 %14, a14\n\t" "v_accvgpr_read_b32 %15, a15\n\t"
#define PT_R1 "v_accvgpr_read_b32 %0, a16\n\t" "v_accvgpr_read_b32 %1, a17\n\t" "v_accvgpr_read_b32 %2, a18\n\t" "v_accvgpr_read_b32 %3, a19\n\t" "v_accvgpr_read_b32 %4, a20\n\t" "v_accvgpr_read_b32 %5, a21\n\t" "v_accvgpr_read_b32 %6, a22\n\t" "v_accvgpr_read_b32 %7, a23\n\t" "v_accvgpr_read_b32 %8, a24\n\t" "v_accvgpr_read_b32 %9, a25\n\t" "v_accvgpr_read_b32 %10, a26\n\t" "v_accvgpr_read_b32 %11, a27\n\t" "v_accvgpr_read_b32 %12, a28\n\t" "v_accvgpr_read_b32 %13, a29\n\t" "v_accvgpr_read_b32 %14, a30\n\t" "v_accvgpr_read_b32 %15, a31\n\t"
#define PT_R2 "v_accvgpr_read_b32 %0, a32\n\t" "v_accvgpr_read_b32 %1, a33\n\t" "v_accvgpr_read_b32 %2, a34\n\t" "v_accvgpr_read_b32 %3, a35\n\t" "v_accvgpr_read_b32 %4, a36\n\t" "v_accvgpr_read_b32 %5, a37\n\t" "v_accvgpr_read_b32 %6, a38\n\t" "v_accvgpr_read_b32 %7, a39\n\t" "v_accvgpr_read_b32 %8, a40\n\t" "v_accvgpr_read_b32 %9, a41\n\t" "v_accvgpr_read_b32 %10, a42\n\t" "v_accvgpr_read_b32 %11, a43\n\t" "v_accvgpr_read_b32 %12, a44\n\t" "v_accvgpr_read_b32 %13, a45\n\t" "v_accvgpr_read_b32 %14, a46\n\t" "v_accvgpr_read_b32 %15, a47\n\t"
#define PT_R3 "v_accvgpr_read_b32 %0, a48\n\t" "v_accvgpr_read_b32 %1, a49\n\t" "v_accvgpr_read_b32 %2, a50\n\t" "v_accvgpr_read_b32 %3, a51\n\t" "v_accvgpr_read_b32 %4, a52\n\t" "v_accvgpr_read_b32 %5, a53\n\t" "v_accvgpr_read_b32 %6, a54\n\t" "v_accvgpr_read_b32 %7, a55\n\t" "v_accvgpr_read_b32 %8, a56\n\t" "v_accvgpr_read_b32 %9, a57\n\t" "v_accvgpr_read_b32 %10, a58\n\t" "v_accvgpr_read_b32 %11, a59\n\t" "v_accvgpr_read_b32 %12, a60\n\t" "v_accvgpr_read_b32 %13, a61\n\t" "v_accvgpr_read_b32 %14, a62\n\t" "v_accvgpr_read_b32 %15, a63\n\t"
#define PT_R4 "v_accvgpr_read_b32 %0, a64\n\t" "v_accvgpr_read_b32 %1, a65\n\t" "v_accvgpr_read_b32 %2, a66\n\t" "v_accvgpr_read_b32 %3, a67\n\t" "v_accvgpr_read_b32 %4, a68\n\t" "v_accvgpr_read_b32 %5, a69\n\t" "v_accvgpr_read_b32 %6, a70\n\t" "v_accvgpr_read_b32 %7, a71\n\t" "v_accvgpr_read_b32 %8, a72\n\t" "v_accvgpr_read_b32 %9, a73\n\t" "v_accvgpr_read_b32 %10, a74\n\t" "v_accvgpr_read_b32 %11, a75\n\t" "v_accvgpr_read_b32 %12, a76\n\t" "v_accvgpr_read_b32 %13, a77\n\t" "v_accvgpr_read_b32 %14, a78\n\t" "v_accvgpr_read_b32 %15, a79\n\t"
#define PT_R5 "v_accvgpr_read_b32 %0, a80\n\t" "v_accvgpr_read_b32 %1, a81\n\t" "v_accvgpr_read_b32 %2, a82\n\t" "v_accvgpr_read_b32 %3, a83\n\t" "v_accvgpr_read_b32 %4, a84\n\t" "v_accvgpr_read_b32 %5, a85\n\t" "v_accvgpr_read_b32 %6, a86\n\t" "v_accvgpr_read_b32 %7, a87\n\t" "v_accvgpr_read_b32 %8, a88\n\t" "v_accvgpr_read_b32 %9, a89\n\t" "v_accvgpr_read_b32 %10, a90\n\t" "v_accvgpr_read_b32 %11, a91\n\t" "v_accvgpr_read_b32 %12, a92\n\t" "v_accvgpr_read_b32 %13, a93\n\t" "v_accvgpr_read_b32 %14, a94\n\t" "v_accvgpr_read_b32 %15, a95\n\t"
#define PT_R6 "v_accvgpr_read_b32 %0, a96\n\t" "v_accvgpr_read_b32 %1, a97\n\t" "v_accvgpr_read_b32 %2, a98\n\t" "v_accvgpr_read_b32 %3, a99\n\t" "v_accvgpr_read_b32 %4, a100\n\t" "v_accvgpr_read_b32 %5, a101\n\t" "v_accvgpr_read_b32 %6, a102\n\t" "v_accvgpr_read_b32 %7, a103\n\t" "v_accvgpr_read_b32 %8, a104\n\t" "v_accvgpr_read_b32 %9, a105\n\t" "v_accvgpr_read_b32 %10, a106\n\t" "v_accvgpr_read_b32 %11, a107\n\t" "v_accvgpr_read_b32 %12, a108\n\t" "v_accvgpr_read_b32 %13, a109\n\t" "v_accvgpr_read_b32 %14, a110\n\t" "v_accvgpr_read_b32 %15, a111\n\t"
#define PT_R7 "v_accvgpr_read_b32 %0, a112\n\t" "v_accvgpr_read_b32 %1, a113\n\t" "v_accvgpr_read_b32 %2, a114\n\t" "v_accvgpr_read_b32 %3, a115\n\t" "v_accvgpr_read_b32 %4, a116\n\t" "v_accvgpr_read_b32 %5, a117\n\t" "v_accvgpr_read_b32 %6, a118\n\t" "v_accvgpr_read_b32 %7, a119\n\t" "v_accvgpr_read_b32 %8, a120\n\t" "v_accvgpr_read_b32 %9, a121\n\t" "v_accvgpr_read_b32 %10, a122\n\t" "v_accvgpr_read_b32 %11, a123\n\t" "v_accvgpr_read_b32 %12, a124\n\t" "v_accvgpr_read_b32 %13, a125\n\t" "v_accvgpr_read_b32 %14, a126\n\t" "v_accvgpr_read_b32 %15, a127\n\t"
DI void pack_tile(Params& p, LAS unsigned char* tail, unsigned call, int ct) {
    const int lane = lane_now(), rank = p.rank;
    char* const ws = p.ws[rank];
    const char* const out = (const char*)p.out[rank];
    const bool ok = wait_words(ws, ws + OFF_AVC + ct * 4, 1, 4, call, call);
    const unsigned s0 = 8 * ct, vb = slab_out(s0, rank) + (lane >> 5) * ROWB + (lane & 31) * 16;
    for (int i = -8; i < 96; ++i) {
        const unsigned n = i + 8 < 96 ? i + 8 : 95, v = vb + n / 3 * (8 * ROWB) + n % 3 * 512;
        unsigned r[16];
        if (i >= 0 && i < 8) WAITVM(28);
        switch (i & 7) {
            case 0: PT_SET("0:3", "4:7", "8:11", "12:15", PT_R0); break;
            case 1: PT_SET("16:19", "20:23", "24:27", "28:31", PT_R1); break;
            case 2: PT_SET("32:35", "36:39", "40:43", "44:47", PT_R2); break;
            case 3: PT_SET("48:51", "52:55", "56:59", "60:63", PT_R3); break;
            case 4: PT_SET("64:67", "68:71", "72:75", "76:79", PT_R4); break;
            case 5: PT_SET("80:83", "84:87", "88:91", "92:95", PT_R5); break;
            case 6: PT_SET("96:99", "100:103", "104:107", "108:111", PT_R6); break;
            default: PT_SET("112:115", "116:119", "120:123", "124:127", PT_R7); break;
        }
        if (i < 0) continue;
        if (!ok) r[0] = NANS;
        const u32x4 f[4] = {{r[0], r[1], r[2], r[3]}, {r[4], r[5], r[6], r[7]}, {r[8], r[9], r[10], r[11]}, {r[12], r[13], r[14], r[15]}};
        if (!pack32(f, ws, OFF_PK32 + (s0 * 3 + (i >> 2)) * QB, i & 3, lane, tail)) store4(ws, OFF_RAW32 + s0 * WBLK + i * 4096 + lane * 16, f);
    }
    WAITVM(0);
    if (lane < NR) st_sys(p.ws[lane] + OFF_AVP + (rank * 64 + ct) * 4, call);
}

DI f16x8 rdfrag(unsigned base, int off) { return *(const LAS f16x8*)(size_t)(base + off); }
DI void dma_to(unsigned ldsaddr) { asm volatile("s_mov_b32 m0, %0" :: "s"(ldsaddr) : "memory"); }
#define PIN_S(x) asm volatile("" : "+s"(x))
DI float rd_acc(float a) { float v; asm volatile("v_accvgpr_read_b32 %0, %1" : "=v"(v) : "a"(a)); return v; }
DI f32x4 silu4(f32x4 y, float one) {
    f32x4 t = y * -1.44269504088896f;
#pragma unroll
    for (int i = 0; i < 4; ++i) t[i] = __builtin_amdgcn_exp2f(__builtin_amdgcn_fmed3f(t[i], -86.5617024533378f, 86.5617024533378f));
    t = t + one;
#pragma unroll
    for (int i = 0; i < 4; ++i) t[i] = __builtin_amdgcn_rcpf(t[i]);
    return y * t;
}
struct G6 {
    unsigned ldsA, ldsB;
    __amdgpu_buffer_rsrc_t rD;
    unsigned voffA, voffB, voffB2;
    unsigned ard, an;
    unsigned blo, bhi;
    unsigned sA, sB, sa;
    unsigned ringB, nextA, nextB;
    f32x4 acc[8][12];
    f16x8 A[2][8], B0, B1;
};
template <int I> DI void dma6(const G6& g, unsigned v, unsigned src) {
    asm volatile("buffer_load_dwordx4 %0, %1, %2 offen offset:%3 sc0 sc1 lds" :: "v"(v), "s"(g.rD), "s"(src), "i"(I * 1024) : "memory");
}
template <int ST> DI f16x8 rdB6(const G6& g, int i) { return rdfrag(ST < 2 ? g.blo : g.bhi, (ST & 1) * STEPW + i * 1024); }
#define APOS {82, 86, 90, 94}
#define BPOS {6, 18, 30, 42, 55, 67}
template <int J, bool ZERO, int VM> DI void step6(G6& g, int it) {
    constexpr int c = J & 1, NX = (J + 1) & 3, PV = (J + 3) & 3;
    f16x8 B[14];
    B[0] = g.B0; B[1] = g.B1;
#pragma unroll
    for (int n = 0; n < 12; ++n) {
        if (n == 10) { __builtin_amdgcn_s_waitcnt(WCNT(VM, 0)); __builtin_amdgcn_s_barrier(); SCHED; }
#pragma unroll
        for (int k = 0; k < 8; ++k) {
            const int q = n * 8 + k, m = (n & 1) ? 7 - k : k;
            if (n < 8) {
                if constexpr (ZERO) asm volatile("v_mfma_f32_16x16x32_f16 %0, %1, %2, 0" : "=a"(g.acc[m][n]) : "v"(g.A[c][m]), "v"(B[n]));
                else asm volatile("v_mfma_f32_16x16x32_f16 %0, %1, %2, %0" : "+a"(g.acc[m][n]) : "v"(g.A[c][m]), "v"(B[n]));
            } else {
                if constexpr (ZERO) asm volatile("v_mfma_f32_16x16x32_f16 %0, %1, %2, 0" : "=&v"(g.acc[m][n]) : "v"(g.A[c][m]), "v"(B[n]));
                else asm volatile("v_mfma_f32_16x16x32_f16 %0, %1, %2, %0" : "+v"(g.acc[m][n]) : "v"(g.A[c][m]), "v"(B[n]));
            }
            SCHED;
            if (k == 0) { B[n + 2] = n < 10 ? rdB6<J>(g, n + 2) : rdB6<NX>(g, n - 10); SCHED; }
            if (k == 4 && n < 8) { g.A[c ^ 1][n] = rdfrag(g.an, n * 1024); SCHED; }
            constexpr int PA[4] = APOS, PB[6] = BPOS;
            if (q == PB[0] - 1) { dma_to(g.ldsB + PV * STEPW); SCHED; }
            if (q == PB[0]) { dma6<0>(g, g.voffB, g.sB); SCHED; }
            if (q == PB[1]) { dma6<1>(g, g.voffB, g.sB); SCHED; }
            if (q == PB[2]) { dma6<2>(g, g.voffB, g.sB); SCHED; }
            if (q == PB[3]) { dma6<3>(g, g.voffB, g.sB); SCHED; }
            if (q == PB[4] - 1) { dma_to(g.ldsB + PV * STEPW + 4096); SCHED; }
            if (q == PB[4]) { dma6<0>(g, g.voffB2, g.sB); SCHED; }
            if (q == PB[5]) { dma6<1>(g, g.voffB2, g.sB); SCHED; }
            if (q == PA[0] - 1) { PIN_S(g.sa); dma_to(g.ldsA + g.sa); SCHED; }
            if (q == PA[0]) { dma6<0>(g, g.voffA, g.sA); SCHED; }
            if (q == PA[1]) { dma6<1>(g, g.voffA, g.sA); SCHED; }
            if (q == PA[2]) { dma6<2>(g, g.voffA, g.sA); SCHED; }
            if (q == PA[3]) { dma6<3>(g, g.voffA, g.sA); SCHED; }
            if (q == 70) { PIN_S(g.sB); g.sB += STEPW; PIN_S(g.sB); SCHED; }
            if (q == 73 && J == 0) { g.sB = it == 3 ? g.ringB : g.sB; PIN_S(g.sB); SCHED; }
            if (q == 75 && J == 0) { g.sB = it == 31 ? g.nextB : g.sB; PIN_S(g.sB); SCHED; }
            if (q == 2) { PIN_S(g.sA); g.sA += STEPX; PIN_S(g.sA); SCHED; }
            if (q == 10 && J == 0) { g.sA = it == 31 ? g.nextA : g.sA; PIN_S(g.sA); SCHED; }
            if (q == 85) { PIN_S(g.sa); g.sa = g.sa == 32768 ? 0 : g.sa + 16384; PIN_S(g.sa); SCHED; }
            if (q == 92) { g.an = g.ard + g.sa; asm volatile("" : "+v"(g.an)); SCHED; }
        }
    }
    g.B0 = B[12]; g.B1 = B[13];
}
DI void g6_init(G6& g, LAS unsigned char* lds, const char* data, unsigned bytes, int wid) {
    const int lane = lane_now(), wr = wid >> 1, wc = wid & 1;
    const unsigned lbase = (unsigned)(size_t)lds;
    g.rD = __builtin_amdgcn_make_buffer_rsrc((void*)data, 0, bytes, 0x00020000);
    g.ldsA = lbase + LDS_A + wid * 4096; g.ldsB = lbase + wid * 6144;
    g.voffA = wid * 4096 + lane * 16; g.voffB = wid * 6144 + lane * 16; g.voffB2 = g.voffB + 4096;
    g.ard = lbase + LDS_A + wr * 8192 + lane * 16; g.blo = lbase + wc * 12288 + lane * 16; g.bhi = g.blo + 2 * STEPW;
    asm volatile("" : "+v"(g.ard), "+v"(g.blo), "+v"(g.bhi), "+v"(g.voffA), "+v"(g.voffB), "+v"(g.voffB2));
}
DI void g6_fillA(G6& g, unsigned st, unsigned src) { dma_to(g.ldsA + st); asm volatile("s_nop 0"); dma6<0>(g, g.voffA, src); dma6<1>(g, g.voffA, src); dma6<2>(g, g.voffA, src); dma6<3>(g, g.voffA, src); }
DI void g6_prologue(G6& g, unsigned dA, unsigned dB) {
#pragma unroll
    for (int i = 0; i < 3; ++i) {
        g6_fillA(g, i * 16384, dA + i * STEPX);
        dma_to(g.ldsB + i * STEPW); asm volatile("s_nop 0"); dma6<0>(g, g.voffB, dB + i * STEPW); dma6<1>(g, g.voffB, dB + i * STEPW); dma6<2>(g, g.voffB, dB + i * STEPW); dma6<3>(g, g.voffB, dB + i * STEPW);
        dma_to(g.ldsB + i * STEPW + 4096); asm volatile("s_nop 0"); dma6<0>(g, g.voffB2, dB + i * STEPW); dma6<1>(g, g.voffB2, dB + i * STEPW);
    }
    WAITVM(0); __builtin_amdgcn_s_barrier(); SCHED;
#pragma unroll
    for (int i = 0; i < 8; ++i) g.A[0][i] = rdfrag(g.ard, i * 1024);
    g.B0 = rdB6<0>(g, 0); g.B1 = rdB6<0>(g, 1);
    __builtin_amdgcn_s_waitcnt(WCNT(63, 0)); __builtin_amdgcn_s_barrier(); SCHED;
    g6_fillA(g, 0, dA + 3 * STEPX);
    WAITVM(0);
    g.sA = dA + 3 * STEPX; g.sB = dB + 3 * STEPW; g.sa = 16384; g.an = g.ard + 16384;
}
DI void g6_epilogue(G6& g, __amdgpu_buffer_rsrc_t rO, unsigned so, unsigned rowb, float one) {
    const int lz = lane_now();
    const unsigned vo = 4 * (lz >> 4) * rowb + 16 * (lz & 15);
    asm volatile("s_nop 15");
#pragma unroll
    for (int m = 0; m < 8; ++m) {
#pragma unroll
        for (int r = 0; r < 4; ++r) {
#pragma unroll
            for (int q = 0; q < 3; ++q) {
                f32x4 y;
                if (q < 2) y = (f32x4){rd_acc(g.acc[m][4 * q][r]), rd_acc(g.acc[m][4 * q + 1][r]), rd_acc(g.acc[m][4 * q + 2][r]), rd_acc(g.acc[m][4 * q + 3][r])};
                else y = (f32x4){g.acc[m][8][r], g.acc[m][9][r], g.acc[m][10][r], g.acc[m][11][r]};
                const f32x4 v = silu4(y, one);
                asm volatile("buffer_store_dwordx4 %0, %1, %2, %3 offen offset:%4 nt\n\ts_nop 1" :: "v"(v), "v"(vo), "s"(rO), "s"(so + r * rowb), "i"(q * 256) : "memory");
            }
            SCHED;
        }
        so += 16 * rowb;
    }
}
DI int lead_ones(bool ok) { const unsigned long long m = ~__ballot(ok); return m ? __builtin_ctzll(m) : 64; }

struct Ready {
    char* ws; unsigned call;
    __amdgpu_buffer_rsrc_t rF; LAS unsigned char* zone;
    int pa, pb, n; bool pending; float one;
    DI void poll(int need) {
        const unsigned long long t0 = __builtin_amdgcn_s_memrealtime();
        const int lane = lane_now();
        for (;;) {
            const unsigned a = ld_sys(ws + OFF_F + pa * 256 + lane * 4), b = ld_sys(ws + OFF_F + pb * 256 + lane * 4);
            n = lead_ones(lane < 8 || (a >= call && b >= call));
            if (n >= need) break;
            if (give_up(ws, call, t0)) { n = 64; one = __builtin_nanf(""); break; }
            __builtin_amdgcn_s_sleep(16);
        }
    }
    DI void request() {
        const int lane = lane_now();
        __builtin_amdgcn_raw_ptr_buffer_load_lds(rF, (LAS void*)zone, 4, lane * 4, pa * 256, 0, AUX_SYS);
        __builtin_amdgcn_raw_ptr_buffer_load_lds(rF, (LAS void*)(zone + 256), 4, lane * 4, pb * 256, 0, AUX_SYS);
        pending = true;
    }
    DI int plan(int it) {
        if (n == 64) return 32;
        const int safe = (n - 4) >> 1;
        if (safe <= it + 1) { request(); return it + 1; }
        return safe - 1;
    }
    DI void check(int need) {
        if (pending) {
            const int lane = lane_now();
            const unsigned a = *(const volatile LAS unsigned*)(zone + lane * 4), b = *(const volatile LAS unsigned*)(zone + 256 + lane * 4);
            n = lead_ones(lane < 8 || (a >= call && b >= call)); pending = false;
        }
        if (n < need) poll(need);
        asm volatile("" ::: "memory");
    }
};

DI void gemm_wg(Params& p, LAS unsigned char* lds, int wid, int rg, int S) {
    const int wr = wid >> 1, wc = wid & 1, rank = p.rank;
    char* const ws = p.ws[rank];
    const unsigned call = convert_shards(p, lds, wid, S * 4 + rg);
    int T = S;
    STAMP(ws, 2);
    G6 g;
    g6_init(g, lds, ws + OFF_DATA, DATAB, wid);
    Ready R;
    R.ws = ws; R.call = call; R.rF = __builtin_amdgcn_make_buffer_rsrc((void*)(ws + OFF_F), 0, NPAN * 256, 0x00020000);
    R.zone = lds + LDS_ZONE + wid * 512; R.pending = false; R.one = 1.0f;
    if (!wait_words(ws, ws + OFF_AV + rank * 1024, NGEMM, 4, 4 * call + 3, call)) R.one = __builtin_nanf("");
    R.pa = 4 * rg + (T & 3); R.pb = 16 + colblock(T >> 2, rank); R.n = 8;
    STAMP(ws, 1);
    g6_prologue(g, rfl(own_off(R.pa, 0)), rfl(own_off(R.pb, 0)));
    for (int ti = 0;; ++ti) {
        const int pm = R.pa, cb = R.pb - 16, li = T >> 2;
        S += 2 * NG;
        const int job = S - PSLOT;
        if ((unsigned)job < (unsigned)CPR) S += 2 * NG;
        const bool has_next = S < NROUND * 2 * NG;
        if (has_next) T = S < PSLOT ? S : S - CPR;
        const int qa = 4 * rg + (T & 3), qb = 16 + colblock(T >> 2, rank);
        g.ringB = rfl(ring_off(li)); g.nextA = rfl(own_off(qa, 0)); g.nextB = rfl(own_off(qb, 0));
        R.check(4);
        int lim = R.plan(0);
        SCHED;
        step6<0, true, 60>(g, 0); step6<1, false, 60>(g, 0); step6<2, false, 16>(g, 0); step6<3, false, 16>(g, 0);
        for (int it = 1; it < 32;) {
            if (it >= lim) {
                R.check(it < 30 ? 2 * it + 4 : 64);
                lim = R.plan(it);
            }
            do {
                SCHED;
                step6<0, false, 16>(g, it); step6<1, false, 16>(g, it); step6<2, false, 16>(g, it); step6<3, false, 16>(g, it);
            } while (++it < lim);
        }
        STAMP(ws, 8 + 2 * ti);
        R.pa = qa; R.pb = qb;
        R.request();
        g6_epilogue(g, __builtin_amdgcn_make_buffer_rsrc((void*)p.out[rank], 0, OUTB, 0x00020000), rfl((pm * 256 + wr * 128) * ROWB + (col0(cb) + wc * 192) * 4), ROWB, R.one);
        WAITVM(60);
        if (wid == 0 && lane_now() == 0) __hip_atomic_fetch_add((gu32*)(ws + OFF_FIN + li * 4), 1u, __ATOMIC_RELAXED, __HIP_MEMORY_SCOPE_AGENT);
        if (li < CPR) {
            WAITVM(0);
            BARRIER;
            STAMP(ws, 4);
            if (wid == 0) {
                __builtin_amdgcn_fence(__ATOMIC_RELEASE, "agent");
                WAITVM(0);
                if (lane_now() == 0) st_sys(ws + OFF_AVC + (li * 16 + pm) * 4, call);
            }
        }
        if ((unsigned)job < (unsigned)CPR) { pack_tile(p, lds + LDS_ZONE + 4096 + wid * 1024, call, job * 16 + 4 * rg + wid); STAMP(ws, 5); }
        STAMP(ws, 9 + 2 * ti);
        if (!has_next) break;
    }
    __builtin_amdgcn_s_waitcnt(WCNT(0, 0));
}

__global__ __launch_bounds__(256, 1) void fused(Args) {
    extern __shared__ __attribute__((aligned(16))) unsigned char shm[];
    Params& p = *(Params*)__builtin_amdgcn_kernarg_segment_ptr();
    LAS unsigned char* lds = (LAS unsigned char*)shm;
    char* const ws = p.ws[p.rank];
    const int wid = rfl(threadIdx.x >> 6);
    STAMP(ws, 0);
    STAMPV(ws, 3, __builtin_amdgcn_s_getreg(20 | (0 << 6) | (3 << 11)));
    const int grp = blockIdx.x & 7, idx = blockIdx.x >> 3;
    if (idx < NG) gemm_wg(p, lds, wid, (grp & 1 ? 0 : 2) + (grp >> 2), (grp >> 1 & 1) * NG + idx);
    else {
        const int cw = (idx - NG) * 8 + grp;
        const unsigned call = draw_call(ws, wid);
        if (wid < 3 || cw >= NIDLE) fetch_wave(p, lds + wid * 32768, call, wid, cw * 4 + wid - (cw < NIDLE ? cw : NIDLE));
    }
}
}

extern "C" void kernel_launch(int rank, int nranks, ncclComm_t comm, hipStream_t stream,
                              void* const* d_in, const int* in_sizes, int n_in,
                              void* d_out, int out_size,
                              const int* in_dtypes, int out_dtype,
                              void* const* d_out_all,
                              void* const* d_ws_all, size_t ws_bytes) {
    if (nranks != NR || (unsigned)rank >= (unsigned)NR || n_in != 2 || ws_bytes < WS_NEED || in_sizes[0] != MM * KS || in_sizes[1] != KS * NN || out_size != MM * NN) {
        fprintf(stderr, "kernel_launch: unexpected problem shape\n");
        return;
    }
    static bool attr_set[NR];
    if (!attr_set[rank]) {
        HIP_CHECK(hipFuncSetAttribute((const void*)fused, hipFuncAttributeMaxDynamicSharedMemorySize, LDS_BYTES));
        attr_set[rank] = true;
    }
    Args p;
    p.x = (const float*)d_in[0]; p.w = (const float*)d_in[1];
    for (int i = 0; i < NR; ++i) { p.ws[i] = (char*)d_ws_all[i]; p.out[i] = (float*)d_out_all[i]; }
    p.rank = rank; p.pad = 0;
    hipLaunchKernelGGL(fused, dim3(256), dim3(256), LDS_BYTES, stream, p);
    HIP_CHECK(hipGetLastError());
}
```

```cpp
#include <hip/hip_runtime.h>
#include <rccl/rccl.h>
#include <cstdio>

#define HIP_CHECK(expr) do { hipError_t _e = (expr); if (_e != hipSuccess) { \
    fprintf(stderr, "HIP error %s at %s:%d\n", hipGetErrorString(_e), __FILE__, __LINE__); \
    return; } } while (0)

#define LAS __attribute__((address_space(3)))
#define GAS __attribute__((address_space(1)))
#define DI __device__ __forceinline__
typedef _Float16 f16x8 __attribute__((ext_vector_type(8)));
typedef float f32x4 __attribute__((ext_vector_type(4)));
typedef unsigned u32x4 __attribute__((ext_vector_type(4)));
typedef unsigned u32x2 __attribute__((ext_vector_type(2)));
typedef GAS unsigned gu32;

#define NG 29
#define DBG 0
namespace {
constexpr int MM = 4096, NN = 32768, KS = 512, NR = 8;
constexpr int NGEMM = 8 * NG, NCOMM = 256 - NGEMM, NCW = 4 * NCOMM;
constexpr int TW = 384;
constexpr int CPR = 2, NCB = NR * CPR;
constexpr int BCOLS = NN - NCB * TW, NBB = (BCOLS + TW - 1) / TW;
constexpr int NWP = NBB + NCB, NPAN = 16 + NWP;
constexpr int NL = CPR + NBB;
constexpr int NROUND = 5, PSLOT = 4 * NG - CPR;
static_assert(NROUND * 2 * NG == 4 * NL + CPR, "slots");
constexpr unsigned QUAD = 16384, XBLK = 2 * QUAD, WBLK = 3 * QUAD, XPANEL = 64 * XBLK;
static_assert(BCOLS - (NBB - 1) * TW == 128, "the narrow block has one quad per block: 64 columns for each wave column");
constexpr unsigned NE16 = 48, NE32 = 16, TS16 = 512 + 2 * NE16, TS32 = 512 + 2 * NE32, QB16 = 4 * (TS16 + 3072), QB32 = 4 * (TS32 + 3072);
constexpr unsigned OUTB = (unsigned)MM * NN * 4u, ROWB = NN * 4;
constexpr unsigned OFF_CALL = 0;
constexpr unsigned OFF_TMO = 32768;
constexpr unsigned OFF_FIN = 32768 + 1024;
constexpr unsigned OFF_TICK = 32768 + 4096;
constexpr unsigned OFF_F = 65536;
constexpr unsigned OFF_AV = 131072;
constexpr unsigned OFF_AVC = OFF_AV + 8192;
constexpr unsigned OFF_CNT = 196608;
constexpr unsigned OFF_EV = OFF_CNT + 4096;
constexpr unsigned OFF_DBG = 262144, OFF_DBG2 = 524288;
constexpr int RING = NL - 32;
constexpr unsigned OFF_DATA = 1u << 20;
constexpr unsigned REL_WL = 16 * XPANEL;
constexpr unsigned REL_WR = REL_WL + NWP * 8 * WBLK;
constexpr unsigned DATAB = REL_WR + RING * 56 * WBLK;
constexpr int NQ16 = 16 * 8 * 2 + (NWP - 1) * 8 * 3 + 8;
constexpr unsigned OFF_PK16 = OFF_DATA + DATAB;
constexpr int NCI = CPR * (MM / 32);
constexpr unsigned OFF_PK32 = OFF_PK16 + (NQ16 + 16) * QB16;
constexpr unsigned OFF_RAW32 = OFF_PK32 + NCI * 3 * QB32;
constexpr unsigned WS_NEED = OFF_RAW32 + NCI * WBLK;
static_assert(NPAN * 256 <= 65536, "flags");
constexpr int LDS_A = 6 * QUAD, LDS_ZONE = LDS_A + 3 * QUAD, LDS_BYTES = LDS_ZONE + 8192;
constexpr unsigned long long DEADLINE = 50000000ull;
constexpr int AUX_SYS = 17, AUX_NT = 2;

struct Args {
    const float* x; const float* w;
    char* ws[NR]; float* out[NR];
    int rank, pad;
};
typedef const __attribute__((address_space(4))) Args Params;

constexpr int WCNT(int vm, int lgkm) { return (vm & 15) | (7 << 4) | ((lgkm & 15) << 8) | ((vm >> 4) << 14); }
#define SCHED __builtin_amdgcn_sched_barrier(0)
#define WAITVM(n) asm volatile("s_waitcnt vmcnt(" #n ")" ::: "memory")
#define WAITLG0 asm volatile("s_waitcnt lgkmcnt(0)" ::: "memory")
#define BARRIER do { asm volatile("" ::: "memory"); __builtin_amdgcn_s_barrier(); asm volatile("" ::: "memory"); } while (0)

DI unsigned ld_sys(const char* p) { return __hip_atomic_load((gu32*)p, __ATOMIC_RELAXED, __HIP_MEMORY_SCOPE_SYSTEM); }
DI void st_sys(char* p, unsigned v) { __hip_atomic_store((gu32*)p, v, __ATOMIC_RELAXED, __HIP_MEMORY_SCOPE_SYSTEM); }
DI unsigned ld_ag(const char* p) { return __hip_atomic_load((gu32*)p, __ATOMIC_RELAXED, __HIP_MEMORY_SCOPE_AGENT); }
DI void st_ag(char* p, unsigned v) { __hip_atomic_store((gu32*)p, v, __ATOMIC_RELAXED, __HIP_MEMORY_SCOPE_AGENT); }
DI unsigned rfl(unsigned x) { return __builtin_amdgcn_readfirstlane(x); }
DI int lane_now() { int l; asm volatile("v_mbcnt_lo_u32_b32 %0, -1, 0\n\tv_mbcnt_hi_u32_b32 %0, -1, %0" : "=v"(l)); return l; }
DI bool give_up(char* ws, unsigned call, unsigned long long t0) {
    if (rfl(ld_ag(ws + OFF_TMO)) == call || __builtin_amdgcn_s_memrealtime() - t0 > DEADLINE) { st_ag(ws + OFF_TMO, call); return true; }
    return false;
}
DI bool arrive(char* ws, char* lane_ws, int rank, int kind, unsigned n, unsigned call) {
    const int lane = lane_now();
    unsigned t = 0;
    if (lane == 0) t = __hip_atomic_fetch_add((gu32*)(ws + OFF_CNT + kind * 128), 1u, __ATOMIC_RELAXED, __HIP_MEMORY_SCOPE_AGENT);
    const bool last = rfl(t) + 1 == n * call;
    if (last && lane < NR) st_sys(lane_ws + OFF_EV + (kind * NR + rank) * 4, call);
    return last;
}
DI bool wait_words(char* ws, const char* f, int n, int step, unsigned want, unsigned call) {
    const int lane = lane_now();
    const unsigned long long t0 = __builtin_amdgcn_s_memrealtime();
    for (;;) {
        bool ok = true;
#pragma unroll
        for (int i = 0; i < 4; ++i) { const int k = lane + 64 * i; if (64 * i < n) ok &= ld_sys(f + (k < n ? k : 0) * step) >= want; }
        if (__all(ok)) break;
        if (give_up(ws, call, t0)) return false;
        __builtin_amdgcn_s_sleep(4);
    }
    asm volatile("" ::: "memory");
    return true;
}
#define STAMP(ws, i) do {} while (0)
#define STAMPW(ws, i) do {} while (0)
#define STAMPV(ws, i, v) do {} while (0)

DI void store1(const char* base, unsigned v, const u32x4& f) {
    asm volatile("s_nop 4\n\tglobal_store_dwordx4 %0, %1, %2 sc0 sc1\n\ts_nop 1" :: "v"(v), "v"(f), "s"(base) : "memory");
}
DI void store3(const char* base, unsigned v, const u32x4& a, const u32x4& b, const u32x4& c) {
    asm volatile("s_nop 4\n\tglobal_store_dwordx4 %0, %1, %4 sc0 sc1\n\t"
                 "global_store_dwordx4 %0, %2, %4 offset:1024 sc0 sc1\n\t"
                 "global_store_dwordx4 %0, %3, %4 offset:2048 sc0 sc1\n\ts_nop 1"
                 :: "v"(v), "v"(a), "v"(b), "v"(c), "s"(base) : "memory");
}
DI void store4(const char* base, unsigned v, const u32x4 (&f)[4]) {
    asm volatile("s_nop 4\n\tglobal_store_dwordx4 %0, %1, %5 sc0 sc1\n\t"
                 "global_store_dwordx4 %0, %2, %5 offset:1024 sc0 sc1\n\t"
                 "global_store_dwordx4 %0, %3, %5 offset:2048 sc0 sc1\n\t"
                 "global_store_dwordx4 %0, %4, %5 offset:3072 sc0 sc1\n\ts_nop 1"
                 :: "v"(v), "v"(f[0]), "v"(f[1]), "v"(f[2]), "v"(f[3]), "s"(base) : "memory");
}
DI void store4o(const char* base, unsigned v, const u32x4 (&f)[4]) {
    asm volatile("s_nop 4\n\tglobal_store_dwordx4 %0, %4, %8 sc1 nt\n\t"
                 "global_store_dwordx4 %1, %5, %8 sc1 nt\n\t"
                 "global_store_dwordx4 %2, %6, %8 sc1 nt\n\t"
                 "global_store_dwordx4 %3, %7, %8 sc1 nt\n\ts_nop 1"
                 :: "v"(v), "v"(v + 2 * ROWB), "v"(v + 4 * ROWB), "v"(v + 6 * ROWB), "v"(f[0]), "v"(f[1]), "v"(f[2]), "v"(f[3]), "s"(base) : "memory");
}

DI unsigned perm(unsigned hi, unsigned lo, unsigned sel) { return __builtin_amdgcn_perm(hi, lo, sel); }
template <int CTRL, int ROWMASK> DI unsigned dpp0(unsigned x) { return __builtin_amdgcn_update_dpp(0u, x, CTRL, ROWMASK, 0xf, false); }
DI unsigned wave_sum(unsigned x) {
    x += dpp0<0x111, 0xf>(x); x += dpp0<0x112, 0xf>(x); x += dpp0<0x114, 0xf>(x); x += dpp0<0x118, 0xf>(x); x += dpp0<0x142, 0xa>(x); x += dpp0<0x143, 0xc>(x);
    return __builtin_amdgcn_readlane(x, 63);
}
DI unsigned umax(unsigned a, unsigned b) { return a > b ? a : b; }
DI unsigned umin(unsigned a, unsigned b) { return a < b ? a : b; }
DI unsigned wave_max(unsigned x) {
    x = umax(x, dpp0<0x111, 0xf>(x)); x = umax(x, dpp0<0x112, 0xf>(x)); x = umax(x, dpp0<0x114, 0xf>(x)); x = umax(x, dpp0<0x118, 0xf>(x));
    x = umax(x, dpp0<0x142, 0xa>(x)); x = umax(x, dpp0<0x143, 0xc>(x));
    return __builtin_amdgcn_readlane(x, 63);
}
template <unsigned NE> struct ExcList {
    volatile LAS unsigned short* e; unsigned cnt;
    DI void begin(LAS unsigned char* tail, int lane) { e = (volatile LAS unsigned short*)(tail + 512); e[lane] = 0; cnt = 1; }
    DI bool add(bool act, unsigned entry) {
        const unsigned long long bal = __ballot(act);
        const unsigned slot = cnt + __builtin_amdgcn_mbcnt_hi((unsigned)(bal >> 32), __builtin_amdgcn_mbcnt_lo((unsigned)bal, 0u));
        if (act && slot < NE) e[slot] = (unsigned short)entry;
        cnt += __builtin_popcountll(bal);
        return cnt <= NE;
    }
};
template <unsigned NE> DI bool pack_finish(const char* base, unsigned blk, int c, int lane, LAS unsigned char* tail, ExcList<NE>& x, unsigned ebase,
                    const u32x4& p0, const u32x4& p1, const u32x4& p2, unsigned t0, unsigned t1) {
    if (lane == 0) x.e[0] = (unsigned short)(ebase | (x.cnt > NE ? 0x8000u : 0u));
    *(volatile LAS u32x2*)(tail + lane * 8) = (u32x2){t0, t1};
    constexpr int TS = 512 + 2 * NE;
    store3(base, blk + 4 * TS + c * 3072 + lane * 16, p0, p1, p2);
    const u32x4 t = *(volatile LAS u32x4*)(tail + (lane < TS / 16 ? lane : 0) * 16);
    if (lane < TS / 16) store1(base, blk + c * TS + lane * 16, t);
    return x.cnt <= NE;
}
DI void pack16(const u32x4 (&f)[4], const char* base, unsigned blk, int c, int lane, LAS unsigned char* tail) {
    unsigned lo[8], h[8], g[8], sum = 0;
#pragma unroll
    for (int i = 0; i < 8; ++i) {
        const unsigned a = f[i >> 1][2 * (i & 1)], b = f[i >> 1][2 * (i & 1) + 1];
        lo[i] = perm(b, a, 0x06040200u); h[i] = perm(b, a, 0x07050301u);
        sum = __builtin_amdgcn_udot4(h[i] & 0x7c7c7c7cu, 0x01010101u, sum, false);
    }
    const int top = (int)((wave_sum(sum) + 15155u + 8191u) >> 13);
    const unsigned eb = top < 7 ? 0u : top > 31 ? 24u : (unsigned)(top - 7), b4 = eb * 0x04040404u;
    ExcList<NE16> x; x.begin(tail, lane);
    unsigned m1 = 0, m0 = 0;
#pragma unroll
    for (int i = 0; i < 8; ++i) {
        g[i] = (h[i] & 0x83838383u) | (((h[i] | 0x80808080u) - b4) & 0x1c1c1c1cu);
        const unsigned xr = (g[i] + b4) ^ h[i];
        m1 |= ((xr >> 6) & 0x01010101u) << i; m0 |= ((xr >> 5) & 0x01010101u) << i;
    }
    for (unsigned m = m1 | m0; __any(m != 0); m &= m - 1) {
        const int k = m ? __builtin_ctz(m) : 0, i = k & 7;
        if (!x.add(m != 0, ((i >> 2) * 1024 + lane * 16 + (i & 3) * 4 + (k >> 3)) | ((m0 >> k) & 1u) << 11 | ((m1 >> k) & 1u) << 12)) break;
    }
    unsigned d[6];
#pragma unroll
    for (int q = 0; q < 2; ++q) {
        const unsigned s = g[4 * q + 3];
        d[3 * q] = g[4 * q] | (s & 0x03030303u) << 5; d[3 * q + 1] = g[4 * q + 1] | (s & 0x0c0c0c0cu) << 3;
        d[3 * q + 2] = g[4 * q + 2] | (s & 0x10101010u) << 1 | (s & 0x80808080u) >> 1;
    }
    pack_finish(base, blk, c, lane, tail, x, eb, (u32x4){lo[0], lo[1], lo[2], lo[3]}, (u32x4){lo[4], lo[5], lo[6], lo[7]}, (u32x4){d[0], d[1], d[2], d[3]}, d[4], d[5]);
}
DI bool pack32(const u32x4 (&f)[4], const char* base, unsigned blk, int c, int lane, LAS unsigned char* tail) {
    unsigned t[12], h[4], mx = 0;
#pragma unroll
    for (int j = 0; j < 4; ++j) {
        t[3 * j] = perm(f[j][3], f[j][0], 0x04020100u); t[3 * j + 1] = perm(f[j][3], f[j][1], 0x05020100u); t[3 * j + 2] = perm(f[j][3], f[j][2], 0x06020100u);
        h[j] = perm(perm(f[j][3], f[j][2], 0x07030703u), perm(f[j][1], f[j][0], 0x07030703u), 0x05040100u);
#pragma unroll
        for (int d = 0; d < 4; ++d) mx = umax(mx, f[j][d] & 0x7fffffffu);
    }
    const unsigned top = wave_max(mx) >> 24, eb = top < 7 ? 0u : top - 7, b1 = eb * 0x01010101u;
    ExcList<NE32> x; x.begin(tail, lane);
    unsigned cd[4], xn[2] = {0, 0};
#pragma unroll
    for (int j = 0; j < 4; ++j) {
        cd[j] = ((h[j] | 0x80808080u) - b1) & 0x07070707u;
        const unsigned xr = ((cd[j] + b1) ^ h[j]) & 0x78787878u;
        xn[j >> 1] |= (j & 1) ? xr << 1 : xr >> 3;
    }
    for (unsigned long long m = (unsigned long long)xn[1] << 32 | xn[0]; __any(m != 0);) {
        const int k = m ? __builtin_ctzll(m) >> 2 : 0, j = 2 * (k >> 3) + (k & 1);
        const unsigned v = (unsigned)(m >> (4 * k)) & 15u;
        if (!x.add(m != 0, (lane * 16 + j * 4 + ((k >> 1) & 3)) | v << 11)) break;
        m &= ~(15ull << (4 * k));
    }
    const unsigned n0 = cd[0] | (h[0] & 0x80808080u) >> 4 | cd[1] << 4 | (h[1] & 0x80808080u), n1 = cd[2] | (h[2] & 0x80808080u) >> 4 | cd[3] << 4 | (h[3] & 0x80808080u);
    return pack_finish(base, blk, c, lane, tail, x, eb, (u32x4){t[0], t[1], t[2], t[3]}, (u32x4){t[4], t[5], t[6], t[7]}, (u32x4){t[8], t[9], t[10], t[11]}, n0, n1);
}
DI bool unpack(bool is32, const LAS unsigned char* main3, const LAS unsigned char* tail, LAS unsigned char* patch, int lane, u32x4 (&f)[4]) {
    unsigned e = *(const volatile LAS unsigned short*)(tail + 512 + lane * 2);
    const u32x2 tt = *(const volatile LAS u32x2*)(tail + lane * 8);
    const u32x4 p0 = *(const volatile LAS u32x4*)(main3 + lane * 16), p1 = *(const volatile LAS u32x4*)(main3 + 1024 + lane * 16), p2 = *(const volatile LAS u32x4*)(main3 + 2048 + lane * 16);
    const unsigned hdr = rfl(e);
    if (lane == 0 || lane >= (int)(is32 ? NE32 : NE16)) e = 0;
    LAS unsigned* const pw = (LAS unsigned*)(patch + (e & 0x7fcu));
    __hip_atomic_fetch_or(pw, (e >> 11) << ((is32 ? 3 : 5) + 8 * (e & 3u)), __ATOMIC_RELAXED, __HIP_MEMORY_SCOPE_WAVEFRONT);
    const u32x4 x0 = *(const volatile LAS u32x4*)(patch + lane * 16), x1 = *(const volatile LAS u32x4*)(patch + 1024 + lane * 16);
    *(volatile LAS unsigned*)pw = 0;
    if (is32) {
        const unsigned b1 = (hdr & 127u) * 0x01010101u, t[12] = {p0[0], p0[1], p0[2], p0[3], p1[0], p1[1], p1[2], p1[3], p2[0], p2[1], p2[2], p2[3]};
#pragma unroll
        for (int j = 0; j < 4; ++j) {
            const unsigned n = tt[j >> 1];
            const unsigned h = ((j & 1) ? ((((n >> 4) & 0x07070707u) + b1) | (n & 0x80808080u)) : (((n & 0x07070707u) + b1) | ((n << 4) & 0x80808080u))) ^ x0[j];
            f[j][0] = perm(h, t[3 * j], 0x04020100u); f[j][1] = perm(h, t[3 * j + 1], 0x05020100u); f[j][2] = perm(h, t[3 * j + 2], 0x06020100u);
            f[j][3] = perm(perm(h, t[3 * j + 2], 0x07030703u), perm(t[3 * j + 1], t[3 * j], 0x07030703u), 0x05040100u);
        }
    } else {
        const unsigned b4 = (hdr & 31u) * 0x04040404u, d[6] = {p2[0], p2[1], p2[2], p2[3], tt[0], tt[1]};
#pragma unroll
        for (int q = 0; q < 2; ++q) {
            const unsigned a = d[3 * q], b = d[3 * q + 1], c = d[3 * q + 2];
            const unsigned g[4] = {a & 0x9f9f9f9fu, b & 0x9f9f9f9fu, c & 0x9f9f9f9fu,
                                   ((a >> 5) & 0x03030303u) | ((b >> 3) & 0x0c0c0c0cu) | ((c >> 1) & 0x10101010u) | ((c << 1) & 0x80808080u)};
#pragma unroll
            for (int i = 0; i < 4; ++i) {
                const unsigned h = (g[i] + b4) ^ (q ? x1[i] : x0[i]), lo = q ? p1[i] : p0[i];
                f[2 * q + (i >> 1)][2 * (i & 1)] = perm(h, lo, 0x05010400u); f[2 * q + (i >> 1)][2 * (i & 1) + 1] = perm(h, lo, 0x07030602u);
            }
        }
    }
    return hdr < 0x8000u;
}

constexpr int PNARROW = 16 + NBB - 1;
DI int col0(int cb) { return cb >= NBB ? BCOLS + (cb - NBB) * TW : cb * TW; }
DI int colblock(int li, int rank) { return li < CPR ? NBB + CPR * rank + li : li - CPR; }
DI unsigned own_off(int panel, int kt) { return panel < 16 ? panel * XPANEL + kt * XBLK : REL_WL + ((panel - 16) * 8 + kt) * WBLK; }
DI unsigned ring_off(int li) { return REL_WR + (li >= RING ? li - RING : li) * (56 * WBLK); }
DI unsigned pk16_off(int panel, int kt) { return OFF_PK16 + (panel < 16 ? (panel * 8 + kt) * 2 : 256 + (panel - 16) * 24 + (panel == PNARROW ? kt : kt * 3)) * QB16; }
static_assert(2 * RING >= NL, "ring");
constexpr int round_end(int r) { return (2 * NG * (r + 1) - (r ? CPR : 0) + 3) / 4; }
#define XLAGS {0, 0, 3, 3}
#define BL 0
constexpr int XL[4] = XLAGS, NSTEP0 = 8 + (XL[3] > BL ? XL[3] : BL);
constexpr int x_lo(int s) { int g = 0; while (g < 4 && s - XL[g] >= 8) ++g; return 4 * g; }
constexpr int x_hi(int s) { int g = 0; while (g < 4 && s >= XL[g]) ++g; return 4 * g; }
constexpr int B0 = round_end(0) - CPR, W0Q = 3 * (B0 + NCB);
constexpr int cstep_start(int s) { int n = 0; for (int k = 0; k < s; ++k) n += 2 * (x_hi(k) - x_lo(k)) + (k < 8 ? W0Q : 0); return n; }
constexpr int steps_within(int quads) { int s = 0; while (s < NSTEP0 && cstep_start(s + 1) <= quads) ++s; return s; }
constexpr int ST1 = 3, STEPS1 = steps_within(NGEMM), STEPS2 = steps_within(ST1 * NGEMM);
static_assert(BL == 0 && STEPS1 >= 1 && NQ16 > ST1 * NGEMM + NGEMM, "stages");
DI void conv_quad(int b, int& panel, int& kt, int& qd) {
    if (b < cstep_start(NSTEP0)) {
        int s = 0, i = b, lo = x_lo(0), nx = 2 * (x_hi(0) - x_lo(0));
#pragma unroll
        for (int k = 1; k < NSTEP0; ++k) if (b >= cstep_start(k)) { s = k; i = b - cstep_start(k); lo = x_lo(k); nx = 2 * (x_hi(k) - x_lo(k)); }
        if (i < nx) { panel = lo + (i >> 1); qd = i & 1; kt = s - (panel < 4 ? XL[0] : panel < 8 ? XL[1] : panel < 12 ? XL[2] : XL[3]); }
        else { const int w = (i - nx) / 3; qd = i - nx - 3 * w; panel = 16 + (w < B0 ? w : w - B0 + NBB); kt = s; }
    } else if (b < NQ16 - 8) { const int r = b - cstep_start(NSTEP0), k = r / 3; qd = r - 3 * k; panel = 16 + B0 + (k >> 3); kt = k & 7; }
    else { panel = PNARROW; kt = b - (NQ16 - 8); qd = 0; }
}
DI unsigned draw_call(char* ws, int wid) {
    unsigned c;
    asm volatile("s_nop 4\n\tglobal_load_dword %0, %1, %2 sc1\n\ts_waitcnt vmcnt(0)" : "=&v"(c) : "v"(OFF_CALL + blockIdx.x * 128), "s"(ws) : "memory");
    c = rfl(c) + 1;
    BARRIER;
    if (wid == 0 && lane_now() == 0) st_ag(ws + OFF_CALL + blockIdx.x * 128, c);
    return c;
}
DI unsigned convert_shards(Params& p, LAS unsigned char* lds, int wid, int gi) {
    const int lane = lane_now(), rank = p.rank;
    char* const ws = p.ws[rank];
    const __amdgpu_buffer_rsrc_t rx = __builtin_amdgcn_make_buffer_rsrc((void*)p.x, 0, MM * KS * 4, 0x00020000);
    const __amdgpu_buffer_rsrc_t rw = __builtin_amdgcn_make_buffer_rsrc((void*)p.w, 0, KS * NN * 4, 0x00020000);
    const unsigned vx = ((lane & 15) * KS + 8 * (lane >> 4)) * 4, vw = (8 * (lane >> 4) * NN + 4 * (lane & 15)) * 4;
    LAS unsigned char* const ring = lds + wid * 32768;
    LAS unsigned char* const tail = lds + 131072 + wid * 1024;
    const int nu = (NQ16 - gi + NGEMM - 1) / NGEMM;
    auto load = [&](int u) {
        int pa, kt, qd; conv_quad(gi + (u < nu ? u : 0) * NGEMM, pa, kt, qd);
        LAS unsigned char* const slot = ring + (u & 3) * 8192;
        if (pa < 16) {
            const unsigned s0 = ((pa * 256 + 128 * (wid >> 1) + 64 * qd) * KS + 32 * (2 * kt + (wid & 1))) * 4;
#pragma unroll
            for (int e = 0; e < 8; ++e) __builtin_amdgcn_raw_ptr_buffer_load_lds(rx, (LAS void*)(slot + e * 1024), 16, vx, s0 + (e >> 1) * (16 * KS * 4) + (e & 1) * 16, 0, AUX_NT);
        } else {
            const unsigned s0 = (32 * (2 * kt + (wid & 1)) * NN + col0(pa - 16) + (pa == PNARROW ? 64 : 192) * (wid >> 1) + 64 * qd) * 4;
#pragma unroll
            for (int e = 0; e < 8; ++e) __builtin_amdgcn_raw_ptr_buffer_load_lds(rw, (LAS void*)(slot + e * 1024), 16, vw, s0 + e * ROWB, 0, AUX_NT);
        }
    };
    load(0);
    char* peer_ws = p.ws[lane & 7];
    const unsigned call = draw_call(ws, wid);
    asm volatile("" : "+v"(peer_ws));
    for (int u = 0; u < nu; ++u) {
        if (u > 0) { load(u + 3); WAITVM(24); }
        if (u == ST1 + 1) {
            BARRIER;
            if (wid == 0) arrive(ws, peer_ws, rank, 2, NGEMM, call);
        }
        int pa, kt, qd; conv_quad(gi + u * NGEMM, pa, kt, qd);
        f32x4 v[8]; u32x4 f[4];
#pragma unroll
        for (int e = 0; e < 8; ++e) v[e] = *(const LAS f32x4*)(ring + (u & 3) * 8192 + e * 1024 + lane * 16);
#pragma unroll
        for (int j = 0; j < 4; ++j) { f16x8 h;
#pragma unroll
            for (int e = 0; e < 8; ++e) h[e] = (_Float16)(pa < 16 ? v[2 * j + (e >> 2)][e & 3] : v[e][j]);
            f[j] = __builtin_bit_cast(u32x4, h); }
        pack16(f, ws, pk16_off(pa, kt) + qd * QB16, wid, lane, tail);
        store4(ws, OFF_DATA + own_off(pa, kt) + (4 * qd + wid) * 4096 + lane * 16, f);
        if (u == 0) {
            load(1); load(2); load(3);
            WAITVM(24);
            BARRIER;
            if (wid == 0 && lane < NR) st_sys(peer_ws + OFF_AV + (rank * 256 + gi) * 4, call);
            if (wid == 1) arrive(ws, peer_ws, rank, 1, NGEMM, call);
        }
    }
    WAITVM(0);
    BARRIER;
    if (wid == 0) arrive(ws, peer_ws, rank, 3, NGEMM, call);
    return call;
}

constexpr int G0 = 16 + round_end(0), NBI = (16 + NL - 1) * 8 + 4, NTW = 7 * NBI, NTC = 7 * NCI, NT = NTW + NTC;
struct Item { const char* src; unsigned size, pk, raw, dof, dstep, rstep; int flag; bool three; };
constexpr int w_lo(int s) { return s < 8 ? 16 : 16 + CPR; }
constexpr int w_hi(int s) { return s < BL || s >= 8 + BL ? 16 + CPR : G0; }
constexpr int step_start(int s) { int n = 0; for (int k = 0; k < s; ++k) n += x_hi(k) - x_lo(k) + w_hi(k) - w_lo(k); return n; }
static_assert(step_start(NSTEP0) == 8 * G0, "order");
DI void block_of(unsigned t, int& i, int& kt) {
    if (t < 8u * G0) {
        int s = 0, hi = x_hi(0), skip = w_lo(0) - x_hi(0); i = t;
#pragma unroll
        for (int k = 1; k < NSTEP0; ++k) if (t >= (unsigned)step_start(k)) { s = k; i = t - step_start(k) + x_lo(k); hi = x_hi(k); skip = w_lo(k) - x_hi(k); }
        if (i >= hi) i += skip;
        kt = s - (i < 4 ? XL[0] : i < 8 ? XL[1] : i < 12 ? XL[2] : i < 16 ? XL[3] : i < 16 + CPR ? 0 : BL);
    } else if (t < NBI - 4u) {
        unsigned u = 0, n = 1, b = 0;
#pragma unroll
        for (int r = 1; r < NROUND; ++r)
            if (t >= 8u * (16 + round_end(r - 1))) { b = 16 + round_end(r - 1); u = t - 8u * b; n = round_end(r) - round_end(r - 1) - (r == NROUND - 1); }
        kt = u / n; i = b + (u - kt * n);
    } else { i = 16 + NL - 1; kt = 2 * (t - (NBI - 4)); }
}
DI unsigned slab_out(unsigned s, int from) { return (s & 127u) * 32u * ROWB + (BCOLS + (CPR * from + (s >> 7)) * TW) * 4u; }
DI Item make_item(Params& p, unsigned t) {
    const int rank = p.rank;
    const unsigned tt = t < (unsigned)NTW ? t : t < (unsigned)NT ? t - NTW : 0, u = tt / 7, pj = tt - 7 * u;
    const int peer = (rank + 1 + pj) & 7;
    Item it;
    it.src = p.ws[peer]; it.size = t < (unsigned)NT ? WS_NEED : 0;
    it.dstep = it.rstep = QUAD;
    if (t < (unsigned)NTW) {
        int i, kt; block_of(u, i, kt);
        const int panel = i < 16 ? i : 16 + colblock(i - 16, rank);
        it.three = i >= 16 && panel != PNARROW;
        if (panel == PNARROW) { it.dstep = 7 * WBLK; it.rstep = WBLK; }
        it.pk = pk16_off(panel, kt); it.raw = OFF_DATA + own_off(panel, kt);
        it.dof = OFF_DATA + (i < 16 ? i * XPANEL + (8 + 7 * kt + pj) * XBLK : ring_off(i - 16) + (7 * kt + pj) * WBLK);
        it.flag = panel * 64 + 8 + 7 * kt + pj;
    } else {
        it.three = true;
        it.pk = OFF_PK32 + u * (3 * QB32); it.raw = OFF_RAW32 + u * WBLK; it.dof = slab_out(u, peer); it.flag = -1;
    }
    return it;
}
DI void load_main(const Item& it, unsigned off, LAS unsigned char* to, int lane) {
    const __amdgpu_buffer_rsrc_t r = __builtin_amdgcn_make_buffer_rsrc((void*)it.src, 0, it.size, 0x00020000);
    asm volatile("s_mov_b32 m0, %3\n\ts_nop 4\n\tbuffer_load_dwordx4 %0, %1, %2 offen sc0 sc1 lds\n\tbuffer_load_dwordx4 %0, %1, %2 offen offset:1024 sc0 sc1 lds\n\t"
                 "buffer_load_dwordx4 %0, %1, %2 offen offset:2048 sc0 sc1 lds" :: "v"(lane * 16), "s"(r), "s"(off), "s"((unsigned)(size_t)to) : "memory");
}
DI void load_tails(const Item& it, unsigned off, LAS unsigned char* to, int lane) {
    const __amdgpu_buffer_rsrc_t r = __builtin_amdgcn_make_buffer_rsrc((void*)it.src, 0, it.size, 0x00020000);
    const unsigned long long rest = it.flag < 0 ? (1ull << NE32 / 2) - 1 : (1ull << NE16 / 2) - 1;
    asm volatile("s_mov_b32 m0, %3\n\ts_nop 4\n\tbuffer_load_dwordx4 %0, %1, %2 offen sc0 sc1 lds\n\tbuffer_load_dwordx4 %0, %1, %2 offen offset:1024 sc0 sc1 lds\n\t"
                 "s_mov_b64 exec, %4\n\tbuffer_load_dwordx4 %0, %1, %2 offen offset:2048 sc0 sc1 lds\n\ts_mov_b64 exec, -1"
                 :: "v"(lane * 16), "s"(r), "s"(off), "s"((unsigned)(size_t)to), "s"(rest) : "memory");
}
constexpr unsigned NANS = 0x7fc07fc0u;
constexpr unsigned LQ = 14848, LMAIN = 2560;
struct Fetch {
    const char* ws; const char* out;
    LAS unsigned char* lds;
    unsigned slot; int lane; bool ok;
    template <int C0, bool WAITED> DI void quad(const Item& cur, unsigned qd, const Item& nx, unsigned nqd) {
        LAS unsigned char* const q = lds + slot; LAS unsigned char* const nq = lds + (LQ - slot);
        const unsigned npk = nx.pk + nqd * (nx.flag < 0 ? QB32 : QB16), nmain = npk + 4 * (nx.flag < 0 ? TS32 : TS16), ts = cur.flag < 0 ? TS32 : TS16;
#pragma unroll
        for (int c = 0; c < 4; ++c) {
            if (c > 0 || !WAITED) WAITVM(25);
            if (c == 0) load_tails(nx, npk, nq, lane);
            load_main(nx, nmain + c * 3072, nq + LMAIN + c * 3072, lane);
            u32x4 f[4];
            if (!unpack(cur.flag < 0, q + LMAIN + c * 3072, q + c * ts, lds + 2 * LQ, lane, f)) {
                const char* a = cur.src + (size_t)(cur.raw + qd * cur.rstep + c * 4096 + lane * 16);
                asm volatile("global_load_dwordx4 %0, %4, off sc0 sc1\n\tglobal_load_dwordx4 %1, %4, off offset:1024 sc0 sc1\n\t"
                             "global_load_dwordx4 %2, %4, off offset:2048 sc0 sc1\n\tglobal_load_dwordx4 %3, %4, off offset:3072 sc0 sc1\n\ts_waitcnt vmcnt(0)"
                             : "=&v"(f[0]), "=&v"(f[1]), "=&v"(f[2]), "=&v"(f[3]) : "v"(a) : "memory");
            }
            if (!ok) f[0] = f[1] = f[2] = f[3] = (u32x4){NANS, NANS, NANS, NANS};
            if (cur.flag >= 0) store4(ws, cur.dof + qd * cur.dstep + c * 4096 + lane * 16, f);
            else store4o(out, cur.dof + ((C0 + c) / 3 * 8 + (lane >> 5)) * ROWB + (C0 + c) % 3 * 512 + (lane & 31) * 16, f);
        }
        slot = LQ - slot;
    }
};
DI void fetch_wave(Params& p, LAS unsigned char* lds, unsigned call, int wid, int wv) {
    const int lane = lane_now(), rank = p.rank;
    char* const ws = p.ws[rank];
    Fetch F;
    F.ws = ws; F.out = (const char*)p.out[rank]; F.lds = lds; F.lane = lane;
    *(LAS u32x4*)(lds + 2 * LQ + lane * 16) = (u32x4){0, 0, 0, 0}; *(LAS u32x4*)(lds + 2 * LQ + 1024 + lane * 16) = (u32x4){0, 0, 0, 0};
    const unsigned wbase = (call - 1) * NTW - NCW, cbase = (call - 1) * NTC - NCW;
    static_assert(NL - RING == 32, "probe lanes");
    const unsigned vprobe = lane < 32 ? OFF_EV + lane * 4 : OFF_FIN + (lane - 32) * 4;
    static_assert(NCW <= 7 * step_start(1), "first items");
    {
        int i0, kt0; block_of(wv / 7, i0, kt0);
        const int q0 = i0 < 16 ? 2 * i0 : 2 * x_hi(0) + 3 * (i0 < 16 + CPR ? B0 + CPR * rank + i0 - 16 : i0 - 16 - CPR), peer = (rank + 1 + wv % 7) & 7;
        F.ok = wait_words(ws, ws + OFF_AV + q0 * 4, NR, 1024, call, call) && wait_words(ws, ws + OFF_AV + peer * 1024 + q0 * 4, 3, 4, call, call);
    }
    STAMP(ws, 1);
    unsigned tw = wv, tc = wv;
    auto choose = [&](unsigned probe) -> unsigned {
        const unsigned ev = (unsigned)__ballot(probe >= call);
        if (tw < (unsigned)NTW) {
            const unsigned u = tw / 7;
            int i, kt; block_of(u, i, kt);
            const int stage = u < (unsigned)step_start(STEPS1) ? 1 : u < (unsigned)step_start(STEPS2) ? 2 : 3;
            if ((tw < (unsigned)NCW || (ev >> (8 * stage + ((rank + 1 + tw - 7 * u) & 7)) & 1)) &&
                (i < 16 + RING || __builtin_amdgcn_readlane(probe, 32 + i - 16 - RING) >= 16 * call)) return tw;
        }
        return tc < (unsigned)NTC && (ev >> ((rank + 1 + tc % 7) & 7) & 1) ? NTW + tc : NT;
    };
    auto publish = [&](int flag) {
        if (flag >= 0 && lane < (flag >> 6 == PNARROW ? 2 : 1)) st_sys(ws + OFF_F + (flag + 7 * lane) * 4, call);
    };
    while (tw < (unsigned)NTW || tc < (unsigned)NTC) {
        unsigned t;
        {
            const unsigned long long t0 = __builtin_amdgcn_s_memrealtime();
            while ((t = choose(ld_sys(ws + vprobe))) == (unsigned)NT) {
                if (give_up(ws, call, t0)) { F.ok = false; t = tw < (unsigned)NTW ? tw : NTW + tc; break; }
                __builtin_amdgcn_s_sleep(16);
            }
        }
        Item cur = make_item(p, t);
        F.slot = 0;
        load_tails(cur, cur.pk, lds, lane);
#pragma unroll
        for (int c = 0; c < 4; ++c) load_main(cur, cur.pk + 4 * (cur.flag < 0 ? TS32 : TS16) + c * 3072, lds + LMAIN + c * 3072, lane);
        WAITVM(0);
        int pflag = -1;
        for (;;) {
            unsigned tk, probe; unsigned long long sv;
            asm volatile("s_nop 3\n\ts_waitcnt vmcnt(25)\n\tglobal_load_dword %1, %6, %5 sc0 sc1\n\t"
                         "s_mov_b64 %2, exec\n\ts_mov_b64 exec, 1\n\tglobal_atomic_add %0, %3, %4, %5 sc0\n\ts_mov_b64 exec, %2"
                         : "=&v"(tk), "=&v"(probe), "=&s"(sv) : "v"(OFF_TICK + (cur.flag < 0 ? 128u : 0u)), "v"(1u), "s"(ws), "v"(vprobe) : "memory");
            F.quad<0, true>(cur, 0, cur, 1);
            asm volatile("s_waitcnt vmcnt(25)" : "+v"(tk), "+v"(probe) :: "memory");
            publish(pflag);
            if (cur.flag >= 0) tw = rfl(tk) - wbase; else tc = rfl(tk) - cbase;
            if (cur.three) F.quad<4, true>(cur, 1, cur, 2);
            t = choose(probe);
            const Item nxt = make_item(p, t);
            F.quad<8, false>(cur, cur.three ? 2 : 1, nxt, 0);
            pflag = cur.flag;
            if (t >= (unsigned)NT) break;
            cur = nxt;
        }
        WAITVM(0);
        publish(pflag);
    }
    STAMPW(ws, 44 + wid);
    if (arrive(ws, p.ws[lane & 7], rank, 4, NCW, call) && !wait_words(ws, ws + OFF_EV + 4 * NR * 4, NR, 4, call, call) && lane == 0)
        p.out[rank][BCOLS + CPR * ((rank + 1) & 7) * TW] = __builtin_nanf("");
    STAMPW(ws, 48 + wid);
}
#define PT_SET(A0, A1, A2, A3, R) \
    asm volatile("s_waitcnt vmcnt(60)\n\t" R \
                 "global_load_dwordx4 a[" A0 "], %16, %20 sc1 nt\n\tglobal_load_dwordx4 a[" A1 "], %17, %20 sc1 nt\n\t" \
                 "global_load_dwordx4 a[" A2 "], %18, %20 sc1 nt\n\tglobal_load_dwordx4 a[" A3 "], %19, %20 sc1 nt" \
                 : "=&v"(r[0]), "=&v"(r[1]), "=&v"(r[2]), "=&v"(r[3]), "=&v"(r[4]), "=&v"(r[5]), "=&v"(r[6]), "=&v"(r[7]), \
                   "=&v"(r[8]), "=&v"(r[9]), "=&v"(r[10]), "=&v"(r[11]), "=&v"(r[12]), "=&v"(r[13]), "=&v"(r[14]), "=&v"(r[15]) \
                 : "v"(v), "v"(v + 2 * ROWB), "v"(v + 4 * ROWB), "v"(v + 6 * ROWB), "s"(out) : "memory", PT_CLOB)
#define PT_CLOB "a0", "a1", "a2", "a3", "a4", "a5", "a6", "a7", "a8", "a9", "a10", "a11", "a12", "a13", "a14", "a15", "a16", "a17", "a18", "a19", "a20", "a21", "a22", "a23", "a24", "a25", "a26", "a27", "a28", "a29", "a30", "a31", "a32", "a33", "a34", "a35", "a36", "a37", "a38", "a39", "a40", "a41", "a42", "a43", "a44", "a45", "a46", "a47", "a48", "a49", "a50", "a51", "a52", "a53", "a54", "a55", "a56", "a57", "a58", "a59", "a60", "a61", "a62", "a63", "a64", "a65", "a66", "a67", "a68", "a69", "a70", "a71", "a72", "a73", "a74", "a75", "a76", "a77", "a78", "a79", "a80", "a81", "a82", "a83", "a84", "a85", "a86", "a87", "a88", "a89", "a90", "a91", "a92", "a93", "a94", "a95", "a96", "a97", "a98", "a99", "a100", "a101", "a102", "a103", "a104", "a105", "a106", "a107", "a108", "a109", "a110", "a111", "a112", "a113", "a114", "a115", "a116", "a117", "a118", "a119", "a120", "a121", "a122", "a123", "a124", "a125", "a126", "a127"
#define PT_R0 "v_accvgpr_read_b32 %0, a0\n\t" "v_accvgpr_read_b32 %1, a1\n\t" "v_accvgpr_read_b32 %2, a2\n\t" "v_accvgpr_read_b32 %3, a3\n\t" "v_accvgpr_read_b32 %4, a4\n\t" "v_accvgpr_read_b32 %5, a5\n\t" "v_accvgpr_read_b32 %6, a6\n\t" "v_accvgpr_read_b32 %7, a7\n\t" "v_accvgpr_read_b32 %8, a8\n\t" "v_accvgpr_read_b32 %9, a9\n\t" "v_accvgpr_read_b32 %10, a10\n\t" "v_accvgpr_read_b32 %11, a11\n\t" "v_accvgpr_read_b32 %12, a12\n\t" "v_accvgpr_read_b32 %13, a13\n\t" "v_accvgpr_read_b32 %14, a14\n\t" "v_accvgpr_read_b32 %15, a15\n\t"
#define PT_R1 "v_accvgpr_read_b32 %0, a16\n\t" "v_accvgpr_read_b32 %1, a17\n\t" "v_accvgpr_read_b32 %2, a18\n\t" "v_accvgpr_read_b32 %3, a19\n\t" "v_accvgpr_read_b32 %4, a20\n\t" "v_accvgpr_read_b32 %5, a21\n\t" "v_accvgpr_read_b32 %6, a22\n\t" "v_accvgpr_read_b32 %7, a23\n\t" "v_accvgpr_read_b32 %8, a24\n\t" "v_accvgpr_read_b32 %9, a25\n\t" "v_accvgpr_read_b32 %10, a26\n\t" "v_accvgpr_read_b32 %11, a27\n\t" "v_accvgpr_read_b32 %12, a28\n\t" "v_accvgpr_read_b32 %13, a29\n\t" "v_accvgpr_read_b32 %14, a30\n\t" "v_accvgpr_read_b32 %15, a31\n\t"
#define PT_R2 "v_accvgpr_read_b32 %0, a32\n\t" "v_accvgpr_read_b32 %1, a33\n\t" "v_accvgpr_read_b32 %2, a34\n\t" "v_accvgpr_read_b32 %3, a35\n\t" "v_accvgpr_read_b32 %4, a36\n\t" "v_accvgpr_read_b32 %5, a37\n\t" "v_accvgpr_read_b32 %6, a38\n\t" "v_accvgpr_read_b32 %7, a39\n\t" "v_accvgpr_read_b32 %8, a40\n\t" "v_accvgpr_read_b32 %9, a41\n\t" "v_accvgpr_read_b32 %10, a42\n\t" "v_accvgpr_read_b32 %11, a43\n\t" "v_accvgpr_read_b32 %12, a44\n\t" "v_accvgpr_read_b32 %13, a45\n\t" "v_accvgpr_read_b32 %14, a46\n\t" "v_accvgpr_read_b32 %15, a47\n\t"
#define PT_R3 "v_accvgpr_read_b32 %0, a48\n\t" "v_accvgpr_read_b32 %1, a49\n\t" "v_accvgpr_read_b32 %2, a50\n\t" "v_accvgpr_read_b32 %3, a51\n\t" "v_accvgpr_read_b32 %4, a52\n\t" "v_accvgpr_read_b32 %5, a53\n\t" "v_accvgpr_read_b32 %6, a54\n\t" "v_accvgpr_read_b32 %7, a55\n\t" "v_accvgpr_read_b32 %8, a56\n\t" "v_accvgpr_read_b32 %9, a57\n\t" "v_accvgpr_read_b32 %10, a58\n\t" "v_accvgpr_read_b32 %11, a59\n\t" "v_accvgpr_read_b32 %12, a60\n\t" "v_accvgpr_read_b32 %13, a61\n\t" "v_accvgpr_read_b32 %14, a62\n\t" "v_accvgpr_read_b32 %15, a63\n\t"
#define PT_R4 "v_accvgpr_read_b32 %0, a64\n\t" "v_accvgpr_read_b32 %1, a65\n\t" "v_accvgpr_read_b32 %2, a66\n\t" "v_accvgpr_read_b32 %3, a67\n\t" "v_accvgpr_read_b32 %4, a68\n\t" "v_accvgpr_read_b32 %5, a69\n\t" "v_accvgpr_read_b32 %6, a70\n\t" "v_accvgpr_read_b32 %7, a71\n\t" "v_accvgpr_read_b32 %8, a72\n\t" "v_accvgpr_read_b32 %9, a73\n\t" "v_accvgpr_read_b32 %10, a74\n\t" "v_accvgpr_read_b32 %11, a75\n\t" "v_accvgpr_read_b32 %12, a76\n\t" "v_accvgpr_read_b32 %13, a77\n\t" "v_accvgpr_read_b32 %14, a78\n\t" "v_accvgpr_read_b32 %15, a79\n\t"
#define PT_R5 "v_accvgpr_read_b32 %0, a80\n\t" "v_accvgpr_read_b32 %1, a81\n\t" "v_accvgpr_read_b32 %2, a82\n\t" "v_accvgpr_read_b32 %3, a83\n\t" "v_accvgpr_read_b32 %4, a84\n\t" "v_accvgpr_read_b32 %5, a85\n\t" "v_accvgpr_read_b32 %6, a86\n\t" "v_accvgpr_read_b32 %7, a87\n\t" "v_accvgpr_read_b32 %8, a88\n\t" "v_accvgpr_read_b32 %9, a89\n\t" "v_accvgpr_read_b32 %10, a90\n\t" "v_accvgpr_read_b32 %11, a91\n\t" "v_accvgpr_read_b32 %12, a92\n\t" "v_accvgpr_read_b32 %13, a93\n\t" "v_accvgpr_read_b32 %14, a94\n\t" "v_accvgpr_read_b32 %15, a95\n\t"
#define PT_R6 "v_accvgpr_read_b32 %0, a96\n\t" "v_accvgpr_read_b32 %1, a97\n\t" "v_accvgpr_read_b32 %2, a98\n\t" "v_accvgpr_read_b32 %3, a99\n\t" "v_accvgpr_read_b32 %4, a100\n\t" "v_accvgpr_read_b32 %5, a101\n\t" "v_accvgpr_read_b32 %6, a102\n\t" "v_accvgpr_read_b32 %7, a103\n\t" "v_accvgpr_read_b32 %8, a104\n\t" "v_accvgpr_read_b32 %9, a105\n\t" "v_accvgpr_read_b32 %10, a106\n\t" "v_accvgpr_read_b32 %11, a107\n\t" "v_accvgpr_read_b32 %12, a108\n\t" "v_accvgpr_read_b32 %13, a109\n\t" "v_accvgpr_read_b32 %14, a110\n\t" "v_accvgpr_read_b32 %15, a111\n\t"
#define PT_R7 "v_accvgpr_read_b32 %0, a112\n\t" "v_accvgpr_read_b32 %1, a113\n\t" "v_accvgpr_read_b32 %2, a114\n\t" "v_accvgpr_read_b32 %3, a115\n\t" "v_accvgpr_read_b32 %4, a116\n\t" "v_accvgpr_read_b32 %5, a117\n\t" "v_accvgpr_read_b32 %6, a118\n\t" "v_accvgpr_read_b32 %7, a119\n\t" "v_accvgpr_read_b32 %8, a120\n\t" "v_accvgpr_read_b32 %9, a121\n\t" "v_accvgpr_read_b32 %10, a122\n\t" "v_accvgpr_read_b32 %11, a123\n\t" "v_accvgpr_read_b32 %12, a124\n\t" "v_accvgpr_read_b32 %13, a125\n\t" "v_accvgpr_read_b32 %14, a126\n\t" "v_accvgpr_read_b32 %15, a127\n\t"
DI void pack_tile(Params& p, LAS unsigned char* tail, unsigned call, int ct) {
    const int lane = lane_now(), rank = p.rank;
    char* const ws = p.ws[rank];
    const char* const out = (const char*)p.out[rank];
    const bool ok = wait_words(ws, ws + OFF_AVC + ct * 4, 1, 4, call, call);
    const unsigned s0 = 8 * ct, vb = slab_out(s0, rank) + (lane >> 5) * ROWB + (lane & 31) * 16;
    for (int i = -8; i < 96; ++i) {
        const unsigned n = i + 8 < 96 ? i + 8 : 95, v = vb + n / 3 * (8 * ROWB) + n % 3 * 512;
        unsigned r[16];
        if (i >= 0 && i < 8) WAITVM(28);
        switch (i & 7) {
            case 0: PT_SET("0:3", "4:7", "8:11", "12:15", PT_R0); break;
            case 1: PT_SET("16:19", "20:23", "24:27", "28:31", PT_R1); break;
            case 2: PT_SET("32:35", "36:39", "40:43", "44:47", PT_R2); break;
            case 3: PT_SET("48:51", "52:55", "56:59", "60:63", PT_R3); break;
            case 4: PT_SET("64:67", "68:71", "72:75", "76:79", PT_R4); break;
            case 5: PT_SET("80:83", "84:87", "88:91", "92:95", PT_R5); break;
            case 6: PT_SET("96:99", "100:103", "104:107", "108:111", PT_R6); break;
            default: PT_SET("112:115", "116:119", "120:123", "124:127", PT_R7); break;
        }
        if (i < 0) continue;
        if (!ok) r[0] = NANS;
        const u32x4 f[4] = {{r[0], r[1], r[2], r[3]}, {r[4], r[5], r[6], r[7]}, {r[8], r[9], r[10], r[11]}, {r[12], r[13], r[14], r[15]}};
        if (!pack32(f, ws, OFF_PK32 + (s0 * 3 + (i >> 2)) * QB32, i & 3, lane, tail)) store4(ws, OFF_RAW32 + s0 * WBLK + i * 4096 + lane * 16, f);
    }
    WAITVM(0);
    arrive(ws, p.ws[lane & 7], rank, 0, 16 * CPR, call);
}

DI f16x8 rdfrag(unsigned base, int off) { return *(const LAS f16x8*)(size_t)(base + off); }
DI void dma_to(unsigned ldsaddr) { asm volatile("s_mov_b32 m0, %0" :: "s"(ldsaddr) : "memory"); }
#define PIN_S(x) asm volatile("" : "+s"(x))
DI float rd_acc(float a) { float v; asm volatile("v_accvgpr_read_b32 %0, %1" : "=v"(v) : "a"(a)); return v; }
DI f32x4 silu4(f32x4 y, float one) {
    f32x4 t = y * -1.44269504088896f;
#pragma unroll
    for (int i = 0; i < 4; ++i) t[i] = __builtin_amdgcn_exp2f(__builtin_amdgcn_fmed3f(t[i], -86.5617024533378f, 86.5617024533378f));
    t = t + one;
#pragma unroll
    for (int i = 0; i < 4; ++i) t[i] = __builtin_amdgcn_rcpf(t[i]);
    return y * t;
}
struct G7 {
    unsigned lds;
    __amdgpu_buffer_rsrc_t rD;
    unsigned voff, voffn;
    unsigned ard, an;
    unsigned blo, bhi;
    unsigned sA, sB, sr, sw;
    unsigned ringB, nextA, nextB;
    f32x4 acc[8][12];
    f16x8 L[2][4][2], H[4][2], B0, B1;
};
template <int I, bool N = false> DI void dma7(const G7& g, unsigned src) {
    asm volatile("buffer_load_dwordx4 %0, %1, %2 offen offset:%3 sc0 sc1 lds" :: "v"(N ? g.voffn : g.voff), "s"(g.rD), "s"(src), "i"(I * 1024) : "memory");
}
template <int SL> DI f16x8 rdB7(const G7& g, int j, int k) { return rdfrag(SL < 3 ? g.blo : g.bhi, (SL % 3) * QUAD + k * 4096 + j * 1024); }
DI unsigned next3(unsigned s) { return s == 2 * QUAD ? 0 : s + QUAD; }
template <int J, bool ZERO, int VM> DI void phase(G7& g, int it) {
    constexpr int t = J % 3, c = J / 3, NX = (J + 1) % 6, PV = (J + 5) % 6;
    f16x8 B[5][2];
    B[0][0] = g.B0; B[0][1] = g.B1;
#pragma unroll
    for (int j = 0; j < 4; ++j) {
        const int n = 4 * t + j;
        if (j == 3) { __builtin_amdgcn_s_waitcnt(WCNT(VM, 0)); __builtin_amdgcn_s_barrier(); SCHED; }
#pragma unroll
        for (int u = 0; u < 16; ++u) {
            const int q = j * 16 + u, r = u >> 1, m = (n & 1) ? 7 - r : r, k = (u & 1) ^ (r & 1);
            const f16x8 a = m < 4 ? g.L[c][m][k] : g.H[m - 4][k];
            if (n < 8) {
                if (ZERO && !(u & 1)) asm volatile("v_mfma_f32_16x16x32_f16 %0, %1, %2, 0" : "=a"(g.acc[m][n]) : "v"(a), "v"(B[j][k]));
                else asm volatile("v_mfma_f32_16x16x32_f16 %0, %1, %2, %0" : "+a"(g.acc[m][n]) : "v"(a), "v"(B[j][k]));
            } else {
                if (ZERO && !(u & 1)) asm volatile("v_mfma_f32_16x16x32_f16 %0, %1, %2, 0" : "=&v"(g.acc[m][n]) : "v"(a), "v"(B[j][k]));
                else asm volatile("v_mfma_f32_16x16x32_f16 %0, %1, %2, %0" : "+v"(g.acc[m][n]) : "v"(a), "v"(B[j][k]));
            }
            SCHED;
            const bool lastcol = t == 2 && j == 3;
            if (u == (lastcol ? 9 : 1)) { B[j + 1][0] = j < 3 ? rdB7<J>(g, j + 1, 0) : rdB7<NX>(g, 0, 0); SCHED; }
            if (u == (lastcol ? 10 : 3)) { B[j + 1][1] = j < 3 ? rdB7<J>(g, j + 1, 1) : rdB7<NX>(g, 0, 1); SCHED; }
            if (t == 2 && j < 3 && (u == 5 || u == 9 || u == 13) && j * 3 + (u - 5) / 4 < 8) { const int i = j * 3 + (u - 5) / 4; g.L[c ^ 1][i >> 1][i & 1] = rdfrag(g.an, (i & 1) * 4096 + (i >> 1) * 1024); SCHED; }
            if (lastcol && u >= 1 && u <= 8) { const int row = 3 - ((u - 1) >> 1), kk = (u - 1) & 1; g.H[row][kk] = rdfrag(g.an, kk * 4096 + row * 1024); SCHED; }
            if ((t == 2 && q == 44) || (lastcol && u == 12)) { PIN_S(g.sr); g.sr = next3(g.sr); PIN_S(g.sr); SCHED; }
            if ((t == 2 && q == 46) || (lastcol && u == 14)) { g.an = g.ard + g.sr; asm volatile("" : "+v"(g.an)); SCHED; }
            if (t < 2) {
                if (q == 4) { PIN_S(g.sw); dma_to(g.lds + LDS_A + g.sw); SCHED; }
                if (q == 5) { dma7<0>(g, g.sA); SCHED; }
                if (q == 10) { dma7<1>(g, g.sA); SCHED; }
                if (q == 15) { dma7<2>(g, g.sA); SCHED; }
                if (q == 21) { dma7<3>(g, g.sA); SCHED; }
                if (q == 52) { PIN_S(g.sA); g.sA += QUAD; PIN_S(g.sA); SCHED; }
                if (q == 54 && J == 0) { g.sA = it == 31 ? g.nextA : g.sA; PIN_S(g.sA); SCHED; }
                if (q == 56) { PIN_S(g.sw); g.sw = next3(g.sw); PIN_S(g.sw); SCHED; }
            }
            constexpr int PB0[4] = {27, 32, 38, 43}, PB2[4] = {7, 16, 27, 39};
            if (q == (t < 2 ? PB0[0] : PB2[0]) - 1) { dma_to(g.lds + PV * QUAD); SCHED; }
            if (q == (t < 2 ? PB0[0] : PB2[0])) { dma7<0, t != 1>(g, g.sB); SCHED; }
            if (q == (t < 2 ? PB0[1] : PB2[1])) { dma7<1, t != 1>(g, g.sB); SCHED; }
            if (q == (t < 2 ? PB0[2] : PB2[2])) { dma7<2, t != 1>(g, g.sB); SCHED; }
            if (q == (t < 2 ? PB0[3] : PB2[3])) { dma7<3, t != 1>(g, g.sB); SCHED; }
            if (q == (t < 2 ? 58 : 42)) { PIN_S(g.sB); g.sB += QUAD; PIN_S(g.sB); SCHED; }
            if (q == 60 && J == 0) { g.sB = it == 3 ? g.ringB : g.sB; PIN_S(g.sB); SCHED; }
            if (q == 62 && J == 0) { g.sB = it == 31 ? g.nextB : g.sB; PIN_S(g.sB); SCHED; }
        }
    }
    g.B0 = B[4][0]; g.B1 = B[4][1];
}
template <bool FIRST> DI void trip(G7& g, int it) {
    SCHED;
    phase<0, FIRST, FIRST ? 60 : 28>(g, it); phase<1, FIRST, FIRST ? 60 : 24>(g, it); phase<2, FIRST, 16>(g, it);
    phase<3, false, 28>(g, it); phase<4, false, 24>(g, it); phase<5, false, 16>(g, it);
}
DI void g7_init(G7& g, LAS unsigned char* lds, const char* data, unsigned bytes, int wid) {
    const int lane = lane_now(), wr = wid >> 1, wc = wid & 1;
    const unsigned lbase = (unsigned)(size_t)lds;
    g.rD = __builtin_amdgcn_make_buffer_rsrc((void*)data, 0, bytes, 0x00020000);
    g.lds = lbase + wid * 4096; g.voff = wid * 4096 + lane * 16;
    g.ard = lbase + LDS_A + wr * 8192 + lane * 16; g.blo = lbase + wc * 8192 + lane * 16; g.bhi = g.blo + 3 * QUAD;
    asm volatile("" : "+v"(g.ard), "+v"(g.blo), "+v"(g.bhi), "+v"(g.voff));
}
DI void g7_fill(G7& g, unsigned to, unsigned src) { dma_to(g.lds + to); asm volatile("s_nop 4"); dma7<0>(g, src); dma7<1>(g, src); dma7<2>(g, src); dma7<3>(g, src); }
DI void g7_prologue(G7& g, unsigned dA, unsigned dB) {
#pragma unroll
    for (int i = 0; i < 5; ++i) { if (i < 3) g7_fill(g, LDS_A + i * QUAD, dA + i * QUAD); g7_fill(g, i * QUAD, dB + i * QUAD); }
    WAITVM(0); __builtin_amdgcn_s_barrier(); SCHED;
#pragma unroll
    for (int i = 0; i < 8; ++i) { g.L[0][i >> 1][i & 1] = rdfrag(g.ard, (i & 1) * 4096 + (i >> 1) * 1024); g.H[i >> 1][i & 1] = rdfrag(g.ard, QUAD + (i & 1) * 4096 + (i >> 1) * 1024); }
    g.B0 = rdB7<0>(g, 0, 0); g.B1 = rdB7<0>(g, 0, 1);
    __builtin_amdgcn_s_waitcnt(WCNT(63, 0)); __builtin_amdgcn_s_barrier(); SCHED;
    g.sA = dA + 3 * QUAD; g.sB = dB + 5 * QUAD; g.sr = 2 * QUAD; g.sw = 0; g.an = g.ard + 2 * QUAD;
}
constexpr unsigned FAR = 0x80000000u;
DI void g7_epilogue(G7& g, __amdgpu_buffer_rsrc_t rO, unsigned so, unsigned rowb, float one, bool narrow) {
    const int lz = lane_now();
    const unsigned vo = 4 * (lz >> 4) * rowb + 16 * (lz & 15), von = narrow ? FAR : vo;
    asm volatile("s_nop 15");
#pragma unroll
    for (int m = 0; m < 8; ++m) {
#pragma unroll
        for (int r = 0; r < 4; ++r) {
#pragma unroll
            for (int q = 0; q < 3; ++q) {
                f32x4 y;
                if (q < 2) y = (f32x4){rd_acc(g.acc[m][4 * q][r]), rd_acc(g.acc[m][4 * q + 1][r]), rd_acc(g.acc[m][4 * q + 2][r]), rd_acc(g.acc[m][4 * q + 3][r])};
                else y = (f32x4){g.acc[m][8][r], g.acc[m][9][r], g.acc[m][10][r], g.acc[m][11][r]};
                const f32x4 v = silu4(y, one);
                asm volatile("buffer_store_dwordx4 %0, %1, %2, %3 offen offset:%4 nt\n\ts_nop 1" :: "v"(v), "v"(q ? von : vo), "s"(rO), "s"(so + r * rowb), "i"(q * 256) : "memory");
            }
            SCHED;
        }
        so += 16 * rowb;
    }
}
DI int lead_ones(bool ok) { const unsigned long long m = ~__ballot(ok); return m ? __builtin_ctzll(m) : 64; }

struct Ready {
    char* ws; unsigned call;
    __amdgpu_buffer_rsrc_t rF; LAS unsigned char* zone;
    int pa, pb, n; bool pending; float one;
    DI void poll(int need) {
        const unsigned long long t0 = __builtin_amdgcn_s_memrealtime();
        const int lane = lane_now();
        for (;;) {
            const unsigned a = ld_sys(ws + OFF_F + pa * 256 + lane * 4), b = ld_sys(ws + OFF_F + pb * 256 + lane * 4);
            n = lead_ones(lane < 8 || (a >= call && b >= call));
            if (n >= need) break;
            if (give_up(ws, call, t0)) { n = 64; one = __builtin_nanf(""); break; }
            __builtin_amdgcn_s_sleep(16);
        }
    }
    DI void request() {
        const int lane = lane_now();
        __builtin_amdgcn_raw_ptr_buffer_load_lds(rF, (LAS void*)zone, 4, lane * 4, pa * 256, 0, AUX_SYS);
        __builtin_amdgcn_raw_ptr_buffer_load_lds(rF, (LAS void*)(zone + 256), 4, lane * 4, pb * 256, 0, AUX_SYS);
        pending = true;
    }
    DI int plan(int it) {
        if (n == 64) return 32;
        const int safe = (n - 4) >> 1;
        if (safe <= it + 1) { request(); return it + 1; }
        return safe - 1;
    }
    DI void check(int need) {
        if (pending) {
            const int lane = lane_now();
            const unsigned a = *(const volatile LAS unsigned*)(zone + lane * 4), b = *(const volatile LAS unsigned*)(zone + 256 + lane * 4);
            n = lead_ones(lane < 8 || (a >= call && b >= call)); pending = false;
        }
        if (n < need) poll(need);
        asm volatile("" ::: "memory");
    }
};

DI void gemm_wg(Params& p, LAS unsigned char* lds, int wid, int rg, int S) {
    const int wr = wid >> 1, wc = wid & 1, rank = p.rank;
    char* const ws = p.ws[rank];
    const unsigned call = convert_shards(p, lds, wid, S * 4 + rg);
    int T = S;
    STAMP(ws, 2);
    G7 g;
    g7_init(g, lds, ws + OFF_DATA, DATAB, wid);
    Ready R;
    R.ws = ws; R.call = call; R.rF = __builtin_amdgcn_make_buffer_rsrc((void*)(ws + OFF_F), 0, NPAN * 256, 0x00020000);
    R.zone = lds + LDS_ZONE + wid * 512; R.pending = false; R.one = 1.0f;
    if (!wait_words(ws, ws + OFF_EV + (3 * NR + rank) * 4, 1, 4, call, call)) R.one = __builtin_nanf("");
    R.pa = 4 * rg + (T & 3); R.pb = 16 + colblock(T >> 2, rank); R.n = 8;
    STAMP(ws, 1);
    g7_prologue(g, rfl(own_off(R.pa, 0)), rfl(own_off(R.pb, 0)));
    for (int ti = 0;; ++ti) {
        const int pm = R.pa, cb = R.pb - 16, li = T >> 2;
        const bool narrow = li == NL - 1;
        g.voffn = narrow ? FAR : g.voff;
        asm volatile("" : "+v"(g.voffn));
        S += 2 * NG;
        const int job = S - PSLOT;
        if ((unsigned)job < (unsigned)CPR) S += 2 * NG;
        const bool has_next = S < NROUND * 2 * NG;
        if (has_next) T = S < PSLOT ? S : S - CPR;
        const int qa = 4 * rg + (T & 3), qb = 16 + colblock(T >> 2, rank);
        g.ringB = rfl(ring_off(li)); g.nextA = rfl(own_off(qa, 0)); g.nextB = rfl(own_off(qb, 0));
        R.check(4);
        int lim = R.plan(0);
        trip<true>(g, 0);
        for (int it = 1; it < 32;) {
            if (it >= lim) {
                R.check(it < 30 ? 2 * it + 4 : 64);
                lim = R.plan(it);
            }
            do trip<false>(g, it); while (++it < lim);
        }
        STAMP(ws, 8 + 2 * ti);
        R.pa = qa; R.pb = qb;
        R.request();
        g7_epilogue(g, __builtin_amdgcn_make_buffer_rsrc((void*)p.out[rank], 0, OUTB, 0x00020000), rfl((pm * 256 + wr * 128) * ROWB + (col0(cb) + wc * (narrow ? 64 : 192)) * 4), ROWB, R.one, narrow);
        WAITVM(60);
        if (wid == 0 && lane_now() == 0) __hip_atomic_fetch_add((gu32*)(ws + OFF_FIN + li * 4), 1u, __ATOMIC_RELAXED, __HIP_MEMORY_SCOPE_AGENT);
        if (li < CPR) {
            WAITVM(0);
            BARRIER;
            STAMP(ws, 4);
            if (wid == 0) {
                __builtin_amdgcn_fence(__ATOMIC_RELEASE, "agent");
                WAITVM(0);
                if (lane_now() == 0) st_sys(ws + OFF_AVC + (li * 16 + pm) * 4, call);
            }
        }
        if ((unsigned)job < (unsigned)CPR) { pack_tile(p, lds + LDS_ZONE + 4096 + wid * 1024, call, job * 16 + 4 * rg + wid); STAMP(ws, 5); }
        STAMP(ws, 9 + 2 * ti);
        if (!has_next) break;
    }
    __builtin_amdgcn_s_waitcnt(WCNT(0, 0));
}

__global__ __launch_bounds__(256, 1) void fused(Args) {
    extern __shared__ __attribute__((aligned(16))) unsigned char shm[];
    Params& p = *(Params*)__builtin_amdgcn_kernarg_segment_ptr();
    LAS unsigned char* lds = (LAS unsigned char*)shm;
    char* const ws = p.ws[p.rank];
    const int wid = rfl(threadIdx.x >> 6);
    STAMP(ws, 0);
    STAMPV(ws, 3, __builtin_amdgcn_s_getreg(20 | (0 << 6) | (3 << 11)));
    const int grp = blockIdx.x & 7, idx = blockIdx.x >> 3;
    if (idx < NG) gemm_wg(p, lds, wid, (grp & 1 ? 0 : 2) + (grp >> 2), (grp >> 1 & 1) * NG + idx);
    else {
        const int cw = (idx - NG) * 8 + grp;
        const unsigned call = draw_call(ws, wid);
        fetch_wave(p, lds + wid * 32768, call, wid, cw * 4 + wid);
    }
}
}

extern "C" void kernel_launch(int rank, int nranks, ncclComm_t comm, hipStream_t stream,
                              void* const* d_in, const int* in_sizes, int n_in,
                              void* d_out, int out_size,
                              const int* in_dtypes, int out_dtype,
                              void* const* d_out_all,
                              void* const* d_ws_all, size_t ws_bytes) {
    if (nranks != NR || (unsigned)rank >= (unsigned)NR || n_in != 2 || ws_bytes < WS_NEED || in_sizes[0] != MM * KS || in_sizes[1] != KS * NN || out_size != MM * NN) {
        fprintf(stderr, "kernel_launch: unexpected problem shape\n");
        return;
    }
    static bool attr_set[NR];
    if (!attr_set[rank]) {
        HIP_CHECK(hipFuncSetAttribute((const void*)fused, hipFuncAttributeMaxDynamicSharedMemorySize, LDS_BYTES));
        attr_set[rank] = true;
    }
    Args p;
    p.x = (const float*)d_in[0]; p.w = (const float*)d_in[1];
    for (int i = 0; i < NR; ++i) { p.ws[i] = (char*)d_ws_all[i]; p.out[i] = (float*)d_out_all[i]; }
    p.rank = rank; p.pad = 0;
    hipLaunchKernelGGL(fused, dim3(256), dim3(256), LDS_BYTES, stream, p);
    HIP_CHECK(hipGetLastError());
}
```

```cpp
#include <hip/hip_runtime.h>
#include <rccl/rccl.h>
#include <cstdio>

#define HIP_CHECK(expr) do { hipError_t _e = (expr); if (_e != hipSuccess) { \
    fprintf(stderr, "HIP error %s at %s:%d\n", hipGetErrorString(_e), __FILE__, __LINE__); \
    return; } } while (0)

#define LAS __attribute__((address_space(3)))
#define GAS __attribute__((address_space(1)))
#define DI __device__ __forceinline__
typedef _Float16 f16x8 __attribute__((ext_vector_type(8)));
typedef float f32x4 __attribute__((ext_vector_type(4)));
typedef unsigned u32x4 __attribute__((ext_vector_type(4)));
typedef unsigned u32x2 __attribute__((ext_vector_type(2)));
typedef GAS unsigned gu32;

#define NG 29
#define DBG 0
namespace {
constexpr int MM = 4096, NN = 32768, KS = 512, NR = 8;
constexpr int NGEMM = 8 * NG, NCOMM = 256 - NGEMM, NCW = 4 * NCOMM;
constexpr int TW = 384;
constexpr int CPR = 2, NCB = NR * CPR;
constexpr int BCOLS = NN - NCB * TW, NBB = (BCOLS + TW - 1) / TW;
constexpr int NWP = NBB + NCB, NPAN = 16 + NWP;
constexpr int NL = CPR + NBB;
#define RBL 3
constexpr int RB = 1 << RBL, NWB = NGEMM * RB / 16, NJOB = CPR * RB / 4, NROUND = 5, PSLOT = 2 * NWB - NJOB;
static_assert(NROUND * NWB == RB * NL + NJOB, "slots");
constexpr unsigned QUAD = 16384, XBLK = 2 * QUAD, WBLK = 3 * QUAD, XPANEL = 64 * XBLK;
static_assert(BCOLS - (NBB - 2) * TW == 2 * 256, "short blocks");
constexpr unsigned NE16 = 48, NE32 = 16, TS16 = 512 + 2 * NE16, TS32 = 512 + 2 * NE32, QB16 = 4 * (TS16 + 3072), QB32 = 4 * (TS32 + 3072);
constexpr unsigned OUTB = (unsigned)MM * NN * 4u, ROWB = NN * 4;
constexpr unsigned OFF_CALL = 0;
constexpr unsigned OFF_TMO = 32768;
constexpr unsigned OFF_FIN = 32768 + 1024;
constexpr unsigned OFF_ETA = 32768 + 2048;
constexpr unsigned OFF_TICK = 32768 + 4096;
constexpr unsigned OFF_F = 65536;
constexpr unsigned OFF_AV = 131072;
constexpr unsigned OFF_AVC = OFF_AV + 8192;
constexpr unsigned OFF_CNT = 196608;
constexpr unsigned OFF_EV = OFF_CNT + 4096;
constexpr unsigned OFF_DBG = 262144, OFF_DBG2 = 524288;
constexpr int RING = NL - 32;
constexpr unsigned OFF_DATA = 1u << 20;
constexpr unsigned REL_WL = 16 * XPANEL;
constexpr unsigned REL_WR = REL_WL + NWP * 8 * WBLK;
constexpr unsigned DATAB = REL_WR + RING * 56 * WBLK;
constexpr int NQ16 = 16 * 8 * 2 + (NWP - 2) * 8 * 3 + 32;
constexpr unsigned OFF_PK16 = OFF_DATA + DATAB;
constexpr int NCI = CPR * (MM / 32);
constexpr unsigned OFF_PK32 = OFF_PK16 + (NQ16 + 16) * QB16;
constexpr unsigned OFF_RAW32 = OFF_PK32 + NCI * 3 * QB32;
constexpr unsigned WS_NEED = OFF_RAW32 + NCI * WBLK;
static_assert(NPAN * 256 <= 65536, "flags");
constexpr int LDS_A = 6 * QUAD, LDS_ZONE = LDS_A + 3 * QUAD, LDS_BYTES = LDS_ZONE + 8192;
constexpr unsigned long long DEADLINE = 50000000ull;
#define WSCALE 32.0f
constexpr int AUX_SYS = 17, AUX_NT = 2;

struct Args {
    const float* x; const float* w;
    char* ws[NR]; float* out[NR];
    int rank, pad;
};
typedef const __attribute__((address_space(4))) Args Params;

constexpr int WCNT(int vm, int lgkm) { return (vm & 15) | (7 << 4) | ((lgkm & 15) << 8) | ((vm >> 4) << 14); }
#define SCHED __builtin_amdgcn_sched_barrier(0)
#define WAITVM(n) asm volatile("s_waitcnt vmcnt(" #n ")" ::: "memory")
#define WAITLG0 asm volatile("s_waitcnt lgkmcnt(0)" ::: "memory")
#define BARRIER do { asm volatile("" ::: "memory"); __builtin_amdgcn_s_barrier(); asm volatile("" ::: "memory"); } while (0)

DI unsigned ld_sys(const char* p) { return __hip_atomic_load((gu32*)p, __ATOMIC_RELAXED, __HIP_MEMORY_SCOPE_SYSTEM); }
DI void st_sys(char* p, unsigned v) { __hip_atomic_store((gu32*)p, v, __ATOMIC_RELAXED, __HIP_MEMORY_SCOPE_SYSTEM); }
DI unsigned ld_ag(const char* p) { return __hip_atomic_load((gu32*)p, __ATOMIC_RELAXED, __HIP_MEMORY_SCOPE_AGENT); }
DI void st_ag(char* p, unsigned v) { __hip_atomic_store((gu32*)p, v, __ATOMIC_RELAXED, __HIP_MEMORY_SCOPE_AGENT); }
DI unsigned rfl(unsigned x) { return __builtin_amdgcn_readfirstlane(x); }
DI int lane_now() { int l; asm volatile("v_mbcnt_lo_u32_b32 %0, -1, 0\n\tv_mbcnt_hi_u32_b32 %0, -1, %0" : "=v"(l)); return l; }
DI bool give_up(char* ws, unsigned call, unsigned long long t0) {
    if (rfl(ld_ag(ws + OFF_TMO)) == call || __builtin_amdgcn_s_memrealtime() - t0 > DEADLINE) { st_ag(ws + OFF_TMO, call); return true; }
    return false;
}
DI bool arrive(char* ws, char* lane_ws, int rank, int kind, unsigned n, unsigned call) {
    const int lane = lane_now();
    unsigned t = 0;
    if (lane == 0) t = __hip_atomic_fetch_add((gu32*)(ws + OFF_CNT + kind * 128), 1u, __ATOMIC_RELAXED, __HIP_MEMORY_SCOPE_AGENT);
    const bool last = rfl(t) + 1 == n * call;
    if (last && lane < NR) st_sys(lane_ws + OFF_EV + (kind * NR + rank) * 4, call);
    return last;
}
DI bool wait_words(char* ws, const char* f, int n, int step, unsigned want, unsigned call) {
    const int lane = lane_now();
    const unsigned long long t0 = __builtin_amdgcn_s_memrealtime();
    for (;;) {
        bool ok = true;
#pragma unroll
        for (int i = 0; i < 4; ++i) { const int k = lane + 64 * i; if (64 * i < n) ok &= ld_sys(f + (k < n ? k : 0) * step) >= want; }
        if (__all(ok)) break;
        if (give_up(ws, call, t0)) return false;
        __builtin_amdgcn_s_sleep(4);
    }
    asm volatile("" ::: "memory");
    return true;
}
#define STAMP(ws, i) do {} while (0)
#define STAMPW(ws, i) do {} while (0)
#define STAMPV(ws, i, v) do {} while (0)

DI void store1(const char* base, unsigned v, const u32x4& f) {
    asm volatile("s_nop 4\n\tglobal_store_dwordx4 %0, %1, %2 sc0 sc1\n\ts_nop 1" :: "v"(v), "v"(f), "s"(base) : "memory");
}
DI void store3(const char* base, unsigned v, const u32x4& a, const u32x4& b, const u32x4& c) {
    asm volatile("s_nop 4\n\tglobal_store_dwordx4 %0, %1, %4 sc0 sc1\n\t"
                 "global_store_dwordx4 %0, %2, %4 offset:1024 sc0 sc1\n\t"
                 "global_store_dwordx4 %0, %3, %4 offset:2048 sc0 sc1\n\ts_nop 1"
                 :: "v"(v), "v"(a), "v"(b), "v"(c), "s"(base) : "memory");
}
DI void store4(const char* base, unsigned v, const u32x4 (&f)[4]) {
    asm volatile("s_nop 4\n\tglobal_store_dwordx4 %0, %1, %5 sc0 sc1\n\t"
                 "global_store_dwordx4 %0, %2, %5 offset:1024 sc0 sc1\n\t"
                 "global_store_dwordx4 %0, %3, %5 offset:2048 sc0 sc1\n\t"
                 "global_store_dwordx4 %0, %4, %5 offset:3072 sc0 sc1\n\ts_nop 1"
                 :: "v"(v), "v"(f[0]), "v"(f[1]), "v"(f[2]), "v"(f[3]), "s"(base) : "memory");
}
DI void store4o(const char* base, unsigned v, const u32x4 (&f)[4]) {
    asm volatile("s_nop 4\n\tglobal_store_dwordx4 %0, %4, %8 sc1 nt\n\t"
                 "global_store_dwordx4 %1, %5, %8 sc1 nt\n\t"
                 "global_store_dwordx4 %2, %6, %8 sc1 nt\n\t"
                 "global_store_dwordx4 %3, %7, %8 sc1 nt\n\ts_nop 1"
                 :: "v"(v), "v"(v + 2 * ROWB), "v"(v + 4 * ROWB), "v"(v + 6 * ROWB), "v"(f[0]), "v"(f[1]), "v"(f[2]), "v"(f[3]), "s"(base) : "memory");
}

DI unsigned perm(unsigned hi, unsigned lo, unsigned sel) { return __builtin_amdgcn_perm(hi, lo, sel); }
template <int CTRL, int ROWMASK> DI unsigned dpp0(unsigned x) { return __builtin_amdgcn_update_dpp(0u, x, CTRL, ROWMASK, 0xf, false); }
DI unsigned wave_sum(unsigned x) {
    x += dpp0<0x111, 0xf>(x); x += dpp0<0x112, 0xf>(x); x += dpp0<0x114, 0xf>(x); x += dpp0<0x118, 0xf>(x); x += dpp0<0x142, 0xa>(x); x += dpp0<0x143, 0xc>(x);
    return __builtin_amdgcn_readlane(x, 63);
}
DI unsigned umax(unsigned a, unsigned b) { return a > b ? a : b; }
DI unsigned umin(unsigned a, unsigned b) { return a < b ? a : b; }
DI unsigned wave_max(unsigned x) {
    x = umax(x, dpp0<0x111, 0xf>(x)); x = umax(x, dpp0<0x112, 0xf>(x)); x = umax(x, dpp0<0x114, 0xf>(x)); x = umax(x, dpp0<0x118, 0xf>(x));
    x = umax(x, dpp0<0x142, 0xa>(x)); x = umax(x, dpp0<0x143, 0xc>(x));
    return __builtin_amdgcn_readlane(x, 63);
}
template <unsigned NE> struct ExcList {
    volatile LAS unsigned short* e; unsigned cnt;
    DI void begin(LAS unsigned char* tail, int lane) { e = (volatile LAS unsigned short*)(tail + 512); e[lane] = 0; cnt = 1; }
    DI bool add(bool act, unsigned entry) {
        const unsigned long long bal = __ballot(act);
        const unsigned slot = cnt + __builtin_amdgcn_mbcnt_hi((unsigned)(bal >> 32), __builtin_amdgcn_mbcnt_lo((unsigned)bal, 0u));
        if (act && slot < NE) e[slot] = (unsigned short)entry;
        cnt += __builtin_popcountll(bal);
        return cnt <= NE;
    }
};
template <unsigned NE> DI bool pack_finish(const char* base, unsigned blk, int c, int lane, LAS unsigned char* tail, ExcList<NE>& x, unsigned ebase,
                    const u32x4& p0, const u32x4& p1, const u32x4& p2, unsigned t0, unsigned t1) {
    if (lane == 0) x.e[0] = (unsigned short)(ebase | (x.cnt > NE ? 0x8000u : 0u));
    *(volatile LAS u32x2*)(tail + lane * 8) = (u32x2){t0, t1};
    constexpr int TS = 512 + 2 * NE;
    store3(base, blk + 4 * TS + c * 3072 + lane * 16, p0, p1, p2);
    const u32x4 t = *(volatile LAS u32x4*)(tail + (lane < TS / 16 ? lane : 0) * 16);
    if (lane < TS / 16) store1(base, blk + c * TS + lane * 16, t);
    return x.cnt <= NE;
}
DI void pack16(const u32x4 (&f)[4], const char* base, unsigned blk, int c, int lane, LAS unsigned char* tail) {
    unsigned lo[8], h[8], g[8], sum = 0;
#pragma unroll
    for (int i = 0; i < 8; ++i) {
        const unsigned a = f[i >> 1][2 * (i & 1)], b = f[i >> 1][2 * (i & 1) + 1];
        lo[i] = perm(b, a, 0x06040200u); h[i] = perm(b, a, 0x07050301u);
        sum = __builtin_amdgcn_udot4(h[i] & 0x7c7c7c7cu, 0x01010101u, sum, false);
    }
    const int top = (int)((wave_sum(sum) + 15155u + 8191u) >> 13);
    const unsigned eb = top < 7 ? 0u : top > 31 ? 24u : (unsigned)(top - 7), b4 = eb * 0x04040404u;
    ExcList<NE16> x; x.begin(tail, lane);
    unsigned m1 = 0, m0 = 0;
#pragma unroll
    for (int i = 0; i < 8; ++i) {
        g[i] = (h[i] & 0x83838383u) | (((h[i] | 0x80808080u) - b4) & 0x1c1c1c1cu);
        const unsigned xr = (g[i] + b4) ^ h[i];
        m1 |= ((xr >> 6) & 0x01010101u) << i; m0 |= ((xr >> 5) & 0x01010101u) << i;
    }
    for (unsigned m = m1 | m0; __any(m != 0); m &= m - 1) {
        const int k = m ? __builtin_ctz(m) : 0, i = k & 7;
        if (!x.add(m != 0, ((i >> 2) * 1024 + lane * 16 + (i & 3) * 4 + (k >> 3)) | ((m0 >> k) & 1u) << 11 | ((m1 >> k) & 1u) << 12)) break;
    }
    unsigned d[6];
#pragma unroll
    for (int q = 0; q < 2; ++q) {
        const unsigned s = g[4 * q + 3];
        d[3 * q] = g[4 * q] | (s & 0x03030303u) << 5; d[3 * q + 1] = g[4 * q + 1] | (s & 0x0c0c0c0cu) << 3;
        d[3 * q + 2] = g[4 * q + 2] | (s & 0x10101010u) << 1 | (s & 0x80808080u) >> 1;
    }
    pack_finish(base, blk, c, lane, tail, x, eb, (u32x4){lo[0], lo[1], lo[2], lo[3]}, (u32x4){lo[4], lo[5], lo[6], lo[7]}, (u32x4){d[0], d[1], d[2], d[3]}, d[4], d[5]);
}
DI bool pack32(const u32x4 (&f)[4], const char* base, unsigned blk, int c, int lane, LAS unsigned char* tail) {
    unsigned t[12], h[4], mx = 0;
#pragma unroll
    for (int j = 0; j < 4; ++j) {
        t[3 * j] = perm(f[j][3], f[j][0], 0x04020100u); t[3 * j + 1] = perm(f[j][3], f[j][1], 0x05020100u); t[3 * j + 2] = perm(f[j][3], f[j][2], 0x06020100u);
        h[j] = perm(perm(f[j][3], f[j][2], 0x07030703u), perm(f[j][1], f[j][0], 0x07030703u), 0x05040100u);
#pragma unroll
        for (int d = 0; d < 4; ++d) mx = umax(mx, f[j][d] & 0x7fffffffu);
    }
    const unsigned top = wave_max(mx) >> 24, eb = top < 7 ? 0u : top - 7, b1 = eb * 0x01010101u;
    ExcList<NE32> x; x.begin(tail, lane);
    unsigned cd[4], xn[2] = {0, 0};
#pragma unroll
    for (int j = 0; j < 4; ++j) {
        cd[j] = ((h[j] | 0x80808080u) - b1) & 0x07070707u;
        const unsigned xr = ((cd[j] + b1) ^ h[j]) & 0x78787878u;
        xn[j >> 1] |= (j & 1) ? xr << 1 : xr >> 3;
    }
    for (unsigned long long m = (unsigned long long)xn[1] << 32 | xn[0]; __any(m != 0);) {
        const int k = m ? __builtin_ctzll(m) >> 2 : 0, j = 2 * (k >> 3) + (k & 1);
        const unsigned v = (unsigned)(m >> (4 * k)) & 15u;
        if (!x.add(m != 0, (lane * 16 + j * 4 + ((k >> 1) & 3)) | v << 11)) break;
        m &= ~(15ull << (4 * k));
    }
    const unsigned n0 = cd[0] | (h[0] & 0x80808080u) >> 4 | cd[1] << 4 | (h[1] & 0x80808080u), n1 = cd[2] | (h[2] & 0x80808080u) >> 4 | cd[3] << 4 | (h[3] & 0x80808080u);
    return pack_finish(base, blk, c, lane, tail, x, eb, (u32x4){t[0], t[1], t[2], t[3]}, (u32x4){t[4], t[5], t[6], t[7]}, (u32x4){t[8], t[9], t[10], t[11]}, n0, n1);
}
DI bool unpack(bool is32, const LAS unsigned char* main3, const LAS unsigned char* tail, LAS unsigned char* patch, int lane, u32x4 (&f)[4]) {
    unsigned e = *(const volatile LAS unsigned short*)(tail + 512 + lane * 2);
    const u32x2 tt = *(const volatile LAS u32x2*)(tail + lane * 8);
    const u32x4 p0 = *(const volatile LAS u32x4*)(main3 + lane * 16), p1 = *(const volatile LAS u32x4*)(main3 + 1024 + lane * 16), p2 = *(const volatile LAS u32x4*)(main3 + 2048 + lane * 16);
    const unsigned hdr = rfl(e);
    if (lane == 0 || lane >= (int)(is32 ? NE32 : NE16)) e = 0;
    LAS unsigned* const pw = (LAS unsigned*)(patch + (e & 0x7fcu));
    __hip_atomic_fetch_or(pw, (e >> 11) << ((is32 ? 3 : 5) + 8 * (e & 3u)), __ATOMIC_RELAXED, __HIP_MEMORY_SCOPE_WAVEFRONT);
    const u32x4 x0 = *(const volatile LAS u32x4*)(patch + lane * 16), x1 = *(const volatile LAS u32x4*)(patch + 1024 + lane * 16);
    *(volatile LAS unsigned*)pw = 0;
    if (is32) {
        const unsigned b1 = (hdr & 127u) * 0x01010101u, t[12] = {p0[0], p0[1], p0[2], p0[3], p1[0], p1[1], p1[2], p1[3], p2[0], p2[1], p2[2], p2[3]};
#pragma unroll
        for (int j = 0; j < 4; ++j) {
            const unsigned n = tt[j >> 1];
            const unsigned h = ((j & 1) ? ((((n >> 4) & 0x07070707u) + b1) | (n & 0x80808080u)) : (((n & 0x07070707u) + b1) | ((n << 4) & 0x80808080u))) ^ x0[j];
            f[j][0] = perm(h, t[3 * j], 0x04020100u); f[j][1] = perm(h, t[3 * j + 1], 0x05020100u); f[j][2] = perm(h, t[3 * j + 2], 0x06020100u);
            f[j][3] = perm(perm(h, t[3 * j + 2], 0x07030703u), perm(t[3 * j + 1], t[3 * j], 0x07030703u), 0x05040100u);
        }
    } else {
        const unsigned b4 = (hdr & 31u) * 0x04040404u, d[6] = {p2[0], p2[1], p2[2], p2[3], tt[0], tt[1]};
#pragma unroll
        for (int q = 0; q < 2; ++q) {
            const unsigned a = d[3 * q], b = d[3 * q + 1], c = d[3 * q + 2];
            const unsigned g[4] = {a & 0x9f9f9f9fu, b & 0x9f9f9f9fu, c & 0x9f9f9f9fu,
                                   ((a >> 5) & 0x03030303u) | ((b >> 3) & 0x0c0c0c0cu) | ((c >> 1) & 0x10101010u) | ((c << 1) & 0x80808080u)};
#pragma unroll
            for (int i = 0; i < 4; ++i) {
                const unsigned h = (g[i] + b4) ^ (q ? x1[i] : x0[i]), lo = q ? p1[i] : p0[i];
                f[2 * q + (i >> 1)][2 * (i & 1)] = perm(h, lo, 0x05010400u); f[2 * q + (i >> 1)][2 * (i & 1) + 1] = perm(h, lo, 0x07030602u);
            }
        }
    }
    return hdr < 0x8000u;
}

constexpr int PSHORT = 16 + NBB - 2;
DI bool two_quads(int panel) { return panel < 16 || (unsigned)(panel - PSHORT) < 2u; }
DI int col0(int cb) { return cb >= NBB ? BCOLS + (cb - NBB) * TW : cb == NBB - 1 ? BCOLS - 256 : cb * TW; }
DI int colblock(int li, int rank) { return li < CPR ? NBB + CPR * rank + li : li - CPR; }
DI unsigned own_off(int panel, int kt) { return (panel < 16 ? panel * XPANEL : REL_WL + (panel - 16) * 8 * WBLK) + kt * (two_quads(panel) ? XBLK : WBLK); }
DI unsigned ring_off(int li) { return REL_WR + (li >= RING ? li - RING : li) * (56 * WBLK); }
DI unsigned pk16_off(int panel, int kt) { return OFF_PK16 + ((panel < 16 ? panel * 16 : 256 + (panel - 16) * 24) + kt * (two_quads(panel) ? 2 : 3)) * QB16; }
static_assert(2 * RING >= NL, "ring");
constexpr int round_end(int r) { return (NWB * (r + 1) - (r ? NJOB : 0) + RB - 1) / RB; }
#define XLAGS {0, 0, 3, 3}
#define BL 0
constexpr int XL[4] = XLAGS, NSTEP0 = 8 + (XL[3] > BL ? XL[3] : BL);
constexpr int x_lo(int s) { int g = 0; while (g < 4 && s - XL[g] >= 8) ++g; return 4 * g; }
constexpr int x_hi(int s) { int g = 0; while (g < 4 && s >= XL[g]) ++g; return 4 * g; }
constexpr int B0 = round_end(0) - CPR, W0Q = 3 * (B0 + NCB);
constexpr int cstep_start(int s) { int n = 0; for (int k = 0; k < s; ++k) n += 2 * (x_hi(k) - x_lo(k)) + (k < 8 ? W0Q : 0); return n; }
constexpr int steps_within(int quads) { int s = 0; while (s < NSTEP0 && cstep_start(s + 1) <= quads) ++s; return s; }
constexpr int ST1 = 3, STEPS1 = steps_within(NGEMM), STEPS2 = steps_within(ST1 * NGEMM);
static_assert(BL == 0 && STEPS1 >= 1 && NQ16 > ST1 * NGEMM + NGEMM, "stages");
DI void conv_quad(int b, int& panel, int& kt, int& qd) {
    if (b < cstep_start(NSTEP0)) {
        int s = 0, i = b, lo = x_lo(0), nx = 2 * (x_hi(0) - x_lo(0));
#pragma unroll
        for (int k = 1; k < NSTEP0; ++k) if (b >= cstep_start(k)) { s = k; i = b - cstep_start(k); lo = x_lo(k); nx = 2 * (x_hi(k) - x_lo(k)); }
        if (i < nx) { panel = lo + (i >> 1); qd = i & 1; kt = s - (panel < 4 ? XL[0] : panel < 8 ? XL[1] : panel < 12 ? XL[2] : XL[3]); }
        else { const int w = (i - nx) / 3; qd = i - nx - 3 * w; panel = 16 + (w < B0 ? w : w - B0 + NBB); kt = s; }
    } else if (b < NQ16 - 32) { const int r = b - cstep_start(NSTEP0), k = r / 3; qd = r - 3 * k; panel = 16 + B0 + (k >> 3); kt = k & 7; }
    else { const int r = b - (NQ16 - 32); panel = PSHORT + (r >> 4); kt = r >> 1 & 7; qd = r & 1; }
}
DI unsigned draw_call(char* ws, int wid) {
    unsigned c;
    asm volatile("s_nop 4\n\tglobal_load_dword %0, %1, %2 sc1\n\ts_waitcnt vmcnt(0)" : "=&v"(c) : "v"(OFF_CALL + blockIdx.x * 128), "s"(ws) : "memory");
    c = rfl(c) + 1;
    BARRIER;
    if (wid == 0 && lane_now() == 0) st_ag(ws + OFF_CALL + blockIdx.x * 128, c);
    return c;
}
DI unsigned convert_shards(Params& p, LAS unsigned char* lds, int wid, int gi) {
    const int lane = lane_now(), rank = p.rank;
    char* const ws = p.ws[rank];
    const __amdgpu_buffer_rsrc_t rx = __builtin_amdgcn_make_buffer_rsrc((void*)p.x, 0, MM * KS * 4, 0x00020000);
    const __amdgpu_buffer_rsrc_t rw = __builtin_amdgcn_make_buffer_rsrc((void*)p.w, 0, KS * NN * 4, 0x00020000);
    const unsigned vx = ((lane & 15) * KS + 8 * (lane >> 4)) * 4, vw = (8 * (lane >> 4) * NN + 4 * (lane & 15)) * 4;
    LAS unsigned char* const ring = lds + wid * 32768;
    LAS unsigned char* const tail = lds + 131072 + wid * 1024;
    const int nu = (NQ16 - gi + NGEMM - 1) / NGEMM;
    auto load = [&](int u) {
        int pa, kt, qd; conv_quad(gi + (u < nu ? u : 0) * NGEMM, pa, kt, qd);
        LAS unsigned char* const slot = ring + (u & 3) * 8192;
        if (pa < 16) {
            const unsigned s0 = ((pa * 256 + 128 * (wid >> 1) + 64 * qd) * KS + 32 * (2 * kt + (wid & 1))) * 4;
#pragma unroll
            for (int e = 0; e < 8; ++e) __builtin_amdgcn_raw_ptr_buffer_load_lds(rx, (LAS void*)(slot + e * 1024), 16, vx, s0 + (e >> 1) * (16 * KS * 4) + (e & 1) * 16, 0, AUX_NT);
        } else {
            const unsigned s0 = (32 * (2 * kt + (wid & 1)) * NN + col0(pa - 16) + (two_quads(pa) ? 128 : 192) * (wid >> 1) + 64 * qd) * 4;
#pragma unroll
            for (int e = 0; e < 8; ++e) __builtin_amdgcn_raw_ptr_buffer_load_lds(rw, (LAS void*)(slot + e * 1024), 16, vw, s0 + e * ROWB, 0, AUX_NT);
        }
    };
    load(0);
    char* peer_ws = p.ws[lane & 7];
    const unsigned call = draw_call(ws, wid);
    asm volatile("" : "+v"(peer_ws));
    STAMP(ws, 50);
    for (int u = 0; u < nu; ++u) {
        if (u > 0) { load(u + 3); WAITVM(24); }
        if (u == ST1 + 1) {
            BARRIER;
            if (wid == 0) arrive(ws, peer_ws, rank, 2, NGEMM, call);
        }
        int pa, kt, qd; conv_quad(gi + u * NGEMM, pa, kt, qd);
        f32x4 v[8]; u32x4 f[4];
#pragma unroll
        for (int e = 0; e < 8; ++e) v[e] = *(const LAS f32x4*)(ring + (u & 3) * 8192 + e * 1024 + lane * 16);
#pragma unroll
        for (int j = 0; j < 4; ++j) { f16x8 h;
#pragma unroll
            for (int e = 0; e < 8; ++e) h[e] = (_Float16)(pa < 16 ? v[2 * j + (e >> 2)][e & 3] : v[e][j] * WSCALE);
            f[j] = __builtin_bit_cast(u32x4, h); }
        pack16(f, ws, pk16_off(pa, kt) + qd * QB16, wid, lane, tail);
        store4(ws, OFF_DATA + own_off(pa, kt) + (4 * qd + wid) * 4096 + lane * 16, f);
        if (u == 0) {
            STAMP(ws, 51);
            WAITVM(0);
            BARRIER;
            if (wid == 0 && lane < NR) st_sys(peer_ws + OFF_AV + (rank * 256 + gi) * 4, call);
            load(1); load(2); load(3);
            if (wid == 1) arrive(ws, peer_ws, rank, 1, NGEMM, call);
            STAMP(ws, 52);
        }
    }
    WAITVM(0);
    BARRIER;
    if (wid == 0) arrive(ws, peer_ws, rank, 3, NGEMM, call);
    return call;
}

constexpr int G0 = 16 + round_end(0), NBI = (16 + NL) * 8, NTW = 7 * NBI, NTC = 7 * NCI, NT = NTW + NTC;
struct Item { const char* src; unsigned size, pk, raw, dof; int flag; bool three; };
constexpr int w_lo(int s) { return s < 8 ? 16 : 16 + CPR; }
constexpr int w_hi(int s) { return s < BL || s >= 8 + BL ? 16 + CPR : G0; }
constexpr int step_start(int s) { int n = 0; for (int k = 0; k < s; ++k) n += x_hi(k) - x_lo(k) + w_hi(k) - w_lo(k); return n; }
static_assert(step_start(NSTEP0) == 8 * G0, "order");
DI void block_of(unsigned t, int& i, int& kt) {
    if (t < 8u * G0) {
        int s = 0, hi = x_hi(0), skip = w_lo(0) - x_hi(0); i = t;
#pragma unroll
        for (int k = 1; k < NSTEP0; ++k) if (t >= (unsigned)step_start(k)) { s = k; i = t - step_start(k) + x_lo(k); hi = x_hi(k); skip = w_lo(k) - x_hi(k); }
        if (i >= hi) i += skip;
        kt = s - (i < 4 ? XL[0] : i < 8 ? XL[1] : i < 12 ? XL[2] : i < 16 ? XL[3] : i < 16 + CPR ? 0 : BL);
    } else {
        unsigned u = 0, n = 1, b = 0;
#pragma unroll
        for (int r = 1; r < NROUND; ++r)
            if (t >= 8u * (16 + round_end(r - 1))) { b = 16 + round_end(r - 1); u = t - 8u * b; n = round_end(r) - round_end(r - 1); }
        kt = u / n; i = b + (u - kt * n);
    }
}
DI unsigned slab_out(unsigned s, int from) { return (s & 127u) * 32u * ROWB + (BCOLS + (CPR * from + (s >> 7)) * TW) * 4u; }
DI Item make_item(Params& p, unsigned t) {
    const int rank = p.rank;
    const unsigned tt = t < (unsigned)NTW ? t : t < (unsigned)NT ? t - NTW : 0, u = tt / 7, pj = tt - 7 * u;
    const int peer = (rank + 1 + pj) & 7;
    Item it;
    it.src = p.ws[peer]; it.size = t < (unsigned)NT ? WS_NEED : 0;
    if (t < (unsigned)NTW) {
        int i, kt; block_of(u, i, kt);
        const int panel = i < 16 ? i : 16 + colblock(i - 16, rank);
        it.three = !two_quads(panel);
        it.pk = pk16_off(panel, kt); it.raw = OFF_DATA + own_off(panel, kt);
        it.dof = OFF_DATA + (i < 16 ? i * XPANEL + 8 * XBLK : ring_off(i - 16)) + (7 * kt + pj) * (it.three ? WBLK : XBLK);
        it.flag = panel * 64 + 8 + 7 * kt + pj;
    } else {
        it.three = true;
        it.pk = OFF_PK32 + u * (3 * QB32); it.raw = OFF_RAW32 + u * WBLK; it.dof = slab_out(u, peer); it.flag = -1;
    }
    return it;
}
DI void load_main(const Item& it, unsigned off, LAS unsigned char* to, int lane) {
    const __amdgpu_buffer_rsrc_t r = __builtin_amdgcn_make_buffer_rsrc((void*)it.src, 0, it.size, 0x00020000);
    asm volatile("s_mov_b32 m0, %3\n\ts_nop 4\n\tbuffer_load_dwordx4 %0, %1, %2 offen sc0 sc1 lds\n\tbuffer_load_dwordx4 %0, %1, %2 offen offset:1024 sc0 sc1 lds\n\t"
                 "buffer_load_dwordx4 %0, %1, %2 offen offset:2048 sc0 sc1 lds" :: "v"(lane * 16), "s"(r), "s"(off), "s"((unsigned)(size_t)to) : "memory");
}
DI void load_tails(const Item& it, unsigned off, LAS unsigned char* to, int lane) {
    const __amdgpu_buffer_rsrc_t r = __builtin_amdgcn_make_buffer_rsrc((void*)it.src, 0, it.size, 0x00020000);
    const unsigned long long rest = it.flag < 0 ? (1ull << NE32 / 2) - 1 : (1ull << NE16 / 2) - 1;
    asm volatile("s_mov_b32 m0, %3\n\ts_nop 4\n\tbuffer_load_dwordx4 %0, %1, %2 offen sc0 sc1 lds\n\tbuffer_load_dwordx4 %0, %1, %2 offen offset:1024 sc0 sc1 lds\n\t"
                 "s_mov_b64 exec, %4\n\tbuffer_load_dwordx4 %0, %1, %2 offen offset:2048 sc0 sc1 lds\n\ts_mov_b64 exec, -1"
                 :: "v"(lane * 16), "s"(r), "s"(off), "s"((unsigned)(size_t)to), "s"(rest) : "memory");
}
constexpr unsigned NANS = 0x7fc07fc0u;
constexpr unsigned LQ = 14848, LMAIN = 2560;
struct Fetch {
    const char* ws; const char* out;
    LAS unsigned char* lds;
    unsigned slot; int lane; bool ok;
    template <int C0, bool WAITED> DI void quad(const Item& cur, unsigned qd, const Item& nx, unsigned nqd) {
        LAS unsigned char* const q = lds + slot; LAS unsigned char* const nq = lds + (LQ - slot);
        const unsigned npk = nx.pk + nqd * (nx.flag < 0 ? QB32 : QB16), nmain = npk + 4 * (nx.flag < 0 ? TS32 : TS16), ts = cur.flag < 0 ? TS32 : TS16;
#pragma unroll
        for (int c = 0; c < 4; ++c) {
            if (c > 0 || !WAITED) WAITVM(25);
            if (c == 0) load_tails(nx, npk, nq, lane);
            load_main(nx, nmain + c * 3072, nq + LMAIN + c * 3072, lane);
            u32x4 f[4];
            if (!unpack(cur.flag < 0, q + LMAIN + c * 3072, q + c * ts, lds + 2 * LQ, lane, f)) {
                const char* a = cur.src + (size_t)(cur.raw + qd * QUAD + c * 4096 + lane * 16);
                asm volatile("global_load_dwordx4 %0, %4, off sc0 sc1\n\tglobal_load_dwordx4 %1, %4, off offset:1024 sc0 sc1\n\t"
                             "global_load_dwordx4 %2, %4, off offset:2048 sc0 sc1\n\tglobal_load_dwordx4 %3, %4, off offset:3072 sc0 sc1\n\ts_waitcnt vmcnt(0)"
                             : "=&v"(f[0]), "=&v"(f[1]), "=&v"(f[2]), "=&v"(f[3]) : "v"(a) : "memory");
            }
            if (!ok) f[0] = f[1] = f[2] = f[3] = (u32x4){NANS, NANS, NANS, NANS};
            if (cur.flag >= 0) store4(ws, cur.dof + qd * QUAD + c * 4096 + lane * 16, f);
            else store4o(out, cur.dof + ((C0 + c) / 3 * 8 + (lane >> 5)) * ROWB + (C0 + c) % 3 * 512 + (lane & 31) * 16, f);
        }
        slot = LQ - slot;
    }
};
DI void fetch_wave(Params& p, LAS unsigned char* lds, unsigned call, int wid, int wv) {
    const int lane = lane_now(), rank = p.rank;
    char* const ws = p.ws[rank];
    Fetch F;
    F.ws = ws; F.out = (const char*)p.out[rank]; F.lds = lds; F.lane = lane;
    *(LAS u32x4*)(lds + 2 * LQ + lane * 16) = (u32x4){0, 0, 0, 0}; *(LAS u32x4*)(lds + 2 * LQ + 1024 + lane * 16) = (u32x4){0, 0, 0, 0};
    const unsigned wbase = (call - 1) * NTW - NCW, cbase = (call - 1) * NTC - NCW;
    static_assert(NL - RING == 32, "probe lanes");
    const unsigned vprobe = lane < 32 ? OFF_EV + lane * 4 : OFF_FIN + (lane - 32) * 4;
    static_assert(NCW <= 7 * step_start(1), "first items");
    if (wv == 0 && lane < NR) st_sys(p.ws[lane] + OFF_EV + (5 * NR + rank) * 4, call);
    {
        int i0, kt0; block_of(wv / 7, i0, kt0);
        const int q0 = i0 < 16 ? 2 * i0 : 2 * x_hi(0) + 3 * (i0 < 16 + CPR ? B0 + CPR * rank + i0 - 16 : i0 - 16 - CPR), peer = (rank + 1 + wv % 7) & 7;
        const unsigned v = lane < NR ? OFF_EV + (5 * NR + lane) * 4 : OFF_AV + peer * 1024 + (q0 + lane % (i0 < 16 ? 2 : 3)) * 4;
        const unsigned long long t0 = __builtin_amdgcn_s_memrealtime();
        F.ok = true;
        for (unsigned n = 1; !__all(ld_sys(ws + v) >= call); ++n) if ((n & 15) == 0 && give_up(ws, call, t0)) { F.ok = false; break; }
        asm volatile("" ::: "memory");
    }
    STAMP(ws, 1);
    unsigned tw = wv, tc = wv;
    auto choose = [&](unsigned probe) -> unsigned {
        const unsigned ev = (unsigned)__ballot(probe >= call);
        if (tw < (unsigned)NTW) {
            const unsigned u = tw / 7;
            int i, kt; block_of(u, i, kt);
            const int stage = u < (unsigned)step_start(STEPS1) ? 1 : u < (unsigned)step_start(STEPS2) ? 2 : 3;
            if ((ev >> (8 * stage + ((rank + 1 + tw - 7 * u) & 7)) & 1) &&
                (i < 16 + RING || __builtin_amdgcn_readlane(probe, 32 + i - 16 - RING) >= 16 * call)) return tw;
        }
        return tc < (unsigned)NTC && (ev >> ((rank + 1 + tc % 7) & 7) & 1) ? NTW + tc : NT;
    };
    auto publish = [&](int flag) {
        if (flag >= 0 && lane == 0) st_sys(ws + OFF_F + flag * 4, call);
    };
    while (tw < (unsigned)NTW || tc < (unsigned)NTC) {
        unsigned t = tw;
        if (tw >= (unsigned)NCW) {
            const unsigned long long t0 = __builtin_amdgcn_s_memrealtime();
            while ((t = choose(ld_sys(ws + vprobe))) == (unsigned)NT) {
                if (give_up(ws, call, t0)) { F.ok = false; t = tw < (unsigned)NTW ? tw : NTW + tc; break; }
                __builtin_amdgcn_s_sleep(16);
            }
        }
        Item cur = make_item(p, t);
        F.slot = 0;
        load_tails(cur, cur.pk, lds, lane);
#pragma unroll
        for (int c = 0; c < 4; ++c) load_main(cur, cur.pk + 4 * (cur.flag < 0 ? TS32 : TS16) + c * 3072, lds + LMAIN + c * 3072, lane);
        WAITVM(0);
        int pflag = -1;
        for (;;) {
            unsigned tk, probe; unsigned long long sv;
            asm volatile("s_nop 3\n\ts_waitcnt vmcnt(25)\n\tglobal_load_dword %1, %6, %5 sc0 sc1\n\t"
                         "s_mov_b64 %2, exec\n\ts_mov_b64 exec, 1\n\tglobal_atomic_add %0, %3, %4, %5 sc0\n\ts_mov_b64 exec, %2"
                         : "=&v"(tk), "=&v"(probe), "=&s"(sv) : "v"(OFF_TICK + (cur.flag < 0 ? 128u : 0u)), "v"(1u), "s"(ws), "v"(vprobe) : "memory");
            F.quad<0, true>(cur, 0, cur, 1);
            asm volatile("s_waitcnt vmcnt(25)" : "+v"(tk), "+v"(probe) :: "memory");
            publish(pflag);
            if (cur.flag >= 0) tw = rfl(tk) - wbase; else tc = rfl(tk) - cbase;
            if (cur.three) F.quad<4, true>(cur, 1, cur, 2);
            t = choose(probe);
            if (t == (unsigned)NT) t = choose(ld_sys(ws + vprobe));
            const Item nxt = make_item(p, t);
            F.quad<8, false>(cur, cur.three ? 2 : 1, nxt, 0);
            pflag = cur.flag;
            if (t >= (unsigned)NT) break;
            cur = nxt;
        }
        WAITVM(0);
        publish(pflag);
    }
    STAMPW(ws, 44 + wid);
    if (arrive(ws, p.ws[lane & 7], rank, 4, NCW, call) && !wait_words(ws, ws + OFF_EV + 4 * NR * 4, NR, 4, call, call) && lane == 0)
        p.out[rank][BCOLS + CPR * ((rank + 1) & 7) * TW] = __builtin_nanf("");
    STAMPW(ws, 48 + wid);
}
#define PT_SET(A0, A1, A2, A3, R) \
    asm volatile("s_waitcnt vmcnt(60)\n\t" R \
                 "global_load_dwordx4 a[" A0 "], %16, %20 sc1 nt\n\tglobal_load_dwordx4 a[" A1 "], %17, %20 sc1 nt\n\t" \
                 "global_load_dwordx4 a[" A2 "], %18, %20 sc1 nt\n\tglobal_load_dwordx4 a[" A3 "], %19, %20 sc1 nt" \
                 : "=&v"(r[0]), "=&v"(r[1]), "=&v"(r[2]), "=&v"(r[3]), "=&v"(r[4]), "=&v"(r[5]), "=&v"(r[6]), "=&v"(r[7]), \
                   "=&v"(r[8]), "=&v"(r[9]), "=&v"(r[10]), "=&v"(r[11]), "=&v"(r[12]), "=&v"(r[13]), "=&v"(r[14]), "=&v"(r[15]) \
                 : "v"(v), "v"(v + 2 * ROWB), "v"(v + 4 * ROWB), "v"(v + 6 * ROWB), "s"(out) : "memory", PT_CLOB)
#define PT_CLOB "a0", "a1", "a2", "a3", "a4", "a5", "a6", "a7", "a8", "a9", "a10", "a11", "a12", "a13", "a14", "a15", "a16", "a17", "a18", "a19", "a20", "a21", "a22", "a23", "a24", "a25", "a26", "a27", "a28", "a29", "a30", "a31", "a32", "a33", "a34", "a35", "a36", "a37", "a38", "a39", "a40", "a41", "a42", "a43", "a44", "a45", "a46", "a47", "a48", "a49", "a50", "a51", "a52", "a53", "a54", "a55", "a56", "a57", "a58", "a59", "a60", "a61", "a62", "a63", "a64", "a65", "a66", "a67", "a68", "a69", "a70", "a71", "a72", "a73", "a74", "a75", "a76", "a77", "a78", "a79", "a80", "a81", "a82", "a83", "a84", "a85", "a86", "a87", "a88", "a89", "a90", "a91", "a92", "a93", "a94", "a95", "a96", "a97", "a98", "a99", "a100", "a101", "a102", "a103", "a104", "a105", "a106", "a107", "a108", "a109", "a110", "a111", "a112", "a113", "a114", "a115", "a116", "a117", "a118", "a119", "a120", "a121", "a122", "a123", "a124", "a125", "a126", "a127"
#define PT_R0 "v_accvgpr_read_b32 %0, a0\n\t" "v_accvgpr_read_b32 %1, a1\n\t" "v_accvgpr_read_b32 %2, a2\n\t" "v_accvgpr_read_b32 %3, a3\n\t" "v_accvgpr_read_b32 %4, a4\n\t" "v_accvgpr_read_b32 %5, a5\n\t" "v_accvgpr_read_b32 %6, a6\n\t" "v_accvgpr_read_b32 %7, a7\n\t" "v_accvgpr_read_b32 %8, a8\n\t" "v_accvgpr_read_b32 %9, a9\n\t" "v_accvgpr_read_b32 %10, a10\n\t" "v_accvgpr_read_b32 %11, a11\n\t" "v_accvgpr_read_b32 %12, a12\n\t" "v_accvgpr_read_b32 %13, a13\n\t" "v_accvgpr_read_b32 %14, a14\n\t" "v_accvgpr_read_b32 %15, a15\n\t"
#define PT_R1 "v_accvgpr_read_b32 %0, a16\n\t" "v_accvgpr_read_b32 %1, a17\n\t" "v_accvgpr_read_b32 %2, a18\n\t" "v_accvgpr_read_b32 %3, a19\n\t" "v_accvgpr_read_b32 %4, a20\n\t" "v_accvgpr_read_b32 %5, a21\n\t" "v_accvgpr_read_b32 %6, a22\n\t" "v_accvgpr_read_b32 %7, a23\n\t" "v_accvgpr_read_b32 %8, a24\n\t" "v_accvgpr_read_b32 %9, a25\n\t" "v_accvgpr_read_b32 %10, a26\n\t" "v_accvgpr_read_b32 %11, a27\n\t" "v_accvgpr_read_b32 %12, a28\n\t" "v_accvgpr_read_b32 %13, a29\n\t" "v_accvgpr_read_b32 %14, a30\n\t" "v_accvgpr_read_b32 %15, a31\n\t"
#define PT_R2 "v_accvgpr_read_b32 %0, a32\n\t" "v_accvgpr_read_b32 %1, a33\n\t" "v_accvgpr_read_b32 %2, a34\n\t" "v_accvgpr_read_b32 %3, a35\n\t" "v_accvgpr_read_b32 %4, a36\n\t" "v_accvgpr_read_b32 %5, a37\n\t" "v_accvgpr_read_b32 %6, a38\n\t" "v_accvgpr_read_b32 %7, a39\n\t" "v_accvgpr_read_b32 %8, a40\n\t" "v_accvgpr_read_b32 %9, a41\n\t" "v_accvgpr_read_b32 %10, a42\n\t" "v_accvgpr_read_b32 %11, a43\n\t" "v_accvgpr_read_b32 %12, a44\n\t" "v_accvgpr_read_b32 %13, a45\n\t" "v_accvgpr_read_b32 %14, a46\n\t" "v_accvgpr_read_b32 %15, a47\n\t"
#define PT_R3 "v_accvgpr_read_b32 %0, a48\n\t" "v_accvgpr_read_b32 %1, a49\n\t" "v_accvgpr_read_b32 %2, a50\n\t" "v_accvgpr_read_b32 %3, a51\n\t" "v_accvgpr_read_b32 %4, a52\n\t" "v_accvgpr_read_b32 %5, a53\n\t" "v_accvgpr_read_b32 %6, a54\n\t" "v_accvgpr_read_b32 %7, a55\n\t" "v_accvgpr_read_b32 %8, a56\n\t" "v_accvgpr_read_b32 %9, a57\n\t" "v_accvgpr_read_b32 %10, a58\n\t" "v_accvgpr_read_b32 %11, a59\n\t" "v_accvgpr_read_b32 %12, a60\n\t" "v_accvgpr_read_b32 %13, a61\n\t" "v_accvgpr_read_b32 %14, a62\n\t" "v_accvgpr_read_b32 %15, a63\n\t"
#define PT_R4 "v_accvgpr_read_b32 %0, a64\n\t" "v_accvgpr_read_b32 %1, a65\n\t" "v_accvgpr_read_b32 %2, a66\n\t" "v_accvgpr_read_b32 %3, a67\n\t" "v_accvgpr_read_b32 %4, a68\n\t" "v_accvgpr_read_b32 %5, a69\n\t" "v_accvgpr_read_b32 %6, a70\n\t" "v_accvgpr_read_b32 %7, a71\n\t" "v_accvgpr_read_b32 %8, a72\n\t" "v_accvgpr_read_b32 %9, a73\n\t" "v_accvgpr_read_b32 %10, a74\n\t" "v_accvgpr_read_b32 %11, a75\n\t" "v_accvgpr_read_b32 %12, a76\n\t" "v_accvgpr_read_b32 %13, a77\n\t" "v_accvgpr_read_b32 %14, a78\n\t" "v_accvgpr_read_b32 %15, a79\n\t"
#define PT_R5 "v_accvgpr_read_b32 %0, a80\n\t" "v_accvgpr_read_b32 %1, a81\n\t" "v_accvgpr_read_b32 %2, a82\n\t" "v_accvgpr_read_b32 %3, a83\n\t" "v_accvgpr_read_b32 %4, a84\n\t" "v_accvgpr_read_b32 %5, a85\n\t" "v_accvgpr_read_b32 %6, a86\n\t" "v_accvgpr_read_b32 %7, a87\n\t" "v_accvgpr_read_b32 %8, a88\n\t" "v_accvgpr_read_b32 %9, a89\n\t" "v_accvgpr_read_b32 %10, a90\n\t" "v_accvgpr_read_b32 %11, a91\n\t" "v_accvgpr_read_b32 %12, a92\n\t" "v_accvgpr_read_b32 %13, a93\n\t" "v_accvgpr_read_b32 %14, a94\n\t" "v_accvgpr_read_b32 %15, a95\n\t"
#define PT_R6 "v_accvgpr_read_b32 %0, a96\n\t" "v_accvgpr_read_b32 %1, a97\n\t" "v_accvgpr_read_b32 %2, a98\n\t" "v_accvgpr_read_b32 %3, a99\n\t" "v_accvgpr_read_b32 %4, a100\n\t" "v_accvgpr_read_b32 %5, a101\n\t" "v_accvgpr_read_b32 %6, a102\n\t" "v_accvgpr_read_b32 %7, a103\n\t" "v_accvgpr_read_b32 %8, a104\n\t" "v_accvgpr_read_b32 %9, a105\n\t" "v_accvgpr_read_b32 %10, a106\n\t" "v_accvgpr_read_b32 %11, a107\n\t" "v_accvgpr_read_b32 %12, a108\n\t" "v_accvgpr_read_b32 %13, a109\n\t" "v_accvgpr_read_b32 %14, a110\n\t" "v_accvgpr_read_b32 %15, a111\n\t"
#define PT_R7 "v_accvgpr_read_b32 %0, a112\n\t" "v_accvgpr_read_b32 %1, a113\n\t" "v_accvgpr_read_b32 %2, a114\n\t" "v_accvgpr_read_b32 %3, a115\n\t" "v_accvgpr_read_b32 %4, a116\n\t" "v_accvgpr_read_b32 %5, a117\n\t" "v_accvgpr_read_b32 %6, a118\n\t" "v_accvgpr_read_b32 %7, a119\n\t" "v_accvgpr_read_b32 %8, a120\n\t" "v_accvgpr_read_b32 %9, a121\n\t" "v_accvgpr_read_b32 %10, a122\n\t" "v_accvgpr_read_b32 %11, a123\n\t" "v_accvgpr_read_b32 %12, a124\n\t" "v_accvgpr_read_b32 %13, a125\n\t" "v_accvgpr_read_b32 %14, a126\n\t" "v_accvgpr_read_b32 %15, a127\n\t"
DI void pack_tile(Params& p, LAS unsigned char* tail, unsigned call, int ct) {
    const int lane = lane_now(), rank = p.rank;
    char* const ws = p.ws[rank];
    const char* const out = (const char*)p.out[rank];
    const bool ok = wait_words(ws, ws + OFF_AVC + ct * 4, 1, 4, call, call);
    __builtin_amdgcn_fence(__ATOMIC_ACQUIRE, "agent");
    const unsigned s0 = 8 * ct, vb = slab_out(s0, rank) + (lane >> 5) * ROWB + (lane & 31) * 16;
    for (int i = -8; i < 96; ++i) {
        const unsigned n = i + 8 < 96 ? i + 8 : 95, v = vb + n / 3 * (8 * ROWB) + n % 3 * 512;
        unsigned r[16];
        if (i >= 0 && i < 8) WAITVM(28);
        switch (i & 7) {
            case 0: PT_SET("0:3", "4:7", "8:11", "12:15", PT_R0); break;
            case 1: PT_SET("16:19", "20:23", "24:27", "28:31", PT_R1); break;
            case 2: PT_SET("32:35", "36:39", "40:43", "44:47", PT_R2); break;
            case 3: PT_SET("48:51", "52:55", "56:59", "60:63", PT_R3); break;
            case 4: PT_SET("64:67", "68:71", "72:75", "76:79", PT_R4); break;
            case 5: PT_SET("80:83", "84:87", "88:91", "92:95", PT_R5); break;
            case 6: PT_SET("96:99", "100:103", "104:107", "108:111", PT_R6); break;
            default: PT_SET("112:115", "116:119", "120:123", "124:127", PT_R7); break;
        }
        if (i < 0) continue;
        if (!ok) r[0] = NANS;
        const u32x4 f[4] = {{r[0], r[1], r[2], r[3]}, {r[4], r[5], r[6], r[7]}, {r[8], r[9], r[10], r[11]}, {r[12], r[13], r[14], r[15]}};
        if (!pack32(f, ws, OFF_PK32 + (s0 * 3 + (i >> 2)) * QB32, i & 3, lane, tail)) store4(ws, OFF_RAW32 + s0 * WBLK + i * 4096 + lane * 16, f);
    }
    WAITVM(0);
    arrive(ws, p.ws[lane & 7], rank, 0, 16 * CPR, call);
}

DI f16x8 rdfrag(unsigned base, int off) { return *(const LAS f16x8*)(size_t)(base + off); }
DI void dma_to(unsigned ldsaddr) { asm volatile("s_mov_b32 m0, %0" :: "s"(ldsaddr) : "memory"); }
#define PIN_S(x) asm volatile("" : "+s"(x))
DI float rd_acc(float a) { float v; asm volatile("v_accvgpr_read_b32 %0, %1" : "=v"(v) : "a"(a)); return v; }
DI f32x4 silu4(f32x4 a, float sc) {
    f32x4 t = a * (-1.44269504088896f / WSCALE);
#pragma unroll
    for (int i = 0; i < 4; ++i) t[i] = __builtin_amdgcn_exp2f(__builtin_amdgcn_fmed3f(t[i], -86.5617024533378f, 86.5617024533378f));
    t = __builtin_elementwise_fma(t, (f32x4){sc, sc, sc, sc}, (f32x4){sc, sc, sc, sc});
#pragma unroll
    for (int i = 0; i < 4; ++i) t[i] = __builtin_amdgcn_rcpf(t[i]);
    return a * t;
}
struct G7 {
    unsigned lds;
    __amdgpu_buffer_rsrc_t rD;
    unsigned voff;
    unsigned ard, an;
    unsigned blo, bhi;
    unsigned sA, sB, sr, sw;
    unsigned ringB, nextA, nextB;
    f32x4 acc[8][12];
    f16x8 L[2][4][2], H[4][2], B0, B1;
};
template <int I> DI void dma7(const G7& g, unsigned src) {
    asm volatile("buffer_load_dwordx4 %0, %1, %2 offen offset:%3 sc0 sc1 lds" :: "v"(g.voff), "s"(g.rD), "s"(src), "i"(I * 1024) : "memory");
}
template <int SL> DI f16x8 rdB7(const G7& g, int j, int k) { return rdfrag(SL < 3 ? g.blo : g.bhi, (SL % 3) * QUAD + k * 4096 + j * 1024); }
DI unsigned next3(unsigned s) { return s == 2 * QUAD ? 0 : s + QUAD; }
template <int J, bool ZERO, int VM> DI void phase(G7& g, int it) {
    constexpr int t = J % 3, c = J / 3, NX = (J + 1) % 6, PV = (J + 5) % 6;
    f16x8 B[5][2];
    B[0][0] = g.B0; B[0][1] = g.B1;
#pragma unroll
    for (int j = 0; j < 4; ++j) {
        const int n = 4 * t + j;
        if (j == 3) { __builtin_amdgcn_s_waitcnt(WCNT(VM, 0)); __builtin_amdgcn_s_barrier(); SCHED; }
#pragma unroll
        for (int u = 0; u < 16; ++u) {
            const int q = j * 16 + u, r = u >> 1, m = (n & 1) ? 7 - r : r, k = (u & 1) ^ (r & 1);
            const f16x8 a = m < 4 ? g.L[c][m][k] : g.H[m - 4][k];
            if (n < 8) {
                if (ZERO && !(u & 1)) asm volatile("v_mfma_f32_16x16x32_f16 %0, %1, %2, 0" : "=a"(g.acc[m][n]) : "v"(a), "v"(B[j][k]));
                else asm volatile("v_mfma_f32_16x16x32_f16 %0, %1, %2, %0" : "+a"(g.acc[m][n]) : "v"(a), "v"(B[j][k]));
            } else {
                if (ZERO && !(u & 1)) asm volatile("v_mfma_f32_16x16x32_f16 %0, %1, %2, 0" : "=&v"(g.acc[m][n]) : "v"(a), "v"(B[j][k]));
                else asm volatile("v_mfma_f32_16x16x32_f16 %0, %1, %2, %0" : "+v"(g.acc[m][n]) : "v"(a), "v"(B[j][k]));
            }
            SCHED;
            const bool lastcol = t == 2 && j == 3;
            if (u == (lastcol ? 9 : 1)) { B[j + 1][0] = j < 3 ? rdB7<J>(g, j + 1, 0) : rdB7<NX>(g, 0, 0); SCHED; }
            if (u == (lastcol ? 10 : 3)) { B[j + 1][1] = j < 3 ? rdB7<J>(g, j + 1, 1) : rdB7<NX>(g, 0, 1); SCHED; }
            if (t == 2 && j < 3 && (u == 5 || u == 9 || u == 13) && j * 3 + (u - 5) / 4 < 8) { const int i = j * 3 + (u - 5) / 4; g.L[c ^ 1][i >> 1][i & 1] = rdfrag(g.an, (i & 1) * 4096 + (i >> 1) * 1024); SCHED; }
            if (lastcol && u >= 1 && u <= 8) { const int row = 3 - ((u - 1) >> 1), kk = (u - 1) & 1; g.H[row][kk] = rdfrag(g.an, kk * 4096 + row * 1024); SCHED; }
            if ((t == 2 && q == 44) || (lastcol && u == 12)) { PIN_S(g.sr); g.sr = next3(g.sr); PIN_S(g.sr); SCHED; }
            if ((t == 2 && q == 46) || (lastcol && u == 14)) { g.an = g.ard + g.sr; asm volatile("" : "+v"(g.an)); SCHED; }
            if (t < 2) {
                if (q == 4) { PIN_S(g.sw); dma_to(g.lds + LDS_A + g.sw); SCHED; }
                if (q == 5) { dma7<0>(g, g.sA); SCHED; }
                if (q == 10) { dma7<1>(g, g.sA); SCHED; }
                if (q == 15) { dma7<2>(g, g.sA); SCHED; }
                if (q == 21) { dma7<3>(g, g.sA); SCHED; }
                if (q == 52) { PIN_S(g.sA); g.sA += QUAD; PIN_S(g.sA); SCHED; }
                if (q == 54 && J == 0) { g.sA = it == 31 ? g.nextA : g.sA; PIN_S(g.sA); SCHED; }
                if (q == 56) { PIN_S(g.sw); g.sw = next3(g.sw); PIN_S(g.sw); SCHED; }
            }
            constexpr int PB0[4] = {27, 32, 38, 43}, PB2[4] = {7, 16, 27, 39};
            if (q == (t < 2 ? PB0[0] : PB2[0]) - 1) { dma_to(g.lds + PV * QUAD); SCHED; }
            if (q == (t < 2 ? PB0[0] : PB2[0])) { dma7<0>(g, g.sB); SCHED; }
            if (q == (t < 2 ? PB0[1] : PB2[1])) { dma7<1>(g, g.sB); SCHED; }
            if (q == (t < 2 ? PB0[2] : PB2[2])) { dma7<2>(g, g.sB); SCHED; }
            if (q == (t < 2 ? PB0[3] : PB2[3])) { dma7<3>(g, g.sB); SCHED; }
            if (q == (t < 2 ? 58 : 42)) { PIN_S(g.sB); g.sB += QUAD; PIN_S(g.sB); SCHED; }
            if (q == 60 && J == 0) { g.sB = it == 3 ? g.ringB : g.sB; PIN_S(g.sB); SCHED; }
            if (q == 62 && J == 0) { g.sB = it == 31 ? g.nextB : g.sB; PIN_S(g.sB); SCHED; }
        }
    }
    g.B0 = B[4][0]; g.B1 = B[4][1];
}
template <bool FIRST> DI void trip(G7& g, int it) {
    SCHED;
    phase<0, FIRST, FIRST ? 60 : 28>(g, it); phase<1, FIRST, FIRST ? 60 : 24>(g, it); phase<2, FIRST, 16>(g, it);
    phase<3, false, 28>(g, it); phase<4, false, 24>(g, it); phase<5, false, 16>(g, it);
}
struct Short { unsigned sb, bn, bx; };
DI unsigned next6(unsigned s) { return s == 5 * QUAD ? 0 : s + QUAD; }
template <int J, bool ZERO, int VM> DI void phase2(G7& g, Short& h, int it) {
    constexpr int t = J & 1, c = J >> 1;
    f16x8 B[5][2];
    B[0][0] = g.B0; B[0][1] = g.B1;
    unsigned pv = h.sb == 0 ? 5 * QUAD : h.sb - QUAD;
#pragma unroll
    for (int j = 0; j < 4; ++j) {
        const int n = 4 * t + j;
        if (j == 3) { __builtin_amdgcn_s_waitcnt(WCNT(VM, 0)); __builtin_amdgcn_s_barrier(); SCHED; }
#pragma unroll
        for (int u = 0; u < 16; ++u) {
            const int q = j * 16 + u, r = u >> 1, m = (n & 1) ? 7 - r : r, k = (u & 1) ^ (r & 1);
            const f16x8 a = m < 4 ? g.L[c][m][k] : g.H[m - 4][k];
            if (ZERO && !(u & 1)) asm volatile("v_mfma_f32_16x16x32_f16 %0, %1, %2, 0" : "=a"(g.acc[m][n]) : "v"(a), "v"(B[j][k]));
            else asm volatile("v_mfma_f32_16x16x32_f16 %0, %1, %2, %0" : "+a"(g.acc[m][n]) : "v"(a), "v"(B[j][k]));
            SCHED;
            const bool lastcol = t == 1 && j == 3;
            if (u == (lastcol ? 9 : 1)) { B[j + 1][0] = j < 3 ? rdfrag(h.bn, (j + 1) * 1024) : rdfrag(h.bx, 0); SCHED; }
            if (u == (lastcol ? 10 : 3)) { B[j + 1][1] = j < 3 ? rdfrag(h.bn, 4096 + (j + 1) * 1024) : rdfrag(h.bx, 4096); SCHED; }
            if (t == 1 && j < 3 && (u == 5 || u == 9 || u == 13) && j * 3 + (u - 5) / 4 < 8) { const int i = j * 3 + (u - 5) / 4; g.L[c ^ 1][i >> 1][i & 1] = rdfrag(g.an, (i & 1) * 4096 + (i >> 1) * 1024); SCHED; }
            if (lastcol && u >= 1 && u <= 8) { const int row = 3 - ((u - 1) >> 1), kk = (u - 1) & 1; g.H[row][kk] = rdfrag(g.an, kk * 4096 + row * 1024); SCHED; }
            if ((t == 1 && q == 44) || (lastcol && u == 12)) { PIN_S(g.sr); g.sr = next3(g.sr); PIN_S(g.sr); SCHED; }
            if ((t == 1 && q == 46) || (lastcol && u == 14)) { g.an = g.ard + g.sr; asm volatile("" : "+v"(g.an)); SCHED; }
            if (q == 0) { PIN_S(g.sw); dma_to(g.lds + LDS_A + g.sw); SCHED; }
            if (q == 2) { dma7<0>(g, g.sA); SCHED; }
            if (q == 6) { dma7<1>(g, g.sA); SCHED; }
            if (q == 10) { dma7<2>(g, g.sA); SCHED; }
            if (q == 14) { dma7<3>(g, g.sA); SCHED; }
            if (q == 16) { PIN_S(g.sA); g.sA += QUAD; PIN_S(g.sA); SCHED; }
            if (q == 18) { PIN_S(g.sw); g.sw = next3(g.sw); PIN_S(g.sw); SCHED; }
            if (q == 22) { PIN_S(pv); dma_to(g.lds + pv); SCHED; }
            if (q == 23) { dma7<0>(g, g.sB); SCHED; }
            if (q == 30) { dma7<1>(g, g.sB); SCHED; }
            if (q == 36) { dma7<2>(g, g.sB); SCHED; }
            if (q == 42) { dma7<3>(g, g.sB); SCHED; }
            if (q == 43) { PIN_S(g.sB); g.sB += QUAD; PIN_S(g.sB); SCHED; }
            if (q == 45 && J == 2) { g.sB = it == 2 ? g.ringB : g.sB; PIN_S(g.sB); SCHED; }
            if (q == 59) { PIN_S(h.sb); h.sb = next6(h.sb); PIN_S(h.sb); SCHED; }
            if (q == 63) { h.bn = h.bx; h.bx = g.blo + next6(h.sb); asm volatile("" : "+v"(h.bx)); SCHED; }
        }
    }
    g.B0 = B[4][0]; g.B1 = B[4][1];
}
DI void short_tile(G7& g) {
    Short h;
    h.sb = 0; h.bn = g.blo; h.bx = g.blo + QUAD;
    SCHED;
    phase2<0, true, 60>(g, h, 0); phase2<1, true, 12>(g, h, 0); phase2<2, false, 12>(g, h, 0); phase2<3, false, 12>(g, h, 0);
    for (int it = 1; it < 32; ++it) { SCHED; phase2<0, false, 12>(g, h, it); phase2<1, false, 12>(g, h, it); phase2<2, false, 12>(g, h, it); phase2<3, false, 12>(g, h, it); }
}
DI void g7_init(G7& g, LAS unsigned char* lds, const char* data, unsigned bytes, int wid) {
    const int lane = lane_now(), wr = wid >> 1, wc = wid & 1;
    const unsigned lbase = (unsigned)(size_t)lds;
    g.rD = __builtin_amdgcn_make_buffer_rsrc((void*)data, 0, bytes, 0x00020000);
    g.lds = lbase + wid * 4096; g.voff = wid * 4096 + lane * 16;
    g.ard = lbase + LDS_A + wr * 8192 + lane * 16; g.blo = lbase + wc * 8192 + lane * 16; g.bhi = g.blo + 3 * QUAD;
    asm volatile("" : "+v"(g.ard), "+v"(g.blo), "+v"(g.bhi), "+v"(g.voff));
}
DI void g7_fill(G7& g, unsigned to, unsigned src) { dma_to(g.lds + to); asm volatile("s_nop 4"); dma7<0>(g, src); dma7<1>(g, src); dma7<2>(g, src); dma7<3>(g, src); }
DI void g7_prologue(G7& g, unsigned dA, unsigned dB) {
#pragma unroll
    for (int i = 0; i < 5; ++i) { if (i < 3) g7_fill(g, LDS_A + i * QUAD, dA + i * QUAD); g7_fill(g, i * QUAD, dB + i * QUAD); }
    WAITVM(0); __builtin_amdgcn_s_barrier(); SCHED;
#pragma unroll
    for (int i = 0; i < 8; ++i) { g.L[0][i >> 1][i & 1] = rdfrag(g.ard, (i & 1) * 4096 + (i >> 1) * 1024); g.H[i >> 1][i & 1] = rdfrag(g.ard, QUAD + (i & 1) * 4096 + (i >> 1) * 1024); }
    g.B0 = rdB7<0>(g, 0, 0); g.B1 = rdB7<0>(g, 0, 1);
    __builtin_amdgcn_s_waitcnt(WCNT(63, 0)); __builtin_amdgcn_s_barrier(); SCHED;
    g.sA = dA + 3 * QUAD; g.sB = dB + 5 * QUAD; g.sr = 2 * QUAD; g.sw = 0; g.an = g.ard + 2 * QUAD;
}
#define EBITS "nt"
template <int NQ> DI void g7_epilogue(G7& g, __amdgpu_buffer_rsrc_t rO, unsigned so, unsigned rowb, float one) {
    const int lz = lane_now();
    const unsigned vo = 4 * (lz >> 4) * rowb + 16 * (lz & 15);
    asm volatile("s_nop 15");
#pragma unroll
    for (int m = 0; m < 8; ++m) {
#pragma unroll
        for (int r = 0; r < 4; ++r) {
#pragma unroll
            for (int q = 0; q < NQ; ++q) {
                f32x4 y;
                if (q < 2) y = (f32x4){rd_acc(g.acc[m][4 * q][r]), rd_acc(g.acc[m][4 * q + 1][r]), rd_acc(g.acc[m][4 * q + 2][r]), rd_acc(g.acc[m][4 * q + 3][r])};
                else y = (f32x4){g.acc[m][8][r], g.acc[m][9][r], g.acc[m][10][r], g.acc[m][11][r]};
                const f32x4 v = silu4(y, one);
                asm volatile("buffer_store_dwordx4 %0, %1, %2, %3 offen offset:%4 " EBITS "\n\ts_nop 1" :: "v"(v), "v"(vo), "s"(rO), "s"(so + r * rowb), "i"(q * 256) : "memory");
            }
            SCHED;
        }
        so += 16 * rowb;
    }
}
DI int lead_ones(bool ok) { const unsigned long long m = ~__ballot(ok); return m ? __builtin_ctzll(m) : 64; }

struct Ready {
    char* ws; unsigned call;
    __amdgpu_buffer_rsrc_t rF; LAS unsigned char* zone;
    int pa, pb, n; bool pending; float one;
    DI void poll(int need) {
        const unsigned long long t0 = __builtin_amdgcn_s_memrealtime();
        const int lane = lane_now();
        for (;;) {
            const unsigned a = ld_sys(ws + OFF_F + pa * 256 + lane * 4), b = ld_sys(ws + OFF_F + pb * 256 + lane * 4);
            n = lead_ones(lane < 8 || (a >= call && b >= call));
            if (n >= need) break;
            if (give_up(ws, call, t0)) { n = 64; one = __builtin_nanf(""); break; }
            __builtin_amdgcn_s_sleep(16);
        }
    }
    DI void request() {
        const int lane = lane_now();
        __builtin_amdgcn_raw_ptr_buffer_load_lds(rF, (LAS void*)zone, 4, lane * 4, pa * 256, 0, AUX_SYS);
        __builtin_amdgcn_raw_ptr_buffer_load_lds(rF, (LAS void*)(zone + 256), 4, lane * 4, pb * 256, 0, AUX_SYS);
        pending = true;
    }
    DI int plan(int it, int stop) {
        if (n == 64) return stop;
        const int safe = (n - 4) >> 1;
        if (safe <= it + 1) { request(); return it + 1; }
        return safe - 1 < stop ? safe - 1 : stop;
    }
    DI void check(int need) {
        if (pending) {
            const int lane = lane_now();
            const unsigned a = *(const volatile LAS unsigned*)(zone + lane * 4), b = *(const volatile LAS unsigned*)(zone + 256 + lane * 4);
            n = lead_ones(lane < 8 || (a >= call && b >= call)); pending = false;
        }
        if (n < need) poll(need);
        asm volatile("" ::: "memory");
    }
};

DI void gemm_wg(Params& p, LAS unsigned char* lds, int wid, int band, int S) {
    const int wr = wid >> 1, wc = wid & 1, rank = p.rank;
    char* const ws = p.ws[rank];
    const unsigned call = convert_shards(p, lds, wid, S * (16 / RB) + band);
    int T = S;
    STAMP(ws, 2);
    G7 g;
    g7_init(g, lds, ws + OFF_DATA, DATAB, wid);
    Ready R;
    R.ws = ws; R.call = call; R.rF = __builtin_amdgcn_make_buffer_rsrc((void*)(ws + OFF_F), 0, NPAN * 256, 0x00020000);
    R.zone = lds + LDS_ZONE + wid * 512; R.pending = false; R.one = WSCALE;
    if (!wait_words(ws, ws + OFF_EV + (3 * NR + rank) * 4, 1, 4, call, call)) R.one = __builtin_nanf("");
    R.pa = RB * band + (T & (RB - 1)); R.pb = 16 + colblock(T >> RBL, rank); R.n = 8;
    STAMP(ws, 1);
    g7_prologue(g, rfl(own_off(R.pa, 0)), rfl(own_off(R.pb, 0)));
    const __amdgpu_buffer_rsrc_t rO = __builtin_amdgcn_make_buffer_rsrc((void*)p.out[rank], 0, OUTB, 0x00020000);
    unsigned tprev = 0;
    int li = T >> RBL;
    for (int ti = 0;; ++ti) {
        const int pm = R.pa, cb = R.pb - 16;
        S += NWB;
        const int job = S - PSLOT;
        if ((unsigned)job < (unsigned)NJOB) S += NWB;
        const bool has_next = S < NROUND * NWB, deal = has_next && S >= (NROUND - 1) * NWB;
        T = S < PSLOT ? S : S - NJOB;
        int qa = pm, ql = li;
        if (has_next && !deal) { qa = RB * band + (T & (RB - 1)); ql = T >> RBL; }
        g.ringB = rfl(ring_off(li)); g.nextA = rfl(own_off(qa, 0)); g.nextB = rfl(own_off(16 + colblock(ql, rank), 0));
        {
            const unsigned now = (unsigned)__builtin_amdgcn_s_memrealtime();
            if (deal && blockIdx.x >> 3 == NG / 2 && wid == 0 && lane_now() == 0)
                __hip_atomic_store((GAS unsigned long long*)(ws + OFF_ETA + (blockIdx.x & 7) * 8), (unsigned long long)call << 32 | (3 * now - 2 * tprev), __ATOMIC_RELAXED, __HIP_MEMORY_SCOPE_AGENT);
            tprev = now;
        }
        R.check(4);
        int lim = R.plan(0, deal ? 29 : 32);
        trip<true>(g, 0);
        for (int it = 1; it < 32;) {
            if (it >= lim) {
                R.check(it < 30 ? 2 * it + 4 : 64);
                lim = R.plan(it, deal && it < 29 ? 29 : 32);
                if (deal && it == 29) {
                    const int lane = lane_now();
                    const unsigned long long t0 = __builtin_amdgcn_s_memrealtime();
                    unsigned long long e;
                    int place = blockIdx.x & 7;
                    while (!__all((unsigned)((e = __hip_atomic_load((GAS unsigned long long*)(ws + OFF_ETA + (lane & 7) * 8), __ATOMIC_RELAXED, __HIP_MEMORY_SCOPE_AGENT)) >> 32) == call)) {
                        if (give_up(ws, call, t0)) { R.one = __builtin_nanf(""); e = 0; break; }
                        __builtin_amdgcn_s_sleep(8);
                    }
                    const int d = (int)((unsigned)e - __builtin_amdgcn_readlane((unsigned)e, place));
                    place = __builtin_popcount((unsigned)__ballot(d < 0 || (d == 0 && lane < place)) & 0xffu);
                    constexpr int T4 = (NROUND - 1) * NWB - NJOB, NF = RB * (NL - 2) - T4;
                    const int P = place * NG + (blockIdx.x >> 3), bd = P / NF;
                    if (bd < 16 / RB) { const int t = T4 + P - bd * NF; qa = RB * bd + (t & (RB - 1)); ql = t >> RBL; }
                    else { const int t = P - 16 / RB * NF; qa = t & 15; ql = NL - 2 + (t >> 4); }
                    g.nextA = rfl(own_off(qa, 0)); g.nextB = rfl(own_off(16 + colblock(ql, rank), 0));
                }
            }
            do trip<false>(g, it); while (++it < lim);
        }
        STAMP(ws, 8 + 2 * ti);
        R.pa = qa; R.pb = 16 + colblock(ql, rank);
        R.request();
        g7_epilogue<3>(g, rO, rfl((pm * 256 + wr * 128) * ROWB + (col0(cb) + wc * 192) * 4), ROWB, R.one);
        WAITVM(60);
        if (wid == 0 && lane_now() == 0) __hip_atomic_fetch_add((gu32*)(ws + OFF_FIN + li * 4), 1u, __ATOMIC_RELAXED, __HIP_MEMORY_SCOPE_AGENT);
        if (li < CPR) {
            WAITVM(0);
            BARRIER;
            STAMP(ws, 4);
            if (wid == 0) {
                __builtin_amdgcn_fence(__ATOMIC_RELEASE, "agent");
                WAITVM(0);
                if (lane_now() == 0) st_sys(ws + OFF_AVC + (li * 16 + pm) * 4, call);
            }
        }
        if ((unsigned)job < (unsigned)NJOB) { pack_tile(p, lds + LDS_ZONE + 4096 + wid * 1024, call, job % CPR * 16 + RB * band + job / CPR * 4 + wid); STAMP(ws, 5); }
        STAMP(ws, 9 + 2 * ti);
        if (!has_next) break;
        li = ql;
        if (li >= NL - 2) {
            g.ringB = rfl(ring_off(li));
            R.check(64);
            short_tile(g);
            STAMP(ws, 10 + 2 * ti);
            g7_epilogue<2>(g, rO, rfl((R.pa * 256 + wr * 128) * ROWB + (col0(R.pb - 16) + wc * 128) * 4), ROWB, R.one);
            STAMP(ws, 11 + 2 * ti);
            break;
        }
    }
    __builtin_amdgcn_s_waitcnt(WCNT(0, 0));
}

__global__ __launch_bounds__(256, 1) void fused(Args) {
    extern __shared__ __attribute__((aligned(16))) unsigned char shm[];
    Params& p = *(Params*)__builtin_amdgcn_kernarg_segment_ptr();
    LAS unsigned char* lds = (LAS unsigned char*)shm;
    char* const ws = p.ws[p.rank];
    const int wid = rfl(threadIdx.x >> 6);
    STAMP(ws, 0);
    STAMPV(ws, 3, __builtin_amdgcn_s_getreg(20 | (0 << 6) | (3 << 11)));
    const int grp = blockIdx.x & 7, idx = blockIdx.x >> 3;
    if (idx < NG) gemm_wg(p, lds, wid, (grp & 1 ? 0 : 8 / RB) + (grp >> RBL), (grp >> 1 & (RB / 2 - 1)) * NG + idx);
    else {
        const int cw = (idx - NG) * 8 + grp;
        const unsigned call = draw_call(ws, wid);
        fetch_wave(p, lds + wid * 32768, call, wid, cw * 4 + wid);
    }
}
}

extern "C" void kernel_launch(int rank, int nranks, ncclComm_t comm, hipStream_t stream,
                              void* const* d_in, const int* in_sizes, int n_in,
                              void* d_out, int out_size,
                              const int* in_dtypes, int out_dtype,
                              void* const* d_out_all,
                              void* const* d_ws_all, size_t ws_bytes) {
    if (nranks != NR || (unsigned)rank >= (unsigned)NR || n_in != 2 || ws_bytes < WS_NEED || in_sizes[0] != MM * KS || in_sizes[1] != KS * NN || out_size != MM * NN) {
        fprintf(stderr, "kernel_launch: unexpected problem shape\n");
        return;
    }
    static bool attr_set[NR];
    if (!attr_set[rank]) {
        HIP_CHECK(hipFuncSetAttribute((const void*)fused, hipFuncAttributeMaxDynamicSharedMemorySize, LDS_BYTES));
        attr_set[rank] = true;
    }
    Args p;
    p.x = (const float*)d_in[0]; p.w = (const float*)d_in[1];
    for (int i = 0; i < NR; ++i) { p.ws[i] = (char*)d_ws_all[i]; p.out[i] = (float*)d_out_all[i]; }
    p.rank = rank; p.pad = 0;
    hipLaunchKernelGGL(fused, dim3(256), dim3(256), LDS_BYTES, stream, p);
    HIP_CHECK(hipGetLastError());
}
```

```cpp
#include <hip/hip_runtime.h>
#include <rccl/rccl.h>
#include <cstdio>

#define HIP_CHECK(expr) do { hipError_t _e = (expr); if (_e != hipSuccess) { \
    fprintf(stderr, "HIP error %s at %s:%d\n", hipGetErrorString(_e), __FILE__, __LINE__); \
    return; } } while (0)

#define LAS __attribute__((address_space(3)))
#define GAS __attribute__((address_space(1)))
#define DI __device__ __forceinline__
typedef _Float16 f16x8 __attribute__((ext_vector_type(8)));
typedef float f32x4 __attribute__((ext_vector_type(4)));
typedef unsigned u32x4 __attribute__((ext_vector_type(4)));
typedef unsigned u32x2 __attribute__((ext_vector_type(2)));
typedef GAS unsigned gu32;

#define NG 29
#define DBG 0
namespace {
constexpr int MM = 4096, NN = 32768, KS = 512, NR = 8;
constexpr int NGEMM = 8 * NG, NCOMM = 256 - NGEMM, NCW = 4 * NCOMM;
constexpr int TW = 384;
constexpr int CPR = 2, NCB = NR * CPR;
constexpr int BCOLS = NN - NCB * TW, NBB = (BCOLS + TW - 1) / TW;
constexpr int NWP = NBB + NCB, NPAN = 16 + NWP;
constexpr int NL = CPR + NBB;
#define RBL 3
constexpr int RB = 1 << RBL, NWB = NGEMM * RB / 16, NJOB = CPR * RB / 4, NROUND = 5, PSLOT = 2 * NWB - NJOB;
static_assert(NROUND * NWB == RB * NL + NJOB, "slots");
constexpr unsigned QUAD = 16384, XBLK = 2 * QUAD, WBLK = 3 * QUAD, XPANEL = 64 * XBLK;
static_assert(BCOLS - (NBB - 2) * TW == 2 * 256, "short blocks");
constexpr unsigned NE16 = 48, NE32 = 16, TS16 = 512 + 2 * NE16, TS32 = 512 + 2 * NE32, QB16 = 4 * (TS16 + 3072), QB32 = 4 * (TS32 + 3072);
constexpr unsigned OUTB = (unsigned)MM * NN * 4u, ROWB = NN * 4;
constexpr unsigned OFF_CALL = 0;
constexpr unsigned OFF_TMO = 32768;
constexpr unsigned OFF_FIN = 32768 + 1024;
constexpr unsigned OFF_ETA = 32768 + 2048;
constexpr unsigned OFF_TICK = 32768 + 4096;
constexpr unsigned OFF_F = 65536;
constexpr unsigned OFF_AV = 131072;
constexpr unsigned OFF_AVC = OFF_AV + 8192;
constexpr unsigned OFF_CNT = 196608;
constexpr unsigned OFF_EV = OFF_CNT + 4096;
constexpr unsigned OFF_DBG = 262144, OFF_DBG2 = 524288;
constexpr int RING = NL - 32;
constexpr unsigned OFF_DATA = 1u << 20;
constexpr unsigned REL_WL = 16 * XPANEL;
constexpr unsigned REL_WR = REL_WL + NWP * 8 * WBLK;
constexpr unsigned DATAB = REL_WR + RING * 56 * WBLK;
constexpr int NQ16 = 16 * 8 * 2 + (NWP - 2) * 8 * 3 + 32;
constexpr unsigned OFF_PK16 = OFF_DATA + DATAB;
constexpr int NCI = CPR * (MM / 32);
constexpr unsigned OFF_PK32 = OFF_PK16 + (NQ16 + 16) * QB16;
constexpr unsigned OFF_RAW32 = OFF_PK32 + NCI * 3 * QB32;
constexpr unsigned WS_NEED = OFF_RAW32 + NCI * WBLK;
static_assert(NPAN * 256 <= 65536, "flags");
constexpr int LDS_A = 6 * QUAD, LDS_ZONE = LDS_A + 3 * QUAD, LDS_BYTES = LDS_ZONE + 8192;
constexpr unsigned long long DEADLINE = 50000000ull;
#define WSCALE 32.0f
constexpr int AUX_SYS = 17, AUX_NT = 2;

struct Args {
    const float* x; const float* w;
    char* ws[NR]; float* out[NR];
    int rank, pad;
};
typedef const __attribute__((address_space(4))) Args Params;

constexpr int WCNT(int vm, int lgkm) { return (vm & 15) | (7 << 4) | ((lgkm & 15) << 8) | ((vm >> 4) << 14); }
#define SCHED __builtin_amdgcn_sched_barrier(0)
#define WAITVM(n) asm volatile("s_waitcnt vmcnt(" #n ")" ::: "memory")
#define WAITLG0 asm volatile("s_waitcnt lgkmcnt(0)" ::: "memory")
#define BARRIER do { asm volatile("" ::: "memory"); __builtin_amdgcn_s_barrier(); asm volatile("" ::: "memory"); } while (0)

DI unsigned ld_sys(const char* p) { return __hip_atomic_load((gu32*)p, __ATOMIC_RELAXED, __HIP_MEMORY_SCOPE_SYSTEM); }
DI void st_sys(char* p, unsigned v) { __hip_atomic_store((gu32*)p, v, __ATOMIC_RELAXED, __HIP_MEMORY_SCOPE_SYSTEM); }
DI unsigned ld_ag(const char* p) { return __hip_atomic_load((gu32*)p, __ATOMIC_RELAXED, __HIP_MEMORY_SCOPE_AGENT); }
DI void st_ag(char* p, unsigned v) { __hip_atomic_store((gu32*)p, v, __ATOMIC_RELAXED, __HIP_MEMORY_SCOPE_AGENT); }
DI unsigned rfl(unsigned x) { return __builtin_amdgcn_readfirstlane(x); }
DI int lane_now() { int l; asm volatile("v_mbcnt_lo_u32_b32 %0, -1, 0\n\tv_mbcnt_hi_u32_b32 %0, -1, %0" : "=v"(l)); return l; }
DI bool give_up(char* ws, unsigned call, unsigned long long t0) {
    if (rfl(ld_ag(ws + OFF_TMO)) == call || __builtin_amdgcn_s_memrealtime() - t0 > DEADLINE) { st_ag(ws + OFF_TMO, call); return true; }
    return false;
}
DI bool arrive(char* ws, char* lane_ws, int rank, int kind, unsigned n, unsigned call) {
    const int lane = lane_now();
    unsigned t = 0;
    if (lane == 0) t = __hip_atomic_fetch_add((gu32*)(ws + OFF_CNT + kind * 128), 1u, __ATOMIC_RELAXED, __HIP_MEMORY_SCOPE_AGENT);
    const bool last = rfl(t) + 1 == n * call;
    if (last && lane < NR) st_sys(lane_ws + OFF_EV + (kind * NR + rank) * 4, call);
    return last;
}
DI bool wait_words(char* ws, const char* f, int n, int step, unsigned want, unsigned call) {
    const int lane = lane_now();
    const unsigned long long t0 = __builtin_amdgcn_s_memrealtime();
    for (;;) {
        bool ok = true;
#pragma unroll
        for (int i = 0; i < 4; ++i) { const int k = lane + 64 * i; if (64 * i < n) ok &= ld_sys(f + (k < n ? k : 0) * step) >= want; }
        if (__all(ok)) break;
        if (give_up(ws, call, t0)) return false;
        __builtin_amdgcn_s_sleep(4);
    }
    asm volatile("" ::: "memory");
    return true;
}
#define STAMP(ws, i) do {} while (0)
#define STAMPW(ws, i) do {} while (0)
#define STAMPV(ws, i, v) do {} while (0)

DI void store1(const char* base, unsigned v, const u32x4& f) {
    asm volatile("s_nop 4\n\tglobal_store_dwordx4 %0, %1, %2 sc0 sc1\n\ts_nop 1" :: "v"(v), "v"(f), "s"(base) : "memory");
}
DI void store3(const char* base, unsigned v, const u32x4& a, const u32x4& b, const u32x4& c) {
    asm volatile("s_nop 4\n\tglobal_store_dwordx4 %0, %1, %4 sc0 sc1\n\t"
                 "global_store_dwordx4 %0, %2, %4 offset:1024 sc0 sc1\n\t"
                 "global_store_dwordx4 %0, %3, %4 offset:2048 sc0 sc1\n\ts_nop 1"
                 :: "v"(v), "v"(a), "v"(b), "v"(c), "s"(base) : "memory");
}
DI void store4(const char* base, unsigned v, const u32x4 (&f)[4]) {
    asm volatile("s_nop 4\n\tglobal_store_dwordx4 %0, %1, %5 sc0 sc1\n\t"
                 "global_store_dwordx4 %0, %2, %5 offset:1024 sc0 sc1\n\t"
                 "global_store_dwordx4 %0, %3, %5 offset:2048 sc0 sc1\n\t"
                 "global_store_dwordx4 %0, %4, %5 offset:3072 sc0 sc1\n\ts_nop 1"
                 :: "v"(v), "v"(f[0]), "v"(f[1]), "v"(f[2]), "v"(f[3]), "s"(base) : "memory");
}
DI void store4o(const char* base, unsigned v, const u32x4 (&f)[4]) {
    asm volatile("s_nop 4\n\tglobal_store_dwordx4 %0, %4, %8 sc1 nt\n\t"
                 "global_store_dwordx4 %1, %5, %8 sc1 nt\n\t"
                 "global_store_dwordx4 %2, %6, %8 sc1 nt\n\t"
                 "global_store_dwordx4 %3, %7, %8 sc1 nt\n\ts_nop 1"
                 :: "v"(v), "v"(v + 2 * ROWB), "v"(v + 4 * ROWB), "v"(v + 6 * ROWB), "v"(f[0]), "v"(f[1]), "v"(f[2]), "v"(f[3]), "s"(base) : "memory");
}

DI unsigned perm(unsigned hi, unsigned lo, unsigned sel) { return __builtin_amdgcn_perm(hi, lo, sel); }
template <int CTRL, int ROWMASK> DI unsigned dpp0(unsigned x) { return __builtin_amdgcn_update_dpp(0u, x, CTRL, ROWMASK, 0xf, false); }
DI unsigned wave_sum(unsigned x) {
    x += dpp0<0x111, 0xf>(x); x += dpp0<0x112, 0xf>(x); x += dpp0<0x114, 0xf>(x); x += dpp0<0x118, 0xf>(x); x += dpp0<0x142, 0xa>(x); x += dpp0<0x143, 0xc>(x);
    return __builtin_amdgcn_readlane(x, 63);
}
DI unsigned umax(unsigned a, unsigned b) { return a > b ? a : b; }
DI unsigned umin(unsigned a, unsigned b) { return a < b ? a : b; }
DI unsigned wave_max(unsigned x) {
    x = umax(x, dpp0<0x111, 0xf>(x)); x = umax(x, dpp0<0x112, 0xf>(x)); x = umax(x, dpp0<0x114, 0xf>(x)); x = umax(x, dpp0<0x118, 0xf>(x));
    x = umax(x, dpp0<0x142, 0xa>(x)); x = umax(x, dpp0<0x143, 0xc>(x));
    return __builtin_amdgcn_readlane(x, 63);
}
template <unsigned NE> struct ExcList {
    volatile LAS unsigned short* e; unsigned cnt;
    DI void begin(LAS unsigned char* tail, int lane) { e = (volatile LAS unsigned short*)(tail + 512); e[lane] = 0; cnt = 1; }
    DI bool add(bool act, unsigned entry) {
        const unsigned long long bal = __ballot(act);
        const unsigned slot = cnt + __builtin_amdgcn_mbcnt_hi((unsigned)(bal >> 32), __builtin_amdgcn_mbcnt_lo((unsigned)bal, 0u));
        if (act && slot < NE) e[slot] = (unsigned short)entry;
        cnt += __builtin_popcountll(bal);
        return cnt <= NE;
    }
};
template <unsigned NE> DI bool pack_finish(const char* base, unsigned blk, int c, int lane, LAS unsigned char* tail, ExcList<NE>& x, unsigned ebase,
                    const u32x4& p0, const u32x4& p1, const u32x4& p2, unsigned t0, unsigned t1) {
    if (lane == 0) x.e[0] = (unsigned short)(ebase | (x.cnt > NE ? 0x8000u : 0u));
    *(volatile LAS u32x2*)(tail + lane * 8) = (u32x2){t0, t1};
    constexpr int TS = 512 + 2 * NE;
    store3(base, blk + 4 * TS + c * 3072 + lane * 16, p0, p1, p2);
    const u32x4 t = *(volatile LAS u32x4*)(tail + (lane < TS / 16 ? lane : 0) * 16);
    if (lane < TS / 16) store1(base, blk + c * TS + lane * 16, t);
    return x.cnt <= NE;
}
DI void pack16(const u32x4 (&f)[4], const char* base, unsigned blk, int c, int lane, LAS unsigned char* tail) {
    unsigned lo[8], h[8], g[8], sum = 0;
#pragma unroll
    for (int i = 0; i < 8; ++i) {
        const unsigned a = f[i >> 1][2 * (i & 1)], b = f[i >> 1][2 * (i & 1) + 1];
        lo[i] = perm(b, a, 0x06040200u); h[i] = perm(b, a, 0x07050301u);
        sum = __builtin_amdgcn_udot4(h[i] & 0x7c7c7c7cu, 0x01010101u, sum, false);
    }
    const int top = (int)((wave_sum(sum) + 15155u + 8191u) >> 13);
    const unsigned eb = top < 7 ? 0u : top > 31 ? 24u : (unsigned)(top - 7), b4 = eb * 0x04040404u;
    ExcList<NE16> x; x.begin(tail, lane);
    unsigned m1 = 0, m0 = 0;
#pragma unroll
    for (int i = 0; i < 8; ++i) {
        g[i] = (h[i] & 0x83838383u) | (((h[i] | 0x80808080u) - b4) & 0x1c1c1c1cu);
        const unsigned xr = (g[i] + b4) ^ h[i];
        m1 |= ((xr >> 6) & 0x01010101u) << i; m0 |= ((xr >> 5) & 0x01010101u) << i;
    }
    for (unsigned m = m1 | m0; __any(m != 0); m &= m - 1) {
        const int k = m ? __builtin_ctz(m) : 0, i = k & 7;
        if (!x.add(m != 0, ((i >> 2) * 1024 + lane * 16 + (i & 3) * 4 + (k >> 3)) | ((m0 >> k) & 1u) << 11 | ((m1 >> k) & 1u) << 12)) break;
    }
    unsigned d[6];
#pragma unroll
    for (int q = 0; q < 2; ++q) {
        const unsigned s = g[4 * q + 3];
        d[3 * q] = g[4 * q] | (s & 0x03030303u) << 5; d[3 * q + 1] = g[4 * q + 1] | (s & 0x0c0c0c0cu) << 3;
        d[3 * q + 2] = g[4 * q + 2] | (s & 0x10101010u) << 1 | (s & 0x80808080u) >> 1;
    }
    pack_finish(base, blk, c, lane, tail, x, eb, (u32x4){lo[0], lo[1], lo[2], lo[3]}, (u32x4){lo[4], lo[5], lo[6], lo[7]}, (u32x4){d[0], d[1], d[2], d[3]}, d[4], d[5]);
}
DI bool pack32(const u32x4 (&f)[4], const char* base, unsigned blk, int c, int lane, LAS unsigned char* tail) {
    unsigned t[12], h[4], mx = 0;
#pragma unroll
    for (int j = 0; j < 4; ++j) {
        t[3 * j] = perm(f[j][3], f[j][0], 0x04020100u); t[3 * j + 1] = perm(f[j][3], f[j][1], 0x05020100u); t[3 * j + 2] = perm(f[j][3], f[j][2], 0x06020100u);
        h[j] = perm(perm(f[j][3], f[j][2], 0x07030703u), perm(f[j][1], f[j][0], 0x07030703u), 0x05040100u);
#pragma unroll
        for (int d = 0; d < 4; ++d) mx = umax(mx, f[j][d] & 0x7fffffffu);
    }
    const unsigned top = wave_max(mx) >> 24, eb = top < 7 ? 0u : top - 7, b1 = eb * 0x01010101u;
    ExcList<NE32> x; x.begin(tail, lane);
    unsigned cd[4], xn[2] = {0, 0};
#pragma unroll
    for (int j = 0; j < 4; ++j) {
        cd[j] = ((h[j] | 0x80808080u) - b1) & 0x07070707u;
        const unsigned xr = ((cd[j] + b1) ^ h[j]) & 0x78787878u;
        xn[j >> 1] |= (j & 1) ? xr << 1 : xr >> 3;
    }
    for (unsigned long long m = (unsigned long long)xn[1] << 32 | xn[0]; __any(m != 0);) {
        const int k = m ? __builtin_ctzll(m) >> 2 : 0, j = 2 * (k >> 3) + (k & 1);
        const unsigned v = (unsigned)(m >> (4 * k)) & 15u;
        if (!x.add(m != 0, (lane * 16 + j * 4 + ((k >> 1) & 3)) | v << 11)) break;
        m &= ~(15ull << (4 * k));
    }
    const unsigned n0 = cd[0] | (h[0] & 0x80808080u) >> 4 | cd[1] << 4 | (h[1] & 0x80808080u), n1 = cd[2] | (h[2] & 0x80808080u) >> 4 | cd[3] << 4 | (h[3] & 0x80808080u);
    return pack_finish(base, blk, c, lane, tail, x, eb, (u32x4){t[0], t[1], t[2], t[3]}, (u32x4){t[4], t[5], t[6], t[7]}, (u32x4){t[8], t[9], t[10], t[11]}, n0, n1);
}
DI bool unpack(bool is32, const LAS unsigned char* main3, const LAS unsigned char* tail, LAS unsigned char* patch, int lane, u32x4 (&f)[4]) {
    unsigned e = *(const volatile LAS unsigned short*)(tail + 512 + lane * 2);
    const u32x2 tt = *(const volatile LAS u32x2*)(tail + lane * 8);
    const u32x4 p0 = *(const volatile LAS u32x4*)(main3 + lane * 16), p1 = *(const volatile LAS u32x4*)(main3 + 1024 + lane * 16), p2 = *(const volatile LAS u32x4*)(main3 + 2048 + lane * 16);
    const unsigned hdr = rfl(e);
    if (lane == 0 || lane >= (int)(is32 ? NE32 : NE16)) e = 0;
    LAS unsigned* const pw = (LAS unsigned*)(patch + (e & 0x7fcu));
    __hip_atomic_fetch_or(pw, (e >> 11) << ((is32 ? 3 : 5) + 8 * (e & 3u)), __ATOMIC_RELAXED, __HIP_MEMORY_SCOPE_WAVEFRONT);
    const u32x4 x0 = *(const volatile LAS u32x4*)(patch + lane * 16), x1 = *(const volatile LAS u32x4*)(patch + 1024 + lane * 16);
    *(volatile LAS unsigned*)pw = 0;
    if (is32) {
        const unsigned b1 = (hdr & 127u) * 0x01010101u, t[12] = {p0[0], p0[1], p0[2], p0[3], p1[0], p1[1], p1[2], p1[3], p2[0], p2[1], p2[2], p2[3]};
#pragma unroll
        for (int j = 0; j < 4; ++j) {
            const unsigned n = tt[j >> 1];
            const unsigned h = ((j & 1) ? ((((n >> 4) & 0x07070707u) + b1) | (n & 0x80808080u)) : (((n & 0x07070707u) + b1) | ((n << 4) & 0x80808080u))) ^ x0[j];
            f[j][0] = perm(h, t[3 * j], 0x04020100u); f[j][1] = perm(h, t[3 * j + 1], 0x05020100u); f[j][2] = perm(h, t[3 * j + 2], 0x06020100u);
            f[j][3] = perm(perm(h, t[3 * j + 2], 0x07030703u), perm(t[3 * j + 1], t[3 * j], 0x07030703u), 0x05040100u);
        }
    } else {
        const unsigned b4 = (hdr & 31u) * 0x04040404u, d[6] = {p2[0], p2[1], p2[2], p2[3], tt[0], tt[1]};
#pragma unroll
        for (int q = 0; q < 2; ++q) {
            const unsigned a = d[3 * q], b = d[3 * q + 1], c = d[3 * q + 2];
            const unsigned g[4] = {a & 0x9f9f9f9fu, b & 0x9f9f9f9fu, c & 0x9f9f9f9fu,
                                   ((a >> 5) & 0x03030303u) | ((b >> 3) & 0x0c0c0c0cu) | ((c >> 1) & 0x10101010u) | ((c << 1) & 0x80808080u)};
#pragma unroll
            for (int i = 0; i < 4; ++i) {
                const unsigned h = (g[i] + b4) ^ (q ? x1[i] : x0[i]), lo = q ? p1[i] : p0[i];
                f[2 * q + (i >> 1)][2 * (i & 1)] = perm(h, lo, 0x05010400u); f[2 * q + (i >> 1)][2 * (i & 1) + 1] = perm(h, lo, 0x07030602u);
            }
        }
    }
    return hdr < 0x8000u;
}

constexpr int PSHORT = 16 + NBB - 2;
DI bool two_quads(int panel) { return panel < 16 || (unsigned)(panel - PSHORT) < 2u; }
DI int col0(int cb) { return cb >= NBB ? BCOLS + (cb - NBB) * TW : cb == NBB - 1 ? BCOLS - 256 : cb * TW; }
DI int colblock(int li, int rank) { return li < CPR ? NBB + CPR * rank + li : li - CPR; }
DI unsigned own_off(int panel, int kt) { return (panel < 16 ? panel * XPANEL : REL_WL + (panel - 16) * 8 * WBLK) + kt * (two_quads(panel) ? XBLK : WBLK); }
DI unsigned ring_off(int li) { return REL_WR + (li >= RING ? li - RING : li) * (56 * WBLK); }
DI unsigned pk16_off(int panel, int kt) { return OFF_PK16 + ((panel < 16 ? panel * 16 : 256 + (panel - 16) * 24) + kt * (two_quads(panel) ? 2 : 3)) * QB16; }
static_assert(2 * RING >= NL, "ring");
constexpr int round_end(int r) { return (NWB * (r + 1) - (r ? NJOB : 0) + RB - 1) / RB; }
#define XLAGS {0, 1, 3, 3}
#define BL 0
constexpr int XL[4] = XLAGS, NSTEP0 = 8 + (XL[3] > BL ? XL[3] : BL);
constexpr int x_lo(int s) { int g = 0; while (g < 4 && s - XL[g] >= 8) ++g; return 4 * g; }
constexpr int x_hi(int s) { int g = 0; while (g < 4 && s >= XL[g]) ++g; return 4 * g; }
constexpr int B0 = round_end(0) - CPR, W0Q = 3 * (B0 + NCB);
constexpr int cstep_start(int s) { int n = 0; for (int k = 0; k < s; ++k) n += 2 * (x_hi(k) - x_lo(k)) + (k < 8 ? W0Q : 0); return n; }
constexpr int steps_within(int quads) { int s = 0; while (s < NSTEP0 && cstep_start(s + 1) <= quads) ++s; return s; }
constexpr int ST1 = 3, STEPS1 = steps_within(NGEMM), STEPS2 = steps_within(ST1 * NGEMM);
static_assert(BL == 0 && STEPS1 >= 1 && NQ16 > ST1 * NGEMM + NGEMM, "stages");
DI void conv_quad(int b, int& panel, int& kt, int& qd) {
    if (b < cstep_start(NSTEP0)) {
        int s = 0, i = b, lo = x_lo(0), nx = 2 * (x_hi(0) - x_lo(0));
#pragma unroll
        for (int k = 1; k < NSTEP0; ++k) if (b >= cstep_start(k)) { s = k; i = b - cstep_start(k); lo = x_lo(k); nx = 2 * (x_hi(k) - x_lo(k)); }
        if (i < nx) { panel = lo + (i >> 1); qd = i & 1; kt = s - (panel < 4 ? XL[0] : panel < 8 ? XL[1] : panel < 12 ? XL[2] : XL[3]); }
        else { const int w = (i - nx) / 3; qd = i - nx - 3 * w; panel = 16 + (w < B0 ? w : w - B0 + NBB); kt = s; }
    } else if (b < NQ16 - 32) { const int r = b - cstep_start(NSTEP0), k = r / 3; qd = r - 3 * k; panel = 16 + B0 + (k >> 3); kt = k & 7; }
    else { const int r = b - (NQ16 - 32); panel = PSHORT + (r >> 4); kt = r >> 1 & 7; qd = r & 1; }
}
DI unsigned draw_call(char* ws, int wid, bool& clear) {
    const int lane = lane_now();
    unsigned v;
    asm volatile("s_nop 4\n\tglobal_load_dword %0, %1, %2 sc0 sc1\n\ts_waitcnt vmcnt(0)"
                 : "=&v"(v) : "v"(lane < 56 ? OFF_CALL + blockIdx.x * 128 : OFF_EV + (4 * NR + lane - 56) * 4), "s"(ws) : "memory");
    const unsigned c = rfl(v) + 1;
    clear = __all(v >= c - 1);
    BARRIER;
    if (wid == 0 && lane == 0) st_ag(ws + OFF_CALL + blockIdx.x * 128, c);
    return c;
}
DI unsigned convert_shards(Params& p, LAS unsigned char* lds, int wid, int gi) {
    const int lane = lane_now(), rank = p.rank;
    char* const ws = p.ws[rank];
    const __amdgpu_buffer_rsrc_t rx = __builtin_amdgcn_make_buffer_rsrc((void*)p.x, 0, MM * KS * 4, 0x00020000);
    const __amdgpu_buffer_rsrc_t rw = __builtin_amdgcn_make_buffer_rsrc((void*)p.w, 0, KS * NN * 4, 0x00020000);
    const unsigned vx = ((lane & 15) * KS + 8 * (lane >> 4)) * 4, vw = (8 * (lane >> 4) * NN + 4 * (lane & 15)) * 4;
    LAS unsigned char* const ring = lds + wid * 32768;
    LAS unsigned char* const tail = lds + 131072 + wid * 1024;
    const int nu = (NQ16 - gi + NGEMM - 1) / NGEMM;
    auto load = [&](int u) {
        int pa, kt, qd; conv_quad(gi + (u < nu ? u : 0) * NGEMM, pa, kt, qd);
        LAS unsigned char* const slot = ring + (u & 3) * 8192;
        if (pa < 16) {
            const unsigned s0 = ((pa * 256 + 128 * (wid >> 1) + 64 * qd) * KS + 32 * (2 * kt + (wid & 1))) * 4;
#pragma unroll
            for (int e = 0; e < 8; ++e) __builtin_amdgcn_raw_ptr_buffer_load_lds(rx, (LAS void*)(slot + e * 1024), 16, vx, s0 + (e >> 1) * (16 * KS * 4) + (e & 1) * 16, 0, AUX_NT);
        } else {
            const unsigned s0 = (32 * (2 * kt + (wid & 1)) * NN + col0(pa - 16) + (two_quads(pa) ? 128 : 192) * (wid >> 1) + 64 * qd) * 4;
#pragma unroll
            for (int e = 0; e < 8; ++e) __builtin_amdgcn_raw_ptr_buffer_load_lds(rw, (LAS void*)(slot + e * 1024), 16, vw, s0 + e * ROWB, 0, AUX_NT);
        }
    };
    load(0);
    char* peer_ws = p.ws[lane & 7];
    bool clear;
    const unsigned call = draw_call(ws, wid, clear);
    if (!clear) wait_words(ws, ws + OFF_EV + 4 * NR * 4, NR, 4, call - 1, call);
    asm volatile("" : "+v"(peer_ws));
    STAMP(ws, 50);
    for (int u = 0; u < nu; ++u) {
        if (u > 0) { load(u + 3); WAITVM(24); }
        if (u == ST1 + 1) {
            BARRIER;
            if (wid == 0) arrive(ws, peer_ws, rank, 2, NGEMM, call);
        }
        int pa, kt, qd; conv_quad(gi + u * NGEMM, pa, kt, qd);
        f32x4 v[8]; u32x4 f[4];
#pragma unroll
        for (int e = 0; e < 8; ++e) v[e] = *(const LAS f32x4*)(ring + (u & 3) * 8192 + e * 1024 + lane * 16);
#pragma unroll
        for (int j = 0; j < 4; ++j) { f16x8 h;
#pragma unroll
            for (int e = 0; e < 8; ++e) h[e] = (_Float16)(pa < 16 ? v[2 * j + (e >> 2)][e & 3] : v[e][j] * WSCALE);
            f[j] = __builtin_bit_cast(u32x4, h); }
        pack16(f, ws, pk16_off(pa, kt) + qd * QB16, wid, lane, tail);
        store4(ws, OFF_DATA + own_off(pa, kt) + (4 * qd + wid) * 4096 + lane * 16, f);
        if (u == 0) {
            STAMP(ws, 51);
            WAITVM(0);
            BARRIER;
            if (wid == 0 && lane < NR) st_sys(peer_ws + OFF_AV + (rank * 256 + gi) * 4, call);
            load(1); load(2); load(3);
            if (wid == 1) arrive(ws, peer_ws, rank, 1, NGEMM, call);
            STAMP(ws, 52);
        }
    }
    WAITVM(0);
    BARRIER;
    if (wid == 0) arrive(ws, peer_ws, rank, 3, NGEMM, call);
    return call;
}

constexpr int G0 = 16 + round_end(0), NBI = (16 + NL) * 8, NTW = 7 * NBI, NTC = 7 * NCI, NT = NTW + NTC;
struct Item { const char* src; unsigned size, pk, raw, dof; int flag; bool three; };
constexpr int w_lo(int s) { return s < 8 ? 16 : 16 + CPR; }
constexpr int w_hi(int s) { return s < BL || s >= 8 + BL ? 16 + CPR : G0; }
constexpr int step_start(int s) { int n = 0; for (int k = 0; k < s; ++k) n += x_hi(k) - x_lo(k) + w_hi(k) - w_lo(k); return n; }
static_assert(step_start(NSTEP0) == 8 * G0, "order");
DI void block_of(unsigned t, int& i, int& kt) {
    if (t < 8u * G0) {
        int s = 0, hi = x_hi(0), skip = w_lo(0) - x_hi(0); i = t;
#pragma unroll
        for (int k = 1; k < NSTEP0; ++k) if (t >= (unsigned)step_start(k)) { s = k; i = t - step_start(k) + x_lo(k); hi = x_hi(k); skip = w_lo(k) - x_hi(k); }
        if (i >= hi) i += skip;
        kt = s - (i < 4 ? XL[0] : i < 8 ? XL[1] : i < 12 ? XL[2] : i < 16 ? XL[3] : i < 16 + CPR ? 0 : BL);
    } else {
        unsigned u = 0, n = 1, b = 0;
#pragma unroll
        for (int r = 1; r < NROUND; ++r)
            if (t >= 8u * (16 + round_end(r - 1))) { b = 16 + round_end(r - 1); u = t - 8u * b; n = round_end(r) - round_end(r - 1); }
        kt = u / n; i = b + (u - kt * n);
    }
}
DI unsigned slab_out(unsigned s, int from) { return (s & 127u) * 32u * ROWB + (BCOLS + (CPR * from + (s >> 7)) * TW) * 4u; }
DI Item make_item(Params& p, unsigned t) {
    const int rank = p.rank;
    const unsigned tt = t < (unsigned)NTW ? t : t < (unsigned)NT ? t - NTW : 0, u = tt / 7, pj = tt - 7 * u;
    const int peer = (rank + 1 + pj) & 7;
    Item it;
    it.src = p.ws[peer]; it.size = t < (unsigned)NT ? WS_NEED : 0;
    if (t < (unsigned)NTW) {
        int i, kt; block_of(u, i, kt);
        const int panel = i < 16 ? i : 16 + colblock(i - 16, rank);
        it.three = !two_quads(panel);
        it.pk = pk16_off(panel, kt); it.raw = OFF_DATA + own_off(panel, kt);
        it.dof = OFF_DATA + (i < 16 ? i * XPANEL + 8 * XBLK : ring_off(i - 16)) + (7 * kt + pj) * (it.three ? WBLK : XBLK);
        it.flag = panel * 64 + 8 + 7 * kt + pj;
    } else {
        it.three = true;
        it.pk = OFF_PK32 + u * (3 * QB32); it.raw = OFF_RAW32 + u * WBLK; it.dof = slab_out(u, peer); it.flag = -1;
    }
    return it;
}
DI void load_main(const Item& it, unsigned off, LAS unsigned char* to, int lane) {
    const __amdgpu_buffer_rsrc_t r = __builtin_amdgcn_make_buffer_rsrc((void*)it.src, 0, it.size, 0x00020000);
    asm volatile("s_mov_b32 m0, %3\n\ts_nop 4\n\tbuffer_load_dwordx4 %0, %1, %2 offen sc0 sc1 lds\n\tbuffer_load_dwordx4 %0, %1, %2 offen offset:1024 sc0 sc1 lds\n\t"
                 "buffer_load_dwordx4 %0, %1, %2 offen offset:2048 sc0 sc1 lds" :: "v"(lane * 16), "s"(r), "s"(off), "s"((unsigned)(size_t)to) : "memory");
}
DI void load_tails(const Item& it, unsigned off, LAS unsigned char* to, int lane) {
    const __amdgpu_buffer_rsrc_t r = __builtin_amdgcn_make_buffer_rsrc((void*)it.src, 0, it.size, 0x00020000);
    const unsigned long long rest = it.flag < 0 ? (1ull << NE32 / 2) - 1 : (1ull << NE16 / 2) - 1;
    asm volatile("s_mov_b32 m0, %3\n\ts_nop 4\n\tbuffer_load_dwordx4 %0, %1, %2 offen sc0 sc1 lds\n\tbuffer_load_dwordx4 %0, %1, %2 offen offset:1024 sc0 sc1 lds\n\t"
                 "s_mov_b64 exec, %4\n\tbuffer_load_dwordx4 %0, %1, %2 offen offset:2048 sc0 sc1 lds\n\ts_mov_b64 exec, -1"
                 :: "v"(lane * 16), "s"(r), "s"(off), "s"((unsigned)(size_t)to), "s"(rest) : "memory");
}
constexpr unsigned NANS = 0x7fc07fc0u;
constexpr unsigned LQ = 14848, LMAIN = 2560;
struct Fetch {
    const char* ws; const char* out;
    LAS unsigned char* lds;
    unsigned slot; int lane; bool ok;
    template <int C0, bool WAITED> DI void quad(const Item& cur, unsigned qd, const Item& nx, unsigned nqd) {
        LAS unsigned char* const q = lds + slot; LAS unsigned char* const nq = lds + (LQ - slot);
        const unsigned npk = nx.pk + nqd * (nx.flag < 0 ? QB32 : QB16), nmain = npk + 4 * (nx.flag < 0 ? TS32 : TS16), ts = cur.flag < 0 ? TS32 : TS16;
#pragma unroll
        for (int c = 0; c < 4; ++c) {
            if (c > 0 || !WAITED) WAITVM(25);
            if (c == 0) load_tails(nx, npk, nq, lane);
            load_main(nx, nmain + c * 3072, nq + LMAIN + c * 3072, lane);
            u32x4 f[4];
            if (!unpack(cur.flag < 0, q + LMAIN + c * 3072, q + c * ts, lds + 2 * LQ, lane, f)) {
                const char* a = cur.src + (size_t)(cur.raw + qd * QUAD + c * 4096 + lane * 16);
                asm volatile("global_load_dwordx4 %0, %4, off sc0 sc1\n\tglobal_load_dwordx4 %1, %4, off offset:1024 sc0 sc1\n\t"
                             "global_load_dwordx4 %2, %4, off offset:2048 sc0 sc1\n\tglobal_load_dwordx4 %3, %4, off offset:3072 sc0 sc1\n\ts_waitcnt vmcnt(0)"
                             : "=&v"(f[0]), "=&v"(f[1]), "=&v"(f[2]), "=&v"(f[3]) : "v"(a) : "memory");
            }
            if (!ok) f[0] = f[1] = f[2] = f[3] = (u32x4){NANS, NANS, NANS, NANS};
            if (cur.flag >= 0) store4(ws, cur.dof + qd * QUAD + c * 4096 + lane * 16, f);
            else store4o(out, cur.dof + ((C0 + c) / 3 * 8 + (lane >> 5)) * ROWB + (C0 + c) % 3 * 512 + (lane & 31) * 16, f);
        }
        slot = LQ - slot;
    }
};
DI void fetch_wave(Params& p, LAS unsigned char* lds, unsigned call, int wid, int wv) {
    const int lane = lane_now(), rank = p.rank;
    char* const ws = p.ws[rank];
    Fetch F;
    F.ws = ws; F.out = (const char*)p.out[rank]; F.lds = lds; F.lane = lane;
    *(LAS u32x4*)(lds + 2 * LQ + lane * 16) = (u32x4){0, 0, 0, 0}; *(LAS u32x4*)(lds + 2 * LQ + 1024 + lane * 16) = (u32x4){0, 0, 0, 0};
    const unsigned wbase = (call - 1) * NTW - NCW, cbase = (call - 1) * NTC - NCW;
    static_assert(NL - RING == 32, "probe lanes");
    const unsigned vprobe = lane < 32 ? OFF_EV + lane * 4 : OFF_FIN + (lane - 32) * 4;
    static_assert(NCW <= 7 * step_start(1), "first items");
    if (wv == 0 && lane < NR) st_sys(p.ws[lane] + OFF_EV + (5 * NR + rank) * 4, call);
    {
        int i0, kt0; block_of(wv / 7, i0, kt0);
        const int q0 = i0 < 16 ? 2 * i0 : 2 * x_hi(0) + 3 * (i0 < 16 + CPR ? B0 + CPR * rank + i0 - 16 : i0 - 16 - CPR), peer = (rank + 1 + wv % 7) & 7;
        const unsigned v = lane < NR ? OFF_EV + (5 * NR + lane) * 4 : OFF_AV + peer * 1024 + (q0 + lane % (i0 < 16 ? 2 : 3)) * 4;
        const unsigned long long t0 = __builtin_amdgcn_s_memrealtime();
        F.ok = true;
        for (unsigned n = 1; !__all(ld_sys(ws + v) >= call); ++n) if ((n & 15) == 0 && give_up(ws, call, t0)) { F.ok = false; break; }
        asm volatile("" ::: "memory");
    }
    STAMP(ws, 1);
    unsigned tw = wv, tc = wv;
    auto choose = [&](unsigned probe) -> unsigned {
        const unsigned ev = (unsigned)__ballot(probe >= call);
        if (tw < (unsigned)NTW) {
            const unsigned u = tw / 7;
            int i, kt; block_of(u, i, kt);
            const int stage = u < (unsigned)step_start(STEPS1) ? 1 : u < (unsigned)step_start(STEPS2) ? 2 : 3;
            if ((ev >> (8 * stage + ((rank + 1 + tw - 7 * u) & 7)) & 1) &&
                (i < 16 + RING || __builtin_amdgcn_readlane(probe, 32 + i - 16 - RING) >= 16 * call)) return tw;
        }
        return tc < (unsigned)NTC && (ev >> ((rank + 1 + tc % 7) & 7) & 1) ? NTW + tc : NT;
    };
    auto publish = [&](int flag) {
        if (flag >= 0 && lane == 0) st_sys(ws + OFF_F + flag * 4, call);
    };
    while (tw < (unsigned)NTW || tc < (unsigned)NTC) {
        unsigned t = tw;
        if (tw >= (unsigned)NCW) {
            const unsigned long long t0 = __builtin_amdgcn_s_memrealtime();
            while ((t = choose(ld_sys(ws + vprobe))) == (unsigned)NT) {
                if (give_up(ws, call, t0)) { F.ok = false; t = tw < (unsigned)NTW ? tw : NTW + tc; break; }
                __builtin_amdgcn_s_sleep(16);
            }
        }
        Item cur = make_item(p, t);
        F.slot = 0;
        load_tails(cur, cur.pk, lds, lane);
#pragma unroll
        for (int c = 0; c < 4; ++c) load_main(cur, cur.pk + 4 * (cur.flag < 0 ? TS32 : TS16) + c * 3072, lds + LMAIN + c * 3072, lane);
        WAITVM(0);
        int pflag = -1;
        for (;;) {
            unsigned tk, probe; unsigned long long sv;
            asm volatile("s_nop 3\n\ts_waitcnt vmcnt(25)\n\tglobal_load_dword %1, %6, %5 sc0 sc1\n\t"
                         "s_mov_b64 %2, exec\n\ts_mov_b64 exec, 1\n\tglobal_atomic_add %0, %3, %4, %5 sc0\n\ts_mov_b64 exec, %2"
                         : "=&v"(tk), "=&v"(probe), "=&s"(sv) : "v"(OFF_TICK + (cur.flag < 0 ? 128u : 0u)), "v"(1u), "s"(ws), "v"(vprobe) : "memory");
            F.quad<0, true>(cur, 0, cur, 1);
            asm volatile("s_waitcnt vmcnt(25)" : "+v"(tk), "+v"(probe) :: "memory");
            publish(pflag);
            if (cur.flag >= 0) tw = rfl(tk) - wbase; else tc = rfl(tk) - cbase;
            if (cur.three) F.quad<4, true>(cur, 1, cur, 2);
            t = choose(probe);
            if (t == (unsigned)NT) t = choose(ld_sys(ws + vprobe));
            const Item nxt = make_item(p, t);
            F.quad<8, false>(cur, cur.three ? 2 : 1, nxt, 0);
            pflag = cur.flag;
            if (t >= (unsigned)NT) break;
            cur = nxt;
        }
        WAITVM(0);
        publish(pflag);
    }
    STAMPW(ws, 44 + wid);
    arrive(ws, p.ws[lane & 7], rank, 4, NCW, call);
    STAMPW(ws, 48 + wid);
}
#define PT_SET(A0, A1, A2, A3, R) \
    asm volatile("s_waitcnt vmcnt(60)\n\t" R \
                 "global_load_dwordx4 a[" A0 "], %16, %20 sc1 nt\n\tglobal_load_dwordx4 a[" A1 "], %17, %20 sc1 nt\n\t" \
                 "global_load_dwordx4 a[" A2 "], %18, %20 sc1 nt\n\tglobal_load_dwordx4 a[" A3 "], %19, %20 sc1 nt" \
                 : "=&v"(r[0]), "=&v"(r[1]), "=&v"(r[2]), "=&v"(r[3]), "=&v"(r[4]), "=&v"(r[5]), "=&v"(r[6]), "=&v"(r[7]), \
                   "=&v"(r[8]), "=&v"(r[9]), "=&v"(r[10]), "=&v"(r[11]), "=&v"(r[12]), "=&v"(r[13]), "=&v"(r[14]), "=&v"(r[15]) \
                 : "v"(v), "v"(v + 2 * ROWB), "v"(v + 4 * ROWB), "v"(v + 6 * ROWB), "s"(out) : "memory", PT_CLOB)
#define PT_CLOB "a0", "a1", "a2", "a3", "a4", "a5", "a6", "a7", "a8", "a9", "a10", "a11", "a12", "a13", "a14", "a15", "a16", "a17", "a18", "a19", "a20", "a21", "a22", "a23", "a24", "a25", "a26", "a27", "a28", "a29", "a30", "a31", "a32", "a33", "a34", "a35", "a36", "a37", "a38", "a39", "a40", "a41", "a42", "a43", "a44", "a45", "a46", "a47", "a48", "a49", "a50", "a51", "a52", "a53", "a54", "a55", "a56", "a57", "a58", "a59", "a60", "a61", "a62", "a63", "a64", "a65", "a66", "a67", "a68", "a69", "a70", "a71", "a72", "a73", "a74", "a75", "a76", "a77", "a78", "a79", "a80", "a81", "a82", "a83", "a84", "a85", "a86", "a87", "a88", "a89", "a90", "a91", "a92", "a93", "a94", "a95", "a96", "a97", "a98", "a99", "a100", "a101", "a102", "a103", "a104", "a105", "a106", "a107", "a108", "a109", "a110", "a111", "a112", "a113", "a114", "a115", "a116", "a117", "a118", "a119", "a120", "a121", "a122", "a123", "a124", "a125", "a126", "a127"
#define PT_R0 "v_accvgpr_read_b32 %0, a0\n\t" "v_accvgpr_read_b32 %1, a1\n\t" "v_accvgpr_read_b32 %2, a2\n\t" "v_accvgpr_read_b32 %3, a3\n\t" "v_accvgpr_read_b32 %4, a4\n\t" "v_accvgpr_read_b32 %5, a5\n\t" "v_accvgpr_read_b32 %6, a6\n\t" "v_accvgpr_read_b32 %7, a7\n\t" "v_accvgpr_read_b32 %8, a8\n\t" "v_accvgpr_read_b32 %9, a9\n\t" "v_accvgpr_read_b32 %10, a10\n\t" "v_accvgpr_read_b32 %11, a11\n\t" "v_accvgpr_read_b32 %12, a12\n\t" "v_accvgpr_read_b32 %13, a13\n\t" "v_accvgpr_read_b32 %14, a14\n\t" "v_accvgpr_read_b32 %15, a15\n\t"
#define PT_R1 "v_accvgpr_read_b32 %0, a16\n\t" "v_accvgpr_read_b32 %1, a17\n\t" "v_accvgpr_read_b32 %2, a18\n\t" "v_accvgpr_read_b32 %3, a19\n\t" "v_accvgpr_read_b32 %4, a20\n\t" "v_accvgpr_read_b32 %5, a21\n\t" "v_accvgpr_read_b32 %6, a22\n\t" "v_accvgpr_read_b32 %7, a23\n\t" "v_accvgpr_read_b32 %8, a24\n\t" "v_accvgpr_read_b32 %9, a25\n\t" "v_accvgpr_read_b32 %10, a26\n\t" "v_accvgpr_read_b32 %11, a27\n\t" "v_accvgpr_read_b32 %12, a28\n\t" "v_accvgpr_read_b32 %13, a29\n\t" "v_accvgpr_read_b32 %14, a30\n\t" "v_accvgpr_read_b32 %15, a31\n\t"
#define PT_R2 "v_accvgpr_read_b32 %0, a32\n\t" "v_accvgpr_read_b32 %1, a33\n\t" "v_accvgpr_read_b32 %2, a34\n\t" "v_accvgpr_read_b32 %3, a35\n\t" "v_accvgpr_read_b32 %4, a36\n\t" "v_accvgpr_read_b32 %5, a37\n\t" "v_accvgpr_read_b32 %6, a38\n\t" "v_accvgpr_read_b32 %7, a39\n\t" "v_accvgpr_read_b32 %8, a40\n\t" "v_accvgpr_read_b32 %9, a41\n\t" "v_accvgpr_read_b32 %10, a42\n\t" "v_accvgpr_read_b32 %11, a43\n\t" "v_accvgpr_read_b32 %12, a44\n\t" "v_accvgpr_read_b32 %13, a45\n\t" "v_accvgpr_read_b32 %14, a46\n\t" "v_accvgpr_read_b32 %15, a47\n\t"
#define PT_R3 "v_accvgpr_read_b32 %0, a48\n\t" "v_accvgpr_read_b32 %1, a49\n\t" "v_accvgpr_read_b32 %2, a50\n\t" "v_accvgpr_read_b32 %3, a51\n\t" "v_accvgpr_read_b32 %4, a52\n\t" "v_accvgpr_read_b32 %5, a53\n\t" "v_accvgpr_read_b32 %6, a54\n\t" "v_accvgpr_read_b32 %7, a55\n\t" "v_accvgpr_read_b32 %8, a56\n\t" "v_accvgpr_read_b32 %9, a57\n\t" "v_accvgpr_read_b32 %10, a58\n\t" "v_accvgpr_read_b32 %11, a59\n\t" "v_accvgpr_read_b32 %12, a60\n\t" "v_accvgpr_read_b32 %13, a61\n\t" "v_accvgpr_read_b32 %14, a62\n\t" "v_accvgpr_read_b32 %15, a63\n\t"
#define PT_R4 "v_accvgpr_read_b32 %0, a64\n\t" "v_accvgpr_read_b32 %1, a65\n\t" "v_accvgpr_read_b32 %2, a66\n\t" "v_accvgpr_read_b32 %3, a67\n\t" "v_accvgpr_read_b32 %4, a68\n\t" "v_accvgpr_read_b32 %5, a69\n\t" "v_accvgpr_read_b32 %6, a70\n\t" "v_accvgpr_read_b32 %7, a71\n\t" "v_accvgpr_read_b32 %8, a72\n\t" "v_accvgpr_read_b32 %9, a73\n\t" "v_accvgpr_read_b32 %10, a74\n\t" "v_accvgpr_read_b32 %11, a75\n\t" "v_accvgpr_read_b32 %12, a76\n\t" "v_accvgpr_read_b32 %13, a77\n\t" "v_accvgpr_read_b32 %14, a78\n\t" "v_accvgpr_read_b32 %15, a79\n\t"
#define PT_R5 "v_accvgpr_read_b32 %0, a80\n\t" "v_accvgpr_read_b32 %1, a81\n\t" "v_accvgpr_read_b32 %2, a82\n\t" "v_accvgpr_read_b32 %3, a83\n\t" "v_accvgpr_read_b32 %4, a84\n\t" "v_accvgpr_read_b32 %5, a85\n\t" "v_accvgpr_read_b32 %6, a86\n\t" "v_accvgpr_read_b32 %7, a87\n\t" "v_accvgpr_read_b32 %8, a88\n\t" "v_accvgpr_read_b32 %9, a89\n\t" "v_accvgpr_read_b32 %10, a90\n\t" "v_accvgpr_read_b32 %11, a91\n\t" "v_accvgpr_read_b32 %12, a92\n\t" "v_accvgpr_read_b32 %13, a93\n\t" "v_accvgpr_read_b32 %14, a94\n\t" "v_accvgpr_read_b32 %15, a95\n\t"
#define PT_R6 "v_accvgpr_read_b32 %0, a96\n\t" "v_accvgpr_read_b32 %1, a97\n\t" "v_accvgpr_read_b32 %2, a98\n\t" "v_accvgpr_read_b32 %3, a99\n\t" "v_accvgpr_read_b32 %4, a100\n\t" "v_accvgpr_read_b32 %5, a101\n\t" "v_accvgpr_read_b32 %6, a102\n\t" "v_accvgpr_read_b32 %7, a103\n\t" "v_accvgpr_read_b32 %8, a104\n\t" "v_accvgpr_read_b32 %9, a105\n\t" "v_accvgpr_read_b32 %10, a106\n\t" "v_accvgpr_read_b32 %11, a107\n\t" "v_accvgpr_read_b32 %12, a108\n\t" "v_accvgpr_read_b32 %13, a109\n\t" "v_accvgpr_read_b32 %14, a110\n\t" "v_accvgpr_read_b32 %15, a111\n\t"
#define PT_R7 "v_accvgpr_read_b32 %0, a112\n\t" "v_accvgpr_read_b32 %1, a113\n\t" "v_accvgpr_read_b32 %2, a114\n\t" "v_accvgpr_read_b32 %3, a115\n\t" "v_accvgpr_read_b32 %4, a116\n\t" "v_accvgpr_read_b32 %5, a117\n\t" "v_accvgpr_read_b32 %6, a118\n\t" "v_accvgpr_read_b32 %7, a119\n\t" "v_accvgpr_read_b32 %8, a120\n\t" "v_accvgpr_read_b32 %9, a121\n\t" "v_accvgpr_read_b32 %10, a122\n\t" "v_accvgpr_read_b32 %11, a123\n\t" "v_accvgpr_read_b32 %12, a124\n\t" "v_accvgpr_read_b32 %13, a125\n\t" "v_accvgpr_read_b32 %14, a126\n\t" "v_accvgpr_read_b32 %15, a127\n\t"
DI void pack_tile(Params& p, LAS unsigned char* tail, unsigned call, int ct) {
    const int lane = lane_now(), rank = p.rank;
    char* const ws = p.ws[rank];
    const char* const out = (const char*)p.out[rank];
    const bool ok = wait_words(ws, ws + OFF_AVC + ct * 4, 1, 4, call, call);
    __builtin_amdgcn_fence(__ATOMIC_ACQUIRE, "agent");
    const unsigned s0 = 8 * ct, vb = slab_out(s0, rank) + (lane >> 5) * ROWB + (lane & 31) * 16;
    for (int i = -8; i < 96; ++i) {
        const unsigned n = i + 8 < 96 ? i + 8 : 95, v = vb + n / 3 * (8 * ROWB) + n % 3 * 512;
        unsigned r[16];
        if (i >= 0 && i < 8) WAITVM(28);
        switch (i & 7) {
            case 0: PT_SET("0:3", "4:7", "8:11", "12:15", PT_R0); break;
            case 1: PT_SET("16:19", "20:23", "24:27", "28:31", PT_R1); break;
            case 2: PT_SET("32:35", "36:39", "40:43", "44:47", PT_R2); break;
            case 3: PT_SET("48:51", "52:55", "56:59", "60:63", PT_R3); break;
            case 4: PT_SET("64:67", "68:71", "72:75", "76:79", PT_R4); break;
            case 5: PT_SET("80:83", "84:87", "88:91", "92:95", PT_R5); break;
            case 6: PT_SET("96:99", "100:103", "104:107", "108:111", PT_R6); break;
            default: PT_SET("112:115", "116:119", "120:123", "124:127", PT_R7); break;
        }
        if (i < 0) continue;
        if (!ok) r[0] = NANS;
        const u32x4 f[4] = {{r[0], r[1], r[2], r[3]}, {r[4], r[5], r[6], r[7]}, {r[8], r[9], r[10], r[11]}, {r[12], r[13], r[14], r[15]}};
        if (!pack32(f, ws, OFF_PK32 + (s0 * 3 + (i >> 2)) * QB32, i & 3, lane, tail)) store4(ws, OFF_RAW32 + s0 * WBLK + i * 4096 + lane * 16, f);
    }
    WAITVM(0);
    arrive(ws, p.ws[lane & 7], rank, 0, 16 * CPR, call);
}

DI f16x8 rdfrag(unsigned base, int off) { return *(const LAS f16x8*)(size_t)(base + off); }
DI void dma_to(unsigned ldsaddr) { asm volatile("s_mov_b32 m0, %0" :: "s"(ldsaddr) : "memory"); }
#define PIN_S(x) asm volatile("" : "+s"(x))
DI float rd_acc(float a) { float v; asm volatile("v_accvgpr_read_b32 %0, %1" : "=v"(v) : "a"(a)); return v; }
DI f32x4 silu4(f32x4 a, float sc) {
    f32x4 t = a * (-1.44269504088896f / WSCALE);
#pragma unroll
    for (int i = 0; i < 4; ++i) t[i] = __builtin_amdgcn_exp2f(__builtin_amdgcn_fmed3f(t[i], -86.5617024533378f, 86.5617024533378f));
    t = __builtin_elementwise_fma(t, (f32x4){sc, sc, sc, sc}, (f32x4){sc, sc, sc, sc});
#pragma unroll
    for (int i = 0; i < 4; ++i) t[i] = __builtin_amdgcn_rcpf(t[i]);
    return a * t;
}
struct G7 {
    unsigned lds;
    __amdgpu_buffer_rsrc_t rD;
    unsigned voff;
    unsigned ard, an;
    unsigned blo, bhi;
    unsigned sA, sB, sr, sw;
    unsigned ringB, nextA, nextB;
    f32x4 acc[8][12];
    f16x8 L[2][4][2], H[4][2], B0, B1;
};
template <int I> DI void dma7(const G7& g, unsigned src) {
    asm volatile("buffer_load_dwordx4 %0, %1, %2 offen offset:%3 sc0 sc1 lds" :: "v"(g.voff), "s"(g.rD), "s"(src), "i"(I * 1024) : "memory");
}
template <int SL> DI f16x8 rdB7(const G7& g, int j, int k) { return rdfrag(SL < 3 ? g.blo : g.bhi, (SL % 3) * QUAD + k * 4096 + j * 1024); }
DI unsigned next3(unsigned s) { return s == 2 * QUAD ? 0 : s + QUAD; }
template <int J, bool ZERO, int VM> DI void phase(G7& g, int it) {
    constexpr int t = J % 3, c = J / 3, NX = (J + 1) % 6, PV = (J + 5) % 6;
    f16x8 B[5][2];
    B[0][0] = g.B0; B[0][1] = g.B1;
#pragma unroll
    for (int j = 0; j < 4; ++j) {
        const int n = 4 * t + j;
        if (j == 3) { __builtin_amdgcn_s_waitcnt(WCNT(VM, 0)); __builtin_amdgcn_s_barrier(); SCHED; }
#pragma unroll
        for (int u = 0; u < 16; ++u) {
            const int q = j * 16 + u, r = u >> 1, m = (n & 1) ? 7 - r : r, k = (u & 1) ^ (r & 1);
            const f16x8 a = m < 4 ? g.L[c][m][k] : g.H[m - 4][k];
            if (n < 8) {
                if (ZERO && !(u & 1)) asm volatile("v_mfma_f32_16x16x32_f16 %0, %1, %2, 0" : "=a"(g.acc[m][n]) : "v"(a), "v"(B[j][k]));
                else asm volatile("v_mfma_f32_16x16x32_f16 %0, %1, %2, %0" : "+a"(g.acc[m][n]) : "v"(a), "v"(B[j][k]));
            } else {
                if (ZERO && !(u & 1)) asm volatile("v_mfma_f32_16x16x32_f16 %0, %1, %2, 0" : "=&v"(g.acc[m][n]) : "v"(a), "v"(B[j][k]));
                else asm volatile("v_mfma_f32_16x16x32_f16 %0, %1, %2, %0" : "+v"(g.acc[m][n]) : "v"(a), "v"(B[j][k]));
            }
            SCHED;
            const bool lastcol = t == 2 && j == 3;
            if (u == (lastcol ? 9 : 1)) { B[j + 1][0] = j < 3 ? rdB7<J>(g, j + 1, 0) : rdB7<NX>(g, 0, 0); SCHED; }
            if (u == (lastcol ? 10 : 3)) { B[j + 1][1] = j < 3 ? rdB7<J>(g, j + 1, 1) : rdB7<NX>(g, 0, 1); SCHED; }
            if (t == 2 && j < 3 && (u == 5 || u == 9 || u == 13) && j * 3 + (u - 5) / 4 < 8) { const int i = j * 3 + (u - 5) / 4; g.L[c ^ 1][i >> 1][i & 1] = rdfrag(g.an, (i & 1) * 4096 + (i >> 1) * 1024); SCHED; }
            if (lastcol && u >= 1 && u <= 8) { const int row = 3 - ((u - 1) >> 1), kk = (u - 1) & 1; g.H[row][kk] = rdfrag(g.an, kk * 4096 + row * 1024); SCHED; }
            if ((t == 2 && q == 44) || (lastcol && u == 12)) { PIN_S(g.sr); g.sr = next3(g.sr); PIN_S(g.sr); SCHED; }
            if ((t == 2 && q == 46) || (lastcol && u == 14)) { g.an = g.ard + g.sr; asm volatile("" : "+v"(g.an)); SCHED; }
            if (t < 2) {
                if (q == 4) { PIN_S(g.sw); dma_to(g.lds + LDS_A + g.sw); SCHED; }
                if (q == 5) { dma7<0>(g, g.sA); SCHED; }
                if (q == 10) { dma7<1>(g, g.sA); SCHED; }
                if (q == 15) { dma7<2>(g, g.sA); SCHED; }
                if (q == 21) { dma7<3>(g, g.sA); SCHED; }
                if (q == 52) { PIN_S(g.sA); g.sA += QUAD; PIN_S(g.sA); SCHED; }
                if (q == 54 && J == 0) { g.sA = it == 31 ? g.nextA : g.sA; PIN_S(g.sA); SCHED; }
                if (q == 56) { PIN_S(g.sw); g.sw = next3(g.sw); PIN_S(g.sw); SCHED; }
            }
            constexpr int PB0[4] = {27, 32, 38, 43}, PB2[4] = {7, 16, 27, 39};
            if (q == (t < 2 ? PB0[0] : PB2[0]) - 1) { dma_to(g.lds + PV * QUAD); SCHED; }
            if (q == (t < 2 ? PB0[0] : PB2[0])) { dma7<0>(g, g.sB); SCHED; }
            if (q == (t < 2 ? PB0[1] : PB2[1])) { dma7<1>(g, g.sB); SCHED; }
            if (q == (t < 2 ? PB0[2] : PB2[2])) { dma7<2>(g, g.sB); SCHED; }
            if (q == (t < 2 ? PB0[3] : PB2[3])) { dma7<3>(g, g.sB); SCHED; }
            if (q == (t < 2 ? 58 : 42)) { PIN_S(g.sB); g.sB += QUAD; PIN_S(g.sB); SCHED; }
            if (q == 60 && J == 0) { g.sB = it == 3 ? g.ringB : g.sB; PIN_S(g.sB); SCHED; }
            if (q == 62 && J == 0) { g.sB = it == 31 ? g.nextB : g.sB; PIN_S(g.sB); SCHED; }
        }
    }
    g.B0 = B[4][0]; g.B1 = B[4][1];
}
template <bool FIRST> DI void trip(G7& g, int it) {
    SCHED;
    phase<0, FIRST, FIRST ? 60 : 28>(g, it); phase<1, FIRST, FIRST ? 60 : 24>(g, it); phase<2, FIRST, 16>(g, it);
    phase<3, false, 28>(g, it); phase<4, false, 24>(g, it); phase<5, false, 16>(g, it);
}
struct Short { unsigned sb, bn, bx; };
DI unsigned next6(unsigned s) { return s == 5 * QUAD ? 0 : s + QUAD; }
template <int J, bool ZERO, int VM> DI void phase2(G7& g, Short& h, int it) {
    constexpr int t = J & 1, c = J >> 1;
    f16x8 B[5][2];
    B[0][0] = g.B0; B[0][1] = g.B1;
    unsigned pv = h.sb == 0 ? 5 * QUAD : h.sb - QUAD;
#pragma unroll
    for (int j = 0; j < 4; ++j) {
        const int n = 4 * t + j;
        if (j == 3) { __builtin_amdgcn_s_waitcnt(WCNT(VM, 0)); __builtin_amdgcn_s_barrier(); SCHED; }
#pragma unroll
        for (int u = 0; u < 16; ++u) {
            const int q = j * 16 + u, r = u >> 1, m = (n & 1) ? 7 - r : r, k = (u & 1) ^ (r & 1);
            const f16x8 a = m < 4 ? g.L[c][m][k] : g.H[m - 4][k];
            if (ZERO && !(u & 1)) asm volatile("v_mfma_f32_16x16x32_f16 %0, %1, %2, 0" : "=a"(g.acc[m][n]) : "v"(a), "v"(B[j][k]));
            else asm volatile("v_mfma_f32_16x16x32_f16 %0, %1, %2, %0" : "+a"(g.acc[m][n]) : "v"(a), "v"(B[j][k]));
            SCHED;
            const bool lastcol = t == 1 && j == 3;
            if (u == (lastcol ? 9 : 1)) { B[j + 1][0] = j < 3 ? rdfrag(h.bn, (j + 1) * 1024) : rdfrag(h.bx, 0); SCHED; }
            if (u == (lastcol ? 10 : 3)) { B[j + 1][1] = j < 3 ? rdfrag(h.bn, 4096 + (j + 1) * 1024) : rdfrag(h.bx, 4096); SCHED; }
            if (t == 1 && j < 3 && (u == 5 || u == 9 || u == 13) && j * 3 + (u - 5) / 4 < 8) { const int i = j * 3 + (u - 5) / 4; g.L[c ^ 1][i >> 1][i & 1] = rdfrag(g.an, (i & 1) * 4096 + (i >> 1) * 1024); SCHED; }
            if (lastcol && u >= 1 && u <= 8) { const int row = 3 - ((u - 1) >> 1), kk = (u - 1) & 1; g.H[row][kk] = rdfrag(g.an, kk * 4096 + row * 1024); SCHED; }
            if ((t == 1 && q == 44) || (lastcol && u == 12)) { PIN_S(g.sr); g.sr = next3(g.sr); PIN_S(g.sr); SCHED; }
            if ((t == 1 && q == 46) || (lastcol && u == 14)) { g.an = g.ard + g.sr; asm volatile("" : "+v"(g.an)); SCHED; }
            if (q == 0) { PIN_S(g.sw); dma_to(g.lds + LDS_A + g.sw); SCHED; }
            if (q == 2) { dma7<0>(g, g.sA); SCHED; }
            if (q == 6) { dma7<1>(g, g.sA); SCHED; }
            if (q == 10) { dma7<2>(g, g.sA); SCHED; }
            if (q == 14) { dma7<3>(g, g.sA); SCHED; }
            if (q == 16) { PIN_S(g.sA); g.sA += QUAD; PIN_S(g.sA); SCHED; }
            if (q == 18) { PIN_S(g.sw); g.sw = next3(g.sw); PIN_S(g.sw); SCHED; }
            if (q == 22) { PIN_S(pv); dma_to(g.lds + pv); SCHED; }
            if (q == 23) { dma7<0>(g, g.sB); SCHED; }
            if (q == 30) { dma7<1>(g, g.sB); SCHED; }
            if (q == 36) { dma7<2>(g, g.sB); SCHED; }
            if (q == 42) { dma7<3>(g, g.sB); SCHED; }
            if (q == 43) { PIN_S(g.sB); g.sB += QUAD; PIN_S(g.sB); SCHED; }
            if (q == 45 && J == 2) { g.sB = it == 2 ? g.ringB : g.sB; PIN_S(g.sB); SCHED; }
            if (q == 59) { PIN_S(h.sb); h.sb = next6(h.sb); PIN_S(h.sb); SCHED; }
            if (q == 63) { h.bn = h.bx; h.bx = g.blo + next6(h.sb); asm volatile("" : "+v"(h.bx)); SCHED; }
        }
    }
    g.B0 = B[4][0]; g.B1 = B[4][1];
}
DI void short_tile(G7& g) {
    Short h;
    h.sb = 0; h.bn = g.blo; h.bx = g.blo + QUAD;
    SCHED;
    phase2<0, true, 60>(g, h, 0); phase2<1, true, 12>(g, h, 0); phase2<2, false, 12>(g, h, 0); phase2<3, false, 12>(g, h, 0);
    for (int it = 1; it < 32; ++it) { SCHED; phase2<0, false, 12>(g, h, it); phase2<1, false, 12>(g, h, it); phase2<2, false, 12>(g, h, it); phase2<3, false, 12>(g, h, it); }
}
DI void g7_init(G7& g, LAS unsigned char* lds, const char* data, unsigned bytes, int wid) {
    const int lane = lane_now(), wr = wid >> 1, wc = wid & 1;
    const unsigned lbase = (unsigned)(size_t)lds;
    g.rD = __builtin_amdgcn_make_buffer_rsrc((void*)data, 0, bytes, 0x00020000);
    g.lds = lbase + wid * 4096; g.voff = wid * 4096 + lane * 16;
    g.ard = lbase + LDS_A + wr * 8192 + lane * 16; g.blo = lbase + wc * 8192 + lane * 16; g.bhi = g.blo + 3 * QUAD;
    asm volatile("" : "+v"(g.ard), "+v"(g.blo), "+v"(g.bhi), "+v"(g.voff));
}
DI void g7_fill(G7& g, unsigned to, unsigned src) { dma_to(g.lds + to); asm volatile("s_nop 4"); dma7<0>(g, src); dma7<1>(g, src); dma7<2>(g, src); dma7<3>(g, src); }
DI void g7_prologue(G7& g, unsigned dA, unsigned dB) {
#pragma unroll
    for (int i = 0; i < 5; ++i) { if (i < 3) g7_fill(g, LDS_A + i * QUAD, dA + i * QUAD); g7_fill(g, i * QUAD, dB + i * QUAD); }
    WAITVM(0); __builtin_amdgcn_s_barrier(); SCHED;
#pragma unroll
    for (int i = 0; i < 8; ++i) { g.L[0][i >> 1][i & 1] = rdfrag(g.ard, (i & 1) * 4096 + (i >> 1) * 1024); g.H[i >> 1][i & 1] = rdfrag(g.ard, QUAD + (i & 1) * 4096 + (i >> 1) * 1024); }
    g.B0 = rdB7<0>(g, 0, 0); g.B1 = rdB7<0>(g, 0, 1);
    __builtin_amdgcn_s_waitcnt(WCNT(63, 0)); __builtin_amdgcn_s_barrier(); SCHED;
    g.sA = dA + 3 * QUAD; g.sB = dB + 5 * QUAD; g.sr = 2 * QUAD; g.sw = 0; g.an = g.ard + 2 * QUAD;
}
template <int NQ> DI void g7_epilogue(G7& g, __amdgpu_buffer_rsrc_t rO, unsigned so, unsigned rowb, float one) {
    const int lz = lane_now();
    const unsigned vo = 4 * (lz >> 4) * rowb + 16 * (lz & 15);
    asm volatile("s_nop 15");
#pragma unroll
    for (int m = 0; m < 8; ++m) {
#pragma unroll
        for (int r = 0; r < 4; ++r) {
#pragma unroll
            for (int q = 0; q < NQ; ++q) {
                f32x4 y;
                if (q < 2) y = (f32x4){rd_acc(g.acc[m][4 * q][r]), rd_acc(g.acc[m][4 * q + 1][r]), rd_acc(g.acc[m][4 * q + 2][r]), rd_acc(g.acc[m][4 * q + 3][r])};
                else y = (f32x4){g.acc[m][8][r], g.acc[m][9][r], g.acc[m][10][r], g.acc[m][11][r]};
                const f32x4 v = silu4(y, one);
                asm volatile("buffer_store_dwordx4 %0, %1, %2, %3 offen offset:%4 nt\n\ts_nop 1" :: "v"(v), "v"(vo), "s"(rO), "s"(so + r * rowb), "i"(q * 256) : "memory");
            }
            SCHED;
        }
        so += 16 * rowb;
    }
}
DI int lead_ones(bool ok) { const unsigned long long m = ~__ballot(ok); return m ? __builtin_ctzll(m) : 64; }

struct Ready {
    char* ws; unsigned call;
    __amdgpu_buffer_rsrc_t rF; LAS unsigned char* zone;
    int pa, pb, n; bool pending; float one;
    DI void poll(int need) {
        const unsigned long long t0 = __builtin_amdgcn_s_memrealtime();
        const int lane = lane_now();
        for (;;) {
            const unsigned a = ld_sys(ws + OFF_F + pa * 256 + lane * 4), b = ld_sys(ws + OFF_F + pb * 256 + lane * 4);
            n = lead_ones(lane < 8 || (a >= call && b >= call));
            if (n >= need) break;
            if (give_up(ws, call, t0)) { n = 64; one = __builtin_nanf(""); break; }
            __builtin_amdgcn_s_sleep(16);
        }
    }
    DI void request() {
        const int lane = lane_now();
        __builtin_amdgcn_raw_ptr_buffer_load_lds(rF, (LAS void*)zone, 4, lane * 4, pa * 256, 0, AUX_SYS);
        __builtin_amdgcn_raw_ptr_buffer_load_lds(rF, (LAS void*)(zone + 256), 4, lane * 4, pb * 256, 0, AUX_SYS);
        pending = true;
    }
    DI int plan(int it, int stop) {
        if (n == 64) return stop;
        const int safe = (n - 4) >> 1;
        if (safe <= it + 1) { request(); return it + 1; }
        return safe - 1 < stop ? safe - 1 : stop;
    }
    DI void check(int need) {
        if (pending) {
            const int lane = lane_now();
            const unsigned a = *(const volatile LAS unsigned*)(zone + lane * 4), b = *(const volatile LAS unsigned*)(zone + 256 + lane * 4);
            n = lead_ones(lane < 8 || (a >= call && b >= call)); pending = false;
        }
        if (n < need) poll(need);
        asm volatile("" ::: "memory");
    }
};

DI void gemm_wg(Params& p, LAS unsigned char* lds, int wid, int band, int S) {
    const int wr = wid >> 1, wc = wid & 1, rank = p.rank;
    char* const ws = p.ws[rank];
    const unsigned call = convert_shards(p, lds, wid, S * (16 / RB) + band);
    int T = S;
    STAMP(ws, 2);
    G7 g;
    g7_init(g, lds, ws + OFF_DATA, DATAB, wid);
    Ready R;
    R.ws = ws; R.call = call; R.rF = __builtin_amdgcn_make_buffer_rsrc((void*)(ws + OFF_F), 0, NPAN * 256, 0x00020000);
    R.zone = lds + LDS_ZONE + wid * 512; R.pending = false; R.one = WSCALE;
    if (!wait_words(ws, ws + OFF_EV + (3 * NR + rank) * 4, 1, 4, call, call) || rfl(ld_ag(ws + OFF_TMO)) == call) R.one = __builtin_nanf("");
    R.pa = RB * band + (T & (RB - 1)); R.pb = 16 + colblock(T >> RBL, rank); R.n = 8;
    STAMP(ws, 1);
    g7_prologue(g, rfl(own_off(R.pa, 0)), rfl(own_off(R.pb, 0)));
    const __amdgpu_buffer_rsrc_t rO = __builtin_amdgcn_make_buffer_rsrc((void*)p.out[rank], 0, OUTB, 0x00020000);
    unsigned tprev = 0;
    int li = T >> RBL;
    for (int ti = 0;; ++ti) {
        const int pm = R.pa, cb = R.pb - 16;
        S += NWB;
        const int job = S - PSLOT;
        if ((unsigned)job < (unsigned)NJOB) S += NWB;
        const bool has_next = S < NROUND * NWB, deal = has_next && S >= (NROUND - 1) * NWB;
        T = S < PSLOT ? S : S - NJOB;
        int qa = pm, ql = li;
        if (has_next && !deal) { qa = RB * band + (T & (RB - 1)); ql = T >> RBL; }
        g.ringB = rfl(ring_off(li)); g.nextA = rfl(own_off(qa, 0)); g.nextB = rfl(own_off(16 + colblock(ql, rank), 0));
        {
            const unsigned now = (unsigned)__builtin_amdgcn_s_memrealtime();
            if (deal && blockIdx.x >> 3 == NG / 2 && wid == 0 && lane_now() == 0)
                __hip_atomic_store((GAS unsigned long long*)(ws + OFF_ETA + (blockIdx.x & 7) * 8), (unsigned long long)call << 32 | (3 * now - 2 * tprev), __ATOMIC_RELAXED, __HIP_MEMORY_SCOPE_AGENT);
            tprev = now;
        }
        R.check(4);
        int lim = R.plan(0, deal ? 29 : 32);
        trip<true>(g, 0);
        for (int it = 1; it < 32;) {
            if (it >= lim) {
                R.check(it < 30 ? 2 * it + 4 : 64);
                lim = R.plan(it, deal && it < 29 ? 29 : 32);
                if (deal && it == 29) {
                    const int lane = lane_now();
                    const unsigned long long t0 = __builtin_amdgcn_s_memrealtime();
                    unsigned long long e;
                    int place = blockIdx.x & 7;
                    while (!__all((unsigned)((e = __hip_atomic_load((GAS unsigned long long*)(ws + OFF_ETA + (lane & 7) * 8), __ATOMIC_RELAXED, __HIP_MEMORY_SCOPE_AGENT)) >> 32) == call)) {
                        if (give_up(ws, call, t0)) { R.one = __builtin_nanf(""); e = 0; break; }
                        __builtin_amdgcn_s_sleep(8);
                    }
                    const int d = (int)((unsigned)e - __builtin_amdgcn_readlane((unsigned)e, place));
                    place = __builtin_popcount((unsigned)__ballot(d < 0 || (d == 0 && lane < place)) & 0xffu);
                    constexpr int T4 = (NROUND - 1) * NWB - NJOB, NF = RB * (NL - 2) - T4;
                    const int P = place * NG + (blockIdx.x >> 3), bd = P / NF;
                    if (bd < 16 / RB) { const int t = T4 + P - bd * NF; qa = RB * bd + (t & (RB - 1)); ql = t >> RBL; }
                    else { const int t = P - 16 / RB * NF; qa = t & 15; ql = NL - 2 + (t >> 4); }
                    g.nextA = rfl(own_off(qa, 0)); g.nextB = rfl(own_off(16 + colblock(ql, rank), 0));
                }
            }
            do trip<false>(g, it); while (++it < lim);
        }
        STAMP(ws, 8 + 2 * ti);
        R.pa = qa; R.pb = 16 + colblock(ql, rank);
        R.request();
        g7_epilogue<3>(g, rO, rfl((pm * 256 + wr * 128) * ROWB + (col0(cb) + wc * 192) * 4), ROWB, R.one);
        WAITVM(60);
        if (wid == 0 && lane_now() == 0) __hip_atomic_fetch_add((gu32*)(ws + OFF_FIN + li * 4), 1u, __ATOMIC_RELAXED, __HIP_MEMORY_SCOPE_AGENT);
        if (li < CPR) {
            WAITVM(0);
            BARRIER;
            STAMP(ws, 4);
            if (wid == 0) {
                __builtin_amdgcn_fence(__ATOMIC_RELEASE, "agent");
                WAITVM(0);
                if (lane_now() == 0) st_sys(ws + OFF_AVC + (li * 16 + pm) * 4, call);
            }
        }
        if ((unsigned)job < (unsigned)NJOB) { pack_tile(p, lds + LDS_ZONE + 4096 + wid * 1024, call, job % CPR * 16 + RB * band + job / CPR * 4 + wid); STAMP(ws, 5); }
        STAMP(ws, 9 + 2 * ti);
        if (!has_next) break;
        li = ql;
        if (li >= NL - 2) {
            g.ringB = rfl(ring_off(li));
            R.check(64);
            short_tile(g);
            STAMP(ws, 10 + 2 * ti);
            g7_epilogue<2>(g, rO, rfl((R.pa * 256 + wr * 128) * ROWB + (col0(R.pb - 16) + wc * 128) * 4), ROWB, R.one);
            STAMP(ws, 11 + 2 * ti);
            break;
        }
    }
    __builtin_amdgcn_s_waitcnt(WCNT(0, 0));
    STAMP(ws, 60 + (call & 1));
}

__global__ __launch_bounds__(256, 1) void fused(Args) {
    extern __shared__ __attribute__((aligned(16))) unsigned char shm[];
    Params& p = *(Params*)__builtin_amdgcn_kernarg_segment_ptr();
    LAS unsigned char* lds = (LAS unsigned char*)shm;
    char* const ws = p.ws[p.rank];
    const int wid = rfl(threadIdx.x >> 6);
    STAMP(ws, 0);
    STAMPV(ws, 3, __builtin_amdgcn_s_getreg(20 | (0 << 6) | (3 << 11)));
    const int grp = blockIdx.x & 7, idx = blockIdx.x >> 3;
    if (idx < NG) gemm_wg(p, lds, wid, (grp & 1 ? 0 : 8 / RB) + (grp >> RBL), (grp >> 1 & (RB / 2 - 1)) * NG + idx);
    else {
        const int cw = (idx - NG) * 8 + grp;
        bool clear;
        const unsigned call = draw_call(ws, wid, clear);
        fetch_wave(p, lds + wid * 32768, call, wid, cw * 4 + wid);
    }
}
}

extern "C" void kernel_launch(int rank, int nranks, ncclComm_t comm, hipStream_t stream,
                              void* const* d_in, const int* in_sizes, int n_in,
                              void* d_out, int out_size,
                              const int* in_dtypes, int out_dtype,
                              void* const* d_out_all,
                              void* const* d_ws_all, size_t ws_bytes) {
    if (nranks != NR || (unsigned)rank >= (unsigned)NR || n_in != 2 || ws_bytes < WS_NEED || in_sizes[0] != MM * KS || in_sizes[1] != KS * NN || out_size != MM * NN) {
        fprintf(stderr, "kernel_launch: unexpected problem shape\n");
        return;
    }
    static bool attr_set[NR];
    if (!attr_set[rank]) {
        HIP_CHECK(hipFuncSetAttribute((const void*)fused, hipFuncAttributeMaxDynamicSharedMemorySize, LDS_BYTES));
        attr_set[rank] = true;
    }
    Args p;
    p.x = (const float*)d_in[0]; p.w = (const float*)d_in[1];
    for (int i = 0; i < NR; ++i) { p.ws[i] = (char*)d_ws_all[i]; p.out[i] = (float*)d_out_all[i]; }
    p.rank = rank; p.pad = 0;
    hipLaunchKernelGGL(fused, dim3(256), dim3(256), LDS_BYTES, stream, p);
    HIP_CHECK(hipGetLastError());
}
```

```cpp
#include <hip/hip_runtime.h>
#include <rccl/rccl.h>
#include <cstdio>

#define HIP_CHECK(expr) do { hipError_t _e = (expr); if (_e != hipSuccess) { \
    fprintf(stderr, "HIP error %s at %s:%d\n", hipGetErrorString(_e), __FILE__, __LINE__); \
    return; } } while (0)

#define LAS __attribute__((address_space(3)))
#define GAS __attribute__((address_space(1)))
#define DI __device__ __forceinline__
typedef _Float16 f16x8 __attribute__((ext_vector_type(8)));
typedef float f32x4 __attribute__((ext_vector_type(4)));
typedef unsigned u32x4 __attribute__((ext_vector_type(4)));
typedef unsigned u32x2 __attribute__((ext_vector_type(2)));
typedef GAS unsigned gu32;

#define NG 29
namespace {
constexpr int MM = 4096, NN = 32768, KS = 512, NR = 8;
constexpr int NGEMM = 8 * NG, NCOMM = 256 - NGEMM, NCW = 4 * NCOMM;
constexpr int TW = 384;
constexpr int CPR = 2, NCB = NR * CPR;
constexpr int BCOLS = NN - NCB * TW, NBB = (BCOLS + TW - 1) / TW;
constexpr int NWP = NBB + NCB, NPAN = 16 + NWP;
constexpr int NL = CPR + NBB;
constexpr int RBL = 3, RB = 1 << RBL, NWB = NGEMM * RB / 16, NJOB = CPR * RB / 4, NROUND = 5, PSLOT = 2 * NWB - NJOB;
static_assert(NROUND * NWB == RB * NL + NJOB, "slots");
constexpr unsigned QUAD = 16384, XBLK = 2 * QUAD, WBLK = 3 * QUAD, XPANEL = 64 * XBLK;
static_assert(BCOLS - (NBB - 2) * TW == 2 * 256, "short blocks");
constexpr unsigned NE16 = 48, NE32 = 16, TS16 = 512 + 2 * NE16, TS32 = 512 + 2 * NE32, QB16 = 4 * (TS16 + 3072), QB32 = 4 * (TS32 + 3072);
constexpr unsigned OUTB = (unsigned)MM * NN * 4u, ROWB = NN * 4;
constexpr unsigned OFF_CALL = 0;
constexpr unsigned OFF_TMO = 32768;
constexpr unsigned OFF_FIN = 32768 + 1024;
constexpr unsigned OFF_ETA = 32768 + 2048;
constexpr unsigned OFF_TICK = 32768 + 4096;
constexpr unsigned OFF_F = 65536;
constexpr unsigned OFF_AV = 131072;
constexpr unsigned OFF_AVC = OFF_AV + 8192;
constexpr unsigned OFF_CNT = 196608;
constexpr unsigned OFF_EV = OFF_CNT + 4096;
constexpr int RING = NL - 32;
constexpr unsigned OFF_DATA = 1u << 20;
constexpr unsigned REL_WL = 16 * XPANEL;
constexpr unsigned REL_WR = REL_WL + NWP * 8 * WBLK;
constexpr unsigned DATAB = REL_WR + RING * 56 * WBLK;
constexpr int NQ16 = 16 * 8 * 2 + (NWP - 2) * 8 * 3 + 32;
constexpr unsigned OFF_PK16 = OFF_DATA + DATAB;
constexpr int NCI = CPR * (MM / 32);
constexpr unsigned OFF_PK32 = OFF_PK16 + (NQ16 + 16) * QB16;
constexpr unsigned OFF_RAW32 = OFF_PK32 + NCI * 3 * QB32;
constexpr unsigned WS_NEED = OFF_RAW32 + NCI * WBLK;
static_assert(NPAN * 256 <= 65536, "flags");
constexpr int LDS_A = 6 * QUAD, LDS_ZONE = LDS_A + 3 * QUAD, LDS_BYTES = LDS_ZONE + 8192;
constexpr unsigned long long DEADLINE = 50000000ull;
constexpr float WSCALE = 32.0f;
constexpr int AUX_SYS = 17, AUX_NT = 2;

struct Args {
    const float* x; const float* w;
    char* ws[NR]; float* out[NR];
    int rank, pad;
};
typedef const __attribute__((address_space(4))) Args Params;

constexpr int WCNT(int vm, int lgkm) { return (vm & 15) | (7 << 4) | ((lgkm & 15) << 8) | ((vm >> 4) << 14); }
#define SCHED __builtin_amdgcn_sched_barrier(0)
#define WAITVM(n) asm volatile("s_waitcnt vmcnt(" #n ")" ::: "memory")
#define BARRIER do { asm volatile("" ::: "memory"); __builtin_amdgcn_s_barrier(); asm volatile("" ::: "memory"); } while (0)

DI unsigned ld_sys(const char* p) { return __hip_atomic_load((gu32*)p, __ATOMIC_RELAXED, __HIP_MEMORY_SCOPE_SYSTEM); }
DI void st_sys(char* p, unsigned v) { __hip_atomic_store((gu32*)p, v, __ATOMIC_RELAXED, __HIP_MEMORY_SCOPE_SYSTEM); }
DI unsigned ld_ag(const char* p) { return __hip_atomic_load((gu32*)p, __ATOMIC_RELAXED, __HIP_MEMORY_SCOPE_AGENT); }
DI void st_ag(char* p, unsigned v) { __hip_atomic_store((gu32*)p, v, __ATOMIC_RELAXED, __HIP_MEMORY_SCOPE_AGENT); }
DI unsigned rfl(unsigned x) { return __builtin_amdgcn_readfirstlane(x); }
DI int lane_now() { int l; asm volatile("v_mbcnt_lo_u32_b32 %0, -1, 0\n\tv_mbcnt_hi_u32_b32 %0, -1, %0" : "=v"(l)); return l; }
DI bool give_up(char* ws, unsigned call, unsigned long long t0) {
    if (rfl(ld_ag(ws + OFF_TMO)) == call || __builtin_amdgcn_s_memrealtime() - t0 > DEADLINE) { st_ag(ws + OFF_TMO, call); return true; }
    return false;
}
DI bool arrive(char* ws, char* lane_ws, int rank, int kind, unsigned n, unsigned call) {
    const int lane = lane_now();
    unsigned t = 0;
    if (lane == 0) t = __hip_atomic_fetch_add((gu32*)(ws + OFF_CNT + kind * 128), 1u, __ATOMIC_RELAXED, __HIP_MEMORY_SCOPE_AGENT);
    const bool last = rfl(t) + 1 == n * call;
    if (last && lane < NR) st_sys(lane_ws + OFF_EV + (kind * NR + rank) * 4, call);
    return last;
}
DI bool wait_words(char* ws, const char* f, int n, int step, unsigned want, unsigned call) {
    const int lane = lane_now();
    const unsigned long long t0 = __builtin_amdgcn_s_memrealtime();
    for (;;) {
        bool ok = true;
#pragma unroll
        for (int i = 0; i < 4; ++i) { const int k = lane + 64 * i; if (64 * i < n) ok &= ld_sys(f + (k < n ? k : 0) * step) >= want; }
        if (__all(ok)) break;
        if (give_up(ws, call, t0)) return false;
        __builtin_amdgcn_s_sleep(4);
    }
    asm volatile("" ::: "memory");
    return true;
}

DI void store1(const char* base, unsigned v, const u32x4& f) {
    asm volatile("s_nop 4\n\tglobal_store_dwordx4 %0, %1, %2 sc0 sc1\n\ts_nop 1" :: "v"(v), "v"(f), "s"(base) : "memory");
}
DI void store3(const char* base, unsigned v, const u32x4& a, const u32x4& b, const u32x4& c) {
    asm volatile("s_nop 4\n\tglobal_store_dwordx4 %0, %1, %4 sc0 sc1\n\t"
                 "global_store_dwordx4 %0, %2, %4 offset:1024 sc0 sc1\n\t"
                 "global_store_dwordx4 %0, %3, %4 offset:2048 sc0 sc1\n\ts_nop 1"
                 :: "v"(v), "v"(a), "v"(b), "v"(c), "s"(base) : "memory");
}
DI void store4(const char* base, unsigned v, const u32x4 (&f)[4]) {
    asm volatile("s_nop 4\n\tglobal_store_dwordx4 %0, %1, %5 sc0 sc1\n\t"
                 "global_store_dwordx4 %0, %2, %5 offset:1024 sc0 sc1\n\t"
                 "global_store_dwordx4 %0, %3, %5 offset:2048 sc0 sc1\n\t"
                 "global_store_dwordx4 %0, %4, %5 offset:3072 sc0 sc1\n\ts_nop 1"
                 :: "v"(v), "v"(f[0]), "v"(f[1]), "v"(f[2]), "v"(f[3]), "s"(base) : "memory");
}
DI void store4o(const char* base, unsigned v, const u32x4 (&f)[4]) {
    asm volatile("s_nop 4\n\tglobal_store_dwordx4 %0, %4, %8 sc1 nt\n\t"
                 "global_store_dwordx4 %1, %5, %8 sc1 nt\n\t"
                 "global_store_dwordx4 %2, %6, %8 sc1 nt\n\t"
                 "global_store_dwordx4 %3, %7, %8 sc1 nt\n\ts_nop 1"
                 :: "v"(v), "v"(v + 2 * ROWB), "v"(v + 4 * ROWB), "v"(v + 6 * ROWB), "v"(f[0]), "v"(f[1]), "v"(f[2]), "v"(f[3]), "s"(base) : "memory");
}

DI unsigned perm(unsigned hi, unsigned lo, unsigned sel) { return __builtin_amdgcn_perm(hi, lo, sel); }
template <int CTRL, int ROWMASK> DI unsigned dpp0(unsigned x) { return __builtin_amdgcn_update_dpp(0u, x, CTRL, ROWMASK, 0xf, false); }
DI unsigned wave_sum(unsigned x) {
    x += dpp0<0x111, 0xf>(x); x += dpp0<0x112, 0xf>(x); x += dpp0<0x114, 0xf>(x); x += dpp0<0x118, 0xf>(x); x += dpp0<0x142, 0xa>(x); x += dpp0<0x143, 0xc>(x);
    return __builtin_amdgcn_readlane(x, 63);
}
DI unsigned umax(unsigned a, unsigned b) { return a > b ? a : b; }
DI unsigned wave_max(unsigned x) {
    x = umax(x, dpp0<0x111, 0xf>(x)); x = umax(x, dpp0<0x112, 0xf>(x)); x = umax(x, dpp0<0x114, 0xf>(x)); x = umax(x, dpp0<0x118, 0xf>(x));
    x = umax(x, dpp0<0x142, 0xa>(x)); x = umax(x, dpp0<0x143, 0xc>(x));
    return __builtin_amdgcn_readlane(x, 63);
}
template <unsigned NE> struct ExcList {
    volatile LAS unsigned short* e; unsigned cnt;
    DI void begin(LAS unsigned char* tail, int lane) { e = (volatile LAS unsigned short*)(tail + 512); e[lane] = 0; cnt = 1; }
    DI bool add(bool act, unsigned entry) {
        const unsigned long long bal = __ballot(act);
        const unsigned slot = cnt + __builtin_amdgcn_mbcnt_hi((unsigned)(bal >> 32), __builtin_amdgcn_mbcnt_lo((unsigned)bal, 0u));
        if (act && slot < NE) e[slot] = (unsigned short)entry;
        cnt += __builtin_popcountll(bal);
        return cnt <= NE;
    }
};
template <unsigned NE> DI bool pack_finish(const char* base, unsigned blk, int c, int lane, LAS unsigned char* tail, ExcList<NE>& x, unsigned ebase,
                    const u32x4& p0, const u32x4& p1, const u32x4& p2, unsigned t0, unsigned t1) {
    if (lane == 0) x.e[0] = (unsigned short)(ebase | (x.cnt > NE ? 0x8000u : 0u));
    *(volatile LAS u32x2*)(tail + lane * 8) = (u32x2){t0, t1};
    constexpr int TS = 512 + 2 * NE;
    store3(base, blk + 4 * TS + c * 3072 + lane * 16, p0, p1, p2);
    const u32x4 t = *(volatile LAS u32x4*)(tail + (lane < TS / 16 ? lane : 0) * 16);
    if (lane < TS / 16) store1(base, blk + c * TS + lane * 16, t);
    return x.cnt <= NE;
}
DI void pack16(const u32x4 (&f)[4], const char* base, unsigned blk, int c, int lane, LAS unsigned char* tail) {
    unsigned lo[8], h[8], g[8], sum = 0;
#pragma unroll
    for (int i = 0; i < 8; ++i) {
        const unsigned a = f[i >> 1][2 * (i & 1)], b = f[i >> 1][2 * (i & 1) + 1];
        lo[i] = perm(b, a, 0x06040200u); h[i] = perm(b, a, 0x07050301u);
        sum = __builtin_amdgcn_udot4(h[i] & 0x7c7c7c7cu, 0x01010101u, sum, false);
    }
    const int top = (int)((wave_sum(sum) + 15155u + 8191u) >> 13);
    const unsigned eb = top < 7 ? 0u : top > 31 ? 24u : (unsigned)(top - 7), b4 = eb * 0x04040404u;
    ExcList<NE16> x; x.begin(tail, lane);
    unsigned m1 = 0, m0 = 0;
#pragma unroll
    for (int i = 0; i < 8; ++i) {
        g[i] = (h[i] & 0x83838383u) | (((h[i] | 0x80808080u) - b4) & 0x1c1c1c1cu);
        const unsigned xr = (g[i] + b4) ^ h[i];
        m1 |= ((xr >> 6) & 0x01010101u) << i; m0 |= ((xr >> 5) & 0x01010101u) << i;
    }
    for (unsigned m = m1 | m0; __any(m != 0); m &= m - 1) {
        const int k = m ? __builtin_ctz(m) : 0, i = k & 7;
        if (!x.add(m != 0, ((i >> 2) * 1024 + lane * 16 + (i & 3) * 4 + (k >> 3)) | ((m0 >> k) & 1u) << 11 | ((m1 >> k) & 1u) << 12)) break;
    }
    unsigned d[6];
#pragma unroll
    for (int q = 0; q < 2; ++q) {
        const unsigned s = g[4 * q + 3];
        d[3 * q] = g[4 * q] | (s & 0x03030303u) << 5; d[3 * q + 1] = g[4 * q + 1] | (s & 0x0c0c0c0cu) << 3;
        d[3 * q + 2] = g[4 * q + 2] | (s & 0x10101010u) << 1 | (s & 0x80808080u) >> 1;
    }
    pack_finish(base, blk, c, lane, tail, x, eb, (u32x4){lo[0], lo[1], lo[2], lo[3]}, (u32x4){lo[4], lo[5], lo[6], lo[7]}, (u32x4){d[0], d[1], d[2], d[3]}, d[4], d[5]);
}
DI bool pack32(const u32x4 (&f)[4], const char* base, unsigned blk, int c, int lane, LAS unsigned char* tail) {
    unsigned t[12], h[4], mx = 0;
#pragma unroll
    for (int j = 0; j < 4; ++j) {
        t[3 * j] = perm(f[j][3], f[j][0], 0x04020100u); t[3 * j + 1] = perm(f[j][3], f[j][1], 0x05020100u); t[3 * j + 2] = perm(f[j][3], f[j][2], 0x06020100u);
        h[j] = perm(perm(f[j][3], f[j][2], 0x07030703u), perm(f[j][1], f[j][0], 0x07030703u), 0x05040100u);
#pragma unroll
        for (int d = 0; d < 4; ++d) mx = umax(mx, f[j][d] & 0x7fffffffu);
    }
    const unsigned top = wave_max(mx) >> 24, eb = top < 7 ? 0u : top - 7, b1 = eb * 0x01010101u;
    ExcList<NE32> x; x.begin(tail, lane);
    unsigned cd[4], xn[2] = {0, 0};
#pragma unroll
    for (int j = 0; j < 4; ++j) {
        cd[j] = ((h[j] | 0x80808080u) - b1) & 0x07070707u;
        const unsigned xr = ((cd[j] + b1) ^ h[j]) & 0x78787878u;
        xn[j >> 1] |= (j & 1) ? xr << 1 : xr >> 3;
    }
    for (unsigned long long m = (unsigned long long)xn[1] << 32 | xn[0]; __any(m != 0);) {
        const int k = m ? __builtin_ctzll(m) >> 2 : 0, j = 2 * (k >> 3) + (k & 1);
        const unsigned v = (unsigned)(m >> (4 * k)) & 15u;
        if (!x.add(m != 0, (lane * 16 + j * 4 + ((k >> 1) & 3)) | v << 11)) break;
        m &= ~(15ull << (4 * k));
    }
    const unsigned n0 = cd[0] | (h[0] & 0x80808080u) >> 4 | cd[1] << 4 | (h[1] & 0x80808080u), n1 = cd[2] | (h[2] & 0x80808080u) >> 4 | cd[3] << 4 | (h[3] & 0x80808080u);
    return pack_finish(base, blk, c, lane, tail, x, eb, (u32x4){t[0], t[1], t[2], t[3]}, (u32x4){t[4], t[5], t[6], t[7]}, (u32x4){t[8], t[9], t[10], t[11]}, n0, n1);
}
DI bool unpack(bool is32, const LAS unsigned char* main3, const LAS unsigned char* tail, LAS unsigned char* patch, int lane, u32x4 (&f)[4]) {
    unsigned e = *(const volatile LAS unsigned short*)(tail + 512 + lane * 2);
    const u32x2 tt = *(const volatile LAS u32x2*)(tail + lane * 8);
    const u32x4 p0 = *(const volatile LAS u32x4*)(main3 + lane * 16), p1 = *(const volatile LAS u32x4*)(main3 + 1024 + lane * 16), p2 = *(const volatile LAS u32x4*)(main3 + 2048 + lane * 16);
    const unsigned hdr = rfl(e);
    if (lane == 0 || lane >= (int)(is32 ? NE32 : NE16)) e = 0;
    LAS unsigned* const pw = (LAS unsigned*)(patch + (e & 0x7fcu));
    __hip_atomic_fetch_or(pw, (e >> 11) << ((is32 ? 3 : 5) + 8 * (e & 3u)), __ATOMIC_RELAXED, __HIP_MEMORY_SCOPE_WAVEFRONT);
    const u32x4 x0 = *(const volatile LAS u32x4*)(patch + lane * 16), x1 = *(const volatile LAS u32x4*)(patch + 1024 + lane * 16);
    *(volatile LAS unsigned*)pw = 0;
    if (is32) {
        const unsigned b1 = (hdr & 127u) * 0x01010101u, t[12] = {p0[0], p0[1], p0[2], p0[3], p1[0], p1[1], p1[2], p1[3], p2[0], p2[1], p2[2], p2[3]};
#pragma unroll
        for (int j = 0; j < 4; ++j) {
            const unsigned n = tt[j >> 1];
            const unsigned h = ((j & 1) ? ((((n >> 4) & 0x07070707u) + b1) | (n & 0x80808080u)) : (((n & 0x07070707u) + b1) | ((n << 4) & 0x80808080u))) ^ x0[j];
            f[j][0] = perm(h, t[3 * j], 0x04020100u); f[j][1] = perm(h, t[3 * j + 1], 0x05020100u); f[j][2] = perm(h, t[3 * j + 2], 0x06020100u);
            f[j][3] = perm(perm(h, t[3 * j + 2], 0x07030703u), perm(t[3 * j + 1], t[3 * j], 0x07030703u), 0x05040100u);
        }
    } else {
        const unsigned b4 = (hdr & 31u) * 0x04040404u, d[6] = {p2[0], p2[1], p2[2], p2[3], tt[0], tt[1]};
#pragma unroll
        for (int q = 0; q < 2; ++q) {
            const unsigned a = d[3 * q], b = d[3 * q + 1], c = d[3 * q + 2];
            const unsigned g[4] = {a & 0x9f9f9f9fu, b & 0x9f9f9f9fu, c & 0x9f9f9f9fu,
                                   ((a >> 5) & 0x03030303u) | ((b >> 3) & 0x0c0c0c0cu) | ((c >> 1) & 0x10101010u) | ((c << 1) & 0x80808080u)};
#pragma unroll
            for (int i = 0; i < 4; ++i) {
                const unsigned h = (g[i] + b4) ^ (q ? x1[i] : x0[i]), lo = q ? p1[i] : p0[i];
                f[2 * q + (i >> 1)][2 * (i & 1)] = perm(h, lo, 0x05010400u); f[2 * q + (i >> 1)][2 * (i & 1) + 1] = perm(h, lo, 0x07030602u);
            }
        }
    }
    return hdr < 0x8000u;
}

constexpr int PSHORT = 16 + NBB - 2;
DI bool two_quads(int panel) { return panel < 16 || (unsigned)(panel - PSHORT) < 2u; }
DI int col0(int cb) { return cb >= NBB ? BCOLS + (cb - NBB) * TW : cb == NBB - 1 ? BCOLS - 256 : cb * TW; }
DI int colblock(int li, int rank) { return li < CPR ? NBB + CPR * rank + li : li - CPR; }
DI unsigned own_off(int panel, int kt) { return (panel < 16 ? panel * XPANEL : REL_WL + (panel - 16) * 8 * WBLK) + kt * (two_quads(panel) ? XBLK : WBLK); }
DI unsigned ring_off(int li) { return REL_WR + (li >= RING ? li - RING : li) * (56 * WBLK); }
DI unsigned pk16_off(int panel, int kt) { return OFF_PK16 + ((panel < 16 ? panel * 16 : 256 + (panel - 16) * 24) + kt * (two_quads(panel) ? 2 : 3)) * QB16; }
static_assert(2 * RING >= NL, "ring");
constexpr int round_end(int r) { return (NWB * (r + 1) - (r ? NJOB : 0) + RB - 1) / RB; }
constexpr int XL[4] = {0, 1, 3, 3}, NSTEP0 = 8 + XL[3];
constexpr int x_lo(int s) { int g = 0; while (g < 4 && s - XL[g] >= 8) ++g; return 4 * g; }
constexpr int x_hi(int s) { int g = 0; while (g < 4 && s >= XL[g]) ++g; return 4 * g; }
constexpr int B0 = round_end(0) - CPR, W0Q = 3 * (B0 + NCB);
constexpr int cstep_start(int s) { int n = 0; for (int k = 0; k < s; ++k) n += 2 * (x_hi(k) - x_lo(k)) + (k < 8 ? W0Q : 0); return n; }
constexpr int steps_within(int quads) { int s = 0; while (s < NSTEP0 && cstep_start(s + 1) <= quads) ++s; return s; }
constexpr int ST1 = 3, STEPS1 = steps_within(NGEMM), STEPS2 = steps_within(ST1 * NGEMM);
static_assert(STEPS1 >= 1 && NQ16 > ST1 * NGEMM + NGEMM, "stages");
DI void conv_quad(int b, int& panel, int& kt, int& qd) {
    if (b < cstep_start(NSTEP0)) {
        int s = 0, i = b, lo = x_lo(0), nx = 2 * (x_hi(0) - x_lo(0));
#pragma unroll
        for (int k = 1; k < NSTEP0; ++k) if (b >= cstep_start(k)) { s = k; i = b - cstep_start(k); lo = x_lo(k); nx = 2 * (x_hi(k) - x_lo(k)); }
        if (i < nx) { panel = lo + (i >> 1); qd = i & 1; kt = s - (panel < 4 ? XL[0] : panel < 8 ? XL[1] : panel < 12 ? XL[2] : XL[3]); }
        else { const int w = (i - nx) / 3; qd = i - nx - 3 * w; panel = 16 + (w < B0 ? w : w - B0 + NBB); kt = s; }
    } else if (b < NQ16 - 32) { const int r = b - cstep_start(NSTEP0), k = r / 3; qd = r - 3 * k; panel = 16 + B0 + (k >> 3); kt = k & 7; }
    else { const int r = b - (NQ16 - 32); panel = PSHORT + (r >> 4); kt = r >> 1 & 7; qd = r & 1; }
}
DI unsigned draw_call(char* ws, int wid, bool& clear) {
    const int lane = lane_now();
    unsigned v;
    asm volatile("s_nop 4\n\tglobal_load_dword %0, %1, %2 sc0 sc1\n\ts_waitcnt vmcnt(0)"
                 : "=&v"(v) : "v"(lane < 56 ? OFF_CALL + blockIdx.x * 128 : OFF_EV + (4 * NR + lane - 56) * 4), "s"(ws) : "memory");
    const unsigned c = rfl(v) + 1;
    clear = __all(v >= c - 1);
    BARRIER;
    if (wid == 0 && lane == 0) st_ag(ws + OFF_CALL + blockIdx.x * 128, c);
    return c;
}
DI unsigned convert_shards(Params& p, LAS unsigned char* lds, int wid, int gi) {
    const int lane = lane_now(), rank = p.rank;
    char* const ws = p.ws[rank];
    const __amdgpu_buffer_rsrc_t rx = __builtin_amdgcn_make_buffer_rsrc((void*)p.x, 0, MM * KS * 4, 0x00020000);
    const __amdgpu_buffer_rsrc_t rw = __builtin_amdgcn_make_buffer_rsrc((void*)p.w, 0, KS * NN * 4, 0x00020000);
    const unsigned vx = ((lane & 15) * KS + 8 * (lane >> 4)) * 4, vw = (8 * (lane >> 4) * NN + 4 * (lane & 15)) * 4;
    LAS unsigned char* const ring = lds + wid * 32768;
    LAS unsigned char* const tail = lds + 131072 + wid * 1024;
    const int nu = (NQ16 - gi + NGEMM - 1) / NGEMM;
    auto load = [&](int u) {
        int pa, kt, qd; conv_quad(gi + (u < nu ? u : 0) * NGEMM, pa, kt, qd);
        LAS unsigned char* const slot = ring + (u & 3) * 8192;
        if (pa < 16) {
            const unsigned s0 = ((pa * 256 + 128 * (wid >> 1) + 64 * qd) * KS + 32 * (2 * kt + (wid & 1))) * 4;
#pragma unroll
            for (int e = 0; e < 8; ++e) __builtin_amdgcn_raw_ptr_buffer_load_lds(rx, (LAS void*)(slot + e * 1024), 16, vx, s0 + (e >> 1) * (16 * KS * 4) + (e & 1) * 16, 0, AUX_NT);
        } else {
            const unsigned s0 = (32 * (2 * kt + (wid & 1)) * NN + col0(pa - 16) + (two_quads(pa) ? 128 : 192) * (wid >> 1) + 64 * qd) * 4;
#pragma unroll
            for (int e = 0; e < 8; ++e) __builtin_amdgcn_raw_ptr_buffer_load_lds(rw, (LAS void*)(slot + e * 1024), 16, vw, s0 + e * ROWB, 0, AUX_NT);
        }
    };
    load(0);
    char* peer_ws = p.ws[lane & 7];
    bool clear;
    const unsigned call = draw_call(ws, wid, clear);
    if (!clear) wait_words(ws, ws + OFF_EV + 4 * NR * 4, NR, 4, call - 1, call);
    asm volatile("" : "+v"(peer_ws));
    for (int u = 0; u < nu; ++u) {
        if (u > 0) { load(u + 3); WAITVM(24); }
        if (u == ST1 + 1) {
            BARRIER;
            if (wid == 0) arrive(ws, peer_ws, rank, 2, NGEMM, call);
        }
        int pa, kt, qd; conv_quad(gi + u * NGEMM, pa, kt, qd);
        f32x4 v[8]; u32x4 f[4];
#pragma unroll
        for (int e = 0; e < 8; ++e) v[e] = *(const LAS f32x4*)(ring + (u & 3) * 8192 + e * 1024 + lane * 16);
#pragma unroll
        for (int j = 0; j < 4; ++j) { f16x8 h;
#pragma unroll
            for (int e = 0; e < 8; ++e) h[e] = (_Float16)(pa < 16 ? v[2 * j + (e >> 2)][e & 3] : v[e][j] * WSCALE);
            f[j] = __builtin_bit_cast(u32x4, h); }
        pack16(f, ws, pk16_off(pa, kt) + qd * QB16, wid, lane, tail);
        store4(ws, OFF_DATA + own_off(pa, kt) + (4 * qd + wid) * 4096 + lane * 16, f);
        if (u == 0) {
            WAITVM(0);
            BARRIER;
            if (wid == 0 && lane < NR) st_sys(peer_ws + OFF_AV + (rank * 256 + gi) * 4, call);
            load(1); load(2); load(3);
            if (wid == 1) arrive(ws, peer_ws, rank, 1, NGEMM, call);
        }
    }
    WAITVM(0);
    BARRIER;
    if (wid == 0) arrive(ws, peer_ws, rank, 3, NGEMM, call);
    return call;
}

constexpr int G0 = 16 + round_end(0), NBI = (16 + NL) * 8, NTW = 7 * NBI, NTC = 7 * NCI, NT = NTW + NTC;
struct Item { const char* src; unsigned size, pk, raw, dof; int flag; bool three; };
constexpr int step_start(int s) { int n = 0; for (int k = 0; k < s; ++k) n += x_hi(k) - x_lo(k) + (k < 8 ? G0 - 16 : 0); return n; }
static_assert(step_start(NSTEP0) == 8 * G0, "order");
DI void block_of(unsigned t, int& i, int& kt) {
    if (t < 8u * G0) {
        int s = 0, hi = x_hi(0); i = t;
#pragma unroll
        for (int k = 1; k < NSTEP0; ++k) if (t >= (unsigned)step_start(k)) { s = k; i = t - step_start(k) + x_lo(k); hi = x_hi(k); }
        if (i >= hi) i += 16 - hi;
        kt = s - (i < 4 ? XL[0] : i < 8 ? XL[1] : i < 12 ? XL[2] : i < 16 ? XL[3] : 0);
    } else {
        unsigned u = 0, n = 1, b = 0;
#pragma unroll
        for (int r = 1; r < NROUND; ++r)
            if (t >= 8u * (16 + round_end(r - 1))) { b = 16 + round_end(r - 1); u = t - 8u * b; n = round_end(r) - round_end(r - 1); }
        kt = u / n; i = b + (u - kt * n);
    }
}
DI unsigned slab_out(unsigned s, int from) { return (s & 127u) * 32u * ROWB + (BCOLS + (CPR * from + (s >> 7)) * TW) * 4u; }
DI Item make_item(Params& p, unsigned t) {
    const int rank = p.rank;
    const unsigned tt = t < (unsigned)NTW ? t : t < (unsigned)NT ? t - NTW : 0, u = tt / 7, pj = tt - 7 * u;
    const int peer = (rank + 1 + pj) & 7;
    Item it;
    it.src = p.ws[peer]; it.size = t < (unsigned)NT ? WS_NEED : 0;
    if (t < (unsigned)NTW) {
        int i, kt; block_of(u, i, kt);
        const int panel = i < 16 ? i : 16 + colblock(i - 16, rank);
        it.three = !two_quads(panel);
        it.pk = pk16_off(panel, kt); it.raw = OFF_DATA + own_off(panel, kt);
        it.dof = OFF_DATA + (i < 16 ? i * XPANEL + 8 * XBLK : ring_off(i - 16)) + (7 * kt + pj) * (it.three ? WBLK : XBLK);
        it.flag = panel * 64 + 8 + 7 * kt + pj;
    } else {
        it.three = true;
        it.pk = OFF_PK32 + u * (3 * QB32); it.raw = OFF_RAW32 + u * WBLK; it.dof = slab_out(u, peer); it.flag = -1;
    }
    return it;
}
DI void load_main(const Item& it, unsigned off, LAS unsigned char* to, int lane) {
    const __amdgpu_buffer_rsrc_t r = __builtin_amdgcn_make_buffer_rsrc((void*)it.src, 0, it.size, 0x00020000);
    asm volatile("s_mov_b32 m0, %3\n\ts_nop 4\n\tbuffer_load_dwordx4 %0, %1, %2 offen sc0 sc1 lds\n\tbuffer_load_dwordx4 %0, %1, %2 offen offset:1024 sc0 sc1 lds\n\t"
                 "buffer_load_dwordx4 %0, %1, %2 offen offset:2048 sc0 sc1 lds" :: "v"(lane * 16), "s"(r), "s"(off), "s"((unsigned)(size_t)to) : "memory");
}
DI void load_tails(const Item& it, unsigned off, LAS unsigned char* to, int lane) {
    const __amdgpu_buffer_rsrc_t r = __builtin_amdgcn_make_buffer_rsrc((void*)it.src, 0, it.size, 0x00020000);
    const unsigned long long rest = it.flag < 0 ? (1ull << NE32 / 2) - 1 : (1ull << NE16 / 2) - 1;
    asm volatile("s_mov_b32 m0, %3\n\ts_nop 4\n\tbuffer_load_dwordx4 %0, %1, %2 offen sc0 sc1 lds\n\tbuffer_load_dwordx4 %0, %1, %2 offen offset:1024 sc0 sc1 lds\n\t"
                 "s_mov_b64 exec, %4\n\tbuffer_load_dwordx4 %0, %1, %2 offen offset:2048 sc0 sc1 lds\n\ts_mov_b64 exec, -1"
                 :: "v"(lane * 16), "s"(r), "s"(off), "s"((unsigned)(size_t)to), "s"(rest) : "memory");
}
constexpr unsigned NANS = 0x7fc07fc0u;
constexpr unsigned LQ = 14848, LMAIN = 2560;
struct Fetch {
    const char* ws; const char* out;
    LAS unsigned char* lds;
    unsigned slot; int lane; bool ok;
    template <int C0, bool WAITED> DI void quad(const Item& cur, unsigned qd, const Item& nx, unsigned nqd) {
        LAS unsigned char* const q = lds + slot; LAS unsigned char* const nq = lds + (LQ - slot);
        const unsigned npk = nx.pk + nqd * (nx.flag < 0 ? QB32 : QB16), nmain = npk + 4 * (nx.flag < 0 ? TS32 : TS16), ts = cur.flag < 0 ? TS32 : TS16;
#pragma unroll
        for (int c = 0; c < 4; ++c) {
            if (c > 0 || !WAITED) WAITVM(25);
            if (c == 0) load_tails(nx, npk, nq, lane);
            load_main(nx, nmain + c * 3072, nq + LMAIN + c * 3072, lane);
            u32x4 f[4];
            if (!unpack(cur.flag < 0, q + LMAIN + c * 3072, q + c * ts, lds + 2 * LQ, lane, f)) {
                const char* a = cur.src + (size_t)(cur.raw + qd * QUAD + c * 4096 + lane * 16);
                asm volatile("global_load_dwordx4 %0, %4, off sc0 sc1\n\tglobal_load_dwordx4 %1, %4, off offset:1024 sc0 sc1\n\t"
                             "global_load_dwordx4 %2, %4, off offset:2048 sc0 sc1\n\tglobal_load_dwordx4 %3, %4, off offset:3072 sc0 sc1\n\ts_waitcnt vmcnt(0)"
                             : "=&v"(f[0]), "=&v"(f[1]), "=&v"(f[2]), "=&v"(f[3]) : "v"(a) : "memory");
            }
            if (!ok) f[0] = f[1] = f[2] = f[3] = (u32x4){NANS, NANS, NANS, NANS};
            if (cur.flag >= 0) store4(ws, cur.dof + qd * QUAD + c * 4096 + lane * 16, f);
            else store4o(out, cur.dof + ((C0 + c) / 3 * 8 + (lane >> 5)) * ROWB + (C0 + c) % 3 * 512 + (lane & 31) * 16, f);
        }
        slot = LQ - slot;
    }
};
DI void fetch_wave(Params& p, LAS unsigned char* lds, unsigned call, int wid, int wv) {
    const int lane = lane_now(), rank = p.rank;
    char* const ws = p.ws[rank];
    Fetch F;
    F.ws = ws; F.out = (const char*)p.out[rank]; F.lds = lds; F.lane = lane;
    *(LAS u32x4*)(lds + 2 * LQ + lane * 16) = (u32x4){0, 0, 0, 0}; *(LAS u32x4*)(lds + 2 * LQ + 1024 + lane * 16) = (u32x4){0, 0, 0, 0};
    const unsigned wbase = (call - 1) * NTW - NCW, cbase = (call - 1) * NTC - NCW;
    static_assert(NL - RING == 32, "probe lanes");
    const unsigned vprobe = lane < 32 ? OFF_EV + lane * 4 : OFF_FIN + (lane - 32) * 4;
    static_assert(NCW <= 7 * step_start(1), "first items");
    if (wv == 0 && lane < NR) st_sys(p.ws[lane] + OFF_EV + (5 * NR + rank) * 4, call);
    {
        int i0, kt0; block_of(wv / 7, i0, kt0);
        const int q0 = i0 < 16 ? 2 * i0 : 2 * x_hi(0) + 3 * (i0 < 16 + CPR ? B0 + CPR * rank + i0 - 16 : i0 - 16 - CPR), peer = (rank + 1 + wv % 7) & 7;
        const unsigned v = lane < NR ? OFF_EV + (5 * NR + lane) * 4 : OFF_AV + peer * 1024 + (q0 + lane % (i0 < 16 ? 2 : 3)) * 4;
        const unsigned long long t0 = __builtin_amdgcn_s_memrealtime();
        F.ok = true;
        for (unsigned n = 1; !__all(ld_sys(ws + v) >= call); ++n) if ((n & 15) == 0 && give_up(ws, call, t0)) { F.ok = false; break; }
        asm volatile("" ::: "memory");
    }
    unsigned tw = wv, tc = wv;
    auto choose = [&](unsigned probe) -> unsigned {
        const unsigned ev = (unsigned)__ballot(probe >= call);
        if (tw < (unsigned)NTW) {
            const unsigned u = tw / 7;
            int i, kt; block_of(u, i, kt);
            const int stage = u < (unsigned)step_start(STEPS1) ? 1 : u < (unsigned)step_start(STEPS2) ? 2 : 3;
            if ((ev >> (8 * stage + ((rank + 1 + tw - 7 * u) & 7)) & 1) &&
                (i < 16 + RING || __builtin_amdgcn_readlane(probe, 32 + i - 16 - RING) >= 16 * call)) return tw;
        }
        return tc < (unsigned)NTC && (ev >> ((rank + 1 + tc % 7) & 7) & 1) ? NTW + tc : NT;
    };
    auto publish = [&](int flag) {
        if (flag >= 0 && lane == 0) st_sys(ws + OFF_F + flag * 4, call);
    };
    while (tw < (unsigned)NTW || tc < (unsigned)NTC) {
        unsigned t = tw;
        if (tw >= (unsigned)NCW) {
            const unsigned long long t0 = __builtin_amdgcn_s_memrealtime();
            while ((t = choose(ld_sys(ws + vprobe))) == (unsigned)NT) {
                if (give_up(ws, call, t0)) { F.ok = false; t = tw < (unsigned)NTW ? tw : NTW + tc; break; }
                __builtin_amdgcn_s_sleep(16);
            }
        }
        Item cur = make_item(p, t);
        F.slot = 0;
        load_tails(cur, cur.pk, lds, lane);
#pragma unroll
        for (int c = 0; c < 4; ++c) load_main(cur, cur.pk + 4 * (cur.flag < 0 ? TS32 : TS16) + c * 3072, lds + LMAIN + c * 3072, lane);
        WAITVM(0);
        int pflag = -1;
        for (;;) {
            unsigned tk, probe; unsigned long long sv;
            asm volatile("s_nop 3\n\ts_waitcnt vmcnt(25)\n\tglobal_load_dword %1, %6, %5 sc0 sc1\n\t"
                         "s_mov_b64 %2, exec\n\ts_mov_b64 exec, 1\n\tglobal_atomic_add %0, %3, %4, %5 sc0\n\ts_mov_b64 exec, %2"
                         : "=&v"(tk), "=&v"(probe), "=&s"(sv) : "v"(OFF_TICK + (cur.flag < 0 ? 128u : 0u)), "v"(1u), "s"(ws), "v"(vprobe) : "memory");
            F.quad<0, true>(cur, 0, cur, 1);
            asm volatile("s_waitcnt vmcnt(25)" : "+v"(tk), "+v"(probe) :: "memory");
            publish(pflag);
            if (cur.flag >= 0) tw = rfl(tk) - wbase; else tc = rfl(tk) - cbase;
            if (cur.three) F.quad<4, true>(cur, 1, cur, 2);
            t = choose(probe);
            if (t == (unsigned)NT) t = choose(ld_sys(ws + vprobe));
            const Item nxt = make_item(p, t);
            F.quad<8, false>(cur, cur.three ? 2 : 1, nxt, 0);
            pflag = cur.flag;
            if (t >= (unsigned)NT) break;
            cur = nxt;
        }
        WAITVM(0);
        publish(pflag);
    }
    arrive(ws, p.ws[lane & 7], rank, 4, NCW, call);
}
#define PT_SET(A0, A1, A2, A3, R) \
    asm volatile("s_waitcnt vmcnt(60)\n\t" R \
                 "global_load_dwordx4 a[" A0 "], %16, %20 sc1 nt\n\tglobal_load_dwordx4 a[" A1 "], %17, %20 sc1 nt\n\t" \
                 "global_load_dwordx4 a[" A2 "], %18, %20 sc1 nt\n\tglobal_load_dwordx4 a[" A3 "], %19, %20 sc1 nt" \
                 : "=&v"(r[0]), "=&v"(r[1]), "=&v"(r[2]), "=&v"(r[3]), "=&v"(r[4]), "=&v"(r[5]), "=&v"(r[6]), "=&v"(r[7]), \
                   "=&v"(r[8]), "=&v"(r[9]), "=&v"(r[10]), "=&v"(r[11]), "=&v"(r[12]), "=&v"(r[13]), "=&v"(r[14]), "=&v"(r[15]) \
                 : "v"(v), "v"(v + 2 * ROWB), "v"(v + 4 * ROWB), "v"(v + 6 * ROWB), "s"(out) : "memory", PT_CLOB)
#define PT_CLOB "a0", "a1", "a2", "a3", "a4", "a5", "a6", "a7", "a8", "a9", "a10", "a11", "a12", "a13", "a14", "a15", "a16", "a17", "a18", "a19", "a20", "a21", "a22", "a23", "a24", "a25", "a26", "a27", "a28", "a29", "a30", "a31", "a32", "a33", "a34", "a35", "a36", "a37", "a38", "a39", "a40", "a41", "a42", "a43", "a44", "a45", "a46", "a47", "a48", "a49", "a50", "a51", "a52", "a53", "a54", "a55", "a56", "a57", "a58", "a59", "a60", "a61", "a62", "a63", "a64", "a65", "a66", "a67", "a68", "a69", "a70", "a71", "a72", "a73", "a74", "a75", "a76", "a77", "a78", "a79", "a80", "a81", "a82", "a83", "a84", "a85", "a86", "a87", "a88", "a89", "a90", "a91", "a92", "a93", "a94", "a95", "a96", "a97", "a98", "a99", "a100", "a101", "a102", "a103", "a104", "a105", "a106", "a107", "a108", "a109", "a110", "a111", "a112", "a113", "a114", "a115", "a116", "a117", "a118", "a119", "a120", "a121", "a122", "a123", "a124", "a125", "a126", "a127"
#define PT_R0 "v_accvgpr_read_b32 %0, a0\n\t" "v_accvgpr_read_b32 %1, a1\n\t" "v_accvgpr_read_b32 %2, a2\n\t" "v_accvgpr_read_b32 %3, a3\n\t" "v_accvgpr_read_b32 %4, a4\n\t" "v_accvgpr_read_b32 %5, a5\n\t" "v_accvgpr_read_b32 %6, a6\n\t" "v_accvgpr_read_b32 %7, a7\n\t" "v_accvgpr_read_b32 %8, a8\n\t" "v_accvgpr_read_b32 %9, a9\n\t" "v_accvgpr_read_b32 %10, a10\n\t" "v_accvgpr_read_b32 %11, a11\n\t" "v_accvgpr_read_b32 %12, a12\n\t" "v_accvgpr_read_b32 %13, a13\n\t" "v_accvgpr_read_b32 %14, a14\n\t" "v_accvgpr_read_b32 %15, a15\n\t"
#define PT_R1 "v_accvgpr_read_b32 %0, a16\n\t" "v_accvgpr_read_b32 %1, a17\n\t" "v_accvgpr_read_b32 %2, a18\n\t" "v_accvgpr_read_b32 %3, a19\n\t" "v_accvgpr_read_b32 %4, a20\n\t" "v_accvgpr_read_b32 %5, a21\n\t" "v_accvgpr_read_b32 %6, a22\n\t" "v_accvgpr_read_b32 %7, a23\n\t" "v_accvgpr_read_b32 %8, a24\n\t" "v_accvgpr_read_b32 %9, a25\n\t" "v_accvgpr_read_b32 %10, a26\n\t" "v_accvgpr_read_b32 %11, a27\n\t" "v_accvgpr_read_b32 %12, a28\n\t" "v_accvgpr_read_b32 %13, a29\n\t" "v_accvgpr_read_b32 %14, a30\n\t" "v_accvgpr_read_b32 %15, a31\n\t"
#define PT_R2 "v_accvgpr_read_b32 %0, a32\n\t" "v_accvgpr_read_b32 %1, a33\n\t" "v_accvgpr_read_b32 %2, a34\n\t" "v_accvgpr_read_b32 %3, a35\n\t" "v_accvgpr_read_b32 %4, a36\n\t" "v_accvgpr_read_b32 %5, a37\n\t" "v_accvgpr_read_b32 %6, a38\n\t" "v_accvgpr_read_b32 %7, a39\n\t" "v_accvgpr_read_b32 %8, a40\n\t" "v_accvgpr_read_b32 %9, a41\n\t" "v_accvgpr_read_b32 %10, a42\n\t" "v_accvgpr_read_b32 %11, a43\n\t" "v_accvgpr_read_b32 %12, a44\n\t" "v_accvgpr_read_b32 %13, a45\n\t" "v_accvgpr_read_b32 %14, a46\n\t" "v_accvgpr_read_b32 %15, a47\n\t"
#define PT_R3 "v_accvgpr_read_b32 %0, a48\n\t" "v_accvgpr_read_b32 %1, a49\n\t" "v_accvgpr_read_b32 %2, a50\n\t" "v_accvgpr_read_b32 %3, a51\n\t" "v_accvgpr_read_b32 %4, a52\n\t" "v_accvgpr_read_b32 %5, a53\n\t" "v_accvgpr_read_b32 %6, a54\n\t" "v_accvgpr_read_b32 %7, a55\n\t" "v_accvgpr_read_b32 %8, a56\n\t" "v_accvgpr_read_b32 %9, a57\n\t" "v_accvgpr_read_b32 %10, a58\n\t" "v_accvgpr_read_b32 %11, a59\n\t" "v_accvgpr_read_b32 %12, a60\n\t" "v_accvgpr_read_b32 %13, a61\n\t" "v_accvgpr_read_b32 %14, a62\n\t" "v_accvgpr_read_b32 %15, a63\n\t"
#define PT_R4 "v_accvgpr_read_b32 %0, a64\n\t" "v_accvgpr_read_b32 %1, a65\n\t" "v_accvgpr_read_b32 %2, a66\n\t" "v_accvgpr_read_b32 %3, a67\n\t" "v_accvgpr_read_b32 %4, a68\n\t" "v_accvgpr_read_b32 %5, a69\n\t" "v_accvgpr_read_b32 %6, a70\n\t" "v_accvgpr_read_b32 %7, a71\n\t" "v_accvgpr_read_b32 %8, a72\n\t" "v_accvgpr_read_b32 %9, a73\n\t" "v_accvgpr_read_b32 %10, a74\n\t" "v_accvgpr_read_b32 %11, a75\n\t" "v_accvgpr_read_b32 %12, a76\n\t" "v_accvgpr_read_b32 %13, a77\n\t" "v_accvgpr_read_b32 %14, a78\n\t" "v_accvgpr_read_b32 %15, a79\n\t"
#define PT_R5 "v_accvgpr_read_b32 %0, a80\n\t" "v_accvgpr_read_b32 %1, a81\n\t" "v_accvgpr_read_b32 %2, a82\n\t" "v_accvgpr_read_b32 %3, a83\n\t" "v_accvgpr_read_b32 %4, a84\n\t" "v_accvgpr_read_b32 %5, a85\n\t" "v_accvgpr_read_b32 %6, a86\n\t" "v_accvgpr_read_b32 %7, a87\n\t" "v_accvgpr_read_b32 %8, a88\n\t" "v_accvgpr_read_b32 %9, a89\n\t" "v_accvgpr_read_b32 %10, a90\n\t" "v_accvgpr_read_b32 %11, a91\n\t" "v_accvgpr_read_b32 %12, a92\n\t" "v_accvgpr_read_b32 %13, a93\n\t" "v_accvgpr_read_b32 %14, a94\n\t" "v_accvgpr_read_b32 %15, a95\n\t"
#define PT_R6 "v_accvgpr_read_b32 %0, a96\n\t" "v_accvgpr_read_b32 %1, a97\n\t" "v_accvgpr_read_b32 %2, a98\n\t" "v_accvgpr_read_b32 %3, a99\n\t" "v_accvgpr_read_b32 %4, a100\n\t" "v_accvgpr_read_b32 %5, a101\n\t" "v_accvgpr_read_b32 %6, a102\n\t" "v_accvgpr_read_b32 %7, a103\n\t" "v_accvgpr_read_b32 %8, a104\n\t" "v_accvgpr_read_b32 %9, a105\n\t" "v_accvgpr_read_b32 %10, a106\n\t" "v_accvgpr_read_b32 %11, a107\n\t" "v_accvgpr_read_b32 %12, a108\n\t" "v_accvgpr_read_b32 %13, a109\n\t" "v_accvgpr_read_b32 %14, a110\n\t" "v_accvgpr_read_b32 %15, a111\n\t"
#define PT_R7 "v_accvgpr_read_b32 %0, a112\n\t" "v_accvgpr_read_b32 %1, a113\n\t" "v_accvgpr_read_b32 %2, a114\n\t" "v_accvgpr_read_b32 %3, a115\n\t" "v_accvgpr_read_b32 %4, a116\n\t" "v_accvgpr_read_b32 %5, a117\n\t" "v_accvgpr_read_b32 %6, a118\n\t" "v_accvgpr_read_b32 %7, a119\n\t" "v_accvgpr_read_b32 %8, a120\n\t" "v_accvgpr_read_b32 %9, a121\n\t" "v_accvgpr_read_b32 %10, a122\n\t" "v_accvgpr_read_b32 %11, a123\n\t" "v_accvgpr_read_b32 %12, a124\n\t" "v_accvgpr_read_b32 %13, a125\n\t" "v_accvgpr_read_b32 %14, a126\n\t" "v_accvgpr_read_b32 %15, a127\n\t"
DI void pack_tile(Params& p, LAS unsigned char* tail, unsigned call, int ct) {
    const int lane = lane_now(), rank = p.rank;
    char* const ws = p.ws[rank];
    const char* const out = (const char*)p.out[rank];
    const bool ok = wait_words(ws, ws + OFF_AVC + ct * 4, 1, 4, call, call);
    __builtin_amdgcn_fence(__ATOMIC_ACQUIRE, "agent");
    const unsigned s0 = 8 * ct, vb = slab_out(s0, rank) + (lane >> 5) * ROWB + (lane & 31) * 16;
    for (int i = -8; i < 96; ++i) {
        const unsigned n = i + 8 < 96 ? i + 8 : 95, v = vb + n / 3 * (8 * ROWB) + n % 3 * 512;
        unsigned r[16];
        if (i >= 0 && i < 8) WAITVM(28);
        switch (i & 7) {
            case 0: PT_SET("0:3", "4:7", "8:11", "12:15", PT_R0); break;
            case 1: PT_SET("16:19", "20:23", "24:27", "28:31", PT_R1); break;
            case 2: PT_SET("32:35", "36:39", "40:43", "44:47", PT_R2); break;
            case 3: PT_SET("48:51", "52:55", "56:59", "60:63", PT_R3); break;
            case 4: PT_SET("64:67", "68:71", "72:75", "76:79", PT_R4); break;
            case 5: PT_SET("80:83", "84:87", "88:91", "92:95", PT_R5); break;
            case 6: PT_SET("96:99", "100:103", "104:107", "108:111", PT_R6); break;
            default: PT_SET("112:115", "116:119", "120:123", "124:127", PT_R7); break;
        }
        if (i < 0) continue;
        if (!ok) r[0] = NANS;
        const u32x4 f[4] = {{r[0], r[1], r[2], r[3]}, {r[4], r[5], r[6], r[7]}, {r[8], r[9], r[10], r[11]}, {r[12], r[13], r[14], r[15]}};
        if (!pack32(f, ws, OFF_PK32 + (s0 * 3 + (i >> 2)) * QB32, i & 3, lane, tail)) store4(ws, OFF_RAW32 + s0 * WBLK + i * 4096 + lane * 16, f);
    }
    WAITVM(0);
    arrive(ws, p.ws[lane & 7], rank, 0, 16 * CPR, call);
}

DI f16x8 rdfrag(unsigned base, int off) { return *(const LAS f16x8*)(size_t)(base + off); }
DI void dma_to(unsigned ldsaddr) { asm volatile("s_mov_b32 m0, %0" :: "s"(ldsaddr) : "memory"); }
#define PIN_S(x) asm volatile("" : "+s"(x))
DI float rd_acc(float a) { float v; asm volatile("v_accvgpr_read_b32 %0, %1" : "=v"(v) : "a"(a)); return v; }
DI f32x4 silu4(f32x4 a, float sc) {
    f32x4 t = a * (-1.44269504088896f / WSCALE);
#pragma unroll
    for (int i = 0; i < 4; ++i) t[i] = __builtin_amdgcn_exp2f(__builtin_amdgcn_fmed3f(t[i], -86.5617024533378f, 86.5617024533378f));
    t = __builtin_elementwise_fma(t, (f32x4){sc, sc, sc, sc}, (f32x4){sc, sc, sc, sc});
#pragma unroll
    for (int i = 0; i < 4; ++i) t[i] = __builtin_amdgcn_rcpf(t[i]);
    return a * t;
}
struct G7 {
    unsigned lds;
    __amdgpu_buffer_rsrc_t rD;
    unsigned voff;
    unsigned ard, an;
    unsigned blo, bhi;
    unsigned sA, sB, sr, sw;
    unsigned ringB, nextA, nextB;
    f32x4 acc[8][12];
    f16x8 L[2][4][2], H[4][2], B0, B1;
};
template <int I> DI void dma7(const G7& g, unsigned src) {
    asm volatile("buffer_load_dwordx4 %0, %1, %2 offen offset:%3 sc0 sc1 lds" :: "v"(g.voff), "s"(g.rD), "s"(src), "i"(I * 1024) : "memory");
}
template <int SL> DI f16x8 rdB7(const G7& g, int j, int k) { return rdfrag(SL < 3 ? g.blo : g.bhi, (SL % 3) * QUAD + k * 4096 + j * 1024); }
DI unsigned next3(unsigned s) { return s == 2 * QUAD ? 0 : s + QUAD; }
template <int J, bool ZERO, int VM> DI void phase(G7& g, int it) {
    constexpr int t = J % 3, c = J / 3, NX = (J + 1) % 6, PV = (J + 5) % 6;
    f16x8 B[5][2];
    B[0][0] = g.B0; B[0][1] = g.B1;
#pragma unroll
    for (int j = 0; j < 4; ++j) {
        const int n = 4 * t + j;
        if (j == 3) { __builtin_amdgcn_s_waitcnt(WCNT(VM, 0)); __builtin_amdgcn_s_barrier(); SCHED; }
#pragma unroll
        for (int u = 0; u < 16; ++u) {
            const int q = j * 16 + u, r = u >> 1, m = (n & 1) ? 7 - r : r, k = (u & 1) ^ (r & 1);
            const f16x8 a = m < 4 ? g.L[c][m][k] : g.H[m - 4][k];
            if (n < 8) {
                if (ZERO && !(u & 1)) asm volatile("v_mfma_f32_16x16x32_f16 %0, %1, %2, 0" : "=a"(g.acc[m][n]) : "v"(a), "v"(B[j][k]));
                else asm volatile("v_mfma_f32_16x16x32_f16 %0, %1, %2, %0" : "+a"(g.acc[m][n]) : "v"(a), "v"(B[j][k]));
            } else {
                if (ZERO && !(u & 1)) asm volatile("v_mfma_f32_16x16x32_f16 %0, %1, %2, 0" : "=&v"(g.acc[m][n]) : "v"(a), "v"(B[j][k]));
                else asm volatile("v_mfma_f32_16x16x32_f16 %0, %1, %2, %0" : "+v"(g.acc[m][n]) : "v"(a), "v"(B[j][k]));
            }
            SCHED;
            const bool lastcol = t == 2 && j == 3;
            if (u == (lastcol ? 9 : 1)) { B[j + 1][0] = j < 3 ? rdB7<J>(g, j + 1, 0) : rdB7<NX>(g, 0, 0); SCHED; }
            if (u == (lastcol ? 10 : 3)) { B[j + 1][1] = j < 3 ? rdB7<J>(g, j + 1, 1) : rdB7<NX>(g, 0, 1); SCHED; }
            if (t == 2 && j < 3 && (u == 5 || u == 9 || u == 13) && j * 3 + (u - 5) / 4 < 8) { const int i = j * 3 + (u - 5) / 4; g.L[c ^ 1][i >> 1][i & 1] = rdfrag(g.an, (i & 1) * 4096 + (i >> 1) * 1024); SCHED; }
            if (lastcol && u >= 1 && u <= 8) { const int row = 3 - ((u - 1) >> 1), kk = (u - 1) & 1; g.H[row][kk] = rdfrag(g.an, kk * 4096 + row * 1024); SCHED; }
            if ((t == 2 && q == 44) || (lastcol && u == 12)) { PIN_S(g.sr); g.sr = next3(g.sr); PIN_S(g.sr); SCHED; }
            if ((t == 2 && q == 46) || (lastcol && u == 14)) { g.an = g.ard + g.sr; asm volatile("" : "+v"(g.an)); SCHED; }
            if (t < 2) {
                if (q == 4) { PIN_S(g.sw); dma_to(g.lds + LDS_A + g.sw); SCHED; }
                if (q == 5) { dma7<0>(g, g.sA); SCHED; }
                if (q == 10) { dma7<1>(g, g.sA); SCHED; }
                if (q == 15) { dma7<2>(g, g.sA); SCHED; }
                if (q == 21) { dma7<3>(g, g.sA); SCHED; }
                if (q == 52) { PIN_S(g.sA); g.sA += QUAD; PIN_S(g.sA); SCHED; }
                if (q == 54 && J == 0) { g.sA = it == 31 ? g.nextA : g.sA; PIN_S(g.sA); SCHED; }
                if (q == 56) { PIN_S(g.sw); g.sw = next3(g.sw); PIN_S(g.sw); SCHED; }
            }
            constexpr int PB0[4] = {27, 32, 38, 43}, PB2[4] = {7, 16, 27, 39};
            if (q == (t < 2 ? PB0[0] : PB2[0]) - 1) { dma_to(g.lds + PV * QUAD); SCHED; }
            if (q == (t < 2 ? PB0[0] : PB2[0])) { dma7<0>(g, g.sB); SCHED; }
            if (q == (t < 2 ? PB0[1] : PB2[1])) { dma7<1>(g, g.sB); SCHED; }
            if (q == (t < 2 ? PB0[2] : PB2[2])) { dma7<2>(g, g.sB); SCHED; }
            if (q == (t < 2 ? PB0[3] : PB2[3])) { dma7<3>(g, g.sB); SCHED; }
            if (q == (t < 2 ? 58 : 42)) { PIN_S(g.sB); g.sB += QUAD; PIN_S(g.sB); SCHED; }
            if (q == 60 && J == 0) { g.sB = it == 3 ? g.ringB : g.sB; PIN_S(g.sB); SCHED; }
            if (q == 62 && J == 0) { g.sB = it == 31 ? g.nextB : g.sB; PIN_S(g.sB); SCHED; }
        }
    }
    g.B0 = B[4][0]; g.B1 = B[4][1];
}
template <bool FIRST> DI void trip(G7& g, int it) {
    SCHED;
    phase<0, FIRST, FIRST ? 60 : 28>(g, it); phase<1, FIRST, FIRST ? 60 : 24>(g, it); phase<2, FIRST, 16>(g, it);
    phase<3, false, 28>(g, it); phase<4, false, 24>(g, it); phase<5, false, 16>(g, it);
}
struct Short { unsigned sb, bn, bx; };
DI unsigned next6(unsigned s) { return s == 5 * QUAD ? 0 : s + QUAD; }
template <int J, bool ZERO, int VM> DI void phase2(G7& g, Short& h, int it) {
    constexpr int t = J & 1, c = J >> 1;
    f16x8 B[5][2];
    B[0][0] = g.B0; B[0][1] = g.B1;
    unsigned pv = h.sb == 0 ? 5 * QUAD : h.sb - QUAD;
#pragma unroll
    for (int j = 0; j < 4; ++j) {
        const int n = 4 * t + j;
        if (j == 3) { __builtin_amdgcn_s_waitcnt(WCNT(VM, 0)); __builtin_amdgcn_s_barrier(); SCHED; }
#pragma unroll
        for (int u = 0; u < 16; ++u) {
            const int q = j * 16 + u, r = u >> 1, m = (n & 1) ? 7 - r : r, k = (u & 1) ^ (r & 1);
            const f16x8 a = m < 4 ? g.L[c][m][k] : g.H[m - 4][k];
            if (ZERO && !(u & 1)) asm volatile("v_mfma_f32_16x16x32_f16 %0, %1, %2, 0" : "=a"(g.acc[m][n]) : "v"(a), "v"(B[j][k]));
            else asm volatile("v_mfma_f32_16x16x32_f16 %0, %1, %2, %0" : "+a"(g.acc[m][n]) : "v"(a), "v"(B[j][k]));
            SCHED;
            const bool lastcol = t == 1 && j == 3;
            if (u == (lastcol ? 9 : 1)) { B[j + 1][0] = j < 3 ? rdfrag(h.bn, (j + 1) * 1024) : rdfrag(h.bx, 0); SCHED; }
            if (u == (lastcol ? 10 : 3)) { B[j + 1][1] = j < 3 ? rdfrag(h.bn, 4096 + (j + 1) * 1024) : rdfrag(h.bx, 4096); SCHED; }
            if (t == 1 && j < 3 && (u == 5 || u == 9 || u == 13) && j * 3 + (u - 5) / 4 < 8) { const int i = j * 3 + (u - 5) / 4; g.L[c ^ 1][i >> 1][i & 1] = rdfrag(g.an, (i & 1) * 4096 + (i >> 1) * 1024); SCHED; }
            if (lastcol && u >= 1 && u <= 8) { const int row = 3 - ((u - 1) >> 1), kk = (u - 1) & 1; g.H[row][kk] = rdfrag(g.an, kk * 4096 + row * 1024); SCHED; }
            if ((t == 1 && q == 44) || (lastcol && u == 12)) { PIN_S(g.sr); g.sr = next3(g.sr); PIN_S(g.sr); SCHED; }
            if ((t == 1 && q == 46) || (lastcol && u == 14)) { g.an = g.ard + g.sr; asm volatile("" : "+v"(g.an)); SCHED; }
            if (q == 0) { PIN_S(g.sw); dma_to(g.lds + LDS_A + g.sw); SCHED; }
            if (q == 2) { dma7<0>(g, g.sA); SCHED; }
            if (q == 6) { dma7<1>(g, g.sA); SCHED; }
            if (q == 10) { dma7<2>(g, g.sA); SCHED; }
            if (q == 14) { dma7<3>(g, g.sA); SCHED; }
            if (q == 16) { PIN_S(g.sA); g.sA += QUAD; PIN_S(g.sA); SCHED; }
            if (q == 18) { PIN_S(g.sw); g.sw = next3(g.sw); PIN_S(g.sw); SCHED; }
            if (q == 22) { PIN_S(pv); dma_to(g.lds + pv); SCHED; }
            if (q == 23) { dma7<0>(g, g.sB); SCHED; }
            if (q == 30) { dma7<1>(g, g.sB); SCHED; }
            if (q == 36) { dma7<2>(g, g.sB); SCHED; }
            if (q == 42) { dma7<3>(g, g.sB); SCHED; }
            if (q == 43) { PIN_S(g.sB); g.sB += QUAD; PIN_S(g.sB); SCHED; }
            if (q == 45 && J == 2) { g.sB = it == 2 ? g.ringB : g.sB; PIN_S(g.sB); SCHED; }
            if (q == 59) { PIN_S(h.sb); h.sb = next6(h.sb); PIN_S(h.sb); SCHED; }
            if (q == 63) { h.bn = h.bx; h.bx = g.blo + next6(h.sb); asm volatile("" : "+v"(h.bx)); SCHED; }
        }
    }
    g.B0 = B[4][0]; g.B1 = B[4][1];
}
DI void short_tile(G7& g) {
    Short h;
    h.sb = 0; h.bn = g.blo; h.bx = g.blo + QUAD;
    SCHED;
    phase2<0, true, 60>(g, h, 0); phase2<1, true, 12>(g, h, 0); phase2<2, false, 12>(g, h, 0); phase2<3, false, 12>(g, h, 0);
    for (int it = 1; it < 32; ++it) { SCHED; phase2<0, false, 12>(g, h, it); phase2<1, false, 12>(g, h, it); phase2<2, false, 12>(g, h, it); phase2<3, false, 12>(g, h, it); }
}
DI void g7_init(G7& g, LAS unsigned char* lds, const char* data, unsigned bytes, int wid) {
    const int lane = lane_now(), wr = wid >> 1, wc = wid & 1;
    const unsigned lbase = (unsigned)(size_t)lds;
    g.rD = __builtin_amdgcn_make_buffer_rsrc((void*)data, 0, bytes, 0x00020000);
    g.lds = lbase + wid * 4096; g.voff = wid * 4096 + lane * 16;
    g.ard = lbase + LDS_A + wr * 8192 + lane * 16; g.blo = lbase + wc * 8192 + lane * 16; g.bhi = g.blo + 3 * QUAD;
    asm volatile("" : "+v"(g.ard), "+v"(g.blo), "+v"(g.bhi), "+v"(g.voff));
}
DI void g7_fill(G7& g, unsigned to, unsigned src) { dma_to(g.lds + to); asm volatile("s_nop 4"); dma7<0>(g, src); dma7<1>(g, src); dma7<2>(g, src); dma7<3>(g, src); }
DI void g7_prologue(G7& g, unsigned dA, unsigned dB) {
#pragma unroll
    for (int i = 0; i < 5; ++i) { if (i < 3) g7_fill(g, LDS_A + i * QUAD, dA + i * QUAD); g7_fill(g, i * QUAD, dB + i * QUAD); }
    WAITVM(0); __builtin_amdgcn_s_barrier(); SCHED;
#pragma unroll
    for (int i = 0; i < 8; ++i) { g.L[0][i >> 1][i & 1] = rdfrag(g.ard, (i & 1) * 4096 + (i >> 1) * 1024); g.H[i >> 1][i & 1] = rdfrag(g.ard, QUAD + (i & 1) * 4096 + (i >> 1) * 1024); }
    g.B0 = rdB7<0>(g, 0, 0); g.B1 = rdB7<0>(g, 0, 1);
    __builtin_amdgcn_s_waitcnt(WCNT(63, 0)); __builtin_amdgcn_s_barrier(); SCHED;
    g.sA = dA + 3 * QUAD; g.sB = dB + 5 * QUAD; g.sr = 2 * QUAD; g.sw = 0; g.an = g.ard + 2 * QUAD;
}
template <int NQ> DI void g7_epilogue(G7& g, __amdgpu_buffer_rsrc_t rO, unsigned so, unsigned rowb, float one) {
    const int lz = lane_now();
    const unsigned vo = 4 * (lz >> 4) * rowb + 16 * (lz & 15);
    if (NQ == 3) asm volatile("s_nop 15" : "+v"(g.acc[0][11]), "+v"(g.acc[1][11]), "+v"(g.acc[2][11]), "+v"(g.acc[3][11]));
    else asm volatile("s_nop 15");
#pragma unroll
    for (int m = 0; m < 8; ++m) {
#pragma unroll
        for (int r = 0; r < 4; ++r) {
#pragma unroll
            for (int q = 0; q < NQ; ++q) {
                f32x4 y;
                if (q < 2) y = (f32x4){rd_acc(g.acc[m][4 * q][r]), rd_acc(g.acc[m][4 * q + 1][r]), rd_acc(g.acc[m][4 * q + 2][r]), rd_acc(g.acc[m][4 * q + 3][r])};
                else y = (f32x4){g.acc[m][8][r], g.acc[m][9][r], g.acc[m][10][r], g.acc[m][11][r]};
                const f32x4 v = silu4(y, one);
                asm volatile("buffer_store_dwordx4 %0, %1, %2, %3 offen offset:%4 nt\n\ts_nop 1" :: "v"(v), "v"(vo), "s"(rO), "s"(so + r * rowb), "i"(q * 256) : "memory");
            }
            SCHED;
        }
        so += 16 * rowb;
    }
}
DI int lead_ones(bool ok) { const unsigned long long m = ~__ballot(ok); return m ? __builtin_ctzll(m) : 64; }

struct Ready {
    char* ws; unsigned call;
    __amdgpu_buffer_rsrc_t rF; LAS unsigned char* zone;
    int pa, pb, n; bool pending; float one;
    DI void poll(int need) {
        const unsigned long long t0 = __builtin_amdgcn_s_memrealtime();
        const int lane = lane_now();
        for (;;) {
            const unsigned a = ld_sys(ws + OFF_F + pa * 256 + lane * 4), b = ld_sys(ws + OFF_F + pb * 256 + lane * 4);
            n = lead_ones(lane < 8 || (a >= call && b >= call));
            if (n >= need) break;
            if (give_up(ws, call, t0)) { n = 64; one = __builtin_nanf(""); break; }
            __builtin_amdgcn_s_sleep(16);
        }
    }
    DI void request() {
        const int lane = lane_now();
        __builtin_amdgcn_raw_ptr_buffer_load_lds(rF, (LAS void*)zone, 4, lane * 4, pa * 256, 0, AUX_SYS);
        __builtin_amdgcn_raw_ptr_buffer_load_lds(rF, (LAS void*)(zone + 256), 4, lane * 4, pb * 256, 0, AUX_SYS);
        pending = true;
    }
    DI int plan(int it, int stop) {
        if (n == 64) return stop;
        const int safe = (n - 4) >> 1;
        if (safe <= it + 1) { request(); return it + 1; }
        return safe - 1 < stop ? safe - 1 : stop;
    }
    DI void check(int need) {
        if (pending) {
            const int lane = lane_now();
            const unsigned a = *(const volatile LAS unsigned*)(zone + lane * 4), b = *(const volatile LAS unsigned*)(zone + 256 + lane * 4);
            n = lead_ones(lane < 8 || (a >= call && b >= call)); pending = false;
        }
        if (n < need) poll(need);
        asm volatile("" ::: "memory");
    }
};

DI void gemm_wg(Params& p, LAS unsigned char* lds, int wid, int band, int S) {
    const int wr = wid >> 1, wc = wid & 1, rank = p.rank;
    char* const ws = p.ws[rank];
    const unsigned call = convert_shards(p, lds, wid, S * (16 / RB) + band);
    int T = S;
    G7 g;
    g7_init(g, lds, ws + OFF_DATA, DATAB, wid);
    Ready R;
    R.ws = ws; R.call = call; R.rF = __builtin_amdgcn_make_buffer_rsrc((void*)(ws + OFF_F), 0, NPAN * 256, 0x00020000);
    R.zone = lds + LDS_ZONE + wid * 512; R.pending = false; R.one = WSCALE;
    if (!wait_words(ws, ws + OFF_EV + (3 * NR + rank) * 4, 1, 4, call, call) || rfl(ld_ag(ws + OFF_TMO)) == call) R.one = __builtin_nanf("");
    R.pa = RB * band + (T & (RB - 1)); R.pb = 16 + colblock(T >> RBL, rank); R.n = 8;
    g7_prologue(g, rfl(own_off(R.pa, 0)), rfl(own_off(R.pb, 0)));
    const __amdgpu_buffer_rsrc_t rO = __builtin_amdgcn_make_buffer_rsrc((void*)p.out[rank], 0, OUTB, 0x00020000);
    unsigned tprev = 0;
    int li = T >> RBL;
    for (int ti = 0;; ++ti) {
        const int pm = R.pa, cb = R.pb - 16;
        S += NWB;
        const int job = S - PSLOT;
        if ((unsigned)job < (unsigned)NJOB) S += NWB;
        const bool has_next = S < NROUND * NWB, deal = has_next && S >= (NROUND - 1) * NWB;
        T = S < PSLOT ? S : S - NJOB;
        int qa = pm, ql = li;
        if (has_next && !deal) { qa = RB * band + (T & (RB - 1)); ql = T >> RBL; }
        g.ringB = rfl(ring_off(li)); g.nextA = rfl(own_off(qa, 0)); g.nextB = rfl(own_off(16 + colblock(ql, rank), 0));
        {
            const unsigned now = (unsigned)__builtin_amdgcn_s_memrealtime();
            if (deal && blockIdx.x >> 3 == NG / 2 && wid == 0 && lane_now() == 0)
                __hip_atomic_store((GAS unsigned long long*)(ws + OFF_ETA + (blockIdx.x & 7) * 8), (unsigned long long)call << 32 | (3 * now - 2 * tprev), __ATOMIC_RELAXED, __HIP_MEMORY_SCOPE_AGENT);
            tprev = now;
        }
        R.check(4);
        int lim = R.plan(0, deal ? 29 : 32);
        trip<true>(g, 0);
        for (int it = 1; it < 32;) {
            if (it >= lim) {
                R.check(it < 30 ? 2 * it + 4 : 64);
                lim = R.plan(it, deal && it < 29 ? 29 : 32);
                if (deal && it == 29) {
                    const int lane = lane_now();
                    const unsigned long long t0 = __builtin_amdgcn_s_memrealtime();
                    unsigned long long e;
                    int place = blockIdx.x & 7;
                    while (!__all((unsigned)((e = __hip_atomic_load((GAS unsigned long long*)(ws + OFF_ETA + (lane & 7) * 8), __ATOMIC_RELAXED, __HIP_MEMORY_SCOPE_AGENT)) >> 32) == call)) {
                        if (give_up(ws, call, t0)) { R.one = __builtin_nanf(""); e = 0; break; }
                        __builtin_amdgcn_s_sleep(8);
                    }
                    const int d = (int)((unsigned)e - __builtin_amdgcn_readlane((unsigned)e, place));
                    place = __builtin_popcount((unsigned)__ballot(d < 0 || (d == 0 && lane < place)) & 0xffu);
                    constexpr int T4 = (NROUND - 1) * NWB - NJOB, NF = RB * (NL - 2) - T4;
                    const int P = place * NG + (blockIdx.x >> 3), bd = P / NF;
                    if (bd < 16 / RB) { const int t = T4 + P - bd * NF; qa = RB * bd + (t & (RB - 1)); ql = t >> RBL; }
                    else { const int t = P - 16 / RB * NF; qa = t & 15; ql = NL - 2 + (t >> 4); }
                    g.nextA = rfl(own_off(qa, 0)); g.nextB = rfl(own_off(16 + colblock(ql, rank), 0));
                }
            }
            do trip<false>(g, it); while (++it < lim);
        }
        R.pa = qa; R.pb = 16 + colblock(ql, rank);
        R.request();
        g7_epilogue<3>(g, rO, rfl((pm * 256 + wr * 128) * ROWB + (col0(cb) + wc * 192) * 4), ROWB, R.one);
        WAITVM(60);
        if (wid == 0 && lane_now() == 0) __hip_atomic_fetch_add((gu32*)(ws + OFF_FIN + li * 4), 1u, __ATOMIC_RELAXED, __HIP_MEMORY_SCOPE_AGENT);
        if (li < CPR) {
            WAITVM(0);
            BARRIER;
            if (wid == 0) {
                __builtin_amdgcn_fence(__ATOMIC_RELEASE, "agent");
                WAITVM(0);
                if (lane_now() == 0) st_sys(ws + OFF_AVC + (li * 16 + pm) * 4, call);
            }
        }
        if ((unsigned)job < (unsigned)NJOB) { pack_tile(p, lds + LDS_ZONE + 4096 + wid * 1024, call, job % CPR * 16 + RB * band + job / CPR * 4 + wid); }
        if (!has_next) break;
        li = ql;
        if (li >= NL - 2) {
            g.ringB = rfl(ring_off(li));
            R.check(64);
            short_tile(g);
            g7_epilogue<2>(g, rO, rfl((R.pa * 256 + wr * 128) * ROWB + (col0(R.pb - 16) + wc * 128) * 4), ROWB, R.one);
            break;
        }
    }
    __builtin_amdgcn_s_waitcnt(WCNT(0, 0));
}

__global__ __launch_bounds__(256, 1) void fused(Args) {
    extern __shared__ __attribute__((aligned(16))) unsigned char shm[];
    Params& p = *(Params*)__builtin_amdgcn_kernarg_segment_ptr();
    LAS unsigned char* lds = (LAS unsigned char*)shm;
    char* const ws = p.ws[p.rank];
    const int wid = rfl(threadIdx.x >> 6);
    const int grp = blockIdx.x & 7, idx = blockIdx.x >> 3;
    if (idx < NG) gemm_wg(p, lds, wid, (grp & 1 ? 0 : 8 / RB) + (grp >> RBL), (grp >> 1 & (RB / 2 - 1)) * NG + idx);
    else {
        const int cw = (idx - NG) * 8 + grp;
        bool clear;
        const unsigned call = draw_call(ws, wid, clear);
        fetch_wave(p, lds + wid * 32768, call, wid, cw * 4 + wid);
    }
}
}

extern "C" void kernel_launch(int rank, int nranks, ncclComm_t comm, hipStream_t stream,
                              void* const* d_in, const int* in_sizes, int n_in,
                              void* d_out, int out_size,
                              const int* in_dtypes, int out_dtype,
                              void* const* d_out_all,
                              void* const* d_ws_all, size_t ws_bytes) {
    if (nranks != NR || (unsigned)rank >= (unsigned)NR || n_in != 2 || ws_bytes < WS_NEED || in_sizes[0] != MM * KS || in_sizes[1] != KS * NN || out_size != MM * NN) {
        fprintf(stderr, "kernel_launch: unexpected problem shape\n");
        return;
    }
    static bool attr_set[NR];
    if (!attr_set[rank]) {
        HIP_CHECK(hipFuncSetAttribute((const void*)fused, hipFuncAttributeMaxDynamicSharedMemorySize, LDS_BYTES));
        attr_set[rank] = true;
    }
    Args p;
    p.x = (const float*)d_in[0]; p.w = (const float*)d_in[1];
    for (int i = 0; i < NR; ++i) { p.ws[i] = (char*)d_ws_all[i]; p.out[i] = (float*)d_out_all[i]; }
    p.rank = rank; p.pad = 0;
    hipLaunchKernelGGL(fused, dim3(256), dim3(256), LDS_BYTES, stream, p);
    HIP_CHECK(hipGetLastError());
}
```

```cpp
#include <hip/hip_runtime.h>
#include <rccl/rccl.h>
#include <cstdio>

#define HIP_CHECK(expr) do { hipError_t _e = (expr); if (_e != hipSuccess) { \
    fprintf(stderr, "HIP error %s at %s:%d\n", hipGetErrorString(_e), __FILE__, __LINE__); \
    return; } } while (0)

#define LAS __attribute__((address_space(3)))
#define GAS __attribute__((address_space(1)))
#define DI __device__ __forceinline__
typedef _Float16 f16x8 __attribute__((ext_vector_type(8)));
typedef float f32x4 __attribute__((ext_vector_type(4)));
typedef unsigned u32x4 __attribute__((ext_vector_type(4)));
typedef unsigned u32x2 __attribute__((ext_vector_type(2)));
typedef GAS unsigned gu32;

#define NG 29
namespace {
constexpr int MM = 4096, NN = 32768, KS = 512, NR = 8;
constexpr int NGEMM = 8 * NG, NCOMM = 256 - NGEMM, NCW = 4 * NCOMM;
constexpr int TW = 384;
constexpr int CPR = 2, NCB = NR * CPR;
constexpr int BCOLS = NN - NCB * TW, NBB = (BCOLS + TW - 1) / TW;
constexpr int NWP = NBB + NCB, NPAN = 16 + NWP;
constexpr int NL = CPR + NBB;
constexpr int RBL = 3, RB = 1 << RBL, NWB = NGEMM * RB / 16, NJOB = CPR * RB / 4, NROUND = 5, PSLOT = 2 * NWB - NJOB;
static_assert(NROUND * NWB == RB * NL + NJOB, "slots");
constexpr unsigned QUAD = 16384, XBLK = 2 * QUAD, WBLK = 3 * QUAD, XPANEL = 64 * XBLK;
static_assert(BCOLS - (NBB - 2) * TW == 2 * 256, "short blocks");
constexpr unsigned NE16 = 48, NE32 = 16, TS16 = 512 + 2 * NE16, TS32 = 512 + 2 * NE32, QB16 = 4 * (TS16 + 3072), QB32 = 4 * (TS32 + 3072);
constexpr unsigned OUTB = (unsigned)MM * NN * 4u, ROWB = NN * 4;
constexpr unsigned OFF_CALL = 0;
constexpr unsigned OFF_TMO = 32768;
constexpr unsigned OFF_FIN = 32768 + 1024;
constexpr unsigned OFF_ETA = 32768 + 2048;
constexpr unsigned OFF_TICK = 32768 + 4096;
constexpr unsigned OFF_F = 65536;
constexpr unsigned OFF_AV = 131072;
constexpr unsigned OFF_AVC = OFF_AV + 8192;
constexpr unsigned OFF_CNT = 196608;
constexpr unsigned OFF_EV = OFF_CNT + 4096;
constexpr int RING = NL - 32;
constexpr unsigned OFF_DATA = 1u << 20;
constexpr unsigned REL_WL = 16 * XPANEL;
constexpr unsigned REL_WR = REL_WL + NWP * 8 * WBLK;
constexpr unsigned DATAB = REL_WR + RING * 56 * WBLK;
constexpr int NQ16 = 16 * 8 * 2 + (NWP - 2) * 8 * 3 + 32;
constexpr unsigned OFF_PK16 = OFF_DATA + DATAB;
constexpr int NCI = CPR * (MM / 32);
constexpr unsigned OFF_PK32 = OFF_PK16 + (NQ16 + 16) * QB16;
constexpr unsigned OFF_RAW32 = OFF_PK32 + NCI * 3 * QB32;
constexpr unsigned WS_NEED = OFF_RAW32 + NCI * WBLK;
static_assert(NPAN * 256 <= 65536, "flags");
constexpr int LDS_A = 6 * QUAD, LDS_ZONE = LDS_A + 3 * QUAD, LDS_BYTES = LDS_ZONE + 8192;
constexpr unsigned long long DEADLINE = 50000000ull;
constexpr float XSCALE = 0.5f, WSCALE = 32.0f, SCALE = XSCALE * WSCALE;
constexpr int AUX_SYS = 17, AUX_NT = 2;

struct Args {
    const float* x; const float* w;
    char* ws[NR]; float* out[NR];
    int rank, pad;
};
typedef const __attribute__((address_space(4))) Args Params;

constexpr int WCNT(int vm, int lgkm) { return (vm & 15) | (7 << 4) | ((lgkm & 15) << 8) | ((vm >> 4) << 14); }
#define SCHED __builtin_amdgcn_sched_barrier(0)
#define WAITVM(n) asm volatile("s_waitcnt vmcnt(" #n ")" ::: "memory")
#define BARRIER do { asm volatile("" ::: "memory"); __builtin_amdgcn_s_barrier(); asm volatile("" ::: "memory"); } while (0)

DI unsigned ld_sys(const char* p) { return __hip_atomic_load((gu32*)p, __ATOMIC_RELAXED, __HIP_MEMORY_SCOPE_SYSTEM); }
DI void st_sys(char* p, unsigned v) { __hip_atomic_store((gu32*)p, v, __ATOMIC_RELAXED, __HIP_MEMORY_SCOPE_SYSTEM); }
DI unsigned ld_ag(const char* p) { return __hip_atomic_load((gu32*)p, __ATOMIC_RELAXED, __HIP_MEMORY_SCOPE_AGENT); }
DI void st_ag(char* p, unsigned v) { __hip_atomic_store((gu32*)p, v, __ATOMIC_RELAXED, __HIP_MEMORY_SCOPE_AGENT); }
DI unsigned rfl(unsigned x) { return __builtin_amdgcn_readfirstlane(x); }
DI int lane_now() { int l; asm volatile("v_mbcnt_lo_u32_b32 %0, -1, 0\n\tv_mbcnt_hi_u32_b32 %0, -1, %0" : "=v"(l)); return l; }
DI bool give_up(char* ws, unsigned call, unsigned long long t0) {
    if (rfl(ld_ag(ws + OFF_TMO)) == call || __builtin_amdgcn_s_memrealtime() - t0 > DEADLINE) { st_ag(ws + OFF_TMO, call); return true; }
    return false;
}
DI bool arrive(char* ws, char* lane_ws, int rank, int kind, unsigned n, unsigned call) {
    const int lane = lane_now();
    unsigned t = 0;
    if (lane == 0) t = __hip_atomic_fetch_add((gu32*)(ws + OFF_CNT + kind * 128), 1u, __ATOMIC_RELAXED, __HIP_MEMORY_SCOPE_AGENT);
    const bool last = rfl(t) + 1 == n * call;
    if (last && lane < NR) st_sys(lane_ws + OFF_EV + (kind * NR + rank) * 4, call);
    return last;
}
DI bool wait_words(char* ws, const char* f, int n, int step, unsigned want, unsigned call) {
    const int lane = lane_now();
    const unsigned long long t0 = __builtin_amdgcn_s_memrealtime();
    for (;;) {
        bool ok = true;
#pragma unroll
        for (int i = 0; i < 4; ++i) { const int k = lane + 64 * i; if (64 * i < n) ok &= ld_sys(f + (k < n ? k : 0) * step) >= want; }
        if (__all(ok)) break;
        if (give_up(ws, call, t0)) return false;
        __builtin_amdgcn_s_sleep(4);
    }
    asm volatile("" ::: "memory");
    return true;
}

DI void store1(const char* base, unsigned v, const u32x4& f) {
    asm volatile("s_nop 4\n\tglobal_store_dwordx4 %0, %1, %2 sc0 sc1\n\ts_nop 1" :: "v"(v), "v"(f), "s"(base) : "memory");
}
DI void store3(const char* base, unsigned v, const u32x4& a, const u32x4& b, const u32x4& c) {
    asm volatile("s_nop 4\n\tglobal_store_dwordx4 %0, %1, %4 sc0 sc1\n\t"
                 "global_store_dwordx4 %0, %2, %4 offset:1024 sc0 sc1\n\t"
                 "global_store_dwordx4 %0, %3, %4 offset:2048 sc0 sc1\n\ts_nop 1"
                 :: "v"(v), "v"(a), "v"(b), "v"(c), "s"(base) : "memory");
}
DI void store4(const char* base, unsigned v, const u32x4 (&f)[4]) {
    asm volatile("s_nop 4\n\tglobal_store_dwordx4 %0, %1, %5 sc0 sc1\n\t"
                 "global_store_dwordx4 %0, %2, %5 offset:1024 sc0 sc1\n\t"
                 "global_store_dwordx4 %0, %3, %5 offset:2048 sc0 sc1\n\t"
                 "global_store_dwordx4 %0, %4, %5 offset:3072 sc0 sc1\n\ts_nop 1"
                 :: "v"(v), "v"(f[0]), "v"(f[1]), "v"(f[2]), "v"(f[3]), "s"(base) : "memory");
}
DI void store4o(const char* base, unsigned v, const u32x4 (&f)[4]) {
    asm volatile("s_nop 4\n\tglobal_store_dwordx4 %0, %4, %8 sc1 nt\n\t"
                 "global_store_dwordx4 %1, %5, %8 sc1 nt\n\t"
                 "global_store_dwordx4 %2, %6, %8 sc1 nt\n\t"
                 "global_store_dwordx4 %3, %7, %8 sc1 nt\n\ts_nop 1"
                 :: "v"(v), "v"(v + 2 * ROWB), "v"(v + 4 * ROWB), "v"(v + 6 * ROWB), "v"(f[0]), "v"(f[1]), "v"(f[2]), "v"(f[3]), "s"(base) : "memory");
}

DI unsigned perm(unsigned hi, unsigned lo, unsigned sel) { return __builtin_amdgcn_perm(hi, lo, sel); }
template <int CTRL, int ROWMASK> DI unsigned dpp0(unsigned x) { return __builtin_amdgcn_update_dpp(0u, x, CTRL, ROWMASK, 0xf, false); }
DI unsigned wave_sum(unsigned x) {
    x += dpp0<0x111, 0xf>(x); x += dpp0<0x112, 0xf>(x); x += dpp0<0x114, 0xf>(x); x += dpp0<0x118, 0xf>(x); x += dpp0<0x142, 0xa>(x); x += dpp0<0x143, 0xc>(x);
    return __builtin_amdgcn_readlane(x, 63);
}
DI unsigned umax(unsigned a, unsigned b) { return a > b ? a : b; }
DI unsigned wave_max(unsigned x) {
    x = umax(x, dpp0<0x111, 0xf>(x)); x = umax(x, dpp0<0x112, 0xf>(x)); x = umax(x, dpp0<0x114, 0xf>(x)); x = umax(x, dpp0<0x118, 0xf>(x));
    x = umax(x, dpp0<0x142, 0xa>(x)); x = umax(x, dpp0<0x143, 0xc>(x));
    return __builtin_amdgcn_readlane(x, 63);
}
template <unsigned NE> struct ExcList {
    volatile LAS unsigned short* e; unsigned cnt;
    DI void begin(LAS unsigned char* tail, int lane) { e = (volatile LAS unsigned short*)(tail + 512); e[lane] = 0; cnt = 1; }
    DI bool add(bool act, unsigned entry) {
        const unsigned long long bal = __ballot(act);
        const unsigned slot = cnt + __builtin_amdgcn_mbcnt_hi((unsigned)(bal >> 32), __builtin_amdgcn_mbcnt_lo((unsigned)bal, 0u));
        if (act && slot < NE) e[slot] = (unsigned short)entry;
        cnt += __builtin_popcountll(bal);
        return cnt <= NE;
    }
};
template <unsigned NE> DI bool pack_finish(const char* base, unsigned blk, int c, int lane, LAS unsigned char* tail, ExcList<NE>& x, unsigned ebase,
                    const u32x4& p0, const u32x4& p1, const u32x4& p2, unsigned t0, unsigned t1) {
    if (lane == 0) x.e[0] = (unsigned short)(ebase | (x.cnt > NE ? 0x8000u : 0u));
    *(volatile LAS u32x2*)(tail + lane * 8) = (u32x2){t0, t1};
    constexpr int TS = 512 + 2 * NE;
    store3(base, blk + 4 * TS + c * 3072 + lane * 16, p0, p1, p2);
    const u32x4 t = *(volatile LAS u32x4*)(tail + (lane < TS / 16 ? lane : 0) * 16);
    if (lane < TS / 16) store1(base, blk + c * TS + lane * 16, t);
    return x.cnt <= NE;
}
DI void pack16(const u32x4 (&f)[4], const char* base, unsigned blk, int c, int lane, LAS unsigned char* tail) {
    unsigned lo[8], h[8], g[8], sum = 0;
#pragma unroll
    for (int i = 0; i < 8; ++i) {
        const unsigned a = f[i >> 1][2 * (i & 1)], b = f[i >> 1][2 * (i & 1) + 1];
        lo[i] = perm(b, a, 0x06040200u); h[i] = perm(b, a, 0x07050301u);
        sum = __builtin_amdgcn_udot4(h[i] & 0x7c7c7c7cu, 0x01010101u, sum, false);
    }
    const int top = (int)((wave_sum(sum) + 15155u + 8191u) >> 13);
    const unsigned eb = top < 7 ? 0u : top > 31 ? 24u : (unsigned)(top - 7), b4 = eb * 0x04040404u;
    ExcList<NE16> x; x.begin(tail, lane);
    unsigned m1 = 0, m0 = 0;
#pragma unroll
    for (int i = 0; i < 8; ++i) {
        g[i] = (h[i] & 0x83838383u) | (((h[i] | 0x80808080u) - b4) & 0x1c1c1c1cu);
        const unsigned xr = (g[i] + b4) ^ h[i];
        m1 |= ((xr >> 6) & 0x01010101u) << i; m0 |= ((xr >> 5) & 0x01010101u) << i;
    }
    for (unsigned m = m1 | m0; __any(m != 0); m &= m - 1) {
        const int k = m ? __builtin_ctz(m) : 0, i = k & 7;
        if (!x.add(m != 0, ((i >> 2) * 1024 + lane * 16 + (i & 3) * 4 + (k >> 3)) | ((m0 >> k) & 1u) << 11 | ((m1 >> k) & 1u) << 12)) break;
    }
    unsigned d[6];
#pragma unroll
    for (int q = 0; q < 2; ++q) {
        const unsigned s = g[4 * q + 3];
        d[3 * q] = g[4 * q] | (s & 0x03030303u) << 5; d[3 * q + 1] = g[4 * q + 1] | (s & 0x0c0c0c0cu) << 3;
        d[3 * q + 2] = g[4 * q + 2] | (s & 0x10101010u) << 1 | (s & 0x80808080u) >> 1;
    }
    pack_finish(base, blk, c, lane, tail, x, eb, (u32x4){lo[0], lo[1], lo[2], lo[3]}, (u32x4){lo[4], lo[5], lo[6], lo[7]}, (u32x4){d[0], d[1], d[2], d[3]}, d[4], d[5]);
}
DI bool pack32(const u32x4 (&f)[4], const char* base, unsigned blk, int c, int lane, LAS unsigned char* tail) {
    unsigned t[12], h[4], mx = 0;
#pragma unroll
    for (int j = 0; j < 4; ++j) {
        t[3 * j] = perm(f[j][3], f[j][0], 0x04020100u); t[3 * j + 1] = perm(f[j][3], f[j][1], 0x05020100u); t[3 * j + 2] = perm(f[j][3], f[j][2], 0x06020100u);
        h[j] = perm(perm(f[j][3], f[j][2], 0x07030703u), perm(f[j][1], f[j][0], 0x07030703u), 0x05040100u);
#pragma unroll
        for (int d = 0; d < 4; ++d) mx = umax(mx, f[j][d] & 0x7fffffffu);
    }
    const unsigned top = wave_max(mx) >> 24, eb = top < 7 ? 0u : top - 7, b1 = eb * 0x01010101u;
    ExcList<NE32> x; x.begin(tail, lane);
    unsigned cd[4], xn[2] = {0, 0};
#pragma unroll
    for (int j = 0; j < 4; ++j) {
        cd[j] = ((h[j] | 0x80808080u) - b1) & 0x07070707u;
        const unsigned xr = ((cd[j] + b1) ^ h[j]) & 0x78787878u;
        xn[j >> 1] |= (j & 1) ? xr << 1 : xr >> 3;
    }
    for (unsigned long long m = (unsigned long long)xn[1] << 32 | xn[0]; __any(m != 0);) {
        const int k = m ? __builtin_ctzll(m) >> 2 : 0, j = 2 * (k >> 3) + (k & 1);
        const unsigned v = (unsigned)(m >> (4 * k)) & 15u;
        if (!x.add(m != 0, (lane * 16 + j * 4 + ((k >> 1) & 3)) | v << 11)) break;
        m &= ~(15ull << (4 * k));
    }
    const unsigned n0 = cd[0] | (h[0] & 0x80808080u) >> 4 | cd[1] << 4 | (h[1] & 0x80808080u), n1 = cd[2] | (h[2] & 0x80808080u) >> 4 | cd[3] << 4 | (h[3] & 0x80808080u);
    return pack_finish(base, blk, c, lane, tail, x, eb, (u32x4){t[0], t[1], t[2], t[3]}, (u32x4){t[4], t[5], t[6], t[7]}, (u32x4){t[8], t[9], t[10], t[11]}, n0, n1);
}
DI bool unpack(bool is32, const LAS unsigned char* main3, const LAS unsigned char* tail, LAS unsigned char* patch, int lane, u32x4 (&f)[4]) {
    unsigned e = *(const volatile LAS unsigned short*)(tail + 512 + lane * 2);
    const u32x2 tt = *(const volatile LAS u32x2*)(tail + lane * 8);
    const u32x4 p0 = *(const volatile LAS u32x4*)(main3 + lane * 16), p1 = *(const volatile LAS u32x4*)(main3 + 1024 + lane * 16), p2 = *(const volatile LAS u32x4*)(main3 + 2048 + lane * 16);
    const unsigned hdr = rfl(e);
    if (lane == 0 || lane >= (int)(is32 ? NE32 : NE16)) e = 0;
    LAS unsigned* const pw = (LAS unsigned*)(patch + (e & 0x7fcu));
    __hip_atomic_fetch_or(pw, (e >> 11) << ((is32 ? 3 : 5) + 8 * (e & 3u)), __ATOMIC_RELAXED, __HIP_MEMORY_SCOPE_WAVEFRONT);
    const u32x4 x0 = *(const volatile LAS u32x4*)(patch + lane * 16), x1 = *(const volatile LAS u32x4*)(patch + 1024 + lane * 16);
    *(volatile LAS unsigned*)pw = 0;
    if (is32) {
        const unsigned b1 = (hdr & 127u) * 0x01010101u, t[12] = {p0[0], p0[1], p0[2], p0[3], p1[0], p1[1], p1[2], p1[3], p2[0], p2[1], p2[2], p2[3]};
#pragma unroll
        for (int j = 0; j < 4; ++j) {
            const unsigned n = tt[j >> 1];
            const unsigned h = ((j & 1) ? ((((n >> 4) & 0x07070707u) + b1) | (n & 0x80808080u)) : (((n & 0x07070707u) + b1) | ((n << 4) & 0x80808080u))) ^ x0[j];
            f[j][0] = perm(h, t[3 * j], 0x04020100u); f[j][1] = perm(h, t[3 * j + 1], 0x05020100u); f[j][2] = perm(h, t[3 * j + 2], 0x06020100u);
            f[j][3] = perm(perm(h, t[3 * j + 2], 0x07030703u), perm(t[3 * j + 1], t[3 * j], 0x07030703u), 0x05040100u);
        }
    } else {
        const unsigned b4 = (hdr & 31u) * 0x04040404u, d[6] = {p2[0], p2[1], p2[2], p2[3], tt[0], tt[1]};
#pragma unroll
        for (int q = 0; q < 2; ++q) {
            const unsigned a = d[3 * q], b = d[3 * q + 1], c = d[3 * q + 2];
            const unsigned g[4] = {a & 0x9f9f9f9fu, b & 0x9f9f9f9fu, c & 0x9f9f9f9fu,
                                   ((a >> 5) & 0x03030303u) | ((b >> 3) & 0x0c0c0c0cu) | ((c >> 1) & 0x10101010u) | ((c << 1) & 0x80808080u)};
#pragma unroll
            for (int i = 0; i < 4; ++i) {
                const unsigned h = (g[i] + b4) ^ (q ? x1[i] : x0[i]), lo = q ? p1[i] : p0[i];
                f[2 * q + (i >> 1)][2 * (i & 1)] = perm(h, lo, 0x05010400u); f[2 * q + (i >> 1)][2 * (i & 1) + 1] = perm(h, lo, 0x07030602u);
            }
        }
    }
    return hdr < 0x8000u;
}

constexpr int PSHORT = 16 + NBB - 2;
DI bool two_quads(int panel) { return panel < 16 || (unsigned)(panel - PSHORT) < 2u; }
DI int col0(int cb) { return cb >= NBB ? BCOLS + (cb - NBB) * TW : cb == NBB - 1 ? BCOLS - 256 : cb * TW; }
DI int colblock(int li, int rank) { return li < CPR ? NBB + CPR * rank + li : li - CPR; }
DI unsigned own_off(int panel, int kt) { return (panel < 16 ? panel * XPANEL : REL_WL + (panel - 16) * 8 * WBLK) + kt * (two_quads(panel) ? XBLK : WBLK); }
DI unsigned ring_off(int li) { return REL_WR + (li >= RING ? li - RING : li) * (56 * WBLK); }
DI unsigned pk16_off(int panel, int kt) { return OFF_PK16 + ((panel < 16 ? panel * 16 : 256 + (panel - 16) * 24) + kt * (two_quads(panel) ? 2 : 3)) * QB16; }
static_assert(2 * RING >= NL, "ring");
constexpr int round_end(int r) { return (NWB * (r + 1) - (r ? NJOB : 0) + RB - 1) / RB; }
constexpr int XL[4] = {0, 1, 2, 2}, NSTEP0 = 8 + XL[3];
constexpr int x_lo(int s) { int g = 0; while (g < 4 && s - XL[g] >= 8) ++g; return 4 * g; }
constexpr int x_hi(int s) { int g = 0; while (g < 4 && s >= XL[g]) ++g; return 4 * g; }
constexpr int B0 = round_end(0) - CPR, W0Q = 3 * (B0 + NCB);
constexpr int cstep_start(int s) { int n = 0; for (int k = 0; k < s; ++k) n += 2 * (x_hi(k) - x_lo(k)) + (k < 8 ? W0Q : 0); return n; }
constexpr int steps_within(int quads) { int s = 0; while (s < NSTEP0 && cstep_start(s + 1) <= quads) ++s; return s; }
constexpr int ST1 = 3, STEPS1 = steps_within(NGEMM), STEPS2 = steps_within(ST1 * NGEMM);
static_assert(STEPS1 >= 1 && NQ16 > ST1 * NGEMM + NGEMM, "stages");
DI void conv_quad(int b, int& panel, int& kt, int& qd) {
    if (b < cstep_start(NSTEP0)) {
        int s = 0, i = b, lo = x_lo(0), nx = 2 * (x_hi(0) - x_lo(0));
#pragma unroll
        for (int k = 1; k < NSTEP0; ++k) if (b >= cstep_start(k)) { s = k; i = b - cstep_start(k); lo = x_lo(k); nx = 2 * (x_hi(k) - x_lo(k)); }
        if (i < nx) { panel = lo + (i >> 1); qd = i & 1; kt = s - (panel < 4 ? XL[0] : panel < 8 ? XL[1] : panel < 12 ? XL[2] : XL[3]); }
        else { const int w = (i - nx) / 3; qd = i - nx - 3 * w; panel = 16 + (w < B0 ? w : w - B0 + NBB); kt = s; }
    } else if (b < NQ16 - 32) { const int r = b - cstep_start(NSTEP0), k = r / 3; qd = r - 3 * k; panel = 16 + B0 + (k >> 3); kt = k & 7; }
    else { const int r = b - (NQ16 - 32); panel = PSHORT + (r >> 4); kt = r >> 1 & 7; qd = r & 1; }
}
DI unsigned draw_call(char* ws, int wid, bool& clear) {
    const int lane = lane_now();
    unsigned v;
    asm volatile("s_nop 4\n\tglobal_load_dword %0, %1, %2 sc0 sc1\n\ts_waitcnt vmcnt(0)"
                 : "=&v"(v) : "v"(lane < 56 ? OFF_CALL + blockIdx.x * 128 : OFF_EV + (4 * NR + lane - 56) * 4), "s"(ws) : "memory");
    const unsigned c = rfl(v) + 1;
    clear = __all(v >= c - 1);
    BARRIER;
    if (wid == 0 && lane == 0) st_ag(ws + OFF_CALL + blockIdx.x * 128, c);
    return c;
}
DI unsigned convert_shards(Params& p, LAS unsigned char* lds, int wid, int gi) {
    const int lane = lane_now(), rank = p.rank;
    char* const ws = p.ws[rank];
    const __amdgpu_buffer_rsrc_t rx = __builtin_amdgcn_make_buffer_rsrc((void*)p.x, 0, MM * KS * 4, 0x00020000);
    const __amdgpu_buffer_rsrc_t rw = __builtin_amdgcn_make_buffer_rsrc((void*)p.w, 0, KS * NN * 4, 0x00020000);
    const unsigned vx = ((lane & 15) * KS + 8 * (lane >> 4)) * 4, vw = (8 * (lane >> 4) * NN + 4 * (lane & 15)) * 4;
    LAS unsigned char* const ring = lds + wid * 32768;
    LAS unsigned char* const tail = lds + 131072 + wid * 1024;
    const int nu = (NQ16 - gi + NGEMM - 1) / NGEMM;
    auto load = [&](int u) {
        int pa, kt, qd; conv_quad(gi + (u < nu ? u : 0) * NGEMM, pa, kt, qd);
        LAS unsigned char* const slot = ring + (u & 3) * 8192;
        if (pa < 16) {
            const unsigned s0 = ((pa * 256 + 128 * (wid >> 1) + 64 * qd) * KS + 32 * (2 * kt + (wid & 1))) * 4;
#pragma unroll
            for (int e = 0; e < 8; ++e) __builtin_amdgcn_raw_ptr_buffer_load_lds(rx, (LAS void*)(slot + e * 1024), 16, vx, s0 + (e >> 1) * (16 * KS * 4) + (e & 1) * 16, 0, AUX_NT);
        } else {
            const unsigned s0 = (32 * (2 * kt + (wid & 1)) * NN + col0(pa - 16) + (two_quads(pa) ? 128 : 192) * (wid >> 1) + 64 * qd) * 4;
#pragma unroll
            for (int e = 0; e < 8; ++e) __builtin_amdgcn_raw_ptr_buffer_load_lds(rw, (LAS void*)(slot + e * 1024), 16, vw, s0 + e * ROWB, 0, AUX_NT);
        }
    };
    load(0);
    char* peer_ws = p.ws[lane & 7];
    bool clear;
    const unsigned call = draw_call(ws, wid, clear);
    if (!clear) wait_words(ws, ws + OFF_EV + 4 * NR * 4, NR, 4, call - 1, call);
    asm volatile("" : "+v"(peer_ws));
    for (int u = 0; u < nu; ++u) {
        if (u > 0) { load(u + 3); WAITVM(24); }
        if (u == ST1 + 1) {
            BARRIER;
            if (wid == 0) arrive(ws, peer_ws, rank, 2, NGEMM, call);
        }
        int pa, kt, qd; conv_quad(gi + u * NGEMM, pa, kt, qd);
        f32x4 v[8]; u32x4 f[4];
#pragma unroll
        for (int e = 0; e < 8; ++e) v[e] = *(const LAS f32x4*)(ring + (u & 3) * 8192 + e * 1024 + lane * 16);
#pragma unroll
        for (int j = 0; j < 4; ++j) { f16x8 h;
#pragma unroll
            for (int e = 0; e < 8; ++e) h[e] = (_Float16)((pa < 16 ? v[2 * j + (e >> 2)][e & 3] : v[e][j]) * (pa < 16 ? XSCALE : WSCALE));
            f[j] = __builtin_bit_cast(u32x4, h); }
        pack16(f, ws, pk16_off(pa, kt) + qd * QB16, wid, lane, tail);
        store4(ws, OFF_DATA + own_off(pa, kt) + (4 * qd + wid) * 4096 + lane * 16, f);
        if (u == 0) {
            WAITVM(0);
            BARRIER;
            if (wid == 0 && lane < NR) st_sys(peer_ws + OFF_AV + (rank * 256 + gi) * 4, call);
            load(1); load(2); load(3);
            if (wid == 1) arrive(ws, peer_ws, rank, 1, NGEMM, call);
        }
    }
    WAITVM(0);
    BARRIER;
    if (wid == 0) arrive(ws, peer_ws, rank, 3, NGEMM, call);
    return call;
}

constexpr int G0 = 16 + round_end(0), NBI = (16 + NL) * 8, NTW = 7 * NBI, NTC = 7 * NCI, NT = NTW + NTC;
struct Item { const char* src; unsigned size, pk, raw, dof; int flag; bool three; };
constexpr int step_start(int s) { int n = 0; for (int k = 0; k < s; ++k) n += x_hi(k) - x_lo(k) + (k < 8 ? G0 - 16 : 0); return n; }
static_assert(step_start(NSTEP0) == 8 * G0, "order");
DI void block_of(unsigned t, int& i, int& kt) {
    if (t < 8u * G0) {
        int s = 0, hi = x_hi(0); i = t;
#pragma unroll
        for (int k = 1; k < NSTEP0; ++k) if (t >= (unsigned)step_start(k)) { s = k; i = t - step_start(k) + x_lo(k); hi = x_hi(k); }
        if (i >= hi) i += 16 - hi;
        kt = s - (i < 4 ? XL[0] : i < 8 ? XL[1] : i < 12 ? XL[2] : i < 16 ? XL[3] : 0);
    } else {
        unsigned u = 0, n = 1, b = 0;
#pragma unroll
        for (int r = 1; r < NROUND; ++r)
            if (t >= 8u * (16 + round_end(r - 1))) { b = 16 + round_end(r - 1); u = t - 8u * b; n = round_end(r) - round_end(r - 1); }
        kt = u / n; i = b + (u - kt * n);
    }
}
DI unsigned slab_out(unsigned s, int from) { return (s & 127u) * 32u * ROWB + (BCOLS + (CPR * from + (s >> 7)) * TW) * 4u; }
DI Item make_item(Params& p, unsigned t) {
    const int rank = p.rank;
    const unsigned tt = t < (unsigned)NTW ? t : t < (unsigned)NT ? t - NTW : 0, u = tt / 7, pj = tt - 7 * u;
    const int peer = (rank + 1 + pj) & 7;
    Item it;
    it.src = p.ws[peer]; it.size = t < (unsigned)NT ? WS_NEED : 0;
    if (t < (unsigned)NTW) {
        int i, kt; block_of(u, i, kt);
        const int panel = i < 16 ? i : 16 + colblock(i - 16, rank);
        it.three = !two_quads(panel);
        it.pk = pk16_off(panel, kt); it.raw = OFF_DATA + own_off(panel, kt);
        it.dof = OFF_DATA + (i < 16 ? i * XPANEL + 8 * XBLK : ring_off(i - 16)) + (7 * kt + pj) * (it.three ? WBLK : XBLK);
        it.flag = panel * 64 + 8 + 7 * kt + pj;
    } else {
        it.three = true;
        it.pk = OFF_PK32 + u * (3 * QB32); it.raw = OFF_RAW32 + u * WBLK; it.dof = slab_out(u, peer); it.flag = -1;
    }
    return it;
}
DI void load_main(const Item& it, unsigned off, LAS unsigned char* to, int lane) {
    const __amdgpu_buffer_rsrc_t r = __builtin_amdgcn_make_buffer_rsrc((void*)it.src, 0, it.size, 0x00020000);
    asm volatile("s_mov_b32 m0, %3\n\ts_nop 4\n\tbuffer_load_dwordx4 %0, %1, %2 offen sc0 sc1 lds\n\tbuffer_load_dwordx4 %0, %1, %2 offen offset:1024 sc0 sc1 lds\n\t"
                 "buffer_load_dwordx4 %0, %1, %2 offen offset:2048 sc0 sc1 lds" :: "v"(lane * 16), "s"(r), "s"(off), "s"((unsigned)(size_t)to) : "memory");
}
DI void load_tails(const Item& it, unsigned off, LAS unsigned char* to, int lane) {
    const __amdgpu_buffer_rsrc_t r = __builtin_amdgcn_make_buffer_rsrc((void*)it.src, 0, it.size, 0x00020000);
    const unsigned long long rest = it.flag < 0 ? (1ull << NE32 / 2) - 1 : (1ull << NE16 / 2) - 1;
    asm volatile("s_mov_b32 m0, %3\n\ts_nop 4\n\tbuffer_load_dwordx4 %0, %1, %2 offen sc0 sc1 lds\n\tbuffer_load_dwordx4 %0, %1, %2 offen offset:1024 sc0 sc1 lds\n\t"
                 "s_mov_b64 exec, %4\n\tbuffer_load_dwordx4 %0, %1, %2 offen offset:2048 sc0 sc1 lds\n\ts_mov_b64 exec, -1"
                 :: "v"(lane * 16), "s"(r), "s"(off), "s"((unsigned)(size_t)to), "s"(rest) : "memory");
}
constexpr unsigned NANS = 0x7fc07fc0u;
constexpr unsigned LQ = 14848, LMAIN = 2560;
struct Fetch {
    const char* ws; const char* out;
    LAS unsigned char* lds;
    unsigned slot; int lane; bool ok;
    template <int C0, bool WAITED> DI void quad(const Item& cur, unsigned qd, const Item& nx, unsigned nqd) {
        LAS unsigned char* const q = lds + slot; LAS unsigned char* const nq = lds + (LQ - slot);
        const unsigned npk = nx.pk + nqd * (nx.flag < 0 ? QB32 : QB16), nmain = npk + 4 * (nx.flag < 0 ? TS32 : TS16), ts = cur.flag < 0 ? TS32 : TS16;
#pragma unroll
        for (int c = 0; c < 4; ++c) {
            if (c > 0 || !WAITED) WAITVM(25);
            if (c == 0) load_tails(nx, npk, nq, lane);
            load_main(nx, nmain + c * 3072, nq + LMAIN + c * 3072, lane);
            u32x4 f[4];
            if (!unpack(cur.flag < 0, q + LMAIN + c * 3072, q + c * ts, lds + 2 * LQ, lane, f)) {
                const char* a = cur.src + (size_t)(cur.raw + qd * QUAD + c * 4096 + lane * 16);
                asm volatile("global_load_dwordx4 %0, %4, off sc0 sc1\n\tglobal_load_dwordx4 %1, %4, off offset:1024 sc0 sc1\n\t"
                             "global_load_dwordx4 %2, %4, off offset:2048 sc0 sc1\n\tglobal_load_dwordx4 %3, %4, off offset:3072 sc0 sc1\n\ts_waitcnt vmcnt(0)"
                             : "=&v"(f[0]), "=&v"(f[1]), "=&v"(f[2]), "=&v"(f[3]) : "v"(a) : "memory");
            }
            if (!ok) f[0] = f[1] = f[2] = f[3] = (u32x4){NANS, NANS, NANS, NANS};
            if (cur.flag >= 0) store4(ws, cur.dof + qd * QUAD + c * 4096 + lane * 16, f);
            else store4o(out, cur.dof + ((C0 + c) / 3 * 8 + (lane >> 5)) * ROWB + (C0 + c) % 3 * 512 + (lane & 31) * 16, f);
        }
        slot = LQ - slot;
    }
};
DI void fetch_wave(Params& p, LAS unsigned char* lds, unsigned call, int wid, int wv) {
    const int lane = lane_now(), rank = p.rank;
    char* const ws = p.ws[rank];
    Fetch F;
    F.ws = ws; F.out = (const char*)p.out[rank]; F.lds = lds; F.lane = lane;
    *(LAS u32x4*)(lds + 2 * LQ + lane * 16) = (u32x4){0, 0, 0, 0}; *(LAS u32x4*)(lds + 2 * LQ + 1024 + lane * 16) = (u32x4){0, 0, 0, 0};
    const unsigned wbase = (call - 1) * NTW - NCW, cbase = (call - 1) * NTC - NCW;
    static_assert(NL - RING == 32, "probe lanes");
    const unsigned vprobe = lane < 32 ? OFF_EV + lane * 4 : OFF_FIN + (lane - 32) * 4;
    static_assert(NCW <= 7 * step_start(1), "first items");
    if (wv == 0 && lane < NR) st_sys(p.ws[lane] + OFF_EV + (5 * NR + rank) * 4, call);
    {
        int i0, kt0; block_of(wv / 7, i0, kt0);
        const int q0 = i0 < 16 ? 2 * i0 : 2 * x_hi(0) + 3 * (i0 < 16 + CPR ? B0 + CPR * rank + i0 - 16 : i0 - 16 - CPR), peer = (rank + 1 + wv % 7) & 7;
        const unsigned v = lane < NR ? OFF_EV + (5 * NR + lane) * 4 : OFF_AV + peer * 1024 + (q0 + lane % (i0 < 16 ? 2 : 3)) * 4;
        const unsigned long long t0 = __builtin_amdgcn_s_memrealtime();
        F.ok = true;
        for (unsigned n = 1; !__all(ld_sys(ws + v) >= call); ++n) if ((n & 15) == 0 && give_up(ws, call, t0)) { F.ok = false; break; }
        asm volatile("" ::: "memory");
    }
    unsigned tw = wv, tc = wv;
    auto choose = [&](unsigned probe) -> unsigned {
        const unsigned ev = (unsigned)__ballot(probe >= call);
        if (tw < (unsigned)NTW) {
            const unsigned u = tw / 7;
            int i, kt; block_of(u, i, kt);
            const int stage = u < (unsigned)step_start(STEPS1) ? 1 : u < (unsigned)step_start(STEPS2) ? 2 : 3;
            if ((ev >> (8 * stage + ((rank + 1 + tw - 7 * u) & 7)) & 1) &&
                (i < 16 + RING || __builtin_amdgcn_readlane(probe, 32 + i - 16 - RING) >= 16 * call)) return tw;
        }
        return tc < (unsigned)NTC && (ev >> ((rank + 1 + tc % 7) & 7) & 1) ? NTW + tc : NT;
    };
    auto publish = [&](int flag) {
        if (flag >= 0 && lane == 0) st_sys(ws + OFF_F + flag * 4, call);
    };
    while (tw < (unsigned)NTW || tc < (unsigned)NTC) {
        unsigned t = tw;
        if (tw >= (unsigned)NCW) {
            const unsigned long long t0 = __builtin_amdgcn_s_memrealtime();
            while ((t = choose(ld_sys(ws + vprobe))) == (unsigned)NT) {
                if (give_up(ws, call, t0)) { F.ok = false; t = tw < (unsigned)NTW ? tw : NTW + tc; break; }
                __builtin_amdgcn_s_sleep(16);
            }
        }
        Item cur = make_item(p, t);
        F.slot = 0;
        load_tails(cur, cur.pk, lds, lane);
#pragma unroll
        for (int c = 0; c < 4; ++c) load_main(cur, cur.pk + 4 * (cur.flag < 0 ? TS32 : TS16) + c * 3072, lds + LMAIN + c * 3072, lane);
        WAITVM(0);
        int pflag = -1;
        for (;;) {
            unsigned tk, probe; unsigned long long sv;
            asm volatile("s_nop 3\n\ts_waitcnt vmcnt(25)\n\tglobal_load_dword %1, %6, %5 sc0 sc1\n\t"
                         "s_mov_b64 %2, exec\n\ts_mov_b64 exec, 1\n\tglobal_atomic_add %0, %3, %4, %5 sc0\n\ts_mov_b64 exec, %2"
                         : "=&v"(tk), "=&v"(probe), "=&s"(sv) : "v"(OFF_TICK + (cur.flag < 0 ? 128u : 0u)), "v"(1u), "s"(ws), "v"(vprobe) : "memory");
            F.quad<0, true>(cur, 0, cur, 1);
            asm volatile("s_waitcnt vmcnt(25)" : "+v"(tk), "+v"(probe) :: "memory");
            publish(pflag);
            if (cur.flag >= 0) tw = rfl(tk) - wbase; else tc = rfl(tk) - cbase;
            if (cur.three) F.quad<4, true>(cur, 1, cur, 2);
            t = choose(probe);
            if (t == (unsigned)NT && (tw < (unsigned)NTW || tc < (unsigned)NTC)) t = choose(ld_sys(ws + vprobe));
            const Item nxt = make_item(p, t);
            F.quad<8, false>(cur, cur.three ? 2 : 1, nxt, 0);
            pflag = cur.flag;
            if (t >= (unsigned)NT) break;
            cur = nxt;
        }
        WAITVM(0);
        publish(pflag);
    }
    arrive(ws, p.ws[lane & 7], rank, 4, NCW, call);
}
#define PT_SET(A0, A1, A2, A3, R) \
    asm volatile("s_waitcnt vmcnt(60)\n\t" R \
                 "global_load_dwordx4 a[" A0 "], %16, %20 sc1 nt\n\tglobal_load_dwordx4 a[" A1 "], %17, %20 sc1 nt\n\t" \
                 "global_load_dwordx4 a[" A2 "], %18, %20 sc1 nt\n\tglobal_load_dwordx4 a[" A3 "], %19, %20 sc1 nt" \
                 : "=&v"(r[0]), "=&v"(r[1]), "=&v"(r[2]), "=&v"(r[3]), "=&v"(r[4]), "=&v"(r[5]), "=&v"(r[6]), "=&v"(r[7]), \
                   "=&v"(r[8]), "=&v"(r[9]), "=&v"(r[10]), "=&v"(r[11]), "=&v"(r[12]), "=&v"(r[13]), "=&v"(r[14]), "=&v"(r[15]) \
                 : "v"(v), "v"(v + 2 * ROWB), "v"(v + 4 * ROWB), "v"(v + 6 * ROWB), "s"(out) : "memory", PT_CLOB)
#define PT_CLOB "a0", "a1", "a2", "a3", "a4", "a5", "a6", "a7", "a8", "a9", "a10", "a11", "a12", "a13", "a14", "a15", "a16", "a17", "a18", "a19", "a20", "a21", "a22", "a23", "a24", "a25", "a26", "a27", "a28", "a29", "a30", "a31", "a32", "a33", "a34", "a35", "a36", "a37", "a38", "a39", "a40", "a41", "a42", "a43", "a44", "a45", "a46", "a47", "a48", "a49", "a50", "a51", "a52", "a53", "a54", "a55", "a56", "a57", "a58", "a59", "a60", "a61", "a62", "a63", "a64", "a65", "a66", "a67", "a68", "a69", "a70", "a71", "a72", "a73", "a74", "a75", "a76", "a77", "a78", "a79", "a80", "a81", "a82", "a83", "a84", "a85", "a86", "a87", "a88", "a89", "a90", "a91", "a92", "a93", "a94", "a95", "a96", "a97", "a98", "a99", "a100", "a101", "a102", "a103", "a104", "a105", "a106", "a107", "a108", "a109", "a110", "a111", "a112", "a113", "a114", "a115", "a116", "a117", "a118", "a119", "a120", "a121", "a122", "a123", "a124", "a125", "a126", "a127"
#define PT_R0 "v_accvgpr_read_b32 %0, a0\n\t" "v_accvgpr_read_b32 %1, a1\n\t" "v_accvgpr_read_b32 %2, a2\n\t" "v_accvgpr_read_b32 %3, a3\n\t" "v_accvgpr_read_b32 %4, a4\n\t" "v_accvgpr_read_b32 %5, a5\n\t" "v_accvgpr_read_b32 %6, a6\n\t" "v_accvgpr_read_b32 %7, a7\n\t" "v_accvgpr_read_b32 %8, a8\n\t" "v_accvgpr_read_b32 %9, a9\n\t" "v_accvgpr_read_b32 %10, a10\n\t" "v_accvgpr_read_b32 %11, a11\n\t" "v_accvgpr_read_b32 %12, a12\n\t" "v_accvgpr_read_b32 %13, a13\n\t" "v_accvgpr_read_b32 %14, a14\n\t" "v_accvgpr_read_b32 %15, a15\n\t"
#define PT_R1 "v_accvgpr_read_b32 %0, a16\n\t" "v_accvgpr_read_b32 %1, a17\n\t" "v_accvgpr_read_b32 %2, a18\n\t" "v_accvgpr_read_b32 %3, a19\n\t" "v_accvgpr_read_b32 %4, a20\n\t" "v_accvgpr_read_b32 %5, a21\n\t" "v_accvgpr_read_b32 %6, a22\n\t" "v_accvgpr_read_b32 %7, a23\n\t" "v_accvgpr_read_b32 %8, a24\n\t" "v_accvgpr_read_b32 %9, a25\n\t" "v_accvgpr_read_b32 %10, a26\n\t" "v_accvgpr_read_b32 %11, a27\n\t" "v_accvgpr_read_b32 %12, a28\n\t" "v_accvgpr_read_b32 %13, a29\n\t" "v_accvgpr_read_b32 %14, a30\n\t" "v_accvgpr_read_b32 %15, a31\n\t"
#define PT_R2 "v_accvgpr_read_b32 %0, a32\n\t" "v_accvgpr_read_b32 %1, a33\n\t" "v_accvgpr_read_b32 %2, a34\n\t" "v_accvgpr_read_b32 %3, a35\n\t" "v_accvgpr_read_b32 %4, a36\n\t" "v_accvgpr_read_b32 %5, a37\n\t" "v_accvgpr_read_b32 %6, a38\n\t" "v_accvgpr_read_b32 %7, a39\n\t" "v_accvgpr_read_b32 %8, a40\n\t" "v_accvgpr_read_b32 %9, a41\n\t" "v_accvgpr_read_b32 %10, a42\n\t" "v_accvgpr_read_b32 %11, a43\n\t" "v_accvgpr_read_b32 %12, a44\n\t" "v_accvgpr_read_b32 %13, a45\n\t" "v_accvgpr_read_b32 %14, a46\n\t" "v_accvgpr_read_b32 %15, a47\n\t"
#define PT_R3 "v_accvgpr_read_b32 %0, a48\n\t" "v_accvgpr_read_b32 %1, a49\n\t" "v_accvgpr_read_b32 %2, a50\n\t" "v_accvgpr_read_b32 %3, a51\n\t" "v_accvgpr_read_b32 %4, a52\n\t" "v_accvgpr_read_b32 %5, a53\n\t" "v_accvgpr_read_b32 %6, a54\n\t" "v_accvgpr_read_b32 %7, a55\n\t" "v_accvgpr_read_b32 %8, a56\n\t" "v_accvgpr_read_b32 %9, a57\n\t" "v_accvgpr_read_b32 %10, a58\n\t" "v_accvgpr_read_b32 %11, a59\n\t" "v_accvgpr_read_b32 %12, a60\n\t" "v_accvgpr_read_b32 %13, a61\n\t" "v_accvgpr_read_b32 %14, a62\n\t" "v_accvgpr_read_b32 %15, a63\n\t"
#define PT_R4 "v_accvgpr_read_b32 %0, a64\n\t" "v_accvgpr_read_b32 %1, a65\n\t" "v_accvgpr_read_b32 %2, a66\n\t" "v_accvgpr_read_b32 %3, a67\n\t" "v_accvgpr_read_b32 %4, a68\n\t" "v_accvgpr_read_b32 %5, a69\n\t" "v_accvgpr_read_b32 %6, a70\n\t" "v_accvgpr_read_b32 %7, a71\n\t" "v_accvgpr_read_b32 %8, a72\n\t" "v_accvgpr_read_b32 %9, a73\n\t" "v_accvgpr_read_b32 %10, a74\n\t" "v_accvgpr_read_b32 %11, a75\n\t" "v_accvgpr_read_b32 %12, a76\n\t" "v_accvgpr_read_b32 %13, a77\n\t" "v_accvgpr_read_b32 %14, a78\n\t" "v_accvgpr_read_b32 %15, a79\n\t"
#define PT_R5 "v_accvgpr_read_b32 %0, a80\n\t" "v_accvgpr_read_b32 %1, a81\n\t" "v_accvgpr_read_b32 %2, a82\n\t" "v_accvgpr_read_b32 %3, a83\n\t" "v_accvgpr_read_b32 %4, a84\n\t" "v_accvgpr_read_b32 %5, a85\n\t" "v_accvgpr_read_b32 %6, a86\n\t" "v_accvgpr_read_b32 %7, a87\n\t" "v_accvgpr_read_b32 %8, a88\n\t" "v_accvgpr_read_b32 %9, a89\n\t" "v_accvgpr_read_b32 %10, a90\n\t" "v_accvgpr_read_b32 %11, a91\n\t" "v_accvgpr_read_b32 %12, a92\n\t" "v_accvgpr_read_b32 %13, a93\n\t" "v_accvgpr_read_b32 %14, a94\n\t" "v_accvgpr_read_b32 %15, a95\n\t"
#define PT_R6 "v_accvgpr_read_b32 %0, a96\n\t" "v_accvgpr_read_b32 %1, a97\n\t" "v_accvgpr_read_b32 %2, a98\n\t" "v_accvgpr_read_b32 %3, a99\n\t" "v_accvgpr_read_b32 %4, a100\n\t" "v_accvgpr_read_b32 %5, a101\n\t" "v_accvgpr_read_b32 %6, a102\n\t" "v_accvgpr_read_b32 %7, a103\n\t" "v_accvgpr_read_b32 %8, a104\n\t" "v_accvgpr_read_b32 %9, a105\n\t" "v_accvgpr_read_b32 %10, a106\n\t" "v_accvgpr_read_b32 %11, a107\n\t" "v_accvgpr_read_b32 %12, a108\n\t" "v_accvgpr_read_b32 %13, a109\n\t" "v_accvgpr_read_b32 %14, a110\n\t" "v_accvgpr_read_b32 %15, a111\n\t"
#define PT_R7 "v_accvgpr_read_b32 %0, a112\n\t" "v_accvgpr_read_b32 %1, a113\n\t" "v_accvgpr_read_b32 %2, a114\n\t" "v_accvgpr_read_b32 %3, a115\n\t" "v_accvgpr_read_b32 %4, a116\n\t" "v_accvgpr_read_b32 %5, a117\n\t" "v_accvgpr_read_b32 %6, a118\n\t" "v_accvgpr_read_b32 %7, a119\n\t" "v_accvgpr_read_b32 %8, a120\n\t" "v_accvgpr_read_b32 %9, a121\n\t" "v_accvgpr_read_b32 %10, a122\n\t" "v_accvgpr_read_b32 %11, a123\n\t" "v_accvgpr_read_b32 %12, a124\n\t" "v_accvgpr_read_b32 %13, a125\n\t" "v_accvgpr_read_b32 %14, a126\n\t" "v_accvgpr_read_b32 %15, a127\n\t"
DI void pack_tile(Params& p, LAS unsigned char* tail, unsigned call, int ct) {
    const int lane = lane_now(), rank = p.rank;
    char* const ws = p.ws[rank];
    const char* const out = (const char*)p.out[rank];
    const bool ok = wait_words(ws, ws + OFF_AVC + ct * 4, 1, 4, call, call);
    __builtin_amdgcn_fence(__ATOMIC_ACQUIRE, "agent");
    const unsigned s0 = 8 * ct, vb = slab_out(s0, rank) + (lane >> 5) * ROWB + (lane & 31) * 16;
    for (int i = -8; i < 96; ++i) {
        const unsigned n = i + 8 < 96 ? i + 8 : 95, v = vb + n / 3 * (8 * ROWB) + n % 3 * 512;
        unsigned r[16];
        if (i >= 0 && i < 8) WAITVM(28);
        switch (i & 7) {
            case 0: PT_SET("0:3", "4:7", "8:11", "12:15", PT_R0); break;
            case 1: PT_SET("16:19", "20:23", "24:27", "28:31", PT_R1); break;
            case 2: PT_SET("32:35", "36:39", "40:43", "44:47", PT_R2); break;
            case 3: PT_SET("48:51", "52:55", "56:59", "60:63", PT_R3); break;
            case 4: PT_SET("64:67", "68:71", "72:75", "76:79", PT_R4); break;
            case 5: PT_SET("80:83", "84:87", "88:91", "92:95", PT_R5); break;
            case 6: PT_SET("96:99", "100:103", "104:107", "108:111", PT_R6); break;
            default: PT_SET("112:115", "116:119", "120:123", "124:127", PT_R7); break;
        }
        if (i < 0) continue;
        if (!ok) r[0] = NANS;
        const u32x4 f[4] = {{r[0], r[1], r[2], r[3]}, {r[4], r[5], r[6], r[7]}, {r[8], r[9], r[10], r[11]}, {r[12], r[13], r[14], r[15]}};
        if (!pack32(f, ws, OFF_PK32 + (s0 * 3 + (i >> 2)) * QB32, i & 3, lane, tail)) store4(ws, OFF_RAW32 + s0 * WBLK + i * 4096 + lane * 16, f);
    }
    WAITVM(0);
    arrive(ws, p.ws[lane & 7], rank, 0, 16 * CPR, call);
}

DI f16x8 rdfrag(unsigned base, int off) { return *(const LAS f16x8*)(size_t)(base + off); }
DI void dma_to(unsigned ldsaddr) { asm volatile("s_mov_b32 m0, %0" :: "s"(ldsaddr) : "memory"); }
#define PIN_S(x) asm volatile("" : "+s"(x))
DI unsigned vadd(unsigned v, unsigned s) { unsigned r; asm volatile("v_add_u32 %0, %1, %2" : "=v"(r) : "s"(s), "v"(v)); return r; }
template <int AT> DI void jump_at(unsigned& x, int it, unsigned to) { asm volatile("s_cmp_eq_u32 %1, %3\n\ts_cselect_b32 %0, %2, %0" : "+s"(x) : "s"(it), "s"(to), "i"(AT) : "scc"); }
struct G7 {
    unsigned lds;
    __amdgpu_buffer_rsrc_t rD;
    unsigned voff;
    unsigned xb, an;
    unsigned blo, bhi;
    unsigned sA, sB;
    unsigned xs[3];
    unsigned ringB, nextA, nextB;
    f32x4 acc[8][12];
    f16x8 L[2][4][2], H[4][2], B0, B1;
};
template <int I> DI void dma7(const G7& g, unsigned src) {
    asm volatile("buffer_load_dwordx4 %0, %1, %2 offen offset:%3 sc0 sc1 lds" :: "v"(g.voff), "s"(g.rD), "s"(src), "i"(I * 1024) : "memory");
}
template <int SL> DI f16x8 rdB7(const G7& g, int j, int k) { return rdfrag(SL < 3 ? g.blo : g.bhi, (SL % 3) * QUAD + k * 4096 + j * 1024); }
#define MFMA(D, A, B, C) "v_mfma_f32_16x16x32_f16 " D ", " A ", " B ", " C
template <bool AG, bool ZERO> DI void mfma1(f32x4& acc, const f16x8& a, const f16x8& b) {
    if (AG) { if (ZERO) asm volatile(MFMA("%0", "%1", "%2", "0") : "=a"(acc) : "v"(a), "v"(b)); else asm volatile(MFMA("%0", "%1", "%2", "%0") : "+a"(acc) : "v"(a), "v"(b)); }
    else { if (ZERO) asm volatile(MFMA("%0", "%1", "%2", "0") : "=&v"(acc) : "v"(a), "v"(b)); else asm volatile(MFMA("%0", "%1", "%2", "%0") : "+v"(acc) : "v"(a), "v"(b)); }
    SCHED;
}
template <bool AG, bool ZERO> DI void mfma2(f32x4& acc, const f16x8& a, const f16x8& b, const f16x8& c, const f16x8& d) {
    if (AG) {
        if (ZERO) asm volatile(MFMA("%0", "%1", "%2", "0") "\n\t" MFMA("%0", "%3", "%4", "%0") : "=a"(acc) : "v"(a), "v"(b), "v"(c), "v"(d));
        else asm volatile(MFMA("%0", "%1", "%2", "%0") "\n\t" MFMA("%0", "%3", "%4", "%0") : "+a"(acc) : "v"(a), "v"(b), "v"(c), "v"(d));
    } else {
        if (ZERO) asm volatile(MFMA("%0", "%1", "%2", "0") "\n\t" MFMA("%0", "%3", "%4", "%0") : "=&v"(acc) : "v"(a), "v"(b), "v"(c), "v"(d));
        else asm volatile(MFMA("%0", "%1", "%2", "%0") "\n\t" MFMA("%0", "%3", "%4", "%0") : "+v"(acc) : "v"(a), "v"(b), "v"(c), "v"(d));
    }
    SCHED;
}
template <int J, bool ZERO, int VM> DI void phase(G7& g, int it) {
    constexpr int t = J % 3, c = J / 3, NX = (J + 1) % 6, PV = (J + 5) % 6;
    f16x8 B[5][2];
    B[0][0] = g.B0; B[0][1] = g.B1;
#pragma unroll
    for (int j = 0; j < 4; ++j) {
        const int n = 4 * t + j;
        const bool lastcol = t == 2 && j == 3;
        if (j == 3) { __builtin_amdgcn_s_waitcnt(WCNT(VM, 0)); __builtin_amdgcn_s_barrier(); SCHED; }
#pragma unroll
        for (int r = 0; r < 8; ++r) {
            const int q = j * 8 + r, m = (n & 1) ? 7 - r : r, k = r & 1;
            const f16x8 a0 = m < 4 ? g.L[c][m][k] : g.H[m - 4][k], a1 = m < 4 ? g.L[c][m][k ^ 1] : g.H[m - 4][k ^ 1];
            const int li = t == 2 && j < 3 && r >= 3 && r <= 5 && j * 3 + r - 3 < 8 ? j * 3 + r - 3 : -1;
            if (r == 1 || r == 2 || li >= 0 || (lastcol && r >= 3 && r <= 5)) {
                mfma1<t < 2, ZERO>(g.acc[m][n], a0, B[j][k]);
                if (r <= 2) B[j + 1][r - 1] = j < 3 ? rdB7<J>(g, j + 1, r - 1) : rdB7<NX>(g, 0, r - 1);
                else if (li >= 0) g.L[c ^ 1][li >> 1][li & 1] = rdfrag(g.an, (li & 1) * 4096 + (li >> 1) * 1024);
                else g.H[r == 3 ? 2 : 5 - r][1] = rdfrag(g.an, 4096 + (r == 3 ? 2 : 5 - r) * 1024);
                SCHED;
                mfma1<t < 2, false>(g.acc[m][n], a1, B[j][k ^ 1]);
            } else mfma2<t < 2, ZERO>(g.acc[m][n], a0, B[j][k], a1, B[j][k ^ 1]);
            if (lastcol && r <= 4) { const int row = r < 2 ? 3 : 4 - r, kk = r == 1; g.H[row][kk] = rdfrag(g.an, kk * 4096 + row * 1024); SCHED; }
            if (t == 2 && q == 22) { g.an = vadd(g.blo, g.xs[(2 * c + 3) % 3]); SCHED; }
            if (lastcol && r == 6) { g.an = vadd(g.blo, g.xs[(2 * c + 4) % 3]); SCHED; }
            constexpr int PA[4] = {1, 4, 6, 9}, PB0[4] = {12, 14, 18, 21}, PB2[4] = {2, 9, 13, 20};
            if (t < 2) {
                if (q == PA[0] - 1) { asm volatile("s_add_u32 m0, %0, %1" :: "s"(g.lds + LDS_A - g.xb), "s"(g.xs[(2 * c + t) % 3]) : "memory", "scc"); SCHED; }
                if (q == PA[0]) { dma7<0>(g, g.sA); SCHED; }
                if (q == PA[1]) { dma7<1>(g, g.sA); SCHED; }
                if (q == PA[2]) { dma7<2>(g, g.sA); SCHED; }
                if (q == PA[3]) { dma7<3>(g, g.sA); SCHED; }
                if (q == 24) { PIN_S(g.sA); g.sA += QUAD; PIN_S(g.sA); SCHED; }
                if (q == 26 && J == 0 && !ZERO) { jump_at<31>(g.sA, it, g.nextA); SCHED; }
            }
            if (q == (t < 2 ? PB0[0] : PB2[0]) - 1) { dma_to(g.lds + PV * QUAD); SCHED; }
            if (q == (t < 2 ? PB0[0] : PB2[0])) { dma7<0>(g, g.sB); SCHED; }
            if (q == (t < 2 ? PB0[1] : PB2[1])) { dma7<1>(g, g.sB); SCHED; }
            if (q == (t < 2 ? PB0[2] : PB2[2])) { dma7<2>(g, g.sB); SCHED; }
            if (q == (t < 2 ? PB0[3] : PB2[3])) { dma7<3>(g, g.sB); SCHED; }
            if (q == (t < 2 ? 25 : 21)) { PIN_S(g.sB); g.sB += QUAD; PIN_S(g.sB); SCHED; }
            if (q == 28 && J == 0 && !ZERO) { jump_at<3>(g.sB, it, g.ringB); SCHED; }
            if (q == 30 && J == 0 && !ZERO) { jump_at<31>(g.sB, it, g.nextB); SCHED; }
        }
    }
    g.B0 = B[4][0]; g.B1 = B[4][1];
}
template <bool FIRST> DI void trip(G7& g, int it) {
    SCHED;
    phase<0, FIRST, FIRST ? 60 : 28>(g, it); phase<1, FIRST, FIRST ? 60 : 24>(g, it); phase<2, FIRST, 16>(g, it);
    phase<3, false, 28>(g, it); phase<4, false, 24>(g, it); phase<5, false, 16>(g, it);
    const unsigned x0 = g.xs[0]; g.xs[0] = g.xs[1]; g.xs[1] = g.xs[2]; g.xs[2] = x0;
}
struct Short { unsigned pv, sb, sx, bn, bx; };
template <int J, bool ZERO, int VM> DI void phase2(G7& g, Short& h, int it) {
    constexpr int t = J & 1, c = J >> 1;
    f16x8 B[5][2];
    B[0][0] = g.B0; B[0][1] = g.B1;
#pragma unroll
    for (int j = 0; j < 4; ++j) {
        const int n = 4 * t + j;
        const bool lastcol = t == 1 && j == 3;
        if (j == 3) { __builtin_amdgcn_s_waitcnt(WCNT(VM, 0)); __builtin_amdgcn_s_barrier(); SCHED; }
#pragma unroll
        for (int r = 0; r < 8; ++r) {
            const int q = j * 8 + r, m = (n & 1) ? 7 - r : r, k = r & 1;
            const f16x8 a0 = m < 4 ? g.L[c][m][k] : g.H[m - 4][k], a1 = m < 4 ? g.L[c][m][k ^ 1] : g.H[m - 4][k ^ 1];
            const int li = t == 1 && j < 3 && r >= 3 && r <= 5 && j * 3 + r - 3 < 8 ? j * 3 + r - 3 : -1;
            if (r == 1 || r == 2 || li >= 0 || (lastcol && r >= 3 && r <= 5)) {
                mfma1<true, ZERO>(g.acc[m][n], a0, B[j][k]);
                if (r <= 2) B[j + 1][r - 1] = j < 3 ? rdfrag(h.bn, (r - 1) * 4096 + (j + 1) * 1024) : rdfrag(h.bx, (r - 1) * 4096);
                else if (li >= 0) g.L[c ^ 1][li >> 1][li & 1] = rdfrag(g.an, (li & 1) * 4096 + (li >> 1) * 1024);
                else g.H[r == 3 ? 2 : 5 - r][1] = rdfrag(g.an, 4096 + (r == 3 ? 2 : 5 - r) * 1024);
                SCHED;
                mfma1<true, false>(g.acc[m][n], a1, B[j][k ^ 1]);
            } else mfma2<true, ZERO>(g.acc[m][n], a0, B[j][k], a1, B[j][k ^ 1]);
            if (lastcol && r <= 4) { const int row = r < 2 ? 3 : 4 - r, kk = r == 1; g.H[row][kk] = rdfrag(g.an, kk * 4096 + row * 1024); SCHED; }
            if (t == 1 && q == 22) { g.an = vadd(g.blo, g.xs[(2 * c + 3) % 3]); SCHED; }
            if (lastcol && r == 6) { g.an = vadd(g.blo, g.xs[(2 * c + 4) % 3]); SCHED; }
            if (q == 0) { asm volatile("s_add_u32 m0, %0, %1" :: "s"(g.lds + LDS_A - g.xb), "s"(g.xs[J % 3]) : "memory", "scc"); SCHED; }
            if (q == 1) { dma7<0>(g, g.sA); SCHED; }
            if (q == 2) { dma7<1>(g, g.sA); SCHED; }
            if (q == 4) { dma7<2>(g, g.sA); SCHED; }
            if (q == 6) { dma7<3>(g, g.sA); SCHED; }
            if (q == 8) { PIN_S(g.sA); g.sA += QUAD; PIN_S(g.sA); SCHED; }
            if (q == 10) { asm volatile("s_add_u32 m0, %0, %1" :: "s"(g.lds), "s"(h.pv) : "memory", "scc"); SCHED; }
            if (q == 11) { dma7<0>(g, g.sB); SCHED; }
            if (q == 13) { dma7<1>(g, g.sB); SCHED; }
            if (q == 16) { dma7<2>(g, g.sB); SCHED; }
            if (q == 18) { dma7<3>(g, g.sB); SCHED; }
            if (q == 19) { PIN_S(g.sB); g.sB += QUAD; PIN_S(g.sB); SCHED; }
            if (q == 20 && J == 2) { jump_at<2>(g.sB, it, g.ringB); SCHED; }
            if (q == 21) { h.pv = h.sb; h.sb = h.sx; PIN_S(h.sx); h.sx = h.sx == 5 * QUAD ? 0 : h.sx + QUAD; PIN_S(h.sx); SCHED; }
            if (q == 29) { h.bn = h.bx; h.bx = vadd(g.blo, h.sx); SCHED; }
        }
    }
    g.B0 = B[4][0]; g.B1 = B[4][1];
    if (J == 3) { const unsigned x0 = g.xs[0]; g.xs[0] = g.xs[1]; g.xs[1] = g.xs[2]; g.xs[2] = x0; }
}
DI void short_tile(G7& g) {
    Short h;
    h.pv = 5 * QUAD; h.sb = 0; h.sx = QUAD; h.bn = g.blo; h.bx = vadd(g.blo, QUAD);
    SCHED;
    phase2<0, true, 60>(g, h, 0); phase2<1, true, 12>(g, h, 0); phase2<2, false, 12>(g, h, 0); phase2<3, false, 12>(g, h, 0);
    for (int it = 1; it < 32; ++it) { SCHED; phase2<0, false, 12>(g, h, it); phase2<1, false, 12>(g, h, it); phase2<2, false, 12>(g, h, it); phase2<3, false, 12>(g, h, it); }
}
DI void g7_init(G7& g, LAS unsigned char* lds, const char* data, unsigned bytes, int wid) {
    const int lane = lane_now(), wr = wid >> 1, wc = wid & 1;
    const unsigned lbase = (unsigned)(size_t)lds;
    g.rD = __builtin_amdgcn_make_buffer_rsrc((void*)data, 0, bytes, 0x00020000);
    g.lds = lbase + wid * 4096; g.voff = wid * 4096 + lane * 16;
    g.xb = LDS_A + (wr - wc) * 8192; g.blo = lbase + wc * 8192 + lane * 16; g.bhi = g.blo + 3 * QUAD;
    asm volatile("" : "+v"(g.blo), "+v"(g.bhi), "+v"(g.voff));
}
DI void g7_fill(G7& g, unsigned to, unsigned src) { dma_to(g.lds + to); asm volatile("s_nop 4"); dma7<0>(g, src); dma7<1>(g, src); dma7<2>(g, src); dma7<3>(g, src); }
DI void g7_prologue(G7& g, unsigned dA, unsigned dB) {
#pragma unroll
    for (int i = 0; i < 5; ++i) { if (i < 3) g7_fill(g, LDS_A + i * QUAD, dA + i * QUAD); g7_fill(g, i * QUAD, dB + i * QUAD); }
    WAITVM(0); __builtin_amdgcn_s_barrier(); SCHED;
    const unsigned a0 = vadd(g.blo, g.xb);
#pragma unroll
    for (int i = 0; i < 8; ++i) { g.L[0][i >> 1][i & 1] = rdfrag(a0, (i & 1) * 4096 + (i >> 1) * 1024); g.H[i >> 1][i & 1] = rdfrag(a0, QUAD + (i & 1) * 4096 + (i >> 1) * 1024); }
    g.B0 = rdB7<0>(g, 0, 0); g.B1 = rdB7<0>(g, 0, 1);
    __builtin_amdgcn_s_waitcnt(WCNT(63, 0)); __builtin_amdgcn_s_barrier(); SCHED;
    g.sA = dA + 3 * QUAD; g.sB = dB + 5 * QUAD; g.xs[0] = g.xb; g.xs[1] = g.xb + QUAD; g.xs[2] = g.xb + 2 * QUAD; g.an = vadd(g.blo, g.xs[2]);
    PIN_S(g.xs[0]); PIN_S(g.xs[1]); PIN_S(g.xs[2]);
}
#define RD8 "v_accvgpr_read_b32 %0, %8\n\tv_accvgpr_read_b32 %1, %9\n\tv_accvgpr_read_b32 %2, %10\n\tv_accvgpr_read_b32 %3, %11\n\t" \
            "v_accvgpr_read_b32 %4, %12\n\tv_accvgpr_read_b32 %5, %13\n\tv_accvgpr_read_b32 %6, %14\n\tv_accvgpr_read_b32 %7, %15"
#define RD8_OUT "=&v"(z[0]), "=&v"(z[1]), "=&v"(z[2]), "=&v"(z[3]), "=&v"(z[4]), "=&v"(z[5]), "=&v"(z[6]), "=&v"(z[7])
#define RD8_IN(m, r) "a"(g.acc[m][0][r]), "a"(g.acc[m][1][r]), "a"(g.acc[m][2][r]), "a"(g.acc[m][3][r]), "a"(g.acc[m][4][r]), "a"(g.acc[m][5][r]), "a"(g.acc[m][6][r]), "a"(g.acc[m][7][r])
template <int NQ> DI void g7_epilogue(G7& g, __amdgpu_buffer_rsrc_t rO, unsigned so, unsigned rowb, float sc) {
    const int lz = lane_now();
    const unsigned vo = 4 * (lz >> 4) * rowb + 16 * (lz & 15);
    if (NQ == 3) asm volatile("s_nop 15" : "+v"(g.acc[0][11]), "+v"(g.acc[1][11]), "+v"(g.acc[2][11]), "+v"(g.acc[3][11]));
    else asm volatile("s_nop 15");
    float z[8];
    asm volatile(RD8 : RD8_OUT : RD8_IN(0, 0));
#pragma unroll
    for (int i = 0; i < 32; ++i) {
        const int m = i >> 2, r = i & 3, m2 = (i + 1 < 32 ? i + 1 : i) >> 2, r2 = (i + 1 < 32 ? i + 1 : i) & 3;
        f32x4 a[3], t[3];
        a[0] = (f32x4){z[0], z[1], z[2], z[3]}; a[1] = (f32x4){z[4], z[5], z[6], z[7]};
        if (NQ == 3) a[2] = (f32x4){g.acc[m][8][r], g.acc[m][9][r], g.acc[m][10][r], g.acc[m][11][r]};
#pragma unroll
        for (int q = 0; q < NQ; ++q) t[q] = a[q] * (-1.44269504088896f / SCALE);
        SCHED;
#pragma unroll
        for (int e = 0; e < 4 * NQ; ++e) t[e >> 2][e & 3] = __builtin_amdgcn_fmed3f(t[e >> 2][e & 3], -86.5617024533378f, 86.5617024533378f);
        SCHED;
#pragma unroll
        for (int e = 0; e < 4 * NQ; ++e) t[e >> 2][e & 3] = __builtin_amdgcn_exp2f(t[e >> 2][e & 3]);
        SCHED;
#pragma unroll
        for (int q = 0; q < NQ; ++q) t[q] = __builtin_elementwise_fma(t[q], (f32x4){sc, sc, sc, sc}, (f32x4){sc, sc, sc, sc});
        SCHED;
#pragma unroll
        for (int e = 0; e < 4 * NQ; ++e) t[e >> 2][e & 3] = __builtin_amdgcn_rcpf(t[e >> 2][e & 3]);
        SCHED;
#pragma unroll
        for (int q = 0; q < NQ; ++q) t[q] = a[q] * t[q];
        const unsigned o = so + (m * 16 + r) * rowb;
        if (NQ == 3) asm volatile("buffer_store_dwordx4 %16, %19, %20, %21 offen nt\n\tbuffer_store_dwordx4 %17, %19, %20, %21 offen offset:256 nt\n\t"
                                  "buffer_store_dwordx4 %18, %19, %20, %21 offen offset:512 nt\n\t" RD8
                                  : RD8_OUT : RD8_IN(m2, r2), "v"(t[0]), "v"(t[1]), "v"(t[2]), "v"(vo), "s"(rO), "s"(o) : "memory");
        else asm volatile("buffer_store_dwordx4 %16, %18, %19, %20 offen nt\n\tbuffer_store_dwordx4 %17, %18, %19, %20 offen offset:256 nt\n\t" RD8
                          : RD8_OUT : RD8_IN(m2, r2), "v"(t[0]), "v"(t[1]), "v"(vo), "s"(rO), "s"(o) : "memory");
        SCHED;
    }
}
DI int lead_ones(bool ok) { const unsigned long long m = ~__ballot(ok); return m ? __builtin_ctzll(m) : 64; }

struct Ready {
    char* ws; unsigned call;
    __amdgpu_buffer_rsrc_t rF; LAS unsigned char* zone;
    int pa, pb, n; bool pending; float one;
    DI void poll(int need) {
        const unsigned long long t0 = __builtin_amdgcn_s_memrealtime();
        const int lane = lane_now();
        for (;;) {
            const unsigned a = ld_sys(ws + OFF_F + pa * 256 + lane * 4), b = ld_sys(ws + OFF_F + pb * 256 + lane * 4);
            n = lead_ones(lane < 8 || (a >= call && b >= call));
            if (n >= need) break;
            if (give_up(ws, call, t0)) { n = 64; one = __builtin_nanf(""); break; }
            __builtin_amdgcn_s_sleep(16);
        }
    }
    DI void request() {
        const int lane = lane_now();
        __builtin_amdgcn_raw_ptr_buffer_load_lds(rF, (LAS void*)zone, 4, lane * 4, pa * 256, 0, AUX_SYS);
        __builtin_amdgcn_raw_ptr_buffer_load_lds(rF, (LAS void*)(zone + 256), 4, lane * 4, pb * 256, 0, AUX_SYS);
        pending = true;
    }
    DI int plan(int it, int stop) {
        if (n == 64) return stop;
        const int safe = (n - 4) >> 1;
        if (safe <= it + 1) { request(); return it + 1; }
        return safe - 1 < stop ? safe - 1 : stop;
    }
    DI void check(int need) {
        if (pending) {
            const int lane = lane_now();
            const unsigned a = *(const volatile LAS unsigned*)(zone + lane * 4), b = *(const volatile LAS unsigned*)(zone + 256 + lane * 4);
            n = lead_ones(lane < 8 || (a >= call && b >= call)); pending = false;
        }
        if (n < need) poll(need);
        asm volatile("" ::: "memory");
    }
};

DI void gemm_wg(Params& p, LAS unsigned char* lds, int wid, int band, int S) {
    const int wr = wid >> 1, wc = wid & 1, rank = p.rank;
    char* const ws = p.ws[rank];
    const unsigned call = convert_shards(p, lds, wid, S * (16 / RB) + band);
    int T = S;
    G7 g;
    g7_init(g, lds, ws + OFF_DATA, DATAB, wid);
    Ready R;
    R.ws = ws; R.call = call; R.rF = __builtin_amdgcn_make_buffer_rsrc((void*)(ws + OFF_F), 0, NPAN * 256, 0x00020000);
    R.zone = lds + LDS_ZONE + wid * 512; R.pending = false; R.one = SCALE;
    if (!wait_words(ws, ws + OFF_EV + (3 * NR + rank) * 4, 1, 4, call, call) || rfl(ld_ag(ws + OFF_TMO)) == call) R.one = __builtin_nanf("");
    R.pa = RB * band + (T & (RB - 1)); R.pb = 16 + colblock(T >> RBL, rank); R.n = 8;
    g7_prologue(g, rfl(own_off(R.pa, 0)), rfl(own_off(R.pb, 0)));
    const __amdgpu_buffer_rsrc_t rO = __builtin_amdgcn_make_buffer_rsrc((void*)p.out[rank], 0, OUTB, 0x00020000);
    unsigned tprev = 0;
    int li = T >> RBL;
    for (int ti = 0;; ++ti) {
        const int pm = R.pa, cb = R.pb - 16;
        S += NWB;
        const int job = S - PSLOT;
        if ((unsigned)job < (unsigned)NJOB) S += NWB;
        const bool has_next = S < NROUND * NWB, deal = has_next && S >= (NROUND - 1) * NWB;
        T = S < PSLOT ? S : S - NJOB;
        int qa = pm, ql = li;
        if (has_next && !deal) { qa = RB * band + (T & (RB - 1)); ql = T >> RBL; }
        g.ringB = rfl(ring_off(li)); g.nextA = rfl(own_off(qa, 0)); g.nextB = rfl(own_off(16 + colblock(ql, rank), 0));
        {
            const unsigned now = (unsigned)__builtin_amdgcn_s_memrealtime();
            if (deal && blockIdx.x >> 3 == NG / 2 && wid == 0 && lane_now() == 0)
                __hip_atomic_store((GAS unsigned long long*)(ws + OFF_ETA + (blockIdx.x & 7) * 8), (unsigned long long)call << 32 | (3 * now - 2 * tprev), __ATOMIC_RELAXED, __HIP_MEMORY_SCOPE_AGENT);
            tprev = now;
        }
        R.check(4);
        int lim = R.plan(0, deal ? 29 : 32);
        trip<true>(g, 0);
        for (int it = 1; it < 32;) {
            if (it >= lim) {
                R.check(it < 30 ? 2 * it + 4 : 64);
                lim = R.plan(it, deal && it < 29 ? 29 : 32);
                if (deal && it == 29) {
                    const int lane = lane_now();
                    const unsigned long long t0 = __builtin_amdgcn_s_memrealtime();
                    unsigned long long e;
                    int place = blockIdx.x & 7;
                    while (!__all((unsigned)((e = __hip_atomic_load((GAS unsigned long long*)(ws + OFF_ETA + (lane & 7) * 8), __ATOMIC_RELAXED, __HIP_MEMORY_SCOPE_AGENT)) >> 32) == call)) {
                        if (give_up(ws, call, t0)) { R.one = __builtin_nanf(""); e = 0; break; }
                        __builtin_amdgcn_s_sleep(8);
                    }
                    const int d = (int)((unsigned)e - __builtin_amdgcn_readlane((unsigned)e, place));
                    place = __builtin_popcount((unsigned)__ballot(d < 0 || (d == 0 && lane < place)) & 0xffu);
                    constexpr int T4 = (NROUND - 1) * NWB - NJOB, NF = RB * (NL - 2) - T4;
                    const int P = place * NG + (blockIdx.x >> 3), bd = P / NF;
                    if (bd < 16 / RB) { const int t = T4 + P - bd * NF; qa = RB * bd + (t & (RB - 1)); ql = t >> RBL; }
                    else { const int t = P - 16 / RB * NF; qa = t & 15; ql = NL - 2 + (t >> 4); }
                    g.nextA = rfl(own_off(qa, 0)); g.nextB = rfl(own_off(16 + colblock(ql, rank), 0));
                }
            }
            do trip<false>(g, it); while (++it < lim);
        }
        R.pa = qa; R.pb = 16 + colblock(ql, rank);
        R.request();
        g7_epilogue<3>(g, rO, rfl((pm * 256 + wr * 128) * ROWB + (col0(cb) + wc * 192) * 4), ROWB, R.one);
        WAITVM(60);
        if (wid == 0 && lane_now() == 0) __hip_atomic_fetch_add((gu32*)(ws + OFF_FIN + li * 4), 1u, __ATOMIC_RELAXED, __HIP_MEMORY_SCOPE_AGENT);
        if (li < CPR) {
            WAITVM(0);
            BARRIER;
            if (wid == 0) {
                __builtin_amdgcn_fence(__ATOMIC_RELEASE, "agent");
                WAITVM(0);
                if (lane_now() == 0) st_sys(ws + OFF_AVC + (li * 16 + pm) * 4, call);
            }
        }
        if ((unsigned)job < (unsigned)NJOB) { pack_tile(p, lds + LDS_ZONE + 4096 + wid * 1024, call, job % CPR * 16 + RB * band + job / CPR * 4 + wid); }
        if (!has_next) break;
        li = ql;
        if (li >= NL - 2) {
            g.ringB = rfl(ring_off(li));
            R.check(64);
            short_tile(g);
            g7_epilogue<2>(g, rO, rfl((R.pa * 256 + wr * 128) * ROWB + (col0(R.pb - 16) + wc * 128) * 4), ROWB, R.one);
            break;
        }
    }
    __builtin_amdgcn_s_waitcnt(WCNT(0, 0));
}

__global__ __launch_bounds__(256, 1) void fused(Args) {
    extern __shared__ __attribute__((aligned(16))) unsigned char shm[];
    Params& p = *(Params*)__builtin_amdgcn_kernarg_segment_ptr();
    LAS unsigned char* lds = (LAS unsigned char*)shm;
    char* const ws = p.ws[p.rank];
    const int wid = rfl(threadIdx.x >> 6);
    const int grp = blockIdx.x & 7, idx = blockIdx.x >> 3;
    if (idx < NG) gemm_wg(p, lds, wid, (grp & 1 ? 0 : 8 / RB) + (grp >> RBL), (grp >> 1 & (RB / 2 - 1)) * NG + idx);
    else {
        const int cw = (idx - NG) * 8 + grp;
        bool clear;
        const unsigned call = draw_call(ws, wid, clear);
        fetch_wave(p, lds + wid * 32768, call, wid, cw * 4 + wid);
    }
}
}

extern "C" void kernel_launch(int rank, int nranks, ncclComm_t comm, hipStream_t stream,
                              void* const* d_in, const int* in_sizes, int n_in,
                              void* d_out, int out_size,
                              const int* in_dtypes, int out_dtype,
                              void* const* d_out_all,
                              void* const* d_ws_all, size_t ws_bytes) {
    if (nranks != NR || (unsigned)rank >= (unsigned)NR || n_in != 2 || ws_bytes < WS_NEED || in_sizes[0] != MM * KS || in_sizes[1] != KS * NN || out_size != MM * NN) {
        fprintf(stderr, "kernel_launch: unexpected problem shape\n");
        return;
    }
    static bool attr_set[NR];
    if (!attr_set[rank]) {
        HIP_CHECK(hipFuncSetAttribute((const void*)fused, hipFuncAttributeMaxDynamicSharedMemorySize, LDS_BYTES));
        attr_set[rank] = true;
    }
    Args p;
    p.x = (const float*)d_in[0]; p.w = (const float*)d_in[1];
    for (int i = 0; i < NR; ++i) { p.ws[i] = (char*)d_ws_all[i]; p.out[i] = (float*)d_out_all[i]; }
    p.rank = rank; p.pad = 0;
    hipLaunchKernelGGL(fused, dim3(256), dim3(256), LDS_BYTES, stream, p);
    HIP_CHECK(hipGetLastError());
}
```

```cpp
#include <hip/hip_runtime.h>
#include <rccl/rccl.h>
#include <cstdio>

#define HIP_CHECK(expr) do { hipError_t _e = (expr); if (_e != hipSuccess) { \
    fprintf(stderr, "HIP error %s at %s:%d\n", hipGetErrorString(_e), __FILE__, __LINE__); \
    return; } } while (0)

#define LAS __attribute__((address_space(3)))
#define GAS __attribute__((address_space(1)))
#define DI __device__ __forceinline__
typedef _Float16 f16x8 __attribute__((ext_vector_type(8)));
typedef float f32x4 __attribute__((ext_vector_type(4)));
typedef unsigned u32x4 __attribute__((ext_vector_type(4)));
typedef unsigned u32x2 __attribute__((ext_vector_type(2)));
typedef GAS unsigned gu32;

#define NG 29
namespace {
constexpr int MM = 4096, NN = 32768, KS = 512, NR = 8;
constexpr int NGEMM = 8 * NG, NCOMM = 256 - NGEMM, NCW = 3 * NCOMM;
constexpr int TW = 384;
constexpr int CPR = 2, NCB = NR * CPR;
constexpr int BCOLS = NN - NCB * TW, NBB = (BCOLS + TW - 1) / TW;
constexpr int NWP = NBB + NCB, NPAN = 16 + NWP;
constexpr int NL = CPR + NBB;
constexpr int RBL = 3, RB = 1 << RBL, NWB = NGEMM * RB / 16, NJOB = CPR * RB / 4, NROUND = 5, PSLOT = 2 * NWB - NJOB;
static_assert(NROUND * NWB == RB * NL + NJOB, "slots");
constexpr unsigned QUAD = 16384, XBLK = 2 * QUAD, WBLK = 3 * QUAD, XPANEL = 64 * XBLK;
static_assert(BCOLS - (NBB - 2) * TW == 2 * 256, "short blocks");
constexpr unsigned NE16 = 48, NE32 = 16, TS16 = 512 + 2 * NE16, TS32 = 512 + 2 * NE32, QB16 = 4 * (TS16 + 3072), QB32 = 4 * (TS32 + 3072);
constexpr unsigned OUTB = (unsigned)MM * NN * 4u, ROWB = NN * 4;
constexpr unsigned OFF_CALL = 0;
constexpr unsigned OFF_TMO = 32768;
constexpr unsigned OFF_FIN = 32768 + 1024;
constexpr unsigned OFF_ETA = 32768 + 2048;
constexpr unsigned OFF_TICK = 32768 + 4096;
constexpr unsigned OFF_F = 65536;
constexpr unsigned OFF_AV = 131072;
constexpr unsigned OFF_AVC = OFF_AV + 8192;
constexpr unsigned OFF_CNT = 196608;
constexpr unsigned OFF_EV = OFF_CNT + 4096;
constexpr int RING = NL - 32;
constexpr unsigned OFF_DATA = 1u << 20;
constexpr unsigned REL_WL = 16 * XPANEL;
constexpr unsigned REL_WR = REL_WL + NWP * 8 * WBLK;
constexpr unsigned DATAB = REL_WR + RING * 56 * WBLK;
constexpr int NQ16 = 16 * 8 * 2 + (NWP - 2) * 8 * 3 + 32;
constexpr unsigned OFF_PK16 = OFF_DATA + DATAB;
constexpr int NCI = CPR * (MM / 32);
constexpr unsigned OFF_PK32 = OFF_PK16 + (NQ16 + 16) * QB16;
constexpr unsigned OFF_RAW32 = OFF_PK32 + NCI * 3 * QB32;
constexpr unsigned WS_NEED = OFF_RAW32 + NCI * WBLK;
static_assert(NPAN * 256 <= 65536, "flags");
constexpr int LDS_A = 6 * QUAD, LDS_ZONE = LDS_A + 3 * QUAD, LDS_BYTES = LDS_ZONE + 8192;
constexpr unsigned long long DEADLINE = 50000000ull;
constexpr float XSCALE = 0.5f, WSCALE = 32.0f, SCALE = XSCALE * WSCALE;
constexpr int AUX_SYS = 17, AUX_NT = 2;

struct Args {
    const float* x; const float* w;
    char* ws[NR]; float* out[NR];
    int rank, pad;
};
typedef const __attribute__((address_space(4))) Args Params;

constexpr int WCNT(int vm, int lgkm) { return (vm & 15) | (7 << 4) | ((lgkm & 15) << 8) | ((vm >> 4) << 14); }
#define SCHED __builtin_amdgcn_sched_barrier(0)
#define WAITVM(n) asm volatile("s_waitcnt vmcnt(" #n ")" ::: "memory")
#define BARRIER do { asm volatile("" ::: "memory"); __builtin_amdgcn_s_barrier(); asm volatile("" ::: "memory"); } while (0)

DI unsigned ld_sys(const char* p) { return __hip_atomic_load((gu32*)p, __ATOMIC_RELAXED, __HIP_MEMORY_SCOPE_SYSTEM); }
DI void st_sys(char* p, unsigned v) { __hip_atomic_store((gu32*)p, v, __ATOMIC_RELAXED, __HIP_MEMORY_SCOPE_SYSTEM); }
DI unsigned ld_ag(const char* p) { return __hip_atomic_load((gu32*)p, __ATOMIC_RELAXED, __HIP_MEMORY_SCOPE_AGENT); }
DI void st_ag(char* p, unsigned v) { __hip_atomic_store((gu32*)p, v, __ATOMIC_RELAXED, __HIP_MEMORY_SCOPE_AGENT); }
DI unsigned rfl(unsigned x) { return __builtin_amdgcn_readfirstlane(x); }
DI int lane_now() { int l; asm volatile("v_mbcnt_lo_u32_b32 %0, -1, 0\n\tv_mbcnt_hi_u32_b32 %0, -1, %0" : "=v"(l)); return l; }
DI bool give_up(char* ws, unsigned call, unsigned long long t0) {
    if (rfl(ld_ag(ws + OFF_TMO)) == call || __builtin_amdgcn_s_memrealtime() - t0 > DEADLINE) { st_ag(ws + OFF_TMO, call); return true; }
    return false;
}
DI bool arrive(char* ws, char* lane_ws, int rank, int kind, unsigned n, unsigned call) {
    const int lane = lane_now();
    unsigned t = 0;
    if (lane == 0) t = __hip_atomic_fetch_add((gu32*)(ws + OFF_CNT + kind * 128), 1u, __ATOMIC_RELAXED, __HIP_MEMORY_SCOPE_AGENT);
    const bool last = rfl(t) + 1 == n * call;
    if (last && lane < NR) st_sys(lane_ws + OFF_EV + (kind * NR + rank) * 4, call);
    return last;
}
DI bool wait_words(char* ws, const char* f, int n, int step, unsigned want, unsigned call) {
    const int lane = lane_now();
    const unsigned long long t0 = __builtin_amdgcn_s_memrealtime();
    for (;;) {
        bool ok = true;
#pragma unroll
        for (int i = 0; i < 4; ++i) { const int k = lane + 64 * i; if (64 * i < n) ok &= ld_sys(f + (k < n ? k : 0) * step) >= want; }
        if (__all(ok)) break;
        if (give_up(ws, call, t0)) return false;
        __builtin_amdgcn_s_sleep(4);
    }
    asm volatile("" ::: "memory");
    return true;
}

DI void store1(const char* base, unsigned v, const u32x4& f) {
    asm volatile("s_nop 4\n\tglobal_store_dwordx4 %0, %1, %2 sc0 sc1\n\ts_nop 1" :: "v"(v), "v"(f), "s"(base) : "memory");
}
DI void store3(const char* base, unsigned v, const u32x4& a, const u32x4& b, const u32x4& c) {
    asm volatile("s_nop 4\n\tglobal_store_dwordx4 %0, %1, %4 sc0 sc1\n\t"
                 "global_store_dwordx4 %0, %2, %4 offset:1024 sc0 sc1\n\t"
                 "global_store_dwordx4 %0, %3, %4 offset:2048 sc0 sc1\n\ts_nop 1"
                 :: "v"(v), "v"(a), "v"(b), "v"(c), "s"(base) : "memory");
}
DI void store4(const char* base, unsigned v, const u32x4 (&f)[4]) {
    asm volatile("s_nop 4\n\tglobal_store_dwordx4 %0, %1, %5 sc0 sc1\n\t"
                 "global_store_dwordx4 %0, %2, %5 offset:1024 sc0 sc1\n\t"
                 "global_store_dwordx4 %0, %3, %5 offset:2048 sc0 sc1\n\t"
                 "global_store_dwordx4 %0, %4, %5 offset:3072 sc0 sc1\n\ts_nop 1"
                 :: "v"(v), "v"(f[0]), "v"(f[1]), "v"(f[2]), "v"(f[3]), "s"(base) : "memory");
}
DI void store4o(const char* base, unsigned v, const u32x4 (&f)[4]) {
    asm volatile("s_nop 4\n\tglobal_store_dwordx4 %0, %4, %8 sc1 nt\n\t"
                 "global_store_dwordx4 %1, %5, %8 sc1 nt\n\t"
                 "global_store_dwordx4 %2, %6, %8 sc1 nt\n\t"
                 "global_store_dwordx4 %3, %7, %8 sc1 nt\n\ts_nop 1"
                 :: "v"(v), "v"(v + 2 * ROWB), "v"(v + 4 * ROWB), "v"(v + 6 * ROWB), "v"(f[0]), "v"(f[1]), "v"(f[2]), "v"(f[3]), "s"(base) : "memory");
}

DI unsigned perm(unsigned hi, unsigned lo, unsigned sel) { return __builtin_amdgcn_perm(hi, lo, sel); }
template <int CTRL, int ROWMASK> DI unsigned dpp0(unsigned x) { return __builtin_amdgcn_update_dpp(0u, x, CTRL, ROWMASK, 0xf, false); }
DI unsigned wave_sum(unsigned x) {
    x += dpp0<0x111, 0xf>(x); x += dpp0<0x112, 0xf>(x); x += dpp0<0x114, 0xf>(x); x += dpp0<0x118, 0xf>(x); x += dpp0<0x142, 0xa>(x); x += dpp0<0x143, 0xc>(x);
    return __builtin_amdgcn_readlane(x, 63);
}
DI unsigned umax(unsigned a, unsigned b) { return a > b ? a : b; }
DI unsigned wave_max(unsigned x) {
    x = umax(x, dpp0<0x111, 0xf>(x)); x = umax(x, dpp0<0x112, 0xf>(x)); x = umax(x, dpp0<0x114, 0xf>(x)); x = umax(x, dpp0<0x118, 0xf>(x));
    x = umax(x, dpp0<0x142, 0xa>(x)); x = umax(x, dpp0<0x143, 0xc>(x));
    return __builtin_amdgcn_readlane(x, 63);
}
template <unsigned NE> struct ExcList {
    volatile LAS unsigned short* e; unsigned cnt;
    DI void begin(LAS unsigned char* tail, int lane) { e = (volatile LAS unsigned short*)(tail + 512); e[lane] = 0; cnt = 1; }
    DI bool add(bool act, unsigned entry) {
        const unsigned long long bal = __ballot(act);
        const unsigned slot = cnt + __builtin_amdgcn_mbcnt_hi((unsigned)(bal >> 32), __builtin_amdgcn_mbcnt_lo((unsigned)bal, 0u));
        if (act && slot < NE) e[slot] = (unsigned short)entry;
        cnt += __builtin_popcountll(bal);
        return cnt <= NE;
    }
};
template <unsigned NE> DI bool pack_finish(const char* base, unsigned blk, int c, int lane, LAS unsigned char* tail, ExcList<NE>& x, unsigned ebase,
                    const u32x4& p0, const u32x4& p1, const u32x4& p2, unsigned t0, unsigned t1) {
    if (lane == 0) x.e[0] = (unsigned short)(ebase | (x.cnt > NE ? 0x8000u : 0u));
    *(volatile LAS u32x2*)(tail + lane * 8) = (u32x2){t0, t1};
    constexpr int TS = 512 + 2 * NE;
    store3(base, blk + 4 * TS + c * 3072 + lane * 16, p0, p1, p2);
    const u32x4 t = *(volatile LAS u32x4*)(tail + (lane < TS / 16 ? lane : 0) * 16);
    if (lane < TS / 16) store1(base, blk + c * TS + lane * 16, t);
    return x.cnt <= NE;
}
DI void pack16(const u32x4 (&f)[4], const char* base, unsigned blk, int c, int lane, LAS unsigned char* tail) {
    unsigned lo[8], h[8], g[8], sum = 0;
#pragma unroll
    for (int i = 0; i < 8; ++i) {
        const unsigned a = f[i >> 1][2 * (i & 1)], b = f[i >> 1][2 * (i & 1) + 1];
        lo[i] = perm(b, a, 0x06040200u); h[i] = perm(b, a, 0x07050301u);
        sum = __builtin_amdgcn_udot4(h[i] & 0x7c7c7c7cu, 0x01010101u, sum, false);
    }
    const int top = (int)((wave_sum(sum) + 15155u + 8191u) >> 13);
    const unsigned eb = top < 7 ? 0u : top > 31 ? 24u : (unsigned)(top - 7), b4 = eb * 0x04040404u;
    ExcList<NE16> x; x.begin(tail, lane);
    unsigned m1 = 0, m0 = 0;
#pragma unroll
    for (int i = 0; i < 8; ++i) {
        g[i] = (h[i] & 0x83838383u) | (((h[i] | 0x80808080u) - b4) & 0x1c1c1c1cu);
        const unsigned xr = (g[i] + b4) ^ h[i];
        m1 |= ((xr >> 6) & 0x01010101u) << i; m0 |= ((xr >> 5) & 0x01010101u) << i;
    }
    for (unsigned m = m1 | m0; __any(m != 0); m &= m - 1) {
        const int k = m ? __builtin_ctz(m) : 0, i = k & 7;
        if (!x.add(m != 0, ((i >> 2) * 1024 + lane * 16 + (i & 3) * 4 + (k >> 3)) | ((m0 >> k) & 1u) << 11 | ((m1 >> k) & 1u) << 12)) break;
    }
    unsigned d[6];
#pragma unroll
    for (int q = 0; q < 2; ++q) {
        const unsigned s = g[4 * q + 3];
        d[3 * q] = g[4 * q] | (s & 0x03030303u) << 5; d[3 * q + 1] = g[4 * q + 1] | (s & 0x0c0c0c0cu) << 3;
        d[3 * q + 2] = g[4 * q + 2] | (s & 0x10101010u) << 1 | (s & 0x80808080u) >> 1;
    }
    pack_finish(base, blk, c, lane, tail, x, eb, (u32x4){lo[0], lo[1], lo[2], lo[3]}, (u32x4){lo[4], lo[5], lo[6], lo[7]}, (u32x4){d[0], d[1], d[2], d[3]}, d[4], d[5]);
}
DI bool pack32(const u32x4 (&f)[4], const char* base, unsigned blk, int c, int lane, LAS unsigned char* tail) {
    unsigned t[12], h[4], mx = 0;
#pragma unroll
    for (int j = 0; j < 4; ++j) {
        t[3 * j] = perm(f[j][3], f[j][0], 0x04020100u); t[3 * j + 1] = perm(f[j][3], f[j][1], 0x05020100u); t[3 * j + 2] = perm(f[j][3], f[j][2], 0x06020100u);
        h[j] = perm(perm(f[j][3], f[j][2], 0x07030703u), perm(f[j][1], f[j][0], 0x07030703u), 0x05040100u);
#pragma unroll
        for (int d = 0; d < 4; ++d) mx = umax(mx, f[j][d] & 0x7fffffffu);
    }
    const unsigned top = wave_max(mx) >> 24, eb = top < 7 ? 0u : top - 7, b1 = eb * 0x01010101u;
    ExcList<NE32> x; x.begin(tail, lane);
    unsigned cd[4], xn[2] = {0, 0};
#pragma unroll
    for (int j = 0; j < 4; ++j) {
        cd[j] = ((h[j] | 0x80808080u) - b1) & 0x07070707u;
        const unsigned xr = ((cd[j] + b1) ^ h[j]) & 0x78787878u;
        xn[j >> 1] |= (j & 1) ? xr << 1 : xr >> 3;
    }
    for (unsigned long long m = (unsigned long long)xn[1] << 32 | xn[0]; __any(m != 0);) {
        const int k = m ? __builtin_ctzll(m) >> 2 : 0, j = 2 * (k >> 3) + (k & 1);
        const unsigned v = (unsigned)(m >> (4 * k)) & 15u;
        if (!x.add(m != 0, (lane * 16 + j * 4 + ((k >> 1) & 3)) | v << 11)) break;
        m &= ~(15ull << (4 * k));
    }
    const unsigned n0 = cd[0] | (h[0] & 0x80808080u) >> 4 | cd[1] << 4 | (h[1] & 0x80808080u), n1 = cd[2] | (h[2] & 0x80808080u) >> 4 | cd[3] << 4 | (h[3] & 0x80808080u);
    return pack_finish(base, blk, c, lane, tail, x, eb, (u32x4){t[0], t[1], t[2], t[3]}, (u32x4){t[4], t[5], t[6], t[7]}, (u32x4){t[8], t[9], t[10], t[11]}, n0, n1);
}
DI bool unpack(bool is32, const LAS unsigned char* main3, const LAS unsigned char* tail, LAS unsigned char* patch, int lane, u32x4 (&f)[4]) {
    unsigned e = *(const volatile LAS unsigned short*)(tail + 512 + lane * 2);
    const u32x2 tt = *(const volatile LAS u32x2*)(tail + lane * 8);
    const u32x4 p0 = *(const volatile LAS u32x4*)(main3 + lane * 16), p1 = *(const volatile LAS u32x4*)(main3 + 1024 + lane * 16), p2 = *(const volatile LAS u32x4*)(main3 + 2048 + lane * 16);
    const unsigned hdr = rfl(e);
    if (lane == 0 || lane >= (int)(is32 ? NE32 : NE16)) e = 0;
    LAS unsigned* const pw = (LAS unsigned*)(patch + (e & 0x7fcu));
    __hip_atomic_fetch_or(pw, (e >> 11) << ((is32 ? 3 : 5) + 8 * (e & 3u)), __ATOMIC_RELAXED, __HIP_MEMORY_SCOPE_WAVEFRONT);
    const u32x4 x0 = *(const volatile LAS u32x4*)(patch + lane * 16), x1 = *(const volatile LAS u32x4*)(patch + 1024 + lane * 16);
    *(volatile LAS unsigned*)pw = 0;
    if (is32) {
        const unsigned b1 = (hdr & 127u) * 0x01010101u, t[12] = {p0[0], p0[1], p0[2], p0[3], p1[0], p1[1], p1[2], p1[3], p2[0], p2[1], p2[2], p2[3]};
#pragma unroll
        for (int j = 0; j < 4; ++j) {
            const unsigned n = tt[j >> 1];
            const unsigned h = ((j & 1) ? ((((n >> 4) & 0x07070707u) + b1) | (n & 0x80808080u)) : (((n & 0x07070707u) + b1) | ((n << 4) & 0x80808080u))) ^ x0[j];
            f[j][0] = perm(h, t[3 * j], 0x04020100u); f[j][1] = perm(h, t[3 * j + 1], 0x05020100u); f[j][2] = perm(h, t[3 * j + 2], 0x06020100u);
            f[j][3] = perm(perm(h, t[3 * j + 2], 0x07030703u), perm(t[3 * j + 1], t[3 * j], 0x07030703u), 0x05040100u);
        }
    } else {
        const unsigned b4 = (hdr & 31u) * 0x04040404u, d[6] = {p2[0], p2[1], p2[2], p2[3], tt[0], tt[1]};
#pragma unroll
        for (int q = 0; q < 2; ++q) {
            const unsigned a = d[3 * q], b = d[3 * q + 1], c = d[3 * q + 2];
            const unsigned g[4] = {a & 0x9f9f9f9fu, b & 0x9f9f9f9fu, c & 0x9f9f9f9fu,
                                   ((a >> 5) & 0x03030303u) | ((b >> 3) & 0x0c0c0c0cu) | ((c >> 1) & 0x10101010u) | ((c << 1) & 0x80808080u)};
#pragma unroll
            for (int i = 0; i < 4; ++i) {
                const unsigned h = (g[i] + b4) ^ (q ? x1[i] : x0[i]), lo = q ? p1[i] : p0[i];
                f[2 * q + (i >> 1)][2 * (i & 1)] = perm(h, lo, 0x05010400u); f[2 * q + (i >> 1)][2 * (i & 1) + 1] = perm(h, lo, 0x07030602u);
            }
        }
    }
    return hdr < 0x8000u;
}

constexpr int PSHORT = 16 + NBB - 2;
DI bool two_quads(int panel) { return panel < 16 || (unsigned)(panel - PSHORT) < 2u; }
DI int col0(int cb) { return cb >= NBB ? BCOLS + (cb - NBB) * TW : cb == NBB - 1 ? BCOLS - 256 : cb * TW; }
DI int colblock(int li, int rank) { return li < CPR ? NBB + CPR * rank + li : li - CPR; }
DI unsigned own_off(int panel, int kt) { return (panel < 16 ? panel * XPANEL : REL_WL + (panel - 16) * 8 * WBLK) + kt * (two_quads(panel) ? XBLK : WBLK); }
DI unsigned ring_off(int li) { return REL_WR + (li >= RING ? li - RING : li) * (56 * WBLK); }
DI unsigned pk16_off(int panel, int kt) { return OFF_PK16 + ((panel < 16 ? panel * 16 : 256 + (panel - 16) * 24) + kt * (two_quads(panel) ? 2 : 3)) * QB16; }
static_assert(2 * RING >= NL, "ring");
constexpr int round_end(int r) { return (NWB * (r + 1) - (r ? NJOB : 0) + RB - 1) / RB; }
constexpr int XL[4] = {0, 1, 2, 2}, NSTEP0 = 8 + XL[3];
constexpr int x_lo(int s) { int g = 0; while (g < 4 && s - XL[g] >= 8) ++g; return 4 * g; }
constexpr int x_hi(int s) { int g = 0; while (g < 4 && s >= XL[g]) ++g; return 4 * g; }
constexpr int B0 = round_end(0) - CPR, W0Q = 3 * (B0 + NCB);
constexpr int cstep_start(int s) { int n = 0; for (int k = 0; k < s; ++k) n += 2 * (x_hi(k) - x_lo(k)) + (k < 8 ? W0Q : 0); return n; }
constexpr int steps_within(int quads) { int s = 0; while (s < NSTEP0 && cstep_start(s + 1) <= quads) ++s; return s; }
constexpr int ST1 = 3, STEPS1 = steps_within(NGEMM), STEPS2 = steps_within(ST1 * NGEMM);
static_assert(STEPS1 >= 1 && NQ16 > ST1 * NGEMM + NGEMM, "stages");
DI void conv_quad(int b, int& panel, int& kt, int& qd) {
    if (b < cstep_start(NSTEP0)) {
        int s = 0, i = b, lo = x_lo(0), nx = 2 * (x_hi(0) - x_lo(0));
#pragma unroll
        for (int k = 1; k < NSTEP0; ++k) if (b >= cstep_start(k)) { s = k; i = b - cstep_start(k); lo = x_lo(k); nx = 2 * (x_hi(k) - x_lo(k)); }
        if (i < nx) { panel = lo + (i >> 1); qd = i & 1; kt = s - (panel < 4 ? XL[0] : panel < 8 ? XL[1] : panel < 12 ? XL[2] : XL[3]); }
        else { const int w = (i - nx) / 3; qd = i - nx - 3 * w; panel = 16 + (w < B0 ? w : w - B0 + NBB); kt = s; }
    } else if (b < NQ16 - 32) { const int r = b - cstep_start(NSTEP0), k = r / 3; qd = r - 3 * k; panel = 16 + B0 + (k >> 3); kt = k & 7; }
    else { const int r = b - (NQ16 - 32); panel = PSHORT + (r >> 4); kt = r >> 1 & 7; qd = r & 1; }
}
DI unsigned draw_call(char* ws, int wid, bool& clear) {
    const int lane = lane_now();
    unsigned v;
    asm volatile("s_nop 4\n\tglobal_load_dword %0, %1, %2 sc0 sc1\n\ts_waitcnt vmcnt(0)"
                 : "=&v"(v) : "v"(lane < 56 ? OFF_CALL + blockIdx.x * 128 : OFF_EV + (4 * NR + lane - 56) * 4), "s"(ws) : "memory");
    const unsigned c = rfl(v) + 1;
    clear = __all(v >= c - 1);
    BARRIER;
    if (wid == 0 && lane == 0) st_ag(ws + OFF_CALL + blockIdx.x * 128, c);
    return c;
}
DI unsigned convert_shards(Params& p, LAS unsigned char* lds, int wid, int gi) {
    const int lane = lane_now(), rank = p.rank;
    char* const ws = p.ws[rank];
    const __amdgpu_buffer_rsrc_t rx = __builtin_amdgcn_make_buffer_rsrc((void*)p.x, 0, MM * KS * 4, 0x00020000);
    const __amdgpu_buffer_rsrc_t rw = __builtin_amdgcn_make_buffer_rsrc((void*)p.w, 0, KS * NN * 4, 0x00020000);
    const unsigned vx = ((lane & 15) * KS + 8 * (lane >> 4)) * 4, vw = (8 * (lane >> 4) * NN + 4 * (lane & 15)) * 4;
    LAS unsigned char* const ring = lds + wid * 32768;
    LAS unsigned char* const tail = lds + 131072 + wid * 1024;
    const int nu = (NQ16 - gi + NGEMM - 1) / NGEMM;
    auto load = [&](int u) {
        int pa, kt, qd; conv_quad(gi + (u < nu ? u : 0) * NGEMM, pa, kt, qd);
        LAS unsigned char* const slot = ring + (u & 3) * 8192;
        if (pa < 16) {
            const unsigned s0 = ((pa * 256 + 128 * (wid >> 1) + 64 * qd) * KS + 32 * (2 * kt + (wid & 1))) * 4;
#pragma unroll
            for (int e = 0; e < 8; ++e) __builtin_amdgcn_raw_ptr_buffer_load_lds(rx, (LAS void*)(slot + e * 1024), 16, vx, s0 + (e >> 1) * (16 * KS * 4) + (e & 1) * 16, 0, AUX_NT);
        } else {
            const unsigned s0 = (32 * (2 * kt + (wid & 1)) * NN + col0(pa - 16) + (two_quads(pa) ? 128 : 192) * (wid >> 1) + 64 * qd) * 4;
#pragma unroll
            for (int e = 0; e < 8; ++e) __builtin_amdgcn_raw_ptr_buffer_load_lds(rw, (LAS void*)(slot + e * 1024), 16, vw, s0 + e * ROWB, 0, AUX_NT);
        }
    };
    load(0);
    char* peer_ws = p.ws[lane & 7];
    bool clear;
    const unsigned call = draw_call(ws, wid, clear);
    if (!clear) wait_words(ws, ws + OFF_EV + 4 * NR * 4, NR, 4, call - 1, call);
    asm volatile("" : "+v"(peer_ws));
    for (int u = 0; u < nu; ++u) {
        if (u > 0) { load(u + 3); WAITVM(24); }
        if (u == ST1 + 1) {
            BARRIER;
            if (wid == 0) arrive(ws, peer_ws, rank, 2, NGEMM, call);
        }
        int pa, kt, qd; conv_quad(gi + u * NGEMM, pa, kt, qd);
        f32x4 v[8]; u32x4 f[4];
#pragma unroll
        for (int e = 0; e < 8; ++e) v[e] = *(const LAS f32x4*)(ring + (u & 3) * 8192 + e * 1024 + lane * 16);
#pragma unroll
        for (int j = 0; j < 4; ++j) { f16x8 h;
#pragma unroll
            for (int e = 0; e < 8; ++e) h[e] = (_Float16)((pa < 16 ? v[2 * j + (e >> 2)][e & 3] : v[e][j]) * (pa < 16 ? XSCALE : WSCALE));
            f[j] = __builtin_bit_cast(u32x4, h); }
        pack16(f, ws, pk16_off(pa, kt) + qd * QB16, wid, lane, tail);
        store4(ws, OFF_DATA + own_off(pa, kt) + (4 * qd + wid) * 4096 + lane * 16, f);
        if (u == 0) {
            WAITVM(0);
            BARRIER;
            if (wid == 0 && lane < NR) st_sys(peer_ws + OFF_AV + (rank * 256 + gi) * 4, call);
            load(1); load(2); load(3);
            if (wid == 1) arrive(ws, peer_ws, rank, 1, NGEMM, call);
        }
    }
    WAITVM(0);
    BARRIER;
    if (wid == 0) arrive(ws, peer_ws, rank, 3, NGEMM, call);
    return call;
}

constexpr int G0 = 16 + round_end(0), NBI = (16 + NL) * 8, NTW = 7 * NBI, NTC = 7 * NCI, NT = NTW + NTC;
struct Item { const char* src; unsigned size, pk, raw, dof; int flag; bool three; };
constexpr int step_start(int s) { int n = 0; for (int k = 0; k < s; ++k) n += x_hi(k) - x_lo(k) + (k < 8 ? G0 - 16 : 0); return n; }
static_assert(step_start(NSTEP0) == 8 * G0, "order");
DI void block_of(unsigned t, int& i, int& kt) {
    if (t < 8u * G0) {
        int s = 0, hi = x_hi(0); i = t;
#pragma unroll
        for (int k = 1; k < NSTEP0; ++k) if (t >= (unsigned)step_start(k)) { s = k; i = t - step_start(k) + x_lo(k); hi = x_hi(k); }
        if (i >= hi) i += 16 - hi;
        kt = s - (i < 4 ? XL[0] : i < 8 ? XL[1] : i < 12 ? XL[2] : i < 16 ? XL[3] : 0);
    } else {
        unsigned u = 0, n = 1, b = 0;
#pragma unroll
        for (int r = 1; r < NROUND; ++r)
            if (t >= 8u * (16 + round_end(r - 1))) { b = 16 + round_end(r - 1); u = t - 8u * b; n = round_end(r) - round_end(r - 1); }
        kt = u / n; i = b + (u - kt * n);
    }
}
DI unsigned slab_out(unsigned s, int from) { return (s & 127u) * 32u * ROWB + (BCOLS + (CPR * from + (s >> 7)) * TW) * 4u; }
DI Item make_item(Params& p, unsigned t) {
    const int rank = p.rank;
    const unsigned tt = t < (unsigned)NTW ? t : t < (unsigned)NT ? t - NTW : 0, u = tt / 7, pj = tt - 7 * u;
    const int peer = (rank + 1 + pj) & 7;
    Item it;
    it.src = p.ws[peer]; it.size = t < (unsigned)NT ? WS_NEED : 0;
    if (t < (unsigned)NTW) {
        int i, kt; block_of(u, i, kt);
        const int panel = i < 16 ? i : 16 + colblock(i - 16, rank);
        it.three = !two_quads(panel);
        it.pk = pk16_off(panel, kt); it.raw = OFF_DATA + own_off(panel, kt);
        it.dof = OFF_DATA + (i < 16 ? i * XPANEL + 8 * XBLK : ring_off(i - 16)) + (7 * kt + pj) * (it.three ? WBLK : XBLK);
        it.flag = panel * 64 + 8 + 7 * kt + pj;
    } else {
        it.three = true;
        it.pk = OFF_PK32 + u * (3 * QB32); it.raw = OFF_RAW32 + u * WBLK; it.dof = slab_out(u, peer); it.flag = -1;
    }
    return it;
}
DI void load_main(const Item& it, unsigned off, LAS unsigned char* to, int lane) {
    const __amdgpu_buffer_rsrc_t r = __builtin_amdgcn_make_buffer_rsrc((void*)it.src, 0, it.size, 0x00020000);
    asm volatile("s_mov_b32 m0, %3\n\ts_nop 4\n\tbuffer_load_dwordx4 %0, %1, %2 offen sc0 sc1 lds\n\tbuffer_load_dwordx4 %0, %1, %2 offen offset:1024 sc0 sc1 lds\n\t"
                 "buffer_load_dwordx4 %0, %1, %2 offen offset:2048 sc0 sc1 lds" :: "v"(lane * 16), "s"(r), "s"(off), "s"((unsigned)(size_t)to) : "memory");
}
DI void load_tails(const Item& it, unsigned off, LAS unsigned char* to, int lane) {
    const __amdgpu_buffer_rsrc_t r = __builtin_amdgcn_make_buffer_rsrc((void*)it.src, 0, it.size, 0x00020000);
    const unsigned long long rest = it.flag < 0 ? (1ull << NE32 / 2) - 1 : (1ull << NE16 / 2) - 1;
    asm volatile("s_mov_b32 m0, %3\n\ts_nop 4\n\tbuffer_load_dwordx4 %0, %1, %2 offen sc0 sc1 lds\n\tbuffer_load_dwordx4 %0, %1, %2 offen offset:1024 sc0 sc1 lds\n\t"
                 "s_mov_b64 exec, %4\n\tbuffer_load_dwordx4 %0, %1, %2 offen offset:2048 sc0 sc1 lds\n\ts_mov_b64 exec, -1"
                 :: "v"(lane * 16), "s"(r), "s"(off), "s"((unsigned)(size_t)to), "s"(rest) : "memory");
}
constexpr unsigned NANS = 0x7fc07fc0u;
constexpr unsigned LQ = 14848, LMAIN = 2560;
struct Fetch {
    const char* ws; const char* out;
    LAS unsigned char* lds;
    unsigned slot; int lane; bool ok;
    template <int C0, bool WAITED> DI void quad(const Item& cur, unsigned qd, const Item& nx, unsigned nqd) {
        LAS unsigned char* const q = lds + slot; LAS unsigned char* const nq = lds + (LQ - slot);
        const unsigned npk = nx.pk + nqd * (nx.flag < 0 ? QB32 : QB16), nmain = npk + 4 * (nx.flag < 0 ? TS32 : TS16), ts = cur.flag < 0 ? TS32 : TS16;
#pragma unroll
        for (int c = 0; c < 4; ++c) {
            if (c > 0 || !WAITED) WAITVM(25);
            if (c == 0) load_tails(nx, npk, nq, lane);
            load_main(nx, nmain + c * 3072, nq + LMAIN + c * 3072, lane);
            u32x4 f[4];
            if (!unpack(cur.flag < 0, q + LMAIN + c * 3072, q + c * ts, lds + 2 * LQ, lane, f)) {
                const char* a = cur.src + (size_t)(cur.raw + qd * QUAD + c * 4096 + lane * 16);
                asm volatile("global_load_dwordx4 %0, %4, off sc0 sc1\n\tglobal_load_dwordx4 %1, %4, off offset:1024 sc0 sc1\n\t"
                             "global_load_dwordx4 %2, %4, off offset:2048 sc0 sc1\n\tglobal_load_dwordx4 %3, %4, off offset:3072 sc0 sc1\n\ts_waitcnt vmcnt(0)"
                             : "=&v"(f[0]), "=&v"(f[1]), "=&v"(f[2]), "=&v"(f[3]) : "v"(a) : "memory");
            }
            if (!ok) f[0] = f[1] = f[2] = f[3] = (u32x4){NANS, NANS, NANS, NANS};
            if (cur.flag >= 0) store4(ws, cur.dof + qd * QUAD + c * 4096 + lane * 16, f);
            else store4o(out, cur.dof + ((C0 + c) / 3 * 8 + (lane >> 5)) * ROWB + (C0 + c) % 3 * 512 + (lane & 31) * 16, f);
        }
        slot = LQ - slot;
    }
};
DI void fetch_wave(Params& p, LAS unsigned char* lds, unsigned call, int wid, int wv) {
    const int lane = lane_now(), rank = p.rank;
    char* const ws = p.ws[rank];
    Fetch F;
    F.ws = ws; F.out = (const char*)p.out[rank]; F.lds = lds; F.lane = lane;
    *(LAS u32x4*)(lds + 2 * LQ + lane * 16) = (u32x4){0, 0, 0, 0}; *(LAS u32x4*)(lds + 2 * LQ + 1024 + lane * 16) = (u32x4){0, 0, 0, 0};
    const unsigned wbase = (call - 1) * NTW - NCW, cbase = (call - 1) * NTC - NCW;
    static_assert(NL - RING == 32, "probe lanes");
    const unsigned vprobe = lane < 32 ? OFF_EV + lane * 4 : OFF_FIN + (lane - 32) * 4;
    static_assert(NCW <= 7 * step_start(1), "first items");
    if (wv == 0 && lane < NR) st_sys(p.ws[lane] + OFF_EV + (5 * NR + rank) * 4, call);
    {
        int i0, kt0; block_of(wv / 7, i0, kt0);
        const int q0 = i0 < 16 ? 2 * i0 : 2 * x_hi(0) + 3 * (i0 < 16 + CPR ? B0 + CPR * rank + i0 - 16 : i0 - 16 - CPR), peer = (rank + 1 + wv % 7) & 7;
        const unsigned v = lane < NR ? OFF_EV + (5 * NR + lane) * 4 : OFF_AV + peer * 1024 + (q0 + lane % (i0 < 16 ? 2 : 3)) * 4;
        const unsigned long long t0 = __builtin_amdgcn_s_memrealtime();
        F.ok = true;
        for (unsigned n = 1; !__all(ld_sys(ws + v) >= call); ++n) if ((n & 15) == 0 && give_up(ws, call, t0)) { F.ok = false; break; }
        asm volatile("" ::: "memory");
    }
    unsigned tw = wv, tc = wv;
    auto choose = [&](unsigned probe) -> unsigned {
        const unsigned ev = (unsigned)__ballot(probe >= call);
        if (tw < (unsigned)NTW) {
            const unsigned u = tw / 7;
            int i, kt; block_of(u, i, kt);
            const int stage = u < (unsigned)step_start(STEPS1) ? 1 : u < (unsigned)step_start(STEPS2) ? 2 : 3;
            if ((ev >> (8 * stage + ((rank + 1 + tw - 7 * u) & 7)) & 1) &&
                (i < 16 + RING || __builtin_amdgcn_readlane(probe, 32 + i - 16 - RING) >= 16 * call)) return tw;
        }
        return tc < (unsigned)NTC && (ev >> ((rank + 1 + tc % 7) & 7) & 1) ? NTW + tc : NT;
    };
    auto publish = [&](int flag) {
        if (flag >= 0 && lane == 0) st_sys(ws + OFF_F + flag * 4, call);
    };
    while (tw < (unsigned)NTW || tc < (unsigned)NTC) {
        unsigned t = tw;
        if (tw >= (unsigned)NCW) {
            const unsigned long long t0 = __builtin_amdgcn_s_memrealtime();
            while ((t = choose(ld_sys(ws + vprobe))) == (unsigned)NT) {
                if (give_up(ws, call, t0)) { F.ok = false; t = tw < (unsigned)NTW ? tw : NTW + tc; break; }
                __builtin_amdgcn_s_sleep(16);
            }
        }
        Item cur = make_item(p, t);
        F.slot = 0;
        load_tails(cur, cur.pk, lds, lane);
#pragma unroll
        for (int c = 0; c < 4; ++c) load_main(cur, cur.pk + 4 * (cur.flag < 0 ? TS32 : TS16) + c * 3072, lds + LMAIN + c * 3072, lane);
        WAITVM(0);
        int pflag = -1;
        for (;;) {
            unsigned tk, probe; unsigned long long sv;
            asm volatile("s_nop 3\n\ts_waitcnt vmcnt(25)\n\tglobal_load_dword %1, %6, %5 sc0 sc1\n\t"
                         "s_mov_b64 %2, exec\n\ts_mov_b64 exec, 1\n\tglobal_atomic_add %0, %3, %4, %5 sc0\n\ts_mov_b64 exec, %2"
                         : "=&v"(tk), "=&v"(probe), "=&s"(sv) : "v"(OFF_TICK + (cur.flag < 0 ? 128u : 0u)), "v"(1u), "s"(ws), "v"(vprobe) : "memory");
            F.quad<0, true>(cur, 0, cur, 1);
            asm volatile("s_waitcnt vmcnt(25)" : "+v"(tk), "+v"(probe) :: "memory");
            publish(pflag);
            if (cur.flag >= 0) tw = rfl(tk) - wbase; else tc = rfl(tk) - cbase;
            if (cur.three) F.quad<4, true>(cur, 1, cur, 2);
            t = choose(probe);
            if (t == (unsigned)NT && (tw < (unsigned)NTW || tc < (unsigned)NTC)) t = choose(ld_sys(ws + vprobe));
            const Item nxt = make_item(p, t);
            F.quad<8, false>(cur, cur.three ? 2 : 1, nxt, 0);
            pflag = cur.flag;
            if (t >= (unsigned)NT) break;
            cur = nxt;
        }
        WAITVM(0);
        publish(pflag);
    }
    arrive(ws, p.ws[lane & 7], rank, 4, NCW, call);
}
#define PT_SET(A0, A1, A2, A3, R) \
    asm volatile("s_waitcnt vmcnt(60)\n\t" R \
                 "global_load_dwordx4 a[" A0 "], %16, %20 sc1 nt\n\tglobal_load_dwordx4 a[" A1 "], %17, %20 sc1 nt\n\t" \
                 "global_load_dwordx4 a[" A2 "], %18, %20 sc1 nt\n\tglobal_load_dwordx4 a[" A3 "], %19, %20 sc1 nt" \
                 : "=&v"(r[0]), "=&v"(r[1]), "=&v"(r[2]), "=&v"(r[3]), "=&v"(r[4]), "=&v"(r[5]), "=&v"(r[6]), "=&v"(r[7]), \
                   "=&v"(r[8]), "=&v"(r[9]), "=&v"(r[10]), "=&v"(r[11]), "=&v"(r[12]), "=&v"(r[13]), "=&v"(r[14]), "=&v"(r[15]) \
                 : "v"(v), "v"(v + 2 * ROWB), "v"(v + 4 * ROWB), "v"(v + 6 * ROWB), "s"(out) : "memory", PT_CLOB)
#define PT_CLOB "a0", "a1", "a2", "a3", "a4", "a5", "a6", "a7", "a8", "a9", "a10", "a11", "a12", "a13", "a14", "a15", "a16", "a17", "a18", "a19", "a20", "a21", "a22", "a23", "a24", "a25", "a26", "a27", "a28", "a29", "a30", "a31", "a32", "a33", "a34", "a35", "a36", "a37", "a38", "a39", "a40", "a41", "a42", "a43", "a44", "a45", "a46", "a47", "a48", "a49", "a50", "a51", "a52", "a53", "a54", "a55", "a56", "a57", "a58", "a59", "a60", "a61", "a62", "a63", "a64", "a65", "a66", "a67", "a68", "a69", "a70", "a71", "a72", "a73", "a74", "a75", "a76", "a77", "a78", "a79", "a80", "a81", "a82", "a83", "a84", "a85", "a86", "a87", "a88", "a89", "a90", "a91", "a92", "a93", "a94", "a95", "a96", "a97", "a98", "a99", "a100", "a101", "a102", "a103", "a104", "a105", "a106", "a107", "a108", "a109", "a110", "a111", "a112", "a113", "a114", "a115", "a116", "a117", "a118", "a119", "a120", "a121", "a122", "a123", "a124", "a125", "a126", "a127"
#define PT_R0 "v_accvgpr_read_b32 %0, a0\n\t" "v_accvgpr_read_b32 %1, a1\n\t" "v_accvgpr_read_b32 %2, a2\n\t" "v_accvgpr_read_b32 %3, a3\n\t" "v_accvgpr_read_b32 %4, a4\n\t" "v_accvgpr_read_b32 %5, a5\n\t" "v_accvgpr_read_b32 %6, a6\n\t" "v_accvgpr_read_b32 %7, a7\n\t" "v_accvgpr_read_b32 %8, a8\n\t" "v_accvgpr_read_b32 %9, a9\n\t" "v_accvgpr_read_b32 %10, a10\n\t" "v_accvgpr_read_b32 %11, a11\n\t" "v_accvgpr_read_b32 %12, a12\n\t" "v_accvgpr_read_b32 %13, a13\n\t" "v_accvgpr_read_b32 %14, a14\n\t" "v_accvgpr_read_b32 %15, a15\n\t"
#define PT_R1 "v_accvgpr_read_b32 %0, a16\n\t" "v_accvgpr_read_b32 %1, a17\n\t" "v_accvgpr_read_b32 %2, a18\n\t" "v_accvgpr_read_b32 %3, a19\n\t" "v_accvgpr_read_b32 %4, a20\n\t" "v_accvgpr_read_b32 %5, a21\n\t" "v_accvgpr_read_b32 %6, a22\n\t" "v_accvgpr_read_b32 %7, a23\n\t" "v_accvgpr_read_b32 %8, a24\n\t" "v_accvgpr_read_b32 %9, a25\n\t" "v_accvgpr_read_b32 %10, a26\n\t" "v_accvgpr_read_b32 %11, a27\n\t" "v_accvgpr_read_b32 %12, a28\n\t" "v_accvgpr_read_b32 %13, a29\n\t" "v_accvgpr_read_b32 %14, a30\n\t" "v_accvgpr_read_b32 %15, a31\n\t"
#define PT_R2 "v_accvgpr_read_b32 %0, a32\n\t" "v_accvgpr_read_b32 %1, a33\n\t" "v_accvgpr_read_b32 %2, a34\n\t" "v_accvgpr_read_b32 %3, a35\n\t" "v_accvgpr_read_b32 %4, a36\n\t" "v_accvgpr_read_b32 %5, a37\n\t" "v_accvgpr_read_b32 %6, a38\n\t" "v_accvgpr_read_b32 %7, a39\n\t" "v_accvgpr_read_b32 %8, a40\n\t" "v_accvgpr_read_b32 %9, a41\n\t" "v_accvgpr_read_b32 %10, a42\n\t" "v_accvgpr_read_b32 %11, a43\n\t" "v_accvgpr_read_b32 %12, a44\n\t" "v_accvgpr_read_b32 %13, a45\n\t" "v_accvgpr_read_b32 %14, a46\n\t" "v_accvgpr_read_b32 %15, a47\n\t"
#define PT_R3 "v_accvgpr_read_b32 %0, a48\n\t" "v_accvgpr_read_b32 %1, a49\n\t" "v_accvgpr_read_b32 %2, a50\n\t" "v_accvgpr_read_b32 %3, a51\n\t" "v_accvgpr_read_b32 %4, a52\n\t" "v_accvgpr_read_b32 %5, a53\n\t" "v_accvgpr_read_b32 %6, a54\n\t" "v_accvgpr_read_b32 %7, a55\n\t" "v_accvgpr_read_b32 %8, a56\n\t" "v_accvgpr_read_b32 %9, a57\n\t" "v_accvgpr_read_b32 %10, a58\n\t" "v_accvgpr_read_b32 %11, a59\n\t" "v_accvgpr_read_b32 %12, a60\n\t" "v_accvgpr_read_b32 %13, a61\n\t" "v_accvgpr_read_b32 %14, a62\n\t" "v_accvgpr_read_b32 %15, a63\n\t"
#define PT_R4 "v_accvgpr_read_b32 %0, a64\n\t" "v_accvgpr_read_b32 %1, a65\n\t" "v_accvgpr_read_b32 %2, a66\n\t" "v_accvgpr_read_b32 %3, a67\n\t" "v_accvgpr_read_b32 %4, a68\n\t" "v_accvgpr_read_b32 %5, a69\n\t" "v_accvgpr_read_b32 %6, a70\n\t" "v_accvgpr_read_b32 %7, a71\n\t" "v_accvgpr_read_b32 %8, a72\n\t" "v_accvgpr_read_b32 %9, a73\n\t" "v_accvgpr_read_b32 %10, a74\n\t" "v_accvgpr_read_b32 %11, a75\n\t" "v_accvgpr_read_b32 %12, a76\n\t" "v_accvgpr_read_b32 %13, a77\n\t" "v_accvgpr_read_b32 %14, a78\n\t" "v_accvgpr_read_b32 %15, a79\n\t"
#define PT_R5 "v_accvgpr_read_b32 %0, a80\n\t" "v_accvgpr_read_b32 %1, a81\n\t" "v_accvgpr_read_b32 %2, a82\n\t" "v_accvgpr_read_b32 %3, a83\n\t" "v_accvgpr_read_b32 %4, a84\n\t" "v_accvgpr_read_b32 %5, a85\n\t" "v_accvgpr_read_b32 %6, a86\n\t" "v_accvgpr_read_b32 %7, a87\n\t" "v_accvgpr_read_b32 %8, a88\n\t" "v_accvgpr_read_b32 %9, a89\n\t" "v_accvgpr_read_b32 %10, a90\n\t" "v_accvgpr_read_b32 %11, a91\n\t" "v_accvgpr_read_b32 %12, a92\n\t" "v_accvgpr_read_b32 %13, a93\n\t" "v_accvgpr_read_b32 %14, a94\n\t" "v_accvgpr_read_b32 %15, a95\n\t"
#define PT_R6 "v_accvgpr_read_b32 %0, a96\n\t" "v_accvgpr_read_b32 %1, a97\n\t" "v_accvgpr_read_b32 %2, a98\n\t" "v_accvgpr_read_b32 %3, a99\n\t" "v_accvgpr_read_b32 %4, a100\n\t" "v_accvgpr_read_b32 %5, a101\n\t" "v_accvgpr_read_b32 %6, a102\n\t" "v_accvgpr_read_b32 %7, a103\n\t" "v_accvgpr_read_b32 %8, a104\n\t" "v_accvgpr_read_b32 %9, a105\n\t" "v_accvgpr_read_b32 %10, a106\n\t" "v_accvgpr_read_b32 %11, a107\n\t" "v_accvgpr_read_b32 %12, a108\n\t" "v_accvgpr_read_b32 %13, a109\n\t" "v_accvgpr_read_b32 %14, a110\n\t" "v_accvgpr_read_b32 %15, a111\n\t"
#define PT_R7 "v_accvgpr_read_b32 %0, a112\n\t" "v_accvgpr_read_b32 %1, a113\n\t" "v_accvgpr_read_b32 %2, a114\n\t" "v_accvgpr_read_b32 %3, a115\n\t" "v_accvgpr_read_b32 %4, a116\n\t" "v_accvgpr_read_b32 %5, a117\n\t" "v_accvgpr_read_b32 %6, a118\n\t" "v_accvgpr_read_b32 %7, a119\n\t" "v_accvgpr_read_b32 %8, a120\n\t" "v_accvgpr_read_b32 %9, a121\n\t" "v_accvgpr_read_b32 %10, a122\n\t" "v_accvgpr_read_b32 %11, a123\n\t" "v_accvgpr_read_b32 %12, a124\n\t" "v_accvgpr_read_b32 %13, a125\n\t" "v_accvgpr_read_b32 %14, a126\n\t" "v_accvgpr_read_b32 %15, a127\n\t"
DI void pack_tile(Params& p, LAS unsigned char* tail, unsigned call, int ct) {
    const int lane = lane_now(), rank = p.rank;
    char* const ws = p.ws[rank];
    const char* const out = (const char*)p.out[rank];
    const bool ok = wait_words(ws, ws + OFF_AVC + ct * 4, 1, 4, call, call);
    __builtin_amdgcn_fence(__ATOMIC_ACQUIRE, "agent");
    const unsigned s0 = 8 * ct, vb = slab_out(s0, rank) + (lane >> 5) * ROWB + (lane & 31) * 16;
    for (int i = -8; i < 96; ++i) {
        const unsigned n = i + 8 < 96 ? i + 8 : 95, v = vb + n / 3 * (8 * ROWB) + n % 3 * 512;
        unsigned r[16];
        if (i >= 0 && i < 8) WAITVM(28);
        switch (i & 7) {
            case 0: PT_SET("0:3", "4:7", "8:11", "12:15", PT_R0); break;
            case 1: PT_SET("16:19", "20:23", "24:27", "28:31", PT_R1); break;
            case 2: PT_SET("32:35", "36:39", "40:43", "44:47", PT_R2); break;
            case 3: PT_SET("48:51", "52:55", "56:59", "60:63", PT_R3); break;
            case 4: PT_SET("64:67", "68:71", "72:75", "76:79", PT_R4); break;
            case 5: PT_SET("80:83", "84:87", "88:91", "92:95", PT_R5); break;
            case 6: PT_SET("96:99", "100:103", "104:107", "108:111", PT_R6); break;
            default: PT_SET("112:115", "116:119", "120:123", "124:127", PT_R7); break;
        }
        if (i < 0) continue;
        if (!ok) r[0] = NANS;
        const u32x4 f[4] = {{r[0], r[1], r[2], r[3]}, {r[4], r[5], r[6], r[7]}, {r[8], r[9], r[10], r[11]}, {r[12], r[13], r[14], r[15]}};
        if (!pack32(f, ws, OFF_PK32 + (s0 * 3 + (i >> 2)) * QB32, i & 3, lane, tail)) store4(ws, OFF_RAW32 + s0 * WBLK + i * 4096 + lane * 16, f);
    }
    WAITVM(0);
    arrive(ws, p.ws[lane & 7], rank, 0, 16 * CPR, call);
}

DI f16x8 rdfrag(unsigned base, int off) { return *(const LAS f16x8*)(size_t)(base + off); }
DI void dma_to(unsigned ldsaddr) { asm volatile("s_mov_b32 m0, %0" :: "s"(ldsaddr) : "memory"); }
#define PIN_S(x) asm volatile("" : "+s"(x))
DI unsigned vadd(unsigned v, unsigned s) { unsigned r; asm volatile("v_add_u32 %0, %1, %2" : "=v"(r) : "s"(s), "v"(v)); return r; }
template <int AT> DI void jump_at(unsigned& x, int it, unsigned to) { asm volatile("s_cmp_eq_u32 %1, %3\n\ts_cselect_b32 %0, %2, %0" : "+s"(x) : "s"(it), "s"(to), "i"(AT) : "scc"); }
struct G7 {
    unsigned lds;
    __amdgpu_buffer_rsrc_t rD;
    unsigned voff;
    unsigned xb, an;
    unsigned blo, bhi;
    unsigned sA, sB;
    unsigned xs[3];
    unsigned ringB, nextA, nextB;
    f32x4 acc[8][12];
    f16x8 L[2][4][2], H[4][2], B0, B1;
};
template <int I> DI void dma7(const G7& g, unsigned src) {
    asm volatile("buffer_load_dwordx4 %0, %1, %2 offen offset:%3 sc0 sc1 lds" :: "v"(g.voff), "s"(g.rD), "s"(src), "i"(I * 1024) : "memory");
}
template <int SL> DI f16x8 rdB7(const G7& g, int j, int k) { return rdfrag(SL < 3 ? g.blo : g.bhi, (SL % 3) * QUAD + k * 4096 + j * 1024); }
#define MFMA(D, A, B, C) "v_mfma_f32_16x16x32_f16 " D ", " A ", " B ", " C
template <bool AG, bool ZERO> DI void mfma1(f32x4& acc, const f16x8& a, const f16x8& b) {
    if (AG) { if (ZERO) asm volatile(MFMA("%0", "%1", "%2", "0") : "=a"(acc) : "v"(a), "v"(b)); else asm volatile(MFMA("%0", "%1", "%2", "%0") : "+a"(acc) : "v"(a), "v"(b)); }
    else { if (ZERO) asm volatile(MFMA("%0", "%1", "%2", "0") : "=&v"(acc) : "v"(a), "v"(b)); else asm volatile(MFMA("%0", "%1", "%2", "%0") : "+v"(acc) : "v"(a), "v"(b)); }
    SCHED;
}
template <bool AG, bool ZERO> DI void mfma2(f32x4& acc, const f16x8& a, const f16x8& b, const f16x8& c, const f16x8& d) {
    if (AG) {
        if (ZERO) asm volatile(MFMA("%0", "%1", "%2", "0") "\n\t" MFMA("%0", "%3", "%4", "%0") : "=a"(acc) : "v"(a), "v"(b), "v"(c), "v"(d));
        else asm volatile(MFMA("%0", "%1", "%2", "%0") "\n\t" MFMA("%0", "%3", "%4", "%0") : "+a"(acc) : "v"(a), "v"(b), "v"(c), "v"(d));
    } else {
        if (ZERO) asm volatile(MFMA("%0", "%1", "%2", "0") "\n\t" MFMA("%0", "%3", "%4", "%0") : "=&v"(acc) : "v"(a), "v"(b), "v"(c), "v"(d));
        else asm volatile(MFMA("%0", "%1", "%2", "%0") "\n\t" MFMA("%0", "%3", "%4", "%0") : "+v"(acc) : "v"(a), "v"(b), "v"(c), "v"(d));
    }
    SCHED;
}
template <int J, bool ZERO, int VM> DI void phase(G7& g, int it) {
    constexpr int t = J % 3, c = J / 3, NX = (J + 1) % 6, PV = (J + 4) % 6;
    f16x8 B[5][2];
    B[0][0] = g.B0; B[0][1] = g.B1;
#pragma unroll
    for (int j = 0; j < 4; ++j) {
        const int n = 4 * t + j;
        const bool lastcol = t == 2 && j == 3;
        if (j == 3 && t != 0) { __builtin_amdgcn_s_waitcnt(WCNT(VM, 0)); __builtin_amdgcn_s_barrier(); SCHED; }
#pragma unroll
        for (int r = 0; r < 8; ++r) {
            const int q = j * 8 + r, m = (n & 1) ? 7 - r : r, k = r & 1;
            const f16x8 a0 = m < 4 ? g.L[c][m][k] : g.H[m - 4][k], a1 = m < 4 ? g.L[c][m][k ^ 1] : g.H[m - 4][k ^ 1];
            const int li = t == 2 && j < 3 && r >= 3 && r <= 5 && j * 3 + r - 3 < 8 ? j * 3 + r - 3 : -1;
            if (r == 1 || r == 2 || li >= 0 || (lastcol && r >= 3 && r <= 5)) {
                mfma1<t < 2, ZERO>(g.acc[m][n], a0, B[j][k]);
                if (r <= 2) B[j + 1][r - 1] = j < 3 ? rdB7<J>(g, j + 1, r - 1) : rdB7<NX>(g, 0, r - 1);
                else if (li >= 0) g.L[c ^ 1][li >> 1][li & 1] = rdfrag(g.an, (li & 1) * 4096 + (li >> 1) * 1024);
                else g.H[r == 3 ? 2 : 5 - r][1] = rdfrag(g.an, 4096 + (r == 3 ? 2 : 5 - r) * 1024);
                SCHED;
                mfma1<t < 2, false>(g.acc[m][n], a1, B[j][k ^ 1]);
            } else mfma2<t < 2, ZERO>(g.acc[m][n], a0, B[j][k], a1, B[j][k ^ 1]);
            if (lastcol && r <= 4) { const int row = r < 2 ? 3 : 4 - r, kk = r == 1; g.H[row][kk] = rdfrag(g.an, kk * 4096 + row * 1024); SCHED; }
            if (t == 2 && q == 22) { g.an = vadd(g.blo, g.xs[(2 * c + 3) % 3]); SCHED; }
            if (lastcol && r == 6) { g.an = vadd(g.blo, g.xs[(2 * c + 4) % 3]); SCHED; }
            constexpr int PA[4] = {1, 4, 6, 9}, PB0[4] = {12, 14, 17, 19}, PB1[4] = {2, 9, 13, 19};
            if (t != 1) {
                if (q == PA[0] - 1) { asm volatile("s_add_u32 m0, %0, %1" :: "s"(g.lds + LDS_A - g.xb), "s"(g.xs[(2 * c + t / 2) % 3]) : "memory", "scc"); SCHED; }
                if (q == PA[0]) { dma7<0>(g, g.sA); SCHED; }
                if (q == PA[1]) { dma7<1>(g, g.sA); SCHED; }
                if (q == PA[2]) { dma7<2>(g, g.sA); SCHED; }
                if (q == PA[3]) { dma7<3>(g, g.sA); SCHED; }
                if (q == 21) { PIN_S(g.sA); g.sA += QUAD; PIN_S(g.sA); SCHED; }
                if (q == 26 && J == 0 && !ZERO) { jump_at<31>(g.sA, it, g.nextA); SCHED; }
            }
            if (q == (t != 1 ? PB0[0] : PB1[0]) - 1) { dma_to(g.lds + PV * QUAD); SCHED; }
            if (q == (t != 1 ? PB0[0] : PB1[0])) { dma7<0>(g, g.sB); SCHED; }
            if (q == (t != 1 ? PB0[1] : PB1[1])) { dma7<1>(g, g.sB); SCHED; }
            if (q == (t != 1 ? PB0[2] : PB1[2])) { dma7<2>(g, g.sB); SCHED; }
            if (q == (t != 1 ? PB0[3] : PB1[3])) { dma7<3>(g, g.sB); SCHED; }
            if (q == 20) { PIN_S(g.sB); g.sB += QUAD; PIN_S(g.sB); SCHED; }
            if (q == 26 && J == 1 && !ZERO) { jump_at<3>(g.sB, it, g.ringB); SCHED; }
            if (q == 29 && J == 1 && !ZERO) { jump_at<31>(g.sB, it, g.nextB); SCHED; }
        }
    }
    g.B0 = B[4][0]; g.B1 = B[4][1];
}
template <bool FIRST> DI void trip(G7& g, int it) {
    SCHED;
    phase<0, FIRST, 0>(g, it); phase<1, FIRST, FIRST ? 60 : 16>(g, it); phase<2, FIRST, 12>(g, it);
    phase<3, false, 0>(g, it); phase<4, false, 16>(g, it); phase<5, false, 12>(g, it);
    const unsigned x0 = g.xs[0]; g.xs[0] = g.xs[1]; g.xs[1] = g.xs[2]; g.xs[2] = x0;
}
struct Short { unsigned pv, sb, sx, bn, bx; };
template <int J, bool ZERO, int VM> DI void phase2(G7& g, Short& h, int it) {
    constexpr int t = J & 1, c = J >> 1;
    f16x8 B[5][2];
    B[0][0] = g.B0; B[0][1] = g.B1;
#pragma unroll
    for (int j = 0; j < 4; ++j) {
        const int n = 4 * t + j;
        const bool lastcol = t == 1 && j == 3;
        if (j == 3) { __builtin_amdgcn_s_waitcnt(WCNT(VM, 0)); __builtin_amdgcn_s_barrier(); SCHED; }
#pragma unroll
        for (int r = 0; r < 8; ++r) {
            const int q = j * 8 + r, m = (n & 1) ? 7 - r : r, k = r & 1;
            const f16x8 a0 = m < 4 ? g.L[c][m][k] : g.H[m - 4][k], a1 = m < 4 ? g.L[c][m][k ^ 1] : g.H[m - 4][k ^ 1];
            const int li = t == 1 && j < 3 && r >= 3 && r <= 5 && j * 3 + r - 3 < 8 ? j * 3 + r - 3 : -1;
            if (r == 1 || r == 2 || li >= 0 || (lastcol && r >= 3 && r <= 5)) {
                mfma1<true, ZERO>(g.acc[m][n], a0, B[j][k]);
                if (r <= 2) B[j + 1][r - 1] = j < 3 ? rdfrag(h.bn, (r - 1) * 4096 + (j + 1) * 1024) : rdfrag(h.bx, (r - 1) * 4096);
                else if (li >= 0) g.L[c ^ 1][li >> 1][li & 1] = rdfrag(g.an, (li & 1) * 4096 + (li >> 1) * 1024);
                else g.H[r == 3 ? 2 : 5 - r][1] = rdfrag(g.an, 4096 + (r == 3 ? 2 : 5 - r) * 1024);
                SCHED;
                mfma1<true, false>(g.acc[m][n], a1, B[j][k ^ 1]);
            } else mfma2<true, ZERO>(g.acc[m][n], a0, B[j][k], a1, B[j][k ^ 1]);
            if (lastcol && r <= 4) { const int row = r < 2 ? 3 : 4 - r, kk = r == 1; g.H[row][kk] = rdfrag(g.an, kk * 4096 + row * 1024); SCHED; }
            if (t == 1 && q == 22) { g.an = vadd(g.blo, g.xs[(2 * c + 3) % 3]); SCHED; }
            if (lastcol && r == 6) { g.an = vadd(g.blo, g.xs[(2 * c + 4) % 3]); SCHED; }
            if (q == 0) { asm volatile("s_add_u32 m0, %0, %1" :: "s"(g.lds + LDS_A - g.xb), "s"(g.xs[J % 3]) : "memory", "scc"); SCHED; }
            if (q == 1) { dma7<0>(g, g.sA); SCHED; }
            if (q == 2) { dma7<1>(g, g.sA); SCHED; }
            if (q == 4) { dma7<2>(g, g.sA); SCHED; }
            if (q == 6) { dma7<3>(g, g.sA); SCHED; }
            if (q == 8) { PIN_S(g.sA); g.sA += QUAD; PIN_S(g.sA); SCHED; }
            if (q == 10) { asm volatile("s_add_u32 m0, %0, %1" :: "s"(g.lds), "s"(h.pv) : "memory", "scc"); SCHED; }
            if (q == 11) { dma7<0>(g, g.sB); SCHED; }
            if (q == 13) { dma7<1>(g, g.sB); SCHED; }
            if (q == 16) { dma7<2>(g, g.sB); SCHED; }
            if (q == 18) { dma7<3>(g, g.sB); SCHED; }
            if (q == 19) { PIN_S(g.sB); g.sB += QUAD; PIN_S(g.sB); SCHED; }
            if (q == 20 && J == 2) { jump_at<2>(g.sB, it, g.ringB); SCHED; }
            if (q == 21) { h.pv = h.sb; h.sb = h.sx; PIN_S(h.sx); h.sx = h.sx == 5 * QUAD ? 0 : h.sx + QUAD; PIN_S(h.sx); SCHED; }
            if (q == 29) { h.bn = h.bx; h.bx = vadd(g.blo, h.sx); SCHED; }
        }
    }
    g.B0 = B[4][0]; g.B1 = B[4][1];
    if (J == 3) { const unsigned x0 = g.xs[0]; g.xs[0] = g.xs[1]; g.xs[1] = g.xs[2]; g.xs[2] = x0; }
}
DI void g7_fill(G7& g, unsigned to, unsigned src) { dma_to(g.lds + to); asm volatile("s_nop 4"); dma7<0>(g, src); dma7<1>(g, src); dma7<2>(g, src); dma7<3>(g, src); }
DI void short_tile(G7& g) {
    Short h;
    h.pv = 5 * QUAD; h.sb = 0; h.sx = QUAD; h.bn = g.blo; h.bx = vadd(g.blo, QUAD);
    g7_fill(g, 4 * QUAD, g.sB); g.sB += QUAD;
    SCHED;
    phase2<0, true, 60>(g, h, 0); phase2<1, true, 12>(g, h, 0); phase2<2, false, 12>(g, h, 0); phase2<3, false, 12>(g, h, 0);
    for (int it = 1; it < 32; ++it) { SCHED; phase2<0, false, 12>(g, h, it); phase2<1, false, 12>(g, h, it); phase2<2, false, 12>(g, h, it); phase2<3, false, 12>(g, h, it); }
}
DI void g7_init(G7& g, LAS unsigned char* lds, const char* data, unsigned bytes, int wid) {
    const int lane = lane_now(), wr = wid >> 1, wc = wid & 1;
    const unsigned lbase = (unsigned)(size_t)lds;
    g.rD = __builtin_amdgcn_make_buffer_rsrc((void*)data, 0, bytes, 0x00020000);
    g.lds = lbase + wid * 4096; g.voff = wid * 4096 + lane * 16;
    g.xb = LDS_A + (wr - wc) * 8192; g.blo = lbase + wc * 8192 + lane * 16; g.bhi = g.blo + 3 * QUAD;
    asm volatile("" : "+v"(g.blo), "+v"(g.bhi), "+v"(g.voff));
}
DI void g7_prologue(G7& g, unsigned dA, unsigned dB) {
#pragma unroll
    for (int i = 0; i < 4; ++i) { if (i < 3) g7_fill(g, LDS_A + i * QUAD, dA + i * QUAD); g7_fill(g, i * QUAD, dB + i * QUAD); }
    WAITVM(0); __builtin_amdgcn_s_barrier(); SCHED;
    const unsigned a0 = vadd(g.blo, g.xb);
#pragma unroll
    for (int i = 0; i < 8; ++i) { g.L[0][i >> 1][i & 1] = rdfrag(a0, (i & 1) * 4096 + (i >> 1) * 1024); g.H[i >> 1][i & 1] = rdfrag(a0, QUAD + (i & 1) * 4096 + (i >> 1) * 1024); }
    g.B0 = rdB7<0>(g, 0, 0); g.B1 = rdB7<0>(g, 0, 1);
    __builtin_amdgcn_s_waitcnt(WCNT(63, 0)); __builtin_amdgcn_s_barrier(); SCHED;
    g.sA = dA + 3 * QUAD; g.sB = dB + 4 * QUAD; g.xs[0] = g.xb; g.xs[1] = g.xb + QUAD; g.xs[2] = g.xb + 2 * QUAD; g.an = vadd(g.blo, g.xs[2]);
    PIN_S(g.xs[0]); PIN_S(g.xs[1]); PIN_S(g.xs[2]);
}
#define RD8 "v_accvgpr_read_b32 %0, %8\n\tv_accvgpr_read_b32 %1, %9\n\tv_accvgpr_read_b32 %2, %10\n\tv_accvgpr_read_b32 %3, %11\n\t" \
            "v_accvgpr_read_b32 %4, %12\n\tv_accvgpr_read_b32 %5, %13\n\tv_accvgpr_read_b32 %6, %14\n\tv_accvgpr_read_b32 %7, %15"
#define RD8_OUT "=&v"(z[0]), "=&v"(z[1]), "=&v"(z[2]), "=&v"(z[3]), "=&v"(z[4]), "=&v"(z[5]), "=&v"(z[6]), "=&v"(z[7])
#define RD8_IN(m, r) "a"(g.acc[m][0][r]), "a"(g.acc[m][1][r]), "a"(g.acc[m][2][r]), "a"(g.acc[m][3][r]), "a"(g.acc[m][4][r]), "a"(g.acc[m][5][r]), "a"(g.acc[m][6][r]), "a"(g.acc[m][7][r])
template <int NQ> DI void g7_epilogue(G7& g, __amdgpu_buffer_rsrc_t rO, unsigned so, unsigned rowb, float sc) {
    const int lz = lane_now();
    const unsigned vo = 4 * (lz >> 4) * rowb + 16 * (lz & 15);
    if (NQ == 3) asm volatile("s_nop 15" : "+v"(g.acc[0][11]), "+v"(g.acc[1][11]), "+v"(g.acc[2][11]), "+v"(g.acc[3][11]));
    else asm volatile("s_nop 15");
    float z[8];
    asm volatile(RD8 : RD8_OUT : RD8_IN(0, 0));
#pragma unroll
    for (int i = 0; i < 32; ++i) {
        const int m = i >> 2, r = i & 3, m2 = (i + 1 < 32 ? i + 1 : i) >> 2, r2 = (i + 1 < 32 ? i + 1 : i) & 3;
        f32x4 a[3], t[3];
        a[0] = (f32x4){z[0], z[1], z[2], z[3]}; a[1] = (f32x4){z[4], z[5], z[6], z[7]};
        if (NQ == 3) a[2] = (f32x4){g.acc[m][8][r], g.acc[m][9][r], g.acc[m][10][r], g.acc[m][11][r]};
#pragma unroll
        for (int q = 0; q < NQ; ++q) t[q] = a[q] * (-1.44269504088896f / SCALE);
        SCHED;
#pragma unroll
        for (int e = 0; e < 4 * NQ; ++e) t[e >> 2][e & 3] = __builtin_amdgcn_fmed3f(t[e >> 2][e & 3], -86.5617024533378f, 86.5617024533378f);
        SCHED;
#pragma unroll
        for (int e = 0; e < 4 * NQ; ++e) t[e >> 2][e & 3] = __builtin_amdgcn_exp2f(t[e >> 2][e & 3]);
        SCHED;
#pragma unroll
        for (int q = 0; q < NQ; ++q) t[q] = __builtin_elementwise_fma(t[q], (f32x4){sc, sc, sc, sc}, (f32x4){sc, sc, sc, sc});
        SCHED;
#pragma unroll
        for (int e = 0; e < 4 * NQ; ++e) t[e >> 2][e & 3] = __builtin_amdgcn_rcpf(t[e >> 2][e & 3]);
        SCHED;
#pragma unroll
        for (int q = 0; q < NQ; ++q) t[q] = a[q] * t[q];
        const unsigned o = so + (m * 16 + r) * rowb;
        if (NQ == 3) asm volatile("buffer_store_dwordx4 %16, %19, %20, %21 offen nt\n\tbuffer_store_dwordx4 %17, %19, %20, %21 offen offset:256 nt\n\t"
                                  "buffer_store_dwordx4 %18, %19, %20, %21 offen offset:512 nt\n\t" RD8
                                  : RD8_OUT : RD8_IN(m2, r2), "v"(t[0]), "v"(t[1]), "v"(t[2]), "v"(vo), "s"(rO), "s"(o) : "memory");
        else asm volatile("buffer_store_dwordx4 %16, %18, %19, %20 offen nt\n\tbuffer_store_dwordx4 %17, %18, %19, %20 offen offset:256 nt\n\t" RD8
                          : RD8_OUT : RD8_IN(m2, r2), "v"(t[0]), "v"(t[1]), "v"(vo), "s"(rO), "s"(o) : "memory");
        SCHED;
    }
}
DI int lead_ones(bool ok) { const unsigned long long m = ~__ballot(ok); return m ? __builtin_ctzll(m) : 64; }

struct Ready {
    char* ws; unsigned call;
    __amdgpu_buffer_rsrc_t rF; LAS unsigned char* zone;
    int pa, pb, n; bool pending; float one;
    DI void poll(int need) {
        const unsigned long long t0 = __builtin_amdgcn_s_memrealtime();
        const int lane = lane_now();
        for (;;) {
            const unsigned a = ld_sys(ws + OFF_F + pa * 256 + lane * 4), b = ld_sys(ws + OFF_F + pb * 256 + lane * 4);
            n = lead_ones(lane < 8 || (a >= call && b >= call));
            if (n >= need) break;
            if (give_up(ws, call, t0)) { n = 64; one = __builtin_nanf(""); break; }
            __builtin_amdgcn_s_sleep(16);
        }
    }
    DI void request() {
        const int lane = lane_now();
        __builtin_amdgcn_raw_ptr_buffer_load_lds(rF, (LAS void*)zone, 4, lane * 4, pa * 256, 0, AUX_SYS);
        __builtin_amdgcn_raw_ptr_buffer_load_lds(rF, (LAS void*)(zone + 256), 4, lane * 4, pb * 256, 0, AUX_SYS);
        pending = true;
    }
    DI int plan(int it, int stop) {
        if (n == 64) return stop;
        const int safe = (n - 4) >> 1;
        if (safe <= it + 1) { request(); return it + 1; }
        return safe - 1 < stop ? safe - 1 : stop;
    }
    DI void check(int need) {
        if (pending) {
            const int lane = lane_now();
            const unsigned a = *(const volatile LAS unsigned*)(zone + lane * 4), b = *(const volatile LAS unsigned*)(zone + 256 + lane * 4);
            n = lead_ones(lane < 8 || (a >= call && b >= call)); pending = false;
        }
        if (n < need) poll(need);
        asm volatile("" ::: "memory");
    }
};

DI void gemm_wg(Params& p, LAS unsigned char* lds, int wid, int band, int S) {
    const int wr = wid >> 1, wc = wid & 1, rank = p.rank;
    char* const ws = p.ws[rank];
    const unsigned call = convert_shards(p, lds, wid, S * (16 / RB) + band);
    int T = S;
    G7 g;
    g7_init(g, lds, ws + OFF_DATA, DATAB, wid);
    Ready R;
    R.ws = ws; R.call = call; R.rF = __builtin_amdgcn_make_buffer_rsrc((void*)(ws + OFF_F), 0, NPAN * 256, 0x00020000);
    R.zone = lds + LDS_ZONE + wid * 512; R.pending = false; R.one = SCALE;
    if (!wait_words(ws, ws + OFF_EV + (3 * NR + rank) * 4, 1, 4, call, call) || rfl(ld_ag(ws + OFF_TMO)) == call) R.one = __builtin_nanf("");
    R.pa = RB * band + (T & (RB - 1)); R.pb = 16 + colblock(T >> RBL, rank); R.n = 8;
    g7_prologue(g, rfl(own_off(R.pa, 0)), rfl(own_off(R.pb, 0)));
    const __amdgpu_buffer_rsrc_t rO = __builtin_amdgcn_make_buffer_rsrc((void*)p.out[rank], 0, OUTB, 0x00020000);
    unsigned tprev = 0;
    int li = T >> RBL;
    for (int ti = 0;; ++ti) {
        const int pm = R.pa, cb = R.pb - 16;
        S += NWB;
        const int job = S - PSLOT;
        if ((unsigned)job < (unsigned)NJOB) S += NWB;
        const bool has_next = S < NROUND * NWB, deal = has_next && S >= (NROUND - 1) * NWB;
        T = S < PSLOT ? S : S - NJOB;
        int qa = pm, ql = li;
        if (has_next && !deal) { qa = RB * band + (T & (RB - 1)); ql = T >> RBL; }
        g.ringB = rfl(ring_off(li)); g.nextA = rfl(own_off(qa, 0)); g.nextB = rfl(own_off(16 + colblock(ql, rank), 0));
        {
            const unsigned now = (unsigned)__builtin_amdgcn_s_memrealtime();
            if (deal && blockIdx.x >> 3 == NG / 2 && wid == 0 && lane_now() == 0)
                __hip_atomic_store((GAS unsigned long long*)(ws + OFF_ETA + (blockIdx.x & 7) * 8), (unsigned long long)call << 32 | (3 * now - 2 * tprev), __ATOMIC_RELAXED, __HIP_MEMORY_SCOPE_AGENT);
            tprev = now;
        }
        R.check(4);
        int lim = R.plan(0, deal ? 29 : 32);
        trip<true>(g, 0);
        for (int it = 1; it < 32;) {
            if (it >= lim) {
                R.check(it < 30 ? 2 * it + 4 : 64);
                lim = R.plan(it, deal && it < 29 ? 29 : 32);
                if (deal && it == 29) {
                    const int lane = lane_now();
                    const unsigned long long t0 = __builtin_amdgcn_s_memrealtime();
                    unsigned long long e;
                    int place = blockIdx.x & 7;
                    while (!__all((unsigned)((e = __hip_atomic_load((GAS unsigned long long*)(ws + OFF_ETA + (lane & 7) * 8), __ATOMIC_RELAXED, __HIP_MEMORY_SCOPE_AGENT)) >> 32) == call)) {
                        if (give_up(ws, call, t0)) { R.one = __builtin_nanf(""); e = 0; break; }
                        __builtin_amdgcn_s_sleep(8);
                    }
                    const int d = (int)((unsigned)e - __builtin_amdgcn_readlane((unsigned)e, place));
                    place = __builtin_popcount((unsigned)__ballot(d < 0 || (d == 0 && lane < place)) & 0xffu);
                    constexpr int T4 = (NROUND - 1) * NWB - NJOB, NF = RB * (NL - 2) - T4;
                    const int P = place * NG + (blockIdx.x >> 3), bd = P / NF;
                    if (bd < 16 / RB) { const int t = T4 + P - bd * NF; qa = RB * bd + (t & (RB - 1)); ql = t >> RBL; }
                    else { const int t = P - 16 / RB * NF; qa = t & 15; ql = NL - 2 + (t >> 4); }
                    g.nextA = rfl(own_off(qa, 0)); g.nextB = rfl(own_off(16 + colblock(ql, rank), 0));
                }
            }
            do trip<false>(g, it); while (++it < lim);
        }
        R.pa = qa; R.pb = 16 + colblock(ql, rank);
        R.request();
        g7_epilogue<3>(g, rO, rfl((pm * 256 + wr * 128) * ROWB + (col0(cb) + wc * 192) * 4), ROWB, R.one);
        WAITVM(60);
        if (wid == 0 && lane_now() == 0) __hip_atomic_fetch_add((gu32*)(ws + OFF_FIN + li * 4), 1u, __ATOMIC_RELAXED, __HIP_MEMORY_SCOPE_AGENT);
        if (li < CPR) {
            WAITVM(0);
            BARRIER;
            if (wid == 0) {
                __builtin_amdgcn_fence(__ATOMIC_RELEASE, "agent");
                WAITVM(0);
                if (lane_now() == 0) st_sys(ws + OFF_AVC + (li * 16 + pm) * 4, call);
            }
        }
        if ((unsigned)job < (unsigned)NJOB) { pack_tile(p, lds + LDS_ZONE + 4096 + wid * 1024, call, job % CPR * 16 + RB * band + job / CPR * 4 + wid); }
        if (!has_next) break;
        li = ql;
        if (li >= NL - 2) {
            g.ringB = rfl(ring_off(li));
            R.check(64);
            short_tile(g);
            g7_epilogue<2>(g, rO, rfl((R.pa * 256 + wr * 128) * ROWB + (col0(R.pb - 16) + wc * 128) * 4), ROWB, R.one);
            break;
        }
    }
    __builtin_amdgcn_s_waitcnt(WCNT(0, 0));
}

__global__ __launch_bounds__(256, 1) void fused(Args) {
    extern __shared__ __attribute__((aligned(16))) unsigned char shm[];
    Params& p = *(Params*)__builtin_amdgcn_kernarg_segment_ptr();
    LAS unsigned char* lds = (LAS unsigned char*)shm;
    char* const ws = p.ws[p.rank];
    const int wid = rfl(threadIdx.x >> 6);
    const int grp = blockIdx.x & 7, idx = blockIdx.x >> 3;
    if (idx < NG) gemm_wg(p, lds, wid, (grp & 1 ? 0 : 8 / RB) + (grp >> RBL), (grp >> 1 & (RB / 2 - 1)) * NG + idx);
    else {
        const int cw = (idx - NG) * 8 + grp;
        bool clear;
        const unsigned call = draw_call(ws, wid, clear);
        if (wid < 3) fetch_wave(p, lds + wid * 32768, call, wid, cw * 3 + wid);
    }
}
}

extern "C" void kernel_launch(int rank, int nranks, ncclComm_t comm, hipStream_t stream,
                              void* const* d_in, const int* in_sizes, int n_in,
                              void* d_out, int out_size,
                              const int* in_dtypes, int out_dtype,
                              void* const* d_out_all,
                              void* const* d_ws_all, size_t ws_bytes) {
    if (nranks != NR || (unsigned)rank >= (unsigned)NR || n_in != 2 || ws_bytes < WS_NEED || in_sizes[0] != MM * KS || in_sizes[1] != KS * NN || out_size != MM * NN) {
        fprintf(stderr, "kernel_launch: unexpected problem shape\n");
        return;
    }
    static bool attr_set[NR];
    if (!attr_set[rank]) {
        HIP_CHECK(hipFuncSetAttribute((const void*)fused, hipFuncAttributeMaxDynamicSharedMemorySize, LDS_BYTES));
        attr_set[rank] = true;
    }
    Args p;
    p.x = (const float*)d_in[0]; p.w = (const float*)d_in[1];
    for (int i = 0; i < NR; ++i) { p.ws[i] = (char*)d_ws_all[i]; p.out[i] = (float*)d_out_all[i]; }
    p.rank = rank; p.pad = 0;
    hipLaunchKernelGGL(fused, dim3(256), dim3(256), LDS_BYTES, stream, p);
    HIP_CHECK(hipGetLastError());
}
```
